```python
import numpy as np

kernel_name = "gpu_ag_gemm_m4096_k4096_n4096_f32_none"


ABSMAX_RTOL = 0.001

COMPUTE_DTYPE = "float32"
EPILOGUE = "none"
IN_QUANT = None
OUT_QUANT = None
IN_DTYPES = {"x": "float32", "w_mat": "float32"}
WORLD_SIZE = 8

def setup_inputs():

    x = np.random.randn(4096, 4096).astype(np.float32)
    w_mat = (np.random.randn(4096, 4096) / np.sqrt(4096)).astype(np.float32)
    return {"x": x, "w_mat": w_mat}

def _epilogue(y):
    return y

def reference(x, w_mat):
    y = (x @ w_mat).astype(np.float32)
    return _epilogue(y)
```

```cpp
#include <hip/hip_runtime.h>
#include <rccl/rccl.h>
#include <cstdio>
#include <cstring>
#include <algorithm>
#include <vector>

#define HIP_CHECK(expr) do { hipError_t _e = (expr); if (_e != hipSuccess) { \
    fprintf(stderr, "HIP error %s at %s:%d\n", hipGetErrorString(_e), __FILE__, __LINE__); \
    return; } } while (0)

#define P_NI 23
#define P_NCOPY 48
#define P_WI 40
#define P_WX 40
#define P_BATCH_I {8, 8, 4, 3}
#define P_BATCH_X {5, 4}
#define P_SMALL_LAST 1
#define P_CLSMODE 0
#define P_NOGEMM 0

namespace agg {

#define GAS __attribute__((address_space(1)))
#define LAS __attribute__((address_space(3)))
typedef unsigned v4u __attribute__((ext_vector_type(4)));
typedef float f32x4 __attribute__((ext_vector_type(4)));
typedef _Float16 h8 __attribute__((ext_vector_type(8)));
typedef GAS unsigned gu32;

constexpr int NR = 8;
constexpr int KD = 4096, SH = 512;
constexpr int PANEL = 131072;
constexpr int NAP = 128, NBP = 64, NPAN = NAP + NBP;
constexpr int NCH = 16, CHUNK = 8192;
constexpr int NI = P_NI, NX = 32 - NI;
constexpr int GRID = 256, NWAVE = 5, NTHR = NWAVE * 64;
constexpr int NCOPY = P_NCOPY;
constexpr int WI = P_WI;
constexpr int WW = NCOPY * NWAVE - 1 - WI;
constexpr int WX = P_WX;
constexpr int MAXT = 4;
static_assert(NCOPY % 8 == 0 && WW >= WX && WX > 0 && WI > 0, "roles");

constexpr unsigned OFF_CNT = 0;
constexpr unsigned OFF_TMO = 16384;
constexpr unsigned OFF_PF = 32768;
constexpr unsigned OFF_YF = 65536;
constexpr unsigned YF_MAX = 4096;
constexpr unsigned OFF_BAR = 98304;
constexpr unsigned OFF_STG = 1u << 20;
constexpr unsigned STG_BYTES = (unsigned)NPAN * PANEL;
constexpr unsigned OFF_DBG = OFF_STG + 2 * STG_BYTES;
constexpr unsigned WS_USED = OFF_DBG + (1u << 20);
constexpr unsigned OUT_BYTES = 4096u * SH * 4u;
constexpr unsigned long long DEADLINE = 50000000ull;
constexpr int AUX_SYS = 17;
constexpr int RSRC3 = 0x00020000;

constexpr int LDS_RED = 65536;
constexpr int LDS_CTL = 2 * LDS_RED;
constexpr int LDS_BYTES = LDS_CTL + 64;

struct Params {
  const float* x;
  const float* w;
  char* out[NR];
  char* ws[NR];
  int rank;
  int ny;
  unsigned short xi_items[NI];
  unsigned short xx_items[NX + 1];
  unsigned tiles[(GRID - NCOPY) * MAXT];
};

__host__ __device__ __forceinline__ int member(int cls, int i) { return P_CLSMODE ? (2 * i + cls) : (4 * cls + i); }
__host__ __device__ __forceinline__ int cls_of(int r) { return P_CLSMODE ? (r & 1) : (r >> 2); }
__host__ __device__ __forceinline__ int idx_of(int r) { return P_CLSMODE ? (r >> 1) : (r & 3); }

#define DRAIN() asm volatile("s_waitcnt vmcnt(0)" ::: "memory")
#define CBAR() asm volatile("" ::: "memory")

__device__ __forceinline__ __amdgpu_buffer_rsrc_t mk(const void* p, unsigned bytes) {
  return __builtin_amdgcn_make_buffer_rsrc((void*)p, 0, (int)bytes, RSRC3);
}
__device__ __forceinline__ unsigned pk(float a, float b) {
  typedef _Float16 h2 __attribute__((ext_vector_type(2)));
  h2 v = {(_Float16)a, (_Float16)b};
  return __builtin_bit_cast(unsigned, v);
}
__device__ __forceinline__ unsigned ld_sys(const char* p) {
  return __hip_atomic_load((gu32*)p, __ATOMIC_RELAXED, __HIP_MEMORY_SCOPE_SYSTEM);
}
__device__ __forceinline__ void st_sys(char* p, unsigned v) {
  __hip_atomic_store((gu32*)p, v, __ATOMIC_RELAXED, __HIP_MEMORY_SCOPE_SYSTEM);
}

#define STAMP(slot) do {} while (0)

__device__ __forceinline__ void copy_x(const Params& p, int rt, int c, int dcls, int idx, unsigned call,
                                       unsigned stg, int lane) {
  const GAS f32x4* src = (const GAS f32x4*)(p.x + (size_t)(rt * 16 + (lane & 15)) * KD + c * 256 + 8 * (lane >> 4));
  f32x4 v[16];
#pragma unroll
  for (int s = 0; s < 8; ++s) { v[2 * s] = src[s * 8]; v[2 * s + 1] = src[s * 8 + 1]; }
  __builtin_amdgcn_sched_barrier(0);
  const int pan = rt * 4 + idx;
  const unsigned so = stg + (unsigned)pan * PANEL + (unsigned)c * CHUNK;
  __amdgpu_buffer_rsrc_t rs[4];
#pragma unroll
  for (int d = 0; d < 4; ++d) rs[d] = mk(p.ws[member(dcls, d) & 7], WS_USED);
#pragma unroll
  for (int s = 0; s < 8; ++s) {
    v4u h = {pk(v[2 * s].x, v[2 * s].y), pk(v[2 * s].z, v[2 * s].w), pk(v[2 * s + 1].x, v[2 * s + 1].y),
             pk(v[2 * s + 1].z, v[2 * s + 1].w)};
#pragma unroll
    for (int d = 0; d < 4; ++d) __builtin_amdgcn_raw_buffer_store_b128(h, rs[d], lane * 16 + s * 1024, so, AUX_SYS);
  }
  DRAIN();
  if (lane == 0) {
#pragma unroll
    for (int d = 0; d < 4; ++d)
      __builtin_amdgcn_raw_buffer_store_b32(call, rs[d], 0, OFF_PF + (pan * NCH + c) * 4, AUX_SYS);
  }
}

__device__ __forceinline__ void copy_w(const Params& p, int g, int c, int cls, int idx, unsigned call,
                                       unsigned stg, int lane) {
  const GAS float* src = (const GAS float*)(p.w + (size_t)(c * 256 + 8 * (lane >> 4)) * SH + g * 16 + (lane & 15));
  float v[64];
#pragma unroll
  for (int s = 0; s < 8; ++s)
#pragma unroll
    for (int j = 0; j < 8; ++j) v[s * 8 + j] = src[(s * 32 + j) * SH];
  __builtin_amdgcn_sched_barrier(0);
  const int pl = NAP + g, pr = NAP + 32 + idx * 8 + (g & 7);
  const __amdgpu_buffer_rsrc_t rl = mk(p.ws[p.rank & 7], WS_USED);
  const __amdgpu_buffer_rsrc_t rr = mk(p.ws[member(cls ^ 1, g >> 3) & 7], WS_USED);
  const unsigned sl = stg + (unsigned)pl * PANEL + (unsigned)c * CHUNK;
  const unsigned sr = stg + (unsigned)pr * PANEL + (unsigned)c * CHUNK;
#pragma unroll
  for (int s = 0; s < 8; ++s) {
    v4u h = {pk(v[s * 8], v[s * 8 + 1]), pk(v[s * 8 + 2], v[s * 8 + 3]), pk(v[s * 8 + 4], v[s * 8 + 5]),
             pk(v[s * 8 + 6], v[s * 8 + 7])};
    __builtin_amdgcn_raw_buffer_store_b128(h, rr, lane * 16 + s * 1024, sr, AUX_SYS);
    __builtin_amdgcn_raw_buffer_store_b128(h, rl, lane * 16 + s * 1024, sl, AUX_SYS);
  }
  DRAIN();
  if (lane == 0) {
    __builtin_amdgcn_raw_buffer_store_b32(call, rr, 0, OFF_PF + (pr * NCH + c) * 4, AUX_SYS);
    __builtin_amdgcn_raw_buffer_store_b32(call, rl, 0, OFF_PF + (pl * NCH + c) * 4, AUX_SYS);
  }
}

struct Ctl { unsigned prog, sync, bad, pad; };

__device__ __forceinline__ void need(volatile LAS unsigned* prog, unsigned& have, unsigned n) {
  while (have < n) {
    have = __builtin_amdgcn_readfirstlane(*prog);
    if (have < n) __builtin_amdgcn_s_sleep(1);
  }
  CBAR();
}

template <int MT, int NT>
__device__ __forceinline__ void gemm_tile(const Params& p, LAS char* lds, unsigned desc, int t, unsigned& have,
                                          unsigned call, unsigned stg, int wave, int lane, int cls, int idx) {
  constexpr int D = (MT + NT <= 4) ? 8 : 4;
  constexpr int NS = 32;
  volatile LAS Ctl* ctl = (volatile LAS Ctl*)(lds + LDS_CTL);
  const int a0 = desc & 127, b0 = (desc >> 7) & 63, yid = (desc >> 13) & 2047;
  const __amdgpu_buffer_rsrc_t rs = mk(p.ws[p.rank & 7], WS_USED);
  const unsigned ab = stg + (unsigned)a0 * PANEL + wave * 1024, bb = stg + (unsigned)(NAP + b0) * PANEL + wave * 1024;
  const int voff = lane * 16;
  const unsigned base = (unsigned)t * NCH;
  f32x4 acc[MT][NT];
#pragma unroll
  for (int m = 0; m < MT; ++m)
#pragma unroll
    for (int n = 0; n < NT; ++n) acc[m][n] = f32x4{0.f, 0.f, 0.f, 0.f};
  v4u ra[D][MT], rb[D][NT];
#define LOADSTEP(d, i) do { \
    _Pragma("unroll") for (int m = 0; m < MT; ++m) \
      ra[d][m] = __builtin_amdgcn_raw_buffer_load_b128(rs, voff, ab + m * PANEL + (i) * 4096, AUX_SYS); \
    _Pragma("unroll") for (int n = 0; n < NT; ++n) \
      rb[d][n] = __builtin_amdgcn_raw_buffer_load_b128(rs, voff, bb + n * PANEL + (i) * 4096, AUX_SYS); \
    __builtin_amdgcn_sched_barrier(0); } while (0)
#define MMASTEP(d) do { \
    _Pragma("unroll") for (int m = 0; m < MT; ++m) \
    _Pragma("unroll") for (int n = 0; n < NT; ++n) \
      acc[m][n] = __builtin_amdgcn_mfma_f32_16x16x32_f16(__builtin_bit_cast(h8, rb[d][n]), \
                                                         __builtin_bit_cast(h8, ra[d][m]), acc[m][n], 0, 0, 0); \
    __builtin_amdgcn_sched_barrier(0); } while (0)
#pragma unroll
  for (int d = 0; d < D; ++d) {
    if ((d & 1) == 0) need(&ctl->prog, have, base + (d >> 1) + 1);
    if (d == 0) STAMP(8 + t * 8 + 1);
    LOADSTEP(d, d);
  }
#pragma unroll 1
  for (int s0 = D; s0 < NS; s0 += D) {
#pragma unroll
    for (int d = 0; d < D; ++d) {
      MMASTEP(d);
      if ((d & 1) == 0) need(&ctl->prog, have, base + ((s0 + d) >> 1) + 1);
      LOADSTEP(d, s0 + d);
    }
  }
#pragma unroll
  for (int d = 0; d < D; ++d) MMASTEP(d);
#undef LOADSTEP
#undef MMASTEP
  STAMP(8 + t * 8 + 2);
  asm volatile("" : "+v"(lane));
  constexpr int C4 = 4 * NT, IMG = 64 * MT * NT;
  LAS f32x4* red = (LAS f32x4*)(lds + (t & 1) * LDS_RED);
#pragma unroll
  for (int m = 0; m < MT; ++m)
#pragma unroll
    for (int n = 0; n < NT; ++n) {
      const int row = m * 16 + (lane & 15), c4 = n * 4 + (lane >> 4);
      red[wave * IMG + row * C4 + (c4 ^ (row & (C4 - 1)))] = acc[m][n];
    }
  asm volatile("s_waitcnt lgkmcnt(0)" ::: "memory");
  if (lane == 0) __hip_atomic_fetch_add((LAS unsigned*)&ctl->sync, 1u, __ATOMIC_RELAXED, __HIP_MEMORY_SCOPE_WORKGROUP);
  while (__builtin_amdgcn_readfirstlane(ctl->sync) < 4u * (t + 1)) __builtin_amdgcn_s_sleep(0);
  CBAR();
  const bool bad = ctl->bad != 0;
  const bool remote = b0 >= 32;
  const int dst = remote ? member(cls ^ 1, (b0 - 32) >> 3) : p.rank;
  const int col0 = remote ? idx * 128 + ((b0 - 32) & 7) * 16 : b0 * 16;
  const __amdgpu_buffer_rsrc_t ro = mk(p.out[dst & 7], OUT_BYTES);
  constexpr int Q = MT * NT / 4;
#pragma unroll
  for (int q = 0; q < Q; ++q) {
    const int e = (wave * Q + q) * 64 + lane, row = e / C4, c4 = e % C4;
    const int li = row * C4 + (c4 ^ (row & (C4 - 1)));
    f32x4 s = (red[li] + red[IMG + li]) + (red[2 * IMG + li] + red[3 * IMG + li]);
    if (bad) s = f32x4{__builtin_nanf(""), __builtin_nanf(""), __builtin_nanf(""), __builtin_nanf("")};
    const int a = a0 + (row >> 4), rt = a >> 2;
    const int grow = member(rt < NI ? cls : cls ^ 1, a & 3) * SH + rt * 16 + (row & 15);
    __builtin_amdgcn_raw_buffer_store_b128(__builtin_bit_cast(v4u, s), ro, (grow * SH + col0 + c4 * 4) * 4, 0, AUX_SYS);
  }
  if (remote) {
    DRAIN();
    if (lane == 0) st_sys(p.ws[dst & 7] + OFF_YF + (unsigned)(idx * p.ny + yid * 4 + wave) % YF_MAX * 4, call);
  }
  STAMP(8 + t * 8 + 3);
}

__global__ __launch_bounds__(NTHR, 2) void ag_gemm(const Params p) {
  extern __shared__ __attribute__((aligned(16))) char smem[];
  LAS char* lds = (LAS char*)smem;
  const int lane = threadIdx.x & 63;
  const int wave = __builtin_amdgcn_readfirstlane(threadIdx.x >> 6);
  const int blk = blockIdx.x;
  char* myws = p.ws[p.rank & 7];
  const int cls = cls_of(p.rank), idx = idx_of(p.rank);
  char* cnt = myws + OFF_CNT + (blk * 8 + wave) * 4;
  const unsigned call = __builtin_amdgcn_readfirstlane(ld_sys(cnt)) + 1u;
  if (lane == 0) st_sys(cnt, call);
  const unsigned stg = OFF_STG + (call & 1u) * STG_BYTES;
  STAMP(0);

  if (blk < NCOPY) {
    const int cw = wave * NCOPY + blk;
    if (cw < WI) {
      for (int it = cw; it < NI * NCH; it += WI) {
        const int e = p.xi_items[it >> 4], f = e >> 8, n = e & 255, r = it - f * NCH;
        copy_x(p, f + r % n, r / n, cls, idx, call, stg, lane);
      }
      STAMP(1);
    } else if (cw < WI + WW) {
      const int u = cw - WI;
      for (int it = u; it < 32 * NCH; it += WW) {
        const int g = it & 31;
        copy_w(p, (g & 3) * 8 + (g >> 2), it >> 5, cls, idx, call, stg, lane);
      }
      STAMP(2);
      if (u < WX) {
        for (int it = u; it < NX * NCH; it += WX) {
          const int e = p.xx_items[it >> 4], f = e >> 8, n = e & 255, r = it - f * NCH;
          copy_x(p, NI + f + r % n, r / n, cls ^ 1, idx, call, stg, lane);
        }
      }
      STAMP(1);
    } else {
      const __amdgpu_buffer_rsrc_t rs = mk(myws, WS_USED);
      const unsigned long long t0 = __builtin_amdgcn_s_memrealtime();
      const int nw = 4 * p.ny; const unsigned fo = OFF_YF;
      int pos = 0;
      while (pos < nw) {
        const int wd = pos + lane * 4;
        const v4u f = __builtin_amdgcn_raw_buffer_load_b128(rs, wd * 4, fo, AUX_SYS);
        const bool ok = wd >= nw || ((int)(f.x - call) >= 0 && (int)(f.y - call) >= 0 && (int)(f.z - call) >= 0 &&
                                     (int)(f.w - call) >= 0);
        if (__all(ok)) { pos += 256; continue; }
        if (ld_sys(myws + OFF_TMO) == call || __builtin_amdgcn_s_memrealtime() - t0 > DEADLINE) {
          if (lane == 0) {
            st_sys(myws + OFF_TMO, call);
            st_sys(p.out[p.rank & 7] + (size_t)p.rank * SH * SH * 4, 0x7fc00000u);
          }
          break;
        }
        __builtin_amdgcn_s_sleep(2);
      }
      STAMP(1);
    }
    return;
  }

  volatile LAS Ctl* ctl = (volatile LAS Ctl*)(lds + LDS_CTL);
  if (threadIdx.x == 0) { ctl->prog = 0; ctl->sync = 0; ctl->bad = 0; }
  __syncthreads();
  const unsigned* tl = p.tiles + (blk - NCOPY) * MAXT;
  if (wave == 4) {
    const unsigned long long t0 = __builtin_amdgcn_s_memrealtime();
    for (int t = 0; t < MAXT; ++t) {
      const unsigned desc = tl[t];
      if (!(desc >> 31)) break;
      const int a0 = desc & 127, b0 = (desc >> 7) & 63, small = (desc >> 24) & 1;
      const int ln = small ? (lane & 31) : lane;
      const char* fa = myws + OFF_PF + (a0 * NCH + ln) * 4;
      const char* fb = myws + OFF_PF + ((NAP + b0) * NCH + ln) * 4;
      unsigned pub = 0;
      while (true) {
        const unsigned va = ld_sys(fa), vb = ld_sys(fb), tm = ld_sys(myws + OFF_TMO);
        unsigned long long m = __ballot((int)(va - call) >= 0 && (int)(vb - call) >= 0);
        m &= m >> 32; m &= m >> 16;
        const unsigned n = __builtin_ctz(~(unsigned)m | 0x10000u);
        if (tm == call || __builtin_amdgcn_s_memrealtime() - t0 > DEADLINE) {
          if (lane == 0) { st_sys(myws + OFF_TMO, call); ctl->bad = 1; }
          asm volatile("s_waitcnt lgkmcnt(0)" ::: "memory");
          if (lane == 0) ctl->prog = 0xffffffffu;
          return;
        }
        if (n > pub) { pub = n; if (lane == 0) ctl->prog = t * NCH + n; }
        if (n == NCH) break;
        __builtin_amdgcn_s_sleep(2);
      }
      STAMP(8 + t * 8 + 4);
    }
    return;
  }
  unsigned have = 0;
  for (int t = 0; t < MAXT; ++t) {
    const unsigned desc = tl[t];
    if (!(desc >> 31)) break;
    STAMP(8 + t * 8);
    if ((desc >> 24) & 1) gemm_tile<2, 2>(p, lds, desc, t, have, call, stg, wave, lane, cls, idx);
    else gemm_tile<4, 4>(p, lds, desc, t, have, call, stg, wave, lane, cls, idx);
  }
  STAMP(1);
}

static bool build_params(Params& p) {
  const int bi[] = P_BATCH_I, bx[] = P_BATCH_X;
  const int nbi = sizeof(bi) / sizeof(int), nbx = sizeof(bx) / sizeof(int);
  struct Batch { int first, n; double t0, t1; bool small; };
  std::vector<Batch> bs;
  const double per = 2.45;
  int f = 0;
  for (int b = 0; b < nbi; ++b) {
    for (int i = 0; i < bi[b]; ++i) if (f + i < NI) p.xi_items[f + i] = (unsigned short)((f << 8) | bi[b]);
    bs.push_back({f, bi[b], f * per, (f + bi[b]) * per, P_SMALL_LAST && b == nbi - 1});
    f += bi[b];
  }
  if (f != NI) return false;
  f = 0;
  for (int b = 0; b < nbx; ++b) {
    for (int i = 0; i < bx[b]; ++i) if (f + i < NX) p.xx_items[f + i] = (unsigned short)((f << 8) | bx[b]);
    bs.push_back({NI + f, bx[b], 8 * per + f * per * 1.4, 8 * per + (f + bx[b]) * per * 1.4, false});
    f += bx[b];
  }
  if (f != NX) return false;
  for (auto& b : bs) b.t1 = std::max(b.t1, 8 * per);
  std::stable_sort(bs.begin(), bs.end(), [](const Batch& a, const Batch& b) { return a.t1 < b.t1; });
  const int G = (GRID - NCOPY) / 8;
  int ycount[4] = {0, 0, 0, 0};
  for (int x = 0; x < 8; ++x) {
    std::vector<double> fr(G, 0.0);
    std::vector<int> cnt(G, 0);
    for (auto& b : bs) {
      for (int h = 1; h >= 0; --h)
        for (int rt = b.first; rt < b.first + b.n; ++rt) {
          const int nsub = b.small ? 4 : 1;
          for (int s = 0; s < nsub; ++s) {
            const int a0 = rt * 4 + (b.small ? (s & 1) * 2 : 0);
            const int b0 = h * 32 + x * 4 + (b.small ? (s >> 1) * 2 : 0);
            int best = -1;
            for (int j = 0; j < G; ++j)
              if (cnt[j] < MAXT && (best < 0 || fr[j] < fr[best])) best = j;
            if (best < 0) return false;
            const double dur = b.small ? 6.0 : 12.0;
            fr[best] = std::max(fr[best] + dur, b.t1 + 2.0);
            unsigned yid = 0;
            if (h) yid = ycount[(b0 - 32) >> 3]++;
            p.tiles[(best * 8 + x) * MAXT + cnt[best]++] =
                0x80000000u | ((b.small ? 1u : 0u) << 24) | (yid << 13) | ((unsigned)b0 << 7) | (unsigned)a0;
          }
        }
    }
  }
  if (ycount[0] != ycount[1] || ycount[0] != ycount[2] || ycount[0] != ycount[3]) return false;
  p.ny = ycount[0] * 4;
  return 4 * p.ny <= (int)YF_MAX;
}

}

extern "C" void kernel_launch(int rank, int nranks, ncclComm_t comm, hipStream_t stream, void* const* d_in,
                              const int* in_sizes, int n_in, void* d_out, int out_size, const int* in_dtypes,
                              int out_dtype, void* const* d_out_all, void* const* d_ws_all, size_t ws_bytes) {
  using namespace agg;
  if (nranks != NR || n_in != 2 || ws_bytes < WS_USED || out_size != 4096 * SH || in_sizes[0] != SH * KD ||
      in_sizes[1] != KD * SH || (unsigned)rank >= (unsigned)NR) {
    fprintf(stderr, "ag_gemm: unexpected problem shape\n");
    return;
  }
  Params p;
  memset(&p, 0, sizeof(p));
  if (!build_params(p)) { fprintf(stderr, "ag_gemm: bad schedule\n"); return; }
  p.x = (const float*)d_in[0];
  p.w = (const float*)d_in[1];
  for (int r = 0; r < NR; ++r) { p.out[r] = (char*)d_out_all[r]; p.ws[r] = (char*)d_ws_all[r]; }
  p.rank = rank;
  HIP_CHECK(hipFuncSetAttribute((const void*)ag_gemm, hipFuncAttributeMaxDynamicSharedMemorySize, LDS_BYTES));
  ag_gemm<<<GRID, NTHR, LDS_BYTES, stream>>>(p);
  HIP_CHECK(hipGetLastError());
}
```

```cpp
#include <hip/hip_runtime.h>
#include <rccl/rccl.h>
#include <cstdio>
#include <cstring>
#include <algorithm>
#include <vector>

#define HIP_CHECK(expr) do { hipError_t _e = (expr); if (_e != hipSuccess) { \
    fprintf(stderr, "HIP error %s at %s:%d\n", hipGetErrorString(_e), __FILE__, __LINE__); \
    return; } } while (0)

#define P_NI 23
#define P_NC 4
#define P_DEPTH 2
#define P_NU 2
#define P_BATCH_I {{8, 4, 4}, {5, 4, 4}, {5, 3, 4}, {5, 3, 2}}
#define P_BATCH_X {{9, 6, 4}}
#define P_NOGEMM 0
#define P_NDR 2
#define P_CDELAY 200
#define P_CDELAY0 0
#define P_DS 4
#define P_DL 2
#define P_PSLEEP 2
#define P_PWIN 2
#define P_PSLOW 24
#define P_XS 1
#define P_K 1.6
#define P_T00 9.5
#define P_PRIO 1
#define P_PACK 1
#define P_NRAW 24
#define P_NRAWX 18
#define P_PER 2.03

namespace agg {

#define GAS __attribute__((address_space(1)))
#define LAS __attribute__((address_space(3)))
typedef unsigned v4u __attribute__((ext_vector_type(4)));
typedef unsigned v2u __attribute__((ext_vector_type(2)));
typedef unsigned short us2 __attribute__((ext_vector_type(2)));
typedef float f32x4 __attribute__((ext_vector_type(4)));
typedef float f32x2 __attribute__((ext_vector_type(2)));
typedef _Float16 h8 __attribute__((ext_vector_type(8)));
typedef GAS unsigned gu32;

constexpr int NR = 8;
constexpr int KD = 4096, SH = 512;
constexpr int PANEL = 131072;
constexpr int NAP = 128, NBP = 64, NPAN = NAP + NBP + 16;
constexpr int NCH = 16, CHUNK = 8192;
constexpr int NI = P_NI, NX = 32 - NI;
constexpr int GRID = 256, NWAVE = 7, NTHR = NWAVE * 64;
constexpr int NC = P_NC, NL = NWAVE - 1, WL = NC * NL;
constexpr int NU = P_PACK ? P_NU : 0, WU = NU * NWAVE + NC + NU;
static_assert(NWAVE == 7, "one wave per link in the group without a link");
constexpr int NG = 32 - NC - NU;
constexpr int MAXT = 6;
constexpr int XN = 1 << P_XS;
constexpr int YMAX = 192;
constexpr int ND = WL * P_NDR;
constexpr int NXY = 128 + NX * NCH;
constexpr int NPI = P_PACK ? NI * NCH - P_NRAW : 0, NPX = P_PACK ? NXY - P_NRAWX : 0;
static_assert(NG * 8 * 5 >= 1024 && NX <= 16 && ND <= 128, "one item per converting wave");
static_assert(NI * NCH <= 63 * WL && NXY + 128 <= 63 * WL, "one lane per forwarded item");
static_assert(!P_PACK || (NI * NCH - ND <= 64 * WU && NXY - ND <= 64 * WU), "one lane per unpacked item");
static_assert((NC + NU) * NWAVE * 64 >= 4 * YMAX, "roles");

constexpr unsigned OFF_CNT = 0;
constexpr unsigned OFF_TMO = 16384;
constexpr unsigned OFF_BAR = 32768;
constexpr unsigned OFF_YF = 65536;
constexpr unsigned OFF_YL = 196608;
constexpr unsigned OFF_PF = 393216;
constexpr unsigned PF_BYTES = NPAN * NCH * 128;
constexpr unsigned OFF_AF = 1u << 20;
constexpr unsigned OFF_YS = 2u << 20;
constexpr unsigned OFF_EXP = OFF_YS + (4u << 20);
constexpr unsigned OFF_STG = OFF_EXP + (8u << 20);
constexpr unsigned STG_BYTES = (unsigned)NPAN * PANEL;
constexpr unsigned OFF_DBG = OFF_STG + 2 * STG_BYTES;
constexpr unsigned WS_USED = OFF_DBG + (1u << 20);
static_assert(OFF_YF + 4 * YMAX * 128 <= OFF_YL && OFF_YL + 1024 * 128 <= OFF_PF && OFF_PF + PF_BYTES <= OFF_AF &&
              OFF_AF + 2 * PF_BYTES <= OFF_YS, "layout");
constexpr unsigned OUT_BYTES = 4096u * SH * 4u;
constexpr unsigned NOWHERE = 0xf0000000u;
constexpr unsigned long long DEADLINE = 50000000ull;
constexpr int AUX_SYS = 17;
constexpr int RSRC3 = 0x00020000;

constexpr int LDS_RED = 65536;
constexpr int LDS_CTL = 2 * LDS_RED;
constexpr int LDS_BYTES = LDS_CTL + 64;

struct Params {
  const float* x;
  const float* w;
  char* out[NR];
  char* ws[NR];
  int idx;
  int ny;
  int nyf;
  unsigned short rowbase[NAP];
  unsigned char ylist[NAP];
  unsigned tiles[2 * XN * NG * MAXT];
};

#define DRAIN() asm volatile("s_waitcnt vmcnt(0)" ::: "memory")
#define LDSWAIT() asm volatile("s_waitcnt lgkmcnt(0)" ::: "memory")
#define CBAR() asm volatile("" ::: "memory")

__device__ __forceinline__ __amdgpu_buffer_rsrc_t mk(const void* p, unsigned bytes) {
  return __builtin_amdgcn_make_buffer_rsrc((void*)p, 0, (int)bytes, RSRC3);
}
__device__ __forceinline__ unsigned long long uniform64(const void* q) {
  const unsigned long long v = (unsigned long long)q;
  return (unsigned long long)(unsigned)__builtin_amdgcn_readfirstlane((unsigned)(v >> 32)) << 32 |
         (unsigned)__builtin_amdgcn_readfirstlane((unsigned)v);
}
__device__ __forceinline__ unsigned pk(float a, float b) {
  typedef _Float16 h2 __attribute__((ext_vector_type(2)));
  h2 v = {(_Float16)a, (_Float16)b};
  return __builtin_bit_cast(unsigned, v);
}
__device__ __forceinline__ unsigned ld_sys(const char* p) {
  return __hip_atomic_load((gu32*)p, __ATOMIC_RELAXED, __HIP_MEMORY_SCOPE_SYSTEM);
}
__device__ __forceinline__ void st_sys(char* p, unsigned v) {
  __hip_atomic_store((gu32*)p, v, __ATOMIC_RELAXED, __HIP_MEMORY_SCOPE_SYSTEM);
}
__device__ __forceinline__ void st16(__amdgpu_buffer_rsrc_t r, unsigned voff, v4u v) {
  __builtin_amdgcn_raw_buffer_store_b128(v, r, (int)voff, 0, AUX_SYS);
}
__device__ __forceinline__ v4u ld16(__amdgpu_buffer_rsrc_t r, unsigned voff, unsigned soff) {
  return __builtin_amdgcn_raw_buffer_load_b128(r, (int)voff, (int)soff, AUX_SYS);
}
__device__ __forceinline__ void flag(bool on, __amdgpu_buffer_rsrc_t r, unsigned off, unsigned call, unsigned hdr = 0) {
  __builtin_amdgcn_raw_buffer_store_b64(v2u{call, hdr}, r, (int)(on ? off : NOWHERE), 0, AUX_SYS);
}
__device__ __forceinline__ v2u ldflag(__amdgpu_buffer_rsrc_t r, unsigned off) {
  return __builtin_amdgcn_raw_buffer_load_b64(r, (int)off, 0, AUX_SYS);
}

#define STAMP(slot) do {} while (0)
#define TREC_DECL() do {} while (0)
#define TREC(i) do {} while (0)
#define TREC_DUMP() do {} while (0)

constexpr unsigned TOP2 = 0xC000C000u;
constexpr int LIST_CAP = 128;
__device__ __forceinline__ unsigned bfi(unsigned m, unsigned a, unsigned b) { return (a & m) | (b & ~m); }
__device__ __forceinline__ void pack8(v4u (&h)[8], unsigned base) {
  const us2 b2 = __builtin_bit_cast(us2, base * 0x08000800u);
#pragma unroll
  for (int d = 0; d < 4; ++d) {
    unsigned t[8], pl = 0;
#pragma unroll
    for (int s = 0; s < 8; ++s) {
      t[s] = __builtin_bit_cast(unsigned, (us2)(__builtin_bit_cast(us2, __builtin_amdgcn_alignbit(h[s][d], h[s][d], 31)) - b2));
      pl |= (t[s] & TOP2) >> (2 * s);
    }
#pragma unroll
    for (int s = 0; s < 7; ++s) h[s][d] = bfi(TOP2, t[7] << (14 - 2 * s), t[s]);
    h[7][d] = pl;
  }
}
__device__ __forceinline__ void unpack8(v4u (&h)[8], unsigned base) {
  const us2 b2 = __builtin_bit_cast(us2, base * 0x08000800u);
#pragma unroll
  for (int d = 0; d < 4; ++d) {
    const unsigned pl = h[7][d];
    unsigned t7 = (pl << 14) & TOP2;
#pragma unroll
    for (int s = 0; s < 7; ++s) {
      t7 |= (h[s][d] & TOP2) >> (14 - 2 * s);
      const unsigned t = __builtin_bit_cast(unsigned, (us2)(__builtin_bit_cast(us2, bfi(TOP2, pl << (2 * s), h[s][d])) + b2));
      h[s][d] = __builtin_amdgcn_alignbit(t, t, 1);
    }
    const unsigned t = __builtin_bit_cast(unsigned, (us2)(__builtin_bit_cast(us2, t7) + b2));
    h[7][d] = __builtin_amdgcn_alignbit(t, t, 1);
  }
}
template <int CTRL> __device__ __forceinline__ unsigned dpp0(unsigned v) {
  return (unsigned)__builtin_amdgcn_update_dpp(0, (int)v, CTRL, 0xf, 0xf, true);
}

__device__ __forceinline__ void item_offsets(bool isw, unsigned stg, unsigned pan, int v, unsigned& o0, unsigned& o1,
                                             unsigned& fl) {
  o0 = stg + pan * PANEL + v * (isw ? 4096 : CHUNK);
  o1 = o0 + (isw ? PANEL : 4096);
  fl = OFF_PF + (isw ? (pan + (v & 1)) * NCH + (v >> 1) : pan * NCH + v) * 128;
}
constexpr int kBI[][3] = P_BATCH_I, kBX[][3] = P_BATCH_X;
constexpr int NBI = sizeof(kBI) / sizeof(kBI[0]), NBX = sizeof(kBX) / sizeof(kBX[0]);
template <int N> __device__ __forceinline__ void batch_item(const int (&b)[N][3], int i, int& u, int& c) {
  int f = 0;
#pragma unroll
  for (int k = 0; k < N; ++k) {
    if (k == 0 || i >= f * NCH) { const int r = i - f * NCH; u = f + r % b[k][0]; c = r / b[k][0]; }
    f += b[k][0];
  }
}
__device__ __forceinline__ void link_item(bool cross, int i, bool& isw, int& u, int& c) {
  isw = cross && i < 128;
  if (isw) { u = i & 3; c = i >> 2; }
  else if (cross) batch_item(kBX, i - 128, u, c);
  else batch_item(kBI, i, u, c);
}
__device__ __forceinline__ unsigned landing_panel(bool cross, bool isw, int u, int from, int to) {
  return !cross ? NI + 3 * u + ((from - to - 1) & 3) : isw ? NAP + 32 + from * 8 + 2 * u : 4 * NI + 4 * u + from;
}

__device__ __forceinline__ void load_inputs(const Params& p, bool isw, int g, int c, int lane, f32x4 (&v)[16]) {
  if (isw) {
    const GAS f32x2* src =
        (const GAS f32x2*)(p.w + (size_t)((c & 31) * 128 + 4 * (lane >> 4)) * SH + (g & 15) * 32 + 2 * (lane & 15));
#pragma unroll
    for (int q = 0; q < 16; ++q) {
      const f32x2 a = src[((q >> 1) * 16 + (q & 1) * 2) * (SH / 2)], b = src[((q >> 1) * 16 + (q & 1) * 2 + 1) * (SH / 2)];
      v[q] = f32x4{a.x, b.x, a.y, b.y};
    }
  } else {
    const GAS f32x4* src =
        (const GAS f32x4*)(p.x + (size_t)((g & 31) * 16 + (lane >> 2)) * KD + (c & 15) * 256 + 4 * (lane & 3));
#pragma unroll
    for (int q = 0; q < 16; ++q) v[q] = src[q * 4];
  }
}
__device__ __forceinline__ void to_fragments(bool isw, const f32x4 (&v)[16], LAS v4u* scr, int lane, v4u (&h)[8]) {
  if (isw) {
#pragma unroll
    for (int s = 0; s < 4; ++s) {
      h[s] = v4u{pk(v[4 * s].x, v[4 * s].y), pk(v[4 * s + 1].x, v[4 * s + 1].y), pk(v[4 * s + 2].x, v[4 * s + 2].y),
                 pk(v[4 * s + 3].x, v[4 * s + 3].y)};
      h[4 + s] = v4u{pk(v[4 * s].z, v[4 * s].w), pk(v[4 * s + 1].z, v[4 * s + 1].w), pk(v[4 * s + 2].z, v[4 * s + 2].w),
                     pk(v[4 * s + 3].z, v[4 * s + 3].w)};
    }
  } else {
#pragma unroll
    for (int s = 0; s < 8; ++s)
      scr[s * 64 + lane] = v4u{pk(v[2 * s].x, v[2 * s].y), pk(v[2 * s].z, v[2 * s].w), pk(v[2 * s + 1].x, v[2 * s + 1].y),
                               pk(v[2 * s + 1].z, v[2 * s + 1].w)};
    LDSWAIT();
#pragma unroll
    for (int s = 0; s < 8; ++s) h[s] = scr[s * 64 + (lane & 15) * 4 + (lane >> 4)];
    LDSWAIT();
  }
}

struct Ctl { unsigned prog, bad, full[2], fre[2], conv; };

__device__ __forceinline__ void need(volatile LAS unsigned* word, unsigned& have, unsigned n) {
  while (have < n) {
    have = __builtin_amdgcn_readfirstlane(*word);
    if (have < n) __builtin_amdgcn_s_sleep(1);
  }
  CBAR();
}

template <int MT, int NT>
__device__ __forceinline__ void gemm_tile(const Params& p, LAS char* lds, int a0, int b0, int t, unsigned& have,
                                          unsigned stg, int wave, int lane) {
  constexpr int D = (MT + NT <= 5) ? P_DS : P_DL;
  constexpr int NS = 32;
  volatile LAS Ctl* ctl = (volatile LAS Ctl*)(lds + LDS_CTL);
  const __amdgpu_buffer_rsrc_t rs = mk(p.ws[p.idx & 3], WS_USED);
  const unsigned ab = stg + (unsigned)a0 * PANEL + wave * 1024, bb = stg + (unsigned)(NAP + b0) * PANEL + wave * 1024;
  const int voff = lane * 16;
  const unsigned base = (unsigned)t * NCH;
  f32x4 acc[MT][NT];
#pragma unroll
  for (int m = 0; m < MT; ++m)
#pragma unroll
    for (int n = 0; n < NT; ++n) acc[m][n] = f32x4{0.f, 0.f, 0.f, 0.f};
  v4u ra[D][MT], rb[D][NT];
#define LOADSTEP(d, i) do { \
    _Pragma("unroll") for (int m = 0; m < MT; ++m) ra[d][m] = ld16(rs, voff, ab + m * PANEL + (i) * 4096); \
    _Pragma("unroll") for (int n = 0; n < NT; ++n) rb[d][n] = ld16(rs, voff, bb + n * PANEL + (i) * 4096); \
    __builtin_amdgcn_sched_barrier(0); } while (0)
#define MMASTEP(d) do { \
    _Pragma("unroll") for (int m = 0; m < MT; ++m) \
    _Pragma("unroll") for (int n = 0; n < NT; ++n) \
      acc[m][n] = __builtin_amdgcn_mfma_f32_16x16x32_f16(__builtin_bit_cast(h8, rb[d][n]), \
                                                         __builtin_bit_cast(h8, ra[d][m]), acc[m][n], 0, 0, 0); \
    __builtin_amdgcn_sched_barrier(0); } while (0)
#pragma unroll
  for (int d = 0; d < D; ++d) {
    if ((d & 1) == 0) need(&ctl->prog, have, base + (d >> 1) + 1);
    LOADSTEP(d, d);
  }
#pragma unroll 1
  for (int s0 = D; s0 < NS; s0 += D) {
#pragma unroll
    for (int d = 0; d < D; ++d) {
      MMASTEP(d);
      if ((d & 1) == 0) need(&ctl->prog, have, base + ((s0 + d) >> 1) + 1);
      LOADSTEP(d, s0 + d);
    }
  }
#pragma unroll
  for (int d = 0; d < D; ++d) MMASTEP(d);
#undef LOADSTEP
#undef MMASTEP
  asm volatile("" : "+v"(lane));
  constexpr int C4 = 4 * NT, M1 = MT > 4 ? (MT + 1) / 2 : MT;
#pragma unroll
  for (int part = 0; part < (MT > 4 ? 2 : 1); ++part) {
    const int tp = t + part, m0 = part ? M1 : 0, mn = part ? MT - M1 : M1;
    unsigned f = 0;
    if (tp >= 2) need(&ctl->fre[tp & 1], f, (unsigned)(tp >> 1));
    else need(&ctl->conv, f, NWAVE);
    LAS f32x4* red = (LAS f32x4*)(lds + (tp & 1) * LDS_RED) + wave * (64 * mn * NT);
#pragma unroll
    for (int m = 0; m < mn; ++m)
#pragma unroll
      for (int n = 0; n < NT; n += 2) {
        const int row = m * 16 + (lane & 15), c4 = n * 4 + 2 * (lane >> 4);
        const f32x4 e = acc[m0 + m][n], o = acc[m0 + m][n + 1];
        red[row * C4 + (c4 ^ (row & (C4 - 1)))] = f32x4{e.x, o.x, e.y, o.y};
        red[row * C4 + ((c4 + 1) ^ (row & (C4 - 1)))] = f32x4{e.z, o.z, e.w, o.w};
      }
    LDSWAIT();
    if (lane == 0)
      __hip_atomic_fetch_add((LAS unsigned*)&ctl->full[tp & 1], 1u, __ATOMIC_RELAXED, __HIP_MEMORY_SCOPE_WORKGROUP);
  }
}

template <int MT, int NT>
__device__ __forceinline__ void finish_tile(const Params& p, LAS char* lds, int bfirst, unsigned desc, int t,
                                            unsigned& have, unsigned call, int lane) {
  constexpr int C4 = 4 * NT, IMG = 64 * MT * NT, RQ = 64 / C4;
  volatile LAS Ctl* ctl = (volatile LAS Ctl*)(lds + LDS_CTL);
  const LAS f32x4* red = (const LAS f32x4*)(lds + (t & 1) * LDS_RED);
  const int idx = p.idx & 3, a0 = desc & 127, b = (desc >> 7) & 7, grp = ((bfirst + ((desc >> 7) & 31)) >> 3) & 7;
  const bool staged = NT == 4 && grp >= 4;
  const __amdgpu_buffer_rsrc_t ro = staged ? mk(p.ws[idx], WS_USED) : mk(p.out[grp < 4 ? idx : grp], OUT_BYTES);
  const int r0 = lane / C4, c4 = lane % C4;
  const unsigned slot = ((grp & 3) * NAP + a0) * 2 + (b >> 2);
  unsigned vo[MT];
#pragma unroll
  for (int m = 0; m < MT; ++m)
    vo[m] = staged ? OFF_YS + (slot + 2 * m) * 4096 + lane * 16
                   : ((p.rowbase[(a0 + m) & 127] + r0) * SH + (grp < 4 ? grp : idx) * 128 + b * 16 + c4 * 4) * 4;
  const unsigned qs = staged ? 1024 : RQ * SH * 4;
  need(&ctl->full[t & 1], have, 4u * ((t >> 1) + 1));
  STAMP(8 + t * 8 + 2);
  const bool bad = ctl->bad != 0;
#pragma unroll
  for (int q = 0; q < MT * NT; ++q) {
    const int row = q * RQ + r0;
    const int li = row * C4 + (c4 ^ (row & (C4 - 1)));
    f32x4 s = (red[li] + red[IMG + li]) + (red[2 * IMG + li] + red[3 * IMG + li]);
    if (bad) s = f32x4{__builtin_nanf(""), __builtin_nanf(""), __builtin_nanf(""), __builtin_nanf("")};
    st16(ro, vo[q / NT] + (q % NT) * qs, __builtin_bit_cast(v4u, s));
  }
  LDSWAIT();
  if (lane == 0) ctl->fre[t & 1] = (t >> 1) + 1;
  if (grp >= 4) {
    DRAIN();
    if (staged) flag(lane < MT, ro, OFF_YL + (slot + 2 * lane) * 128, call);
    else flag(lane == 0, mk(p.ws[grp], WS_USED), OFF_YF + (idx * YMAX + ((desc >> 13) & 255)) * 128, call);
  }
  STAMP(8 + t * 8 + 3);
}

#define BY_SHAPE(desc, BIG, F, ...) \
  switch (((desc) >> 24) & 15) { \
    case 1: F<1, 2>(__VA_ARGS__); break; \
    case 2: F<2, 2>(__VA_ARGS__); break; \
    case 3: F<3, 2>(__VA_ARGS__); break; \
    case 9: F<1, 4>(__VA_ARGS__); break; \
    case 10: F<2, 4>(__VA_ARGS__); break; \
    case 11: F<3, 4>(__VA_ARGS__); break; \
    case 12: F<4, 4>(__VA_ARGS__); break; \
    case 13: F<BIG ? 5 : 3, 4>(__VA_ARGS__); break; \
    default: F<BIG ? 6 : 3, 4>(__VA_ARGS__); break; \
  }

__global__ __launch_bounds__(NTHR, 2) void ag_gemm(const Params p) {
  extern __shared__ __attribute__((aligned(16))) char smem[];
  LAS char* lds = (LAS char*)smem;
  if (threadIdx.x < 16) ((volatile LAS unsigned*)(lds + LDS_CTL))[threadIdx.x] = 0;
  LDSWAIT();
  __builtin_amdgcn_s_barrier();
  const int lane = threadIdx.x & 63;
  const int wave = __builtin_amdgcn_readfirstlane(threadIdx.x >> 6);
  const int blk = blockIdx.x, grp = blk & 7, j = blk >> 3;
  const int idx = p.idx & 3;
  const bool cross = grp >= 4;
  char* myws = p.ws[idx];
  const __amdgpu_buffer_rsrc_t rl = mk(myws, WS_USED), rd = mk(p.ws[grp], WS_USED);
  STAMP(0);
  char* counter = myws + OFF_CNT + (blk * 8 + wave) * 4;
  const unsigned last_call = ld_sys(counter);

  if (j < NC && grp != idx) {
    struct Ring { unsigned full[NL], done, abort; };
    volatile LAS Ring* rg = (volatile LAS Ring*)(lds + LDS_CTL);
    volatile LAS v4u* meta = (volatile LAS v4u*)(lds + 2 * NL * 8192);
    const int n = !cross ? NI * NCH : NXY + (P_NOGEMM ? 0 : p.nyf);
    if (wave == 0) {
      const unsigned call = __builtin_amdgcn_readfirstlane(last_call) + 1u;
      if (lane == 0) st_sys(counter, call);
      __builtin_amdgcn_s_setprio(3);
      const unsigned long long peer_out = uniform64(p.out[grp]), peer_ws = uniform64(p.ws[grp]);
      unsigned pl[P_DEPTH], ph[P_DEPTH];
#pragma unroll
      for (int d = 0; d < P_DEPTH; ++d) { pl[d] = NOWHERE; ph[d] = 0; }
      const int cnt = (n - j + NC - 1) / NC;
      int l = 0, k = 0;
      TREC_DECL();
      for (int r = 0; r < cnt; ++r) {
        if (__builtin_amdgcn_readfirstlane(rg->full[l]) <= (unsigned)k) {
          DRAIN();
#pragma unroll
          for (int d = 0; d < P_DEPTH; ++d) { flag(lane == 0, rd, pl[d], call, ph[d]); pl[d] = NOWHERE; }
          while (__builtin_amdgcn_readfirstlane(rg->full[l]) <= (unsigned)k) __builtin_amdgcn_s_sleep(0);
        }
        if (__builtin_amdgcn_readfirstlane(rg->abort)) { rg->done = 0xffffffffu; return; }
        CBAR();
        const int slot = 2 * l + (k & 1);
        const v4u m = meta[slot];
        v4u v[8];
#pragma unroll
        for (int q = 0; q < 8; ++q) v[q] = *(const LAS v4u*)(lds + slot * 8192 + q * 1024 + lane * 16);
        const unsigned hd = __builtin_amdgcn_readfirstlane(m.w);
        const bool isy = (hd >> 17) & 1;
        const __amdgpu_buffer_rsrc_t rs = mk((char*)(isy ? peer_out : peer_ws), isy ? OUT_BYTES : WS_USED);
        const unsigned vb = isy ? (lane >> 4) * (SH * 4) + (lane & 15) * 16 : lane * 16, qs = isy ? 4 * SH * 4 : 1024;
#pragma unroll
        for (int q = 0; q < 8; ++q) asm volatile("" : "+v"(v[q]));
        rg->done = r + 1;
        asm volatile("s_waitcnt vmcnt(%0)" :: "n"(9 * (P_DEPTH - 1)) : "memory");
        flag(lane == 0, rd, pl[0], call, ph[0]);
        if (!(r & 1)) TREC(r >> 1);
        CBAR();
#pragma unroll
        for (int q = 0; q < 4; ++q) st16(rs, vb + m.x + q * qs, v[q]);
#pragma unroll
        for (int q = 0; q < 3; ++q) st16(rs, vb + m.y + q * qs, v[4 + q]);
        st16(rs, lane < (hd >> 24) ? vb + m.y + 3 * qs : NOWHERE, v[7]);
        CBAR();
#pragma unroll
        for (int d = 0; d + 1 < P_DEPTH; ++d) { pl[d] = pl[d + 1]; ph[d] = ph[d + 1]; }
        pl[P_DEPTH - 1] = __builtin_amdgcn_readfirstlane(m.z); ph[P_DEPTH - 1] = hd;
        if (++l == NL) { l = 0; ++k; }
      }
#pragma unroll
      for (int d = 0; d < P_DEPTH; ++d) {
        if (d == 0) asm volatile("s_waitcnt vmcnt(%0)" :: "n"(9 * (P_DEPTH - 1)) : "memory");
        else DRAIN();
        flag(lane == 0, rd, pl[d], call, ph[d]);
      }
      STAMP(1);
      TREC_DUMP();
      return;
    }
    const int l = wave - 1, first = l * NC + j, cnt = (n - first + WL - 1) / WL;
    f32x4 in[16];
    bool isw; int u, c;
    link_item(cross, first, isw, u, c);
    load_inputs(p, isw, isw ? (grp - 4) * 4 + u : u, c, lane, in);
    const unsigned call = __builtin_amdgcn_readfirstlane(last_call) + 1u;
    if (lane == 0) st_sys(counter, call);
    const unsigned stg = OFF_STG + (call & 1u) * STG_BYTES;
    const unsigned long long t0 = __builtin_amdgcn_s_memrealtime();
    for (int k = 0; k < P_NDR; ++k) {
      if (k) {
        link_item(cross, first + k * WL, isw, u, c);
        load_inputs(p, isw, isw ? (grp - 4) * 4 + u : u, c, lane, in);
      }
      LAS v4u* sl = (LAS v4u*)(lds + (2 * l + (k & 1)) * 8192);
      v4u h[8];
      to_fragments(isw, in, sl, lane, h);
      unsigned d0, d1, df;
      item_offsets(isw, stg, landing_panel(cross, isw, u, idx, grp & 3), c, d0, d1, df);
#pragma unroll
      for (int q = 0; q < 8; ++q) sl[q * 64 + lane] = h[q];
      meta[2 * l + (k & 1)] = v4u{d0, d1, df, 64u << 24};
      LDSWAIT();
      rg->full[l] = k + 1;
    }
    STAMP(2);
    const int i = min(first + lane * WL, n - 1);
    unsigned s0, s1, sf, sg, d0, d1, df, kind;
    if (!cross || i < NXY) {
      link_item(cross, i, isw, u, c);
      const int g = isw ? (grp - 4) * 4 + u : cross ? NI + u : u;
      item_offsets(isw, stg, isw ? NAP + 2 * g : cross ? NAP + NBP + u : u, c, s0, s1, sf);
      item_offsets(isw, stg, landing_panel(cross, isw, u, idx, grp & 3), c, d0, d1, df);
      sg = sf;
      kind = i < (cross ? NPX : NPI);
      if (kind) {
        s0 = OFF_EXP + (isw ? 512 + g * 32 + c : g * 16 + c) * 8192; s1 = s0 + 4096;
        df += OFF_AF + (call & 1u) * PF_BYTES - OFF_PF;
      }
    } else {
      const int a = p.ylist[(i - NXY) & 127] & 127, slot = ((grp & 3) * NAP + a) * 2;
      s0 = OFF_YS + slot * 4096; s1 = s0 + 4096;
      sf = OFF_YL + slot * 128; sg = sf + 128;
      d0 = (p.rowbase[a] * SH + idx * 128) * 4; d1 = d0 + 256;
      df = OFF_YF + (idx * YMAX + (i - NXY)) * 128;
      kind = 2;
    }
    if (lane == 63) { sf = OFF_TMO; sg = OFF_TMO; }
    unsigned long long ready = 0;
    unsigned hdv = 64u << 24 | kind << 16;
    for (int k = P_NDR; k < cnt; ++k) {
      while (!((ready >> k) & 1)) {
        const bool ask = ((lane >= k && lane < cnt) || lane == 63) && !((ready >> lane) & 1);
        const v2u f = ldflag(rl, ask ? sf : NOWHERE);
        const unsigned g = __builtin_amdgcn_raw_buffer_load_b32(rl, ask && kind == 2 ? sg : NOWHERE, 0, AUX_SYS);
        const bool ok = f.x == call && (kind != 2 || g == call);
        if (ok && kind == 1) hdv = f.y;
        ready |= __ballot(ok);
        if ((ready >> 63) || __builtin_amdgcn_s_memrealtime() - t0 > DEADLINE) {
          st_sys(myws + OFF_TMO, call);
          rg->abort = 1; LDSWAIT(); rg->full[l] = 0xffffffffu;
          return;
        }
      }
      CBAR();
      const unsigned hd = __builtin_amdgcn_readlane(hdv, k);
      const unsigned a0 = __builtin_amdgcn_readlane(s0, k), a1 = __builtin_amdgcn_readlane(s1, k);
      v4u h[8];
#pragma unroll
      for (int q = 0; q < 4; ++q) h[q] = ld16(rl, lane * 16 + q * 1024, a0);
#pragma unroll
      for (int q = 0; q < 3; ++q) h[4 + q] = ld16(rl, lane * 16 + q * 1024, a1);
      h[7] = ld16(rl, lane < (hd >> 24) ? lane * 16 + 3072 : NOWHERE, a1);
      if (k >= 2) {
        unsigned f = 0;
        need(&rg->done, f, (unsigned)((k - 2) * NL + l + 1));
        if (__builtin_amdgcn_readfirstlane(rg->abort)) return;
      }
      LAS v4u* sl = (LAS v4u*)(lds + (2 * l + (k & 1)) * 8192);
#pragma unroll
      for (int q = 0; q < 8; ++q) sl[q * 64 + lane] = h[q];
      meta[2 * l + (k & 1)] = v4u{(unsigned)__builtin_amdgcn_readlane(d0, k), (unsigned)__builtin_amdgcn_readlane(d1, k),
                                  (unsigned)__builtin_amdgcn_readlane(df, k), hd};
      LDSWAIT();
      rg->full[l] = k + 1;
    }
    STAMP(1);
    return;
  }
  if (j < NC + NU) {
    const unsigned call = __builtin_amdgcn_readfirstlane(last_call) + 1u;
    if (lane == 0) st_sys(counter, call);
    const unsigned stg = OFF_STG + (call & 1u) * STG_BYTES;
    const unsigned aoff = OFF_AF + (call & 1u) * PF_BYTES - OFF_PF;
    const unsigned long long t0 = __builtin_amdgcn_s_memrealtime();
    if (P_PACK) {
      const int lk = grp == idx ? (idx + 1 + wave) & 7 : grp;
      const bool xl = lk >= 4;
      const int n = xl ? NPX : NPI, first = ND + (grp == idx ? NU * NWAVE + j : wave * NU + j - NC);
      const int cnt = (n - first + WU - 1) / WU;
      bool isw; int u, c;
      link_item(xl, min(first + lane * WU, n - 1), isw, u, c);
      unsigned o0, o1, fl;
      item_offsets(isw, stg, landing_panel(xl, isw, u, lk & 3, idx), c, o0, o1, fl);
      LAS unsigned* plane = (LAS unsigned*)(lds + wave * 1024);
      unsigned pend = NOWHERE, hdr = 0;
      bool have_next = false;
      TREC_DECL();
      for (int r = 0; r < cnt; ++r) {
        const unsigned fr = __builtin_amdgcn_readlane(fl, r);
        *(LAS v4u*)(plane + lane * 4) = v4u{0, 0, 0, 0};
        while (!have_next) {
          const v2u f = ldflag(rl, lane == 0 ? fr + aoff : lane == 1 ? OFF_TMO : NOWHERE);
          if (pend != NOWHERE) {
            DRAIN();
            flag(lane == 0, rl, pend, call);
            TREC(4 * r - 1);
            pend = NOWHERE;
          }
          hdr = __builtin_amdgcn_readfirstlane(f.y);
          if ((int)(__builtin_amdgcn_readfirstlane(f.x) - call) >= 0) break;
          if (__builtin_amdgcn_readlane(f.x, 1) == call || __builtin_amdgcn_s_memrealtime() - t0 > DEADLINE) {
            st_sys(myws + OFF_TMO, call);
            st_sys(p.out[idx] + (size_t)p.rowbase[0] * SH * 4, 0x7fc00000u);
            return;
          }
        }
        CBAR();
        TREC(4 * r);
        const unsigned a0 = __builtin_amdgcn_readlane(o0, r), a1 = __builtin_amdgcn_readlane(o1, r);
        const bool dense = (hdr >> 16) & 1;
        const unsigned e2 = __builtin_amdgcn_raw_buffer_load_b32(rl, lane * 4 + 3072, a1, AUX_SYS);
        v4u h[8];
#pragma unroll
        for (int q = 0; q < 4; ++q) h[q] = ld16(rl, lane * 16 + q * 1024, a0);
#pragma unroll
        for (int q = 0; q < 3; ++q) h[4 + q] = ld16(rl, lane * 16 + q * 1024, a1);
        h[7] = ld16(rl, dense ? lane * 16 + 3072 : NOWHERE, a1);
        const v2u fn = ldflag(rl, lane == 0 ? __builtin_amdgcn_readlane(fl, min(r + 1, cnt - 1)) + aoff : NOWHERE);
        DRAIN();
        flag(lane == 0, rl, pend, call);
        if (pend != NOWHERE) TREC(4 * r - 1);
        TREC(4 * r + 1);
        if (!dense) {
          const unsigned cn = (hdr >> 8) & 255;
          LDSWAIT();
#pragma unroll
          for (int k = 0; k < 2; ++k) {
            const unsigned e = (e2 >> (16 * k)) & 0xffffu;
            __hip_atomic_fetch_or(plane + ((e >> 7) & 255), 2 * lane + k < cn ? (e & 15) << ((e >> 2) & 28) : 0u,
                                  __ATOMIC_RELAXED, __HIP_MEMORY_SCOPE_WORKGROUP);
          }
          LDSWAIT();
          h[7] = *(volatile LAS v4u*)(plane + lane * 4);
        }
        unpack8(h, hdr & 31);
#pragma unroll
        for (int q = 0; q < 4; ++q) st16(rl, a0 + lane * 16 + q * 1024, h[q]);
#pragma unroll
        for (int q = 0; q < 4; ++q) st16(rl, a1 + lane * 16 + q * 1024, h[4 + q]);
        pend = fr;
        TREC(4 * r + 2);
        hdr = __builtin_amdgcn_readfirstlane(fn.y);
        have_next = (int)(__builtin_amdgcn_readfirstlane(fn.x) - call) >= 0;
      }
      DRAIN();
      flag(lane == 0, rl, pend, call);
      TREC_DUMP();
      STAMP(2);
    }
    if (grp == idx) {
      const int f = (j * NWAVE + wave) * 64 + lane;
      if (f - lane >= 4 * YMAX) return;
      const bool mine = (f % YMAX) < p.ny;
      while (!P_NOGEMM) {
        const unsigned v = __builtin_amdgcn_raw_buffer_load_b32(rl, mine ? OFF_YF + f * 128 : NOWHERE, 0, AUX_SYS);
        const unsigned tm = __builtin_amdgcn_readfirstlane(ld_sys(myws + OFF_TMO));
        if (__all(!mine || (int)(v - call) >= 0)) break;
        if (tm == call || __builtin_amdgcn_s_memrealtime() - t0 > DEADLINE) {
          st_sys(myws + OFF_TMO, call);
          st_sys(p.out[idx] + (size_t)p.rowbase[0] * SH * 4, 0x7fc00000u);
          break;
        }
        __builtin_amdgcn_s_sleep(1);
      }
      STAMP(1);
    }
    return;
  }

  const int jg = j - NC - NU;
  volatile LAS Ctl* ctl = (volatile LAS Ctl*)(lds + LDS_CTL);
  unsigned call;
  {
    const unsigned long long tw = __builtin_amdgcn_s_memrealtime();
    const int it = min(wave, 4) * (NG * 8) + jg * 8 + grp;
    const bool work = wave != 4 && wave != 6 && it < 1024;
    bool isw = false; int g = 0, c = 0, u = 0;
    f32x4 v[16];
    if (work) {
      const int k = (it / 5 + ND) & 127, m = it % 5;
      isw = it < 640 && m < 4;
      if (isw) { g = m * 4 + (k & 3); c = k >> 2; }
      else {
        const int xi = it < 640 ? k : it - 512;
        bool dummy;
        link_item(xi >= NI * NCH, xi >= NI * NCH ? xi - NI * NCH + 128 : xi, dummy, u, c);
        g = xi >= NI * NCH ? NI + u : u;
      }
      while ((int)(__builtin_amdgcn_s_memrealtime() - tw) < P_CDELAY0 + (it * P_CDELAY >> 8)) __builtin_amdgcn_s_sleep(2);
      load_inputs(p, isw, g, c, lane, v);
    }
    call = __builtin_amdgcn_readfirstlane(last_call) + 1u;
    if (lane == 0) st_sys(counter, call);
    if (work) {
      const unsigned stg = OFF_STG + (call & 1u) * STG_BYTES;
      LAS v4u* scr = (LAS v4u*)(lds + wave * 8192);
      v4u h[8];
      to_fragments(isw, v, scr, lane, h);
      unsigned o0, o1, fl, hdr = 0;
      item_offsets(isw, stg, isw ? NAP + 2 * g : g < NI ? g : NAP + NBP + u, c, o0, o1, fl);
      const __amdgpu_buffer_rsrc_t rs = rl;
      STAMP(7);
#pragma unroll
      for (int s = 0; s < 4; ++s) st16(rs, o0 + lane * 16 + s * 1024, h[s]);
#pragma unroll
      for (int s = 0; s < 4; ++s) st16(rs, o1 + lane * 16 + s * 1024, h[4 + s]);
      if (P_PACK) {
        float ss = 0.f;
#pragma unroll
        for (int q = 0; q < 16; q += 4) ss += v[q].x * v[q].x + v[q].y * v[q].y + v[q].z * v[q].z + v[q].w * v[q].w;
        ss += __builtin_bit_cast(float, dpp0<0x111>(__builtin_bit_cast(unsigned, ss)));
        ss += __builtin_bit_cast(float, dpp0<0x112>(__builtin_bit_cast(unsigned, ss)));
        ss += __builtin_bit_cast(float, dpp0<0x114>(__builtin_bit_cast(unsigned, ss)));
        ss += __builtin_bit_cast(float, dpp0<0x118>(__builtin_bit_cast(unsigned, ss)));
        const unsigned sb = __builtin_bit_cast(unsigned, ss);
        const float tot = (__builtin_bit_cast(float, __builtin_amdgcn_readlane(sb, 15)) +
                           __builtin_bit_cast(float, __builtin_amdgcn_readlane(sb, 31))) +
                          (__builtin_bit_cast(float, __builtin_amdgcn_readlane(sb, 47)) +
                           __builtin_bit_cast(float, __builtin_amdgcn_readlane(sb, 63)));
        const float top = __builtin_amdgcn_sqrtf(tot * (6.76f / 1024.f));
        const int ex = (int)((__builtin_amdgcn_readfirstlane(__builtin_bit_cast(unsigned, top)) >> 23) & 255) - 127 + 15 - 7;
        const unsigned base = (unsigned)min(max(ex, 0), 24);
        pack8(h, base);
        unsigned nz[4], cnt = 0;
#pragma unroll
        for (int d = 0; d < 4; ++d) {
          unsigned z = h[7][d];
          z |= z >> 1; z |= z >> 2;
          nz[d] = z & 0x11111111u;
          cnt += __builtin_popcount(nz[d]);
        }
        unsigned inc = cnt;
        inc += dpp0<0x111>(inc); inc += dpp0<0x112>(inc); inc += dpp0<0x114>(inc); inc += dpp0<0x118>(inc);
        const unsigned r0 = __builtin_amdgcn_readlane(inc, 15), r1 = r0 + __builtin_amdgcn_readlane(inc, 31),
                       r2 = r1 + __builtin_amdgcn_readlane(inc, 47), total = r2 + __builtin_amdgcn_readlane(inc, 63);
        const bool dense = total > (unsigned)LIST_CAP;
        if (!dense) {
          unsigned pos = inc - cnt + (lane < 16 ? 0 : lane < 32 ? r0 : lane < 48 ? r1 : r2);
          LAS unsigned short* list = (LAS unsigned short*)scr;
#pragma unroll
          for (int d = 0; d < 4; ++d) {
            unsigned m = nz[d];
            while (m) {
              const unsigned b = __builtin_ctz(m);
              list[pos++] = (unsigned short)(((lane * 4 + d) << 7) | (b << 2) | ((h[7][d] >> b) & 15));
              m &= m - 1;
            }
          }
          LDSWAIT();
          h[7] = *(volatile LAS v4u*)(scr + (lane & 15));
        }
        const unsigned nl = dense ? 64 : max((total + 31) >> 5, 1u) * 4;
        hdr = base | (dense ? 1u << 16 : total << 8) | nl << 24;
        const unsigned e0 = OFF_EXP + (isw ? 512 + g * 32 + c : g * 16 + c) * 8192 + lane * 16;
#pragma unroll
        for (int s = 0; s < 7; ++s) st16(rs, e0 + s * 1024, h[s]);
        st16(rs, lane < nl ? e0 + 7168 : NOWHERE, h[7]);
      }
      STAMP(6);
      DRAIN();
      flag(lane == 0, rs, fl, call, hdr);
    }
    if (lane == 0) __hip_atomic_fetch_add((LAS unsigned*)&ctl->conv, 1u, __ATOMIC_RELAXED, __HIP_MEMORY_SCOPE_WORKGROUP);
    STAMP(2);
  }
  const unsigned stg = OFF_STG + (call & 1u) * STG_BYTES;
  const unsigned* tl = p.tiles + (((grp >> 2) * XN + (grp & (XN - 1))) * NG + jg) * MAXT;
  const int bfirst = (grp >> P_XS << P_XS) * 8;
  if (P_PRIO && wave >= 4) __builtin_amdgcn_s_setprio(3);
  if (wave == 4) {
    const unsigned long long t0 = __builtin_amdgcn_s_memrealtime();
    unsigned bdone = 0;
    for (int t = 0; t < MAXT && !P_NOGEMM; ++t) {
      const unsigned desc = tl[t];
      if (!(desc >> 31)) break;
      if ((desc >> 28) & 1) continue;
      const int mt = (desc >> 24) & 7, nt = (desc >> 27) & 1 ? 4 : 2, b = (desc >> 7) & 31;
      const unsigned bm = ((1u << nt) - 1) << b;
      const int np = (bdone & bm) == bm ? mt : mt + nt, l2 = np <= 4 ? 2 : np <= 8 ? 3 : 4;
      const int q = lane & ((1 << l2) - 1), ahead = lane >> l2;
      const unsigned fo = OFF_PF + (q < mt ? (desc & 127) + q : NAP + bfirst + b + (q - mt)) * (NCH * 128);
      unsigned pub = 0, win = 16;
      while (pub < NCH) {
        const unsigned c = pub + ahead;
        const bool act = q < np && c < NCH && ahead < win;
        const unsigned v = __builtin_amdgcn_raw_buffer_load_b32(rl, act ? fo + c * 128 : NOWHERE, 0, AUX_SYS);
        const unsigned tm = __builtin_amdgcn_readfirstlane(ld_sys(myws + OFF_TMO));
        const unsigned long long m = __ballot(act ? (int)(v - call) < 0 : ahead >= win);
        if (tm == call || __builtin_amdgcn_s_memrealtime() - t0 > DEADLINE) {
          st_sys(myws + OFF_TMO, call);
          ctl->bad = 1;
          LDSWAIT();
          ctl->prog = 0xffffffffu;
          return;
        }
        const unsigned n = m ? (unsigned)__builtin_ctzll(m) >> l2 : 64u >> l2;
        win = n >= win || n >= (64u >> l2) ? 16 : n + P_PWIN;
        if (n) {
          pub = min(pub + n, (unsigned)NCH);
          ctl->prog = t * NCH + pub;
        }
        if (t > 0 && __builtin_amdgcn_readfirstlane(ctl->full[(t - 1) & 1]) < 4u * ((t - 1) >> 1) + 1u)
          __builtin_amdgcn_s_sleep(P_PSLOW);
        else if (!n) __builtin_amdgcn_s_sleep(P_PSLEEP);
      }
      bdone |= bm;
      STAMP(8 + t * 8 + 4);
    }
    return;
  }
  unsigned have = 0;
  if (wave > 4) {
    for (int t = wave - 5; t < MAXT && !P_NOGEMM; t += 2) {
      const unsigned desc = tl[t];
      if (!(desc >> 31)) break;
      BY_SHAPE(((desc >> 24) & 7) > 4 ? 11u << 24 : desc, false, finish_tile, p, lds, bfirst, desc, t, have, call, lane)
    }
    return;
  }
  for (int t = 0; t < MAXT && !P_NOGEMM; ++t) {
    const unsigned desc = tl[t];
    if (!(desc >> 31)) break;
    if ((desc >> 28) & 1) continue;
    STAMP(8 + t * 8);
    const int a0 = desc & 127, b0 = bfirst + ((desc >> 7) & 31);
    BY_SHAPE(desc, true, gemm_tile, p, lds, a0, b0, t, have, stg, wave, lane)
  }
  STAMP(1);
}

static const int kClass[2][4] = {{0, 2, 3, 6}, {1, 4, 5, 7}};

static bool build_params(Params& p, int rank, int* roles) {
  int cls = 0, idx = 0;
  for (int c = 0; c < 2; ++c)
    for (int i = 0; i < 4; ++i) if (kClass[c][i] == rank) { cls = c; idx = i; }
  for (int i = 0; i < 4; ++i) { roles[i] = kClass[cls][i]; roles[4 + i] = kClass[cls ^ 1][i]; }
  p.idx = idx;
  for (int a = 0; a < NAP; ++a) {
    int src, rt;
    if (a < NI) { src = rank; rt = a; }
    else if (a < 4 * NI) { rt = (a - NI) / 3; src = roles[(idx + 1 + (a - NI) % 3) & 3]; }
    else { rt = NI + (a - 4 * NI) / 4; src = roles[4 + (a - 4 * NI) % 4]; }
    p.rowbase[a] = (unsigned short)(src * SH + rt * 16);
  }
  struct Seg { int a0, n, mt, nt; double t0, t1; };
  std::vector<Seg> segs;
  const double per = P_PER, t00 = P_T00;
  segs.push_back({0, NI, 4, 4, 8.0, 11.0});
  int f = 0;
  for (int b = 0; b < NBI; ++b) {
    segs.push_back({NI + 3 * f, 3 * kBI[b][0], kBI[b][1], kBI[b][2], f ? t00 + f * per : 6.0,
                    t00 + (f + kBI[b][0]) * per - (b == NBI - 1 ? 3.2 : 0.0)});
    f += kBI[b][0];
  }
  if (f != NI) return false;
  for (int b = 0; b < NBX; ++b) {
    segs.push_back({4 * NI + 4 * (f - NI), 4 * kBX[b][0], kBX[b][1], kBX[b][2], t00 + (8 + f - NI) * per,
                    t00 + 1.0 + (8 + f - NI + kBX[b][0]) * per - (b == NBX - 1 && P_NRAWX ? 2.2 : 0.0)});
    f += kBX[b][0];
  }
  if (f != 32) return false;
  auto dur = [](int mt, int nt) { return 0.6 + P_K * (mt + nt); };
  std::stable_sort(segs.begin(), segs.end(), [&](const Seg& a, const Seg& b) {
    return std::max(a.t0, a.t1 + 1.8 - dur(a.mt, a.nt)) < std::max(b.t0, b.t1 + 1.8 - dur(b.mt, b.nt));
  });
  for (int h = 0; h < 2; ++h) {
    std::vector<double> fr(XN * NG, 0.0);
    std::vector<int> cnt(XN * NG, 0);
    std::vector<std::pair<unsigned*, int>> direct;
    int ndirect = 0;
    std::vector<std::pair<double, int>> done;
    for (auto& s : segs) {
      if ((s.nt != 2 && s.nt != 4) || s.mt < 1 || s.mt > (s.nt == 4 ? 6 : 3)) return false;
      const double t1 = h ? std::max(s.t1, t00 + 8 * per) : s.t1;
      const int rows = (s.n + s.mt - 1) / s.mt;
      for (int r = 0, a = s.a0; r < rows; ++r) {
        const int mt = s.n / rows + (r < s.n % rows), ntile = 8 * XN / s.nt;
        auto finish = [&](double free_at) { return std::max(std::max(free_at, s.t0) + dur(mt, s.nt), t1 + 1.8); };
        int home = 0; double bt = 1e9;
        for (int x = 0; x < XN; ++x) {
          std::vector<double> v(fr.begin() + x * NG, fr.begin() + (x + 1) * NG);
          std::sort(v.begin(), v.end());
          double sum = 0;
          for (int q = 0; q < NG; ++q) sum += (q < ntile ? finish(v[q]) : 0.0) + 1e-3 * v[q];
          if (sum < bt) { bt = sum; home = x; }
        }
        double fin = 0;
        for (int b = 0; b < 8 * XN; b += s.nt) {
          int best = -1; double bf = 0;
          for (int g = 0; g < XN * NG; ++g) {
            const double e = finish(fr[g]) + (g / NG == home ? 0.0 : 0.7) + 1e-3 * fr[g];
            if (cnt[g] + (mt > 4) < MAXT && (best < 0 || e < bf)) { best = g; bf = e; }
          }
          if (best < 0) return false;
          fr[best] = finish(fr[best]);
          fin = std::max(fin, fr[best]);
          unsigned* e = &p.tiles[(h * XN * NG + best) * MAXT + cnt[best]++];
          *e = 0x80000000u | (s.nt == 4 ? 1u << 27 : 0u) | ((unsigned)mt << 24) | ((unsigned)b << 7) | (unsigned)a;
          if (mt > 4) {
            e[1] = 0x98000000u | ((unsigned)(mt - 3) << 24) | ((unsigned)b << 7) | (unsigned)(a + 3);
            ++cnt[best];
          }
          if (s.nt != 4) direct.push_back({e, ndirect + (b & 7) / s.nt});
        }
        if (s.nt != 4) ndirect += 8 / s.nt;
        else for (int m = 0; m < mt; ++m) done.push_back({fin, a + m});
        a += mt;
      }
    }
    std::stable_sort(done.begin(), done.end(),
                     [](const std::pair<double, int>& a, const std::pair<double, int>& b) { return a.first < b.first; });
    int nyf = 0;
    for (auto& d : done) p.ylist[nyf++] = (unsigned char)d.second;
    for (auto& d : direct) *d.first |= (unsigned)(nyf + d.second) << 13;
    p.nyf = nyf;
    p.ny = nyf + ndirect;
    if (p.ny > YMAX) return false;
  }
  return true;
}

}

extern "C" void kernel_launch(int rank, int nranks, ncclComm_t comm, hipStream_t stream, void* const* d_in,
                              const int* in_sizes, int n_in, void* d_out, int out_size, const int* in_dtypes,
                              int out_dtype, void* const* d_out_all, void* const* d_ws_all, size_t ws_bytes) {
  using namespace agg;
  if (nranks != NR || n_in != 2 || ws_bytes < WS_USED || out_size != 4096 * SH || in_sizes[0] != SH * KD ||
      in_sizes[1] != KD * SH || (unsigned)rank >= (unsigned)NR) {
    fprintf(stderr, "ag_gemm: unexpected problem shape\n");
    return;
  }
  Params p;
  memset(&p, 0, sizeof(p));
  int roles[NR];
  if (!build_params(p, rank, roles)) { fprintf(stderr, "ag_gemm: bad schedule\n"); return; }
  p.x = (const float*)d_in[0];
  p.w = (const float*)d_in[1];
  for (int r = 0; r < NR; ++r) { p.out[r] = (char*)d_out_all[roles[r]]; p.ws[r] = (char*)d_ws_all[roles[r]]; }
  HIP_CHECK(hipFuncSetAttribute((const void*)ag_gemm, hipFuncAttributeMaxDynamicSharedMemorySize, LDS_BYTES));
  ag_gemm<<<GRID, NTHR, LDS_BYTES, stream>>>(p);
  HIP_CHECK(hipGetLastError());
}
```

```cpp
#include <hip/hip_runtime.h>
#include <rccl/rccl.h>
#include <cstdio>
#include <cstring>
#include <algorithm>
#include <vector>

#define HIP_CHECK(expr) do { hipError_t _e = (expr); if (_e != hipSuccess) { \
    fprintf(stderr, "HIP error %s at %s:%d\n", hipGetErrorString(_e), __FILE__, __LINE__); \
    return; } } while (0)

#define P_NI 23
#define P_NC 4
#define P_DEPTH 2
#define P_NU 3
#define P_BATCH_I {{7, 6, 4, 1}, {6, 4, 4, 1}, {5, 3, 4, 1}, {5, 3, 2, 0}}
#define P_BATCH_X {{9, 6, 4, 1}}
#define P_NOGEMM 0
#define P_ND 48
#define P_NDRAW 14
#define P_LEAD 330
#define P_HOLD 0
#define P_WIN 0
#define P_YWIN 6
#define P_OWN 4
#define P_SWAP 4
#define P_NRAWW 0
#define P_LWIN 3
#define P_PACE 25
#define P_PACE0 140
#define P_TOUCH 1
#define P_CDELAY 200
#define P_CDELAY0 0
#define P_DS 4
#define P_DL 2
#define P_PSLEEP 2
#define P_PWIN 2
#define P_PSLOW 24
#define P_XS 1
#define P_K 1.65
#define P_T00 9.3
#define P_PRIO 1
#define P_PACK 1
#define P_NRAW 24
#define P_NRAWX 18
#define P_PER 2.03

namespace agg {

#define GAS __attribute__((address_space(1)))
#define LAS __attribute__((address_space(3)))
typedef unsigned v4u __attribute__((ext_vector_type(4)));
typedef unsigned v2u __attribute__((ext_vector_type(2)));
typedef unsigned short us2 __attribute__((ext_vector_type(2)));
typedef float f32x4 __attribute__((ext_vector_type(4)));
typedef float f32x2 __attribute__((ext_vector_type(2)));
typedef _Float16 h8 __attribute__((ext_vector_type(8)));
typedef GAS unsigned gu32;

constexpr int NR = 8;
constexpr int KD = 4096, SH = 512;
constexpr int PANEL = 131072;
constexpr int NAP = 128, NBP = 64, NPAN = NAP + NBP + 16;
constexpr int NCH = 16, CHUNK = 8192;
constexpr int NI = P_NI, NX = 32 - NI;
constexpr int GRID = 256, NWAVE = 7, NTHR = NWAVE * 64;
constexpr int NC = P_NC, NL = NWAVE - 1, WL = NC * NL;
constexpr int NU = P_PACK ? P_NU : 0, WU = NU * NWAVE + NC + NU;
static_assert(NWAVE == 7, "one wave per link in the group without a link");
constexpr int NG = 32 - NC - NU;
constexpr int MAXT = 6;
constexpr int XN = 1 << P_XS;
constexpr int YMAX = 192;
constexpr int NDRAW = P_PACK ? P_NDRAW : P_ND;
constexpr int ND = P_ND;
constexpr int NXY = 128 + NX * NCH;
constexpr int NPI = P_PACK ? NI * NCH - P_NRAW : 0, NPX = P_PACK ? NXY - P_NRAWX : 0;
static_assert(NG * 8 * 5 + 2 * NG >= 1024 && NX <= 16 && ND <= 2 * NG && NDRAW <= ND, "one item per converting wave");
static_assert(NI * NCH <= 63 * WL && NXY + 128 <= 63 * WL, "one lane per forwarded item");
static_assert(!P_PACK || (NI * NCH <= 64 * WU && NXY <= 64 * WU), "one lane per unpacked item");
static_assert((NC + NU) * NWAVE * 64 >= 4 * YMAX, "roles");

constexpr unsigned OFF_CNT = 0;
constexpr unsigned OFF_TMO = 16384;
constexpr unsigned OFF_YQ = 20480;
constexpr unsigned OFF_BAR = 32768;
constexpr unsigned OFF_YF = 65536;
constexpr unsigned OFF_YL = 196608;
constexpr unsigned OFF_PF = 393216;
constexpr unsigned PF_BYTES = NPAN * NCH * 128;
constexpr unsigned OFF_AF = 1u << 20;
constexpr unsigned OFF_YS = 2u << 20;
constexpr unsigned OFF_EXP = OFF_YS + (4u << 20);
constexpr unsigned OFF_STG = OFF_EXP + (8u << 20);
constexpr unsigned STG_BYTES = (unsigned)NPAN * PANEL;
constexpr unsigned OFF_DBG = OFF_STG + 2 * STG_BYTES;
constexpr unsigned WS_USED = OFF_DBG + (1u << 20);
static_assert(OFF_YF + 4 * YMAX * 128 <= OFF_YL && OFF_YL + 1024 * 128 <= OFF_PF && OFF_PF + PF_BYTES <= OFF_AF &&
              OFF_AF + 2 * PF_BYTES <= OFF_YS, "layout");
constexpr unsigned OUT_BYTES = 4096u * SH * 4u;
constexpr unsigned NOWHERE = 0xf0000000u;
constexpr unsigned long long DEADLINE = 50000000ull;
constexpr int AUX_SYS = 17;
constexpr int RSRC3 = 0x00020000;

constexpr int LDS_RED = 65536;
constexpr int LDS_CTL = 2 * LDS_RED;
constexpr int LDS_BYTES = LDS_CTL + 64;

struct Params {
  const float* x;
  const float* w;
  char* out[NR];
  char* ws[NR];
  int idx;
  int ny;
  int nyf;
  unsigned short rowbase[NAP];
  unsigned tiles[2 * XN * NG * MAXT];
};

#define DRAIN() asm volatile("s_waitcnt vmcnt(0)" ::: "memory")
#define LDSWAIT() asm volatile("s_waitcnt lgkmcnt(0)" ::: "memory")
#define CBAR() asm volatile("" ::: "memory")

__device__ __forceinline__ __amdgpu_buffer_rsrc_t mk(const void* p, unsigned bytes) {
  return __builtin_amdgcn_make_buffer_rsrc((void*)p, 0, (int)bytes, RSRC3);
}
__device__ __forceinline__ unsigned long long uniform64(const void* q) {
  const unsigned long long v = (unsigned long long)q;
  return (unsigned long long)(unsigned)__builtin_amdgcn_readfirstlane((unsigned)(v >> 32)) << 32 |
         (unsigned)__builtin_amdgcn_readfirstlane((unsigned)v);
}
__device__ __forceinline__ unsigned pk(float a, float b) {
  typedef _Float16 h2 __attribute__((ext_vector_type(2)));
  h2 v = {(_Float16)a, (_Float16)b};
  return __builtin_bit_cast(unsigned, v);
}
__device__ __forceinline__ unsigned ld_sys(const char* p) {
  return __hip_atomic_load((gu32*)p, __ATOMIC_RELAXED, __HIP_MEMORY_SCOPE_SYSTEM);
}
__device__ __forceinline__ void st_sys(char* p, unsigned v) {
  __hip_atomic_store((gu32*)p, v, __ATOMIC_RELAXED, __HIP_MEMORY_SCOPE_SYSTEM);
}
__device__ __forceinline__ void st16(__amdgpu_buffer_rsrc_t r, unsigned voff, v4u v) {
  __builtin_amdgcn_raw_buffer_store_b128(v, r, (int)voff, 0, AUX_SYS);
}
__device__ __forceinline__ v4u ld16(__amdgpu_buffer_rsrc_t r, unsigned voff, unsigned soff) {
  return __builtin_amdgcn_raw_buffer_load_b128(r, (int)voff, (int)soff, AUX_SYS);
}
__device__ __forceinline__ void flag(bool on, __amdgpu_buffer_rsrc_t r, unsigned off, unsigned call, unsigned hdr = 0) {
  __builtin_amdgcn_raw_buffer_store_b64(v2u{call, hdr}, r, (int)(on ? off : NOWHERE), 0, AUX_SYS);
}
__device__ __forceinline__ v2u ldflag(__amdgpu_buffer_rsrc_t r, unsigned off) {
  return __builtin_amdgcn_raw_buffer_load_b64(r, (int)off, 0, AUX_SYS);
}

#define STAMP(slot) do {} while (0)
#define TREC_DECL() do {} while (0)
#define TREC(i) do {} while (0)
#define TREC_DUMP() do {} while (0)

constexpr unsigned TOP2 = 0xC000C000u;
constexpr int LIST_CAP = 128;
__device__ __forceinline__ unsigned bfi(unsigned m, unsigned a, unsigned b) { return (a & m) | (b & ~m); }
__device__ __forceinline__ void pack8(v4u (&h)[8], unsigned base) {
  const us2 b2 = __builtin_bit_cast(us2, base * 0x08000800u);
#pragma unroll
  for (int d = 0; d < 4; ++d) {
    unsigned t[8], pl = 0;
#pragma unroll
    for (int s = 0; s < 8; ++s) {
      t[s] = __builtin_bit_cast(unsigned, (us2)(__builtin_bit_cast(us2, __builtin_amdgcn_alignbit(h[s][d], h[s][d], 31)) - b2));
      pl |= (t[s] & TOP2) >> (2 * s);
    }
#pragma unroll
    for (int s = 0; s < 7; ++s) h[s][d] = bfi(TOP2, t[7] << (14 - 2 * s), t[s]);
    h[7][d] = pl;
  }
}
__device__ __forceinline__ void unpack8(v4u (&h)[8], unsigned base) {
  const us2 b2 = __builtin_bit_cast(us2, base * 0x08000800u);
#pragma unroll
  for (int d = 0; d < 4; ++d) {
    const unsigned pl = h[7][d];
    unsigned t7 = (pl << 14) & TOP2;
#pragma unroll
    for (int s = 0; s < 7; ++s) {
      t7 |= (h[s][d] & TOP2) >> (14 - 2 * s);
      const unsigned t = __builtin_bit_cast(unsigned, (us2)(__builtin_bit_cast(us2, bfi(TOP2, pl << (2 * s), h[s][d])) + b2));
      h[s][d] = __builtin_amdgcn_alignbit(t, t, 1);
    }
    const unsigned t = __builtin_bit_cast(unsigned, (us2)(__builtin_bit_cast(us2, t7) + b2));
    h[7][d] = __builtin_amdgcn_alignbit(t, t, 1);
  }
}
template <int CTRL> __device__ __forceinline__ unsigned dpp0(unsigned v) {
  return (unsigned)__builtin_amdgcn_update_dpp(0, (int)v, CTRL, 0xf, 0xf, true);
}

__device__ __forceinline__ void item_offsets(bool isw, unsigned stg, unsigned pan, int v, unsigned& o0, unsigned& o1,
                                             unsigned& fl) {
  o0 = stg + pan * PANEL + v * (isw ? 4096 : CHUNK);
  o1 = o0 + (isw ? PANEL : 4096);
  fl = OFF_PF + (isw ? (pan + (v & 1)) * NCH + (v >> 1) : pan * NCH + v) * 128;
}
constexpr int kBI[][4] = P_BATCH_I, kBX[][4] = P_BATCH_X;
constexpr int NBI = sizeof(kBI) / sizeof(kBI[0]), NBX = sizeof(kBX) / sizeof(kBX[0]);
template <int N> __device__ __forceinline__ void batch_item(const int (&b)[N][4], int i, int& u, int& c) {
  int f = 0;
#pragma unroll
  for (int k = 0; k < N; ++k) {
    if (k == 0 || i >= f * NCH) { const int r = i - f * NCH; u = f + r % b[k][0]; c = r / b[k][0]; }
    f += b[k][0];
  }
}
__device__ __forceinline__ void link_item(bool cross, int i, bool& isw, int& u, int& c) {
  isw = cross && i < 128;
  if (isw) { u = i & 3; c = i >> 2; }
  else if (cross) batch_item(kBX, i - 128, u, c);
  else batch_item(kBI, i, u, c);
}
__device__ __forceinline__ unsigned landing_panel(bool cross, bool isw, int u, int from, int to) {
  return !cross ? NI + 3 * u + ((from - to - 1) & 3) : isw ? NAP + 32 + from * 8 + 2 * u : 4 * NI + 4 * u + from;
}

__device__ __forceinline__ void load_inputs(const Params& p, bool isw, int g, int c, int lane, f32x4 (&v)[16]) {
  if (isw) {
    const GAS f32x2* src =
        (const GAS f32x2*)(p.w + (size_t)((c & 31) * 128 + 4 * (lane >> 4)) * SH + (g & 15) * 32 + 2 * (lane & 15));
#pragma unroll
    for (int q = 0; q < 16; ++q) {
      const f32x2 a = src[((q >> 1) * 16 + (q & 1) * 2) * (SH / 2)], b = src[((q >> 1) * 16 + (q & 1) * 2 + 1) * (SH / 2)];
      v[q] = f32x4{a.x, b.x, a.y, b.y};
    }
  } else {
    const GAS f32x4* src =
        (const GAS f32x4*)(p.x + (size_t)((g & 31) * 16 + (lane >> 2)) * KD + (c & 15) * 256 + 4 * (lane & 3));
#pragma unroll
    for (int q = 0; q < 16; ++q) v[q] = src[q * 4];
  }
}
__device__ __forceinline__ void to_fragments(bool isw, const f32x4 (&v)[16], LAS v4u* scr, int lane, v4u (&h)[8]) {
  if (isw) {
#pragma unroll
    for (int s = 0; s < 4; ++s) {
      h[s] = v4u{pk(v[4 * s].x, v[4 * s].y), pk(v[4 * s + 1].x, v[4 * s + 1].y), pk(v[4 * s + 2].x, v[4 * s + 2].y),
                 pk(v[4 * s + 3].x, v[4 * s + 3].y)};
      h[4 + s] = v4u{pk(v[4 * s].z, v[4 * s].w), pk(v[4 * s + 1].z, v[4 * s + 1].w), pk(v[4 * s + 2].z, v[4 * s + 2].w),
                     pk(v[4 * s + 3].z, v[4 * s + 3].w)};
    }
  } else {
#pragma unroll
    for (int s = 0; s < 8; ++s)
      scr[s * 64 + lane] = v4u{pk(v[2 * s].x, v[2 * s].y), pk(v[2 * s].z, v[2 * s].w), pk(v[2 * s + 1].x, v[2 * s + 1].y),
                               pk(v[2 * s + 1].z, v[2 * s + 1].w)};
    LDSWAIT();
#pragma unroll
    for (int s = 0; s < 8; ++s) h[s] = scr[s * 64 + (lane & 15) * 4 + (lane >> 4)];
    LDSWAIT();
  }
}

struct Ctl { unsigned prog, bad, full[2], fre[2], conv; };

__device__ __forceinline__ void need(volatile LAS unsigned* word, unsigned& have, unsigned n) {
  while (have < n) {
    have = __builtin_amdgcn_readfirstlane(*word);
    if (have < n) __builtin_amdgcn_s_sleep(1);
  }
  CBAR();
}

template <int MT, int NT>
__device__ __forceinline__ void gemm_tile(const Params& p, LAS char* lds, int a0, int b0, int t, unsigned& have,
                                          unsigned stg, int wave, int lane) {
  constexpr int D = (MT + NT <= 5) ? P_DS : P_DL;
  constexpr int NS = 32;
  volatile LAS Ctl* ctl = (volatile LAS Ctl*)(lds + LDS_CTL);
  const __amdgpu_buffer_rsrc_t rs = mk(p.ws[p.idx & 3], WS_USED);
  const unsigned ab = stg + (unsigned)a0 * PANEL + wave * 1024, bb = stg + (unsigned)(NAP + b0) * PANEL + wave * 1024;
  const int voff = lane * 16;
  const unsigned base = (unsigned)t * NCH;
  f32x4 acc[MT][NT];
#pragma unroll
  for (int m = 0; m < MT; ++m)
#pragma unroll
    for (int n = 0; n < NT; ++n) acc[m][n] = f32x4{0.f, 0.f, 0.f, 0.f};
  v4u ra[D][MT], rb[D][NT];
#define LOADSTEP(d, i) do { \
    _Pragma("unroll") for (int m = 0; m < MT; ++m) ra[d][m] = ld16(rs, voff, ab + m * PANEL + (i) * 4096); \
    _Pragma("unroll") for (int n = 0; n < NT; ++n) rb[d][n] = ld16(rs, voff, bb + n * PANEL + (i) * 4096); \
    __builtin_amdgcn_sched_barrier(0); } while (0)
#define MMASTEP(d) do { \
    _Pragma("unroll") for (int m = 0; m < MT; ++m) \
    _Pragma("unroll") for (int n = 0; n < NT; ++n) \
      acc[m][n] = __builtin_amdgcn_mfma_f32_16x16x32_f16(__builtin_bit_cast(h8, rb[d][n]), \
                                                         __builtin_bit_cast(h8, ra[d][m]), acc[m][n], 0, 0, 0); \
    __builtin_amdgcn_sched_barrier(0); } while (0)
#pragma unroll
  for (int d = 0; d < D; ++d) {
    if ((d & 1) == 0) need(&ctl->prog, have, base + (d >> 1) + 1);
    LOADSTEP(d, d);
  }
#pragma unroll 1
  for (int s0 = D; s0 < NS; s0 += D) {
#pragma unroll
    for (int d = 0; d < D; ++d) {
      MMASTEP(d);
      if ((d & 1) == 0) need(&ctl->prog, have, base + ((s0 + d) >> 1) + 1);
      LOADSTEP(d, s0 + d);
    }
  }
#pragma unroll
  for (int d = 0; d < D; ++d) MMASTEP(d);
#undef LOADSTEP
#undef MMASTEP
  asm volatile("" : "+v"(lane));
  constexpr int C4 = 4 * NT, M1 = MT > 4 ? (MT + 1) / 2 : MT;
#pragma unroll
  for (int part = 0; part < (MT > 4 ? 2 : 1); ++part) {
    const int tp = t + part, m0 = part ? M1 : 0, mn = part ? MT - M1 : M1;
    unsigned f = 0;
    if (tp >= 2) need(&ctl->fre[tp & 1], f, (unsigned)(tp >> 1));
    else need(&ctl->conv, f, NWAVE);
    LAS f32x4* red = (LAS f32x4*)(lds + (tp & 1) * LDS_RED) + wave * (64 * mn * NT);
#pragma unroll
    for (int m = 0; m < mn; ++m)
#pragma unroll
      for (int n = 0; n < NT; n += 2) {
        const int row = m * 16 + (lane & 15), c4 = n * 4 + 2 * (lane >> 4);
        const f32x4 e = acc[m0 + m][n], o = acc[m0 + m][n + 1];
        red[row * C4 + (c4 ^ (row & (C4 - 1)))] = f32x4{e.x, o.x, e.y, o.y};
        red[row * C4 + ((c4 + 1) ^ (row & (C4 - 1)))] = f32x4{e.z, o.z, e.w, o.w};
      }
    LDSWAIT();
    if (lane == 0)
      __hip_atomic_fetch_add((LAS unsigned*)&ctl->full[tp & 1], 1u, __ATOMIC_RELAXED, __HIP_MEMORY_SCOPE_WORKGROUP);
  }
}

template <int MT, int NT>
__device__ __forceinline__ void finish_tile(const Params& p, LAS char* lds, int bfirst, unsigned desc, int t,
                                            unsigned& have, unsigned call, int lane) {
  constexpr int C4 = 4 * NT, IMG = 64 * MT * NT, RQ = 64 / C4;
  volatile LAS Ctl* ctl = (volatile LAS Ctl*)(lds + LDS_CTL);
  const LAS f32x4* red = (const LAS f32x4*)(lds + (t & 1) * LDS_RED);
  const int idx = p.idx & 3, a0 = desc & 127, b = (desc >> 7) & 7, grp = ((bfirst + ((desc >> 7) & 31)) >> 3) & 7;
  const bool staged = NT == 4 && grp >= 4 && ((desc >> 29) & 1);
  const __amdgpu_buffer_rsrc_t ro = staged ? mk(p.ws[idx], WS_USED) : mk(p.out[grp < 4 ? idx : grp], OUT_BYTES);
  const int r0 = lane / C4, c4 = lane % C4;
  const unsigned slot = ((grp & 3) * NAP + a0) * 2 + (b >> 2);
  unsigned vo[MT];
#pragma unroll
  for (int m = 0; m < MT; ++m)
    vo[m] = staged ? OFF_YS + (slot + 2 * m) * 4096 + lane * 16
                   : ((p.rowbase[(a0 + m) & 127] + r0) * SH + (grp < 4 ? grp : idx) * 128 + b * 16 + c4 * 4) * 4;
  const unsigned qs = staged ? 1024 : RQ * SH * 4;
  need(&ctl->full[t & 1], have, 4u * ((t >> 1) + 1));
  STAMP(8 + t * 8 + 2);
  const bool bad = ctl->bad != 0;
  const unsigned ticket = NT != 4 ? 0u : (unsigned)__builtin_amdgcn_raw_ptr_buffer_atomic_add_i32(
      MT, ro, staged && lane == 0 ? OFF_YQ + (grp & 3) * 128 : NOWHERE, 0, 16);
  f32x4 s[MT * NT];
#pragma unroll
  for (int q = 0; q < MT * NT; ++q) {
    const int row = q * RQ + r0;
    const int li = row * C4 + (c4 ^ (row & (C4 - 1)));
    s[q] = (red[li] + red[IMG + li]) + (red[2 * IMG + li] + red[3 * IMG + li]);
    if (bad) s[q] = f32x4{__builtin_nanf(""), __builtin_nanf(""), __builtin_nanf(""), __builtin_nanf("")};
    if (q < P_YWIN) st16(ro, vo[q / NT] + (q % NT) * qs, __builtin_bit_cast(v4u, s[q]));
  }
  LDSWAIT();
  if (lane == 0) ctl->fre[t & 1] = (t >> 1) + 1;
  if (grp >= 4 && !staged) {
#pragma unroll
    for (int q = P_YWIN; q < MT * NT; ++q) {
      asm volatile("s_waitcnt vmcnt(%0)" :: "n"(P_YWIN - 1) : "memory");
      st16(ro, vo[q / NT] + (q % NT) * qs, __builtin_bit_cast(v4u, s[q]));
    }
  } else {
#pragma unroll
    for (int q = P_YWIN; q < MT * NT; ++q) st16(ro, vo[q / NT] + (q % NT) * qs, __builtin_bit_cast(v4u, s[q]));
  }
  if (grp >= 4) {
    DRAIN();
    if (staged) {
      const unsigned pos = __builtin_amdgcn_readfirstlane(ticket) - (call - 1) * 2 * p.nyf + lane;
      flag(lane < MT && pos < 2 * NAP, ro, OFF_YL + ((grp & 3) * 2 * NAP + pos) * 128, call, (slot & 255) + 2 * lane);
    }
    else flag(lane == 0, mk(p.ws[grp], WS_USED), OFF_YF + (idx * YMAX + ((desc >> 13) & 255)) * 128, call);
  }
  STAMP(8 + t * 8 + 3);
}

#define BY_SHAPE(desc, BIG, F, ...) \
  switch (((desc) >> 24) & 15) { \
    case 1: F<1, 2>(__VA_ARGS__); break; \
    case 2: F<2, 2>(__VA_ARGS__); break; \
    case 3: F<3, 2>(__VA_ARGS__); break; \
    case 9: F<1, 4>(__VA_ARGS__); break; \
    case 10: F<2, 4>(__VA_ARGS__); break; \
    case 11: F<3, 4>(__VA_ARGS__); break; \
    case 12: F<4, 4>(__VA_ARGS__); break; \
    case 13: F<BIG ? 5 : 3, 4>(__VA_ARGS__); break; \
    default: F<BIG ? 6 : 3, 4>(__VA_ARGS__); break; \
  }

__global__ __launch_bounds__(NTHR, 2) void ag_gemm(const Params p) {
  extern __shared__ __attribute__((aligned(16))) char smem[];
  LAS char* lds = (LAS char*)smem;
  if (threadIdx.x < 16) ((volatile LAS unsigned*)(lds + LDS_CTL))[threadIdx.x] = 0;
  LDSWAIT();
  __builtin_amdgcn_s_barrier();
  const int lane = threadIdx.x & 63;
  const int wave = __builtin_amdgcn_readfirstlane(threadIdx.x >> 6);
  const int blk = blockIdx.x, grp = blk & 7, j = blk >> 3;
  const int idx = p.idx & 3;
  const bool cross = grp >= 4;
  char* myws = p.ws[idx];
  const __amdgpu_buffer_rsrc_t rl = mk(myws, WS_USED), rd = mk(p.ws[grp], WS_USED);
  STAMP(0);
  char* counter = myws + OFF_CNT + (blk * 8 + wave) * 4;
  const unsigned last_call = ld_sys(counter);

  if (j < NC && grp != idx) {
    struct Ring { unsigned full[NL], done, abort; };
    volatile LAS Ring* rg = (volatile LAS Ring*)(lds + LDS_CTL);
    volatile LAS v4u* meta = (volatile LAS v4u*)(lds + 2 * NL * 8192);
    const int n = !cross ? NI * NCH : NXY + (P_NOGEMM ? 0 : p.nyf);
    if (wave == 0) {
      const unsigned long long tw = __builtin_amdgcn_s_memrealtime();
      const unsigned call = __builtin_amdgcn_readfirstlane(last_call) + 1u;
      if (lane == 0) st_sys(counter, call);
      __builtin_amdgcn_s_setprio(3);
      const unsigned long long peer_out = uniform64(p.out[grp]), peer_ws = uniform64(p.ws[grp]);
      unsigned pl[P_DEPTH], ph[P_DEPTH];
#pragma unroll
      for (int d = 0; d < P_DEPTH; ++d) { pl[d] = NOWHERE; ph[d] = 0; }
      const int cnt = (n - ND - j + NC - 1) / NC;
      int l = 0, k = 0;
      while ((int)(__builtin_amdgcn_s_memrealtime() - tw) < P_PACE0 + ND * P_PACE / 2 - P_HOLD) __builtin_amdgcn_s_sleep(2);
      TREC_DECL();
      for (int r = 0; r < cnt; ++r) {
        if (__builtin_amdgcn_readfirstlane(rg->full[l]) <= (unsigned)k) {
          DRAIN();
#pragma unroll
          for (int d = 0; d < P_DEPTH; ++d) { flag(lane == 0, rd, pl[d], call, ph[d]); pl[d] = NOWHERE; }
          while (__builtin_amdgcn_readfirstlane(rg->full[l]) <= (unsigned)k) __builtin_amdgcn_s_sleep(0);
        }
        if (__builtin_amdgcn_readfirstlane(rg->abort)) { rg->done = 0xffffffffu; return; }
        CBAR();
        const int slot = 2 * l + (k & 1);
        const v4u m = meta[slot];
        v4u v[8];
#pragma unroll
        for (int q = 0; q < 8; ++q) v[q] = *(const LAS v4u*)(lds + slot * 8192 + q * 1024 + lane * 16);
        const unsigned hd = __builtin_amdgcn_readfirstlane(m.w);
        const bool isy = (hd >> 17) & 1;
        const __amdgpu_buffer_rsrc_t rs = mk((char*)(isy ? peer_out : peer_ws), isy ? OUT_BYTES : WS_USED);
        const unsigned vb = isy ? (lane >> 4) * (SH * 4) + (lane & 15) * 16 : lane * 16, qs = isy ? 4 * SH * 4 : 1024;
#pragma unroll
        for (int q = 0; q < 8; ++q) asm volatile("" : "+v"(v[q]));
        rg->done = r + 1;
        asm volatile("s_waitcnt vmcnt(%0)" :: "n"(9 * (P_DEPTH - 1)) : "memory");
        flag(lane == 0, rd, pl[0], call, ph[0]);
        if (!(r & 1)) TREC(r >> 1);
        CBAR();
        if (P_WIN) asm volatile("s_waitcnt vmcnt(%0)" :: "n"(P_WIN + 1) : "memory");
#pragma unroll
        for (int q = 0; q < 4; ++q) st16(rs, vb + m.x + q * qs, v[q]);
        if (P_WIN) asm volatile("s_waitcnt vmcnt(%0)" :: "n"(P_WIN + 1) : "memory");
#pragma unroll
        for (int q = 0; q < 3; ++q) st16(rs, vb + m.y + q * qs, v[4 + q]);
        st16(rs, lane < (hd >> 24) ? vb + m.y + 3 * qs : NOWHERE, v[7]);
        CBAR();
#pragma unroll
        for (int d = 0; d + 1 < P_DEPTH; ++d) { pl[d] = pl[d + 1]; ph[d] = ph[d + 1]; }
        pl[P_DEPTH - 1] = __builtin_amdgcn_readfirstlane(m.z); ph[P_DEPTH - 1] = hd;
        if (++l == NL) { l = 0; ++k; }
      }
#pragma unroll
      for (int d = 0; d < P_DEPTH; ++d) {
        if (d == 0) asm volatile("s_waitcnt vmcnt(%0)" :: "n"(9 * (P_DEPTH - 1)) : "memory");
        else DRAIN();
        flag(lane == 0, rd, pl[d], call, ph[d]);
      }
      STAMP(1);
      TREC_DUMP();
      return;
    }
    const int l = wave - 1, first = ND + l * NC + j, cnt = (n - first + WL - 1) / WL;
    bool isw; int u, c;
    const unsigned call = __builtin_amdgcn_readfirstlane(last_call) + 1u;
    if (lane == 0) st_sys(counter, call);
    const unsigned stg = OFF_STG + (call & 1u) * STG_BYTES;
    const unsigned long long t0 = __builtin_amdgcn_s_memrealtime();
    if (P_TOUCH && wave == NL) {
      unsigned t = __builtin_amdgcn_raw_buffer_load_b32(rd, lane < 26 ? stg + (lane << 20) : lane < 28 ? (lane - 26) << 20 : NOWHERE, 0, AUX_SYS);
      t += __builtin_amdgcn_raw_buffer_load_b32(mk(p.out[grp], OUT_BYTES), cross && lane < 8 ? lane << 20 : NOWHERE, 0, AUX_SYS);
      t += __builtin_amdgcn_raw_buffer_load_b32(rl, lane < 26 ? stg + (lane << 20) : lane < 40 ? (lane - 26) << 20 : NOWHERE, 0, AUX_SYS);
      t += __builtin_amdgcn_raw_buffer_load_b32(mk(p.out[idx], OUT_BYTES), lane < 8 ? lane << 20 : NOWHERE, 0, AUX_SYS);
      asm volatile("" :: "v"(t));
    }
    const int i = min(first + lane * WL, n - 1);
    unsigned s0, s1, sf, sg, d0, d1, df, kind;
    if (!cross || i < NXY) {
      link_item(cross, i, isw, u, c);
      const int g = isw ? (grp - 4) * 4 + u : cross ? NI + u : u;
      item_offsets(isw, stg, isw ? NAP + 2 * g : cross ? NAP + NBP + u : u, c, s0, s1, sf);
      item_offsets(isw, stg, landing_panel(cross, isw, u, idx, grp & 3), c, d0, d1, df);
      sg = sf;
      kind = i < (cross ? NPX : NPI) && !(isw && i >= 128 - P_NRAWW);
      if (kind) {
        s0 = OFF_EXP + (isw ? 512 + g * 32 + c : g * 16 + c) * 8192; s1 = s0 + 4096;
        df += OFF_AF + (call & 1u) * PF_BYTES - OFF_PF;
      }
    } else {
      sf = OFF_YL + ((grp & 3) * NAP + ((i - NXY) & 127)) * 256; sg = sf + 128;
      s0 = s1 = d0 = d1 = 0;
      df = OFF_YF + (idx * YMAX + (i - NXY)) * 128;
      kind = 2;
    }
    const unsigned rb0 = p.rowbase[lane], rb1 = p.rowbase[64 + lane];
    if (lane == 63) { sf = OFF_TMO; sg = OFF_TMO; }
    unsigned long long ready = 0;
    unsigned hdv = 64u << 24 | kind << 16;
    for (int k = 0; k < cnt; ++k) {
      while (!((ready >> k) & 1)) {
        const bool ask = ((lane >= k && lane < min(cnt, k + (k < 2 ? P_LWIN : 64))) || lane == 63) && !((ready >> lane) & 1);
        const v2u f = ldflag(rl, ask ? sf : NOWHERE);
        const v2u g = ldflag(rl, ask && kind == 2 ? sg : NOWHERE);
        const bool ok = f.x == call && (kind != 2 || g.x == call);
        if (ok && kind == 1) hdv = f.y;
        if (ok && kind == 2) { s0 = f.y & 255; s1 = g.y & 255; }
        ready |= __ballot(ok);
        if ((ready >> 63) || __builtin_amdgcn_s_memrealtime() - t0 > DEADLINE) {
          st_sys(myws + OFF_TMO, call);
          rg->abort = 1; LDSWAIT(); rg->full[l] = 0xffffffffu;
          return;
        }
      }
      CBAR();
      const unsigned hd = __builtin_amdgcn_readlane(hdv, k);
      unsigned a0 = __builtin_amdgcn_readlane(s0, k), a1 = __builtin_amdgcn_readlane(s1, k);
      unsigned e0 = __builtin_amdgcn_readlane(d0, k), e1 = __builtin_amdgcn_readlane(d1, k);
      if ((hd >> 17) & 1) {
        e0 = (__builtin_amdgcn_readlane(a0 & 128 ? rb1 : rb0, (a0 >> 1) & 63) * SH + idx * 128) * 4 + (a0 & 1) * 256;
        e1 = (__builtin_amdgcn_readlane(a1 & 128 ? rb1 : rb0, (a1 >> 1) & 63) * SH + idx * 128) * 4 + (a1 & 1) * 256;
        a0 = OFF_YS + ((grp & 3) * 2 * NAP + a0) * 4096;
        a1 = OFF_YS + ((grp & 3) * 2 * NAP + a1) * 4096;
      }
      v4u h[8];
#pragma unroll
      for (int q = 0; q < 4; ++q) h[q] = ld16(rl, lane * 16 + q * 1024, a0);
#pragma unroll
      for (int q = 0; q < 3; ++q) h[4 + q] = ld16(rl, lane * 16 + q * 1024, a1);
      h[7] = ld16(rl, lane < (hd >> 24) ? lane * 16 + 3072 : NOWHERE, a1);
      if (k >= 2) {
        unsigned f = 0;
        need(&rg->done, f, (unsigned)((k - 2) * NL + l + 1));
        if (__builtin_amdgcn_readfirstlane(rg->abort)) return;
      }
      LAS v4u* sl = (LAS v4u*)(lds + (2 * l + (k & 1)) * 8192);
#pragma unroll
      for (int q = 0; q < 8; ++q) sl[q * 64 + lane] = h[q];
      meta[2 * l + (k & 1)] = v4u{e0, e1, (unsigned)__builtin_amdgcn_readlane(df, k), hd};
      LDSWAIT();
      rg->full[l] = k + 1;
    }
    STAMP(1);
    return;
  }
  if (j < NC + NU) {
    const unsigned call = __builtin_amdgcn_readfirstlane(last_call) + 1u;
    if (lane == 0) st_sys(counter, call);
    const unsigned stg = OFF_STG + (call & 1u) * STG_BYTES;
    const unsigned aoff = OFF_AF + (call & 1u) * PF_BYTES - OFF_PF;
    const unsigned long long t0 = __builtin_amdgcn_s_memrealtime();
    if (P_PACK) {
      const int lk = grp == idx ? (idx + 1 + wave) & 7 : grp;
      const bool xl = lk >= 4;
      const int n = xl ? NPX - P_NRAWW : NPI, first = NDRAW + (grp == idx ? NU * NWAVE + j : wave * NU + j - NC);
      const int cnt = (n - first + WU - 1) / WU;
      bool isw; int u, c;
      const int ui = min(first + lane * WU, n - 1);
      link_item(xl, xl && ui >= 128 - P_NRAWW ? ui + P_NRAWW : ui, isw, u, c);
      unsigned o0, o1, fl;
      item_offsets(isw, stg, landing_panel(xl, isw, u, lk & 3, idx), c, o0, o1, fl);
      LAS unsigned* plane = (LAS unsigned*)(lds + wave * 1024);
      unsigned pend = NOWHERE, hdr = 0;
      bool have_next = false;
      TREC_DECL();
      for (int r = 0; r < cnt; ++r) {
        const unsigned fr = __builtin_amdgcn_readlane(fl, r);
        *(LAS v4u*)(plane + lane * 4) = v4u{0, 0, 0, 0};
        while (!have_next) {
          const v2u f = ldflag(rl, lane == 0 ? fr + aoff : lane == 1 ? OFF_TMO : NOWHERE);
          if (pend != NOWHERE) {
            DRAIN();
            flag(lane == 0, rl, pend, call);
            TREC(4 * r - 1);
            pend = NOWHERE;
          }
          hdr = __builtin_amdgcn_readfirstlane(f.y);
          if ((int)(__builtin_amdgcn_readfirstlane(f.x) - call) >= 0) break;
          if (__builtin_amdgcn_readlane(f.x, 1) == call || __builtin_amdgcn_s_memrealtime() - t0 > DEADLINE) {
            st_sys(myws + OFF_TMO, call);
            st_sys(p.out[idx] + (size_t)p.rowbase[0] * SH * 4, 0x7fc00000u);
            return;
          }
        }
        CBAR();
        TREC(4 * r);
        const unsigned a0 = __builtin_amdgcn_readlane(o0, r), a1 = __builtin_amdgcn_readlane(o1, r);
        const bool dense = (hdr >> 16) & 1;
        const unsigned e2 = __builtin_amdgcn_raw_buffer_load_b32(rl, lane * 4 + 3072, a1, AUX_SYS);
        CBAR();
        v4u h[8];
#pragma unroll
        for (int q = 0; q < 4; ++q) h[q] = ld16(rl, lane * 16 + q * 1024, a0);
#pragma unroll
        for (int q = 0; q < 3; ++q) h[4 + q] = ld16(rl, lane * 16 + q * 1024, a1);
        h[7] = ld16(rl, dense ? lane * 16 + 3072 : NOWHERE, a1);
        const v2u fn = ldflag(rl, lane == 0 ? __builtin_amdgcn_readlane(fl, min(r + 1, cnt - 1)) + aoff : NOWHERE);
        CBAR();
        const unsigned cn = dense ? 0 : (hdr >> 8) & 255;
        LDSWAIT();
#pragma unroll
        for (int k = 0; k < 2; ++k) {
          const unsigned e = (e2 >> (16 * k)) & 0xffffu;
          __hip_atomic_fetch_or(plane + ((e >> 7) & 255), 2 * lane + k < cn ? (e & 15) << ((e >> 2) & 28) : 0u,
                                __ATOMIC_RELAXED, __HIP_MEMORY_SCOPE_WORKGROUP);
        }
        LDSWAIT();
        const v4u pl = *(volatile LAS v4u*)(plane + lane * 4);
        DRAIN();
        flag(lane == 0, rl, pend, call);
        if (pend != NOWHERE) TREC(4 * r - 1);
        TREC(4 * r + 1);
        h[7] |= pl;
        unpack8(h, hdr & 31);
#pragma unroll
        for (int q = 0; q < 4; ++q) st16(rl, a0 + lane * 16 + q * 1024, h[q]);
#pragma unroll
        for (int q = 0; q < 4; ++q) st16(rl, a1 + lane * 16 + q * 1024, h[4 + q]);
        pend = fr;
        TREC(4 * r + 2);
        hdr = __builtin_amdgcn_readfirstlane(fn.y);
        have_next = (int)(__builtin_amdgcn_readfirstlane(fn.x) - call) >= 0;
      }
      DRAIN();
      flag(lane == 0, rl, pend, call);
      TREC_DUMP();
      STAMP(2);
    }
    if (grp == idx) {
      const int f = (j * NWAVE + wave) * 64 + lane;
      if (f - lane >= 4 * YMAX) return;
      const bool mine = (f % YMAX) < p.ny;
      while (!P_NOGEMM) {
        const unsigned v = __builtin_amdgcn_raw_buffer_load_b32(rl, mine ? OFF_YF + f * 128 : NOWHERE, 0, AUX_SYS);
        const unsigned tm = __builtin_amdgcn_readfirstlane(ld_sys(myws + OFF_TMO));
        if (__all(!mine || (int)(v - call) >= 0)) break;
        if (tm == call || __builtin_amdgcn_s_memrealtime() - t0 > DEADLINE) {
          st_sys(myws + OFF_TMO, call);
          st_sys(p.out[idx] + (size_t)p.rowbase[0] * SH * 4, 0x7fc00000u);
          break;
        }
        __builtin_amdgcn_s_sleep(1);
      }
      STAMP(1);
    }
    return;
  }

  const int jg = j - NC - NU;
  volatile LAS Ctl* ctl = (volatile LAS Ctl*)(lds + LDS_CTL);
  unsigned call;
  {
    const unsigned long long tw = __builtin_amdgcn_s_memrealtime();
    const bool direct = (wave == 6 || wave == 4) && grp != idx;
    const int di = wave == 6 ? jg : NG + jg, leave = P_PACE0 + di * P_PACE / 2;
    const int it = wave == 6 || wave == 4 ? NG * 40 + di : min(wave, 4) * (NG * 8) + jg * 8 + grp;
    const bool work = direct ? di < ND : it < 1024;
    bool isw = false, raw = di < NDRAW, packed = !raw;
    int g = 0, c = 0, u = 0;
    f32x4 v[16];
    if (P_PRIO) __builtin_amdgcn_s_setprio(3);
    if (work && direct) {
      link_item(cross, di, isw, u, c);
      g = isw ? (grp - 4) * 4 + u : u;
      while ((int)(__builtin_amdgcn_s_memrealtime() - tw) < leave - P_LEAD) __builtin_amdgcn_s_sleep(2);
    } else if (work) {
      const int k = (it / 5 + ND) & 127, m = it % 5;
      isw = it < 640 && m < 4;
      if (isw) { g = m * 4 + (k & 3); c = k >> 2; raw = true; packed = k >= ND && k < 128 - P_NRAWW; }
      else {
        const int xi = it < 640 ? k : it - 512;
        const bool xc = xi >= NI * NCH;
        const int li = xc ? xi - NI * NCH + 128 : xi;
        bool dummy;
        link_item(xc, li, dummy, u, c);
        g = xc ? NI + u : u;
        packed = li >= ND && li < (xc ? NPX : NPI);
        raw = !xc || !packed;
      }
      while ((int)(__builtin_amdgcn_s_memrealtime() - tw) < P_CDELAY0 + (it * P_CDELAY >> 8)) __builtin_amdgcn_s_sleep(2);
    }
    if (work) load_inputs(p, isw, g, c, lane, v);
    call = __builtin_amdgcn_readfirstlane(last_call) + 1u;
    if (lane == 0) st_sys(counter, call);
    if (work) {
      const unsigned stg = OFF_STG + (call & 1u) * STG_BYTES;
      LAS v4u* scr = (LAS v4u*)(lds + wave * 8192);
      v4u h[8];
      to_fragments(isw, v, scr, lane, h);
      unsigned o0, o1, fl, hdr = 0;
      item_offsets(isw, stg,
                   direct ? landing_panel(cross, isw, u, idx, grp & 3) : isw ? NAP + 2 * g : g < NI ? g : NAP + NBP + u, c,
                   o0, o1, fl);
      const __amdgpu_buffer_rsrc_t rs = mk(direct ? p.ws[grp] : myws, WS_USED);
      STAMP(7);
      if (raw) {
        if (direct) while ((int)(__builtin_amdgcn_s_memrealtime() - tw) < leave) __builtin_amdgcn_s_sleep(1);
#pragma unroll
        for (int s = 0; s < 4; ++s) st16(rs, o0 + lane * 16 + s * 1024, h[s]);
#pragma unroll
        for (int s = 0; s < 4; ++s) st16(rs, o1 + lane * 16 + s * 1024, h[4 + s]);
      }
      if (P_PACK && packed) {
        float ss = 0.f;
#pragma unroll
        for (int q = 0; q < 16; q += 4) ss += v[q].x * v[q].x + v[q].y * v[q].y + v[q].z * v[q].z + v[q].w * v[q].w;
        ss += __builtin_bit_cast(float, dpp0<0x111>(__builtin_bit_cast(unsigned, ss)));
        ss += __builtin_bit_cast(float, dpp0<0x112>(__builtin_bit_cast(unsigned, ss)));
        ss += __builtin_bit_cast(float, dpp0<0x114>(__builtin_bit_cast(unsigned, ss)));
        ss += __builtin_bit_cast(float, dpp0<0x118>(__builtin_bit_cast(unsigned, ss)));
        const unsigned sb = __builtin_bit_cast(unsigned, ss);
        const float tot = (__builtin_bit_cast(float, __builtin_amdgcn_readlane(sb, 15)) +
                           __builtin_bit_cast(float, __builtin_amdgcn_readlane(sb, 31))) +
                          (__builtin_bit_cast(float, __builtin_amdgcn_readlane(sb, 47)) +
                           __builtin_bit_cast(float, __builtin_amdgcn_readlane(sb, 63)));
        const float top = __builtin_amdgcn_sqrtf(tot * (6.76f / 1024.f));
        const int ex = (int)((__builtin_amdgcn_readfirstlane(__builtin_bit_cast(unsigned, top)) >> 23) & 255) - 127 + 15 - 7;
        const unsigned base = (unsigned)min(max(ex, 0), 24);
        pack8(h, base);
        unsigned nz[4], cnt = 0;
#pragma unroll
        for (int d = 0; d < 4; ++d) {
          unsigned z = h[7][d];
          z |= z >> 1; z |= z >> 2;
          nz[d] = z & 0x11111111u;
          cnt += __builtin_popcount(nz[d]);
        }
        unsigned inc = cnt;
        inc += dpp0<0x111>(inc); inc += dpp0<0x112>(inc); inc += dpp0<0x114>(inc); inc += dpp0<0x118>(inc);
        const unsigned r0 = __builtin_amdgcn_readlane(inc, 15), r1 = r0 + __builtin_amdgcn_readlane(inc, 31),
                       r2 = r1 + __builtin_amdgcn_readlane(inc, 47), total = r2 + __builtin_amdgcn_readlane(inc, 63);
        const bool dense = total > (unsigned)LIST_CAP;
        if (!dense) {
          unsigned pos = inc - cnt + (lane < 16 ? 0 : lane < 32 ? r0 : lane < 48 ? r1 : r2);
          LAS unsigned short* list = (LAS unsigned short*)scr;
#pragma unroll
          for (int d = 0; d < 4; ++d) {
            unsigned m = nz[d];
            while (m) {
              const unsigned b = __builtin_ctz(m);
              list[pos++] = (unsigned short)(((lane * 4 + d) << 7) | (b << 2) | ((h[7][d] >> b) & 15));
              m &= m - 1;
            }
          }
          LDSWAIT();
          h[7] = *(volatile LAS v4u*)(scr + (lane & 15));
        }
        const unsigned nl = dense ? 64 : max((total + 31) >> 5, 1u) * 4;
        hdr = base | (dense ? 1u << 16 : total << 8) | nl << 24;
        if (direct) {
          fl += OFF_AF + (call & 1u) * PF_BYTES - OFF_PF;
          while ((int)(__builtin_amdgcn_s_memrealtime() - tw) < leave) __builtin_amdgcn_s_sleep(1);
        } else {
          o0 = OFF_EXP + (isw ? 512 + g * 32 + c : g * 16 + c) * 8192;
          o1 = o0 + 4096;
        }
#pragma unroll
        for (int s = 0; s < 4; ++s) st16(rs, o0 + lane * 16 + s * 1024, h[s]);
#pragma unroll
        for (int s = 0; s < 3; ++s) st16(rs, o1 + lane * 16 + s * 1024, h[4 + s]);
        st16(rs, lane < nl ? o1 + lane * 16 + 3072 : NOWHERE, h[7]);
      }
      STAMP(6);
      DRAIN();
      flag(lane == 0, rs, fl, call, hdr);
    }
    if (lane == 0) __hip_atomic_fetch_add((LAS unsigned*)&ctl->conv, 1u, __ATOMIC_RELAXED, __HIP_MEMORY_SCOPE_WORKGROUP);
    STAMP(2);
  }
  const unsigned stg = OFF_STG + (call & 1u) * STG_BYTES;
  const int gg = grp ^ P_SWAP;
  const unsigned* tl = p.tiles + (((gg >> 2) * XN + (gg & (XN - 1))) * NG + jg) * MAXT;
  const int bfirst = (gg >> P_XS << P_XS) * 8;
  if (P_PRIO && wave < 4) __builtin_amdgcn_s_setprio(0);
  if (wave == 4) {
    const unsigned long long t0 = __builtin_amdgcn_s_memrealtime();
    unsigned bdone = 0;
    for (int t = 0; t < MAXT && !P_NOGEMM; ++t) {
      const unsigned desc = tl[t];
      if (!(desc >> 31)) break;
      if ((desc >> 28) & 1) continue;
      const int mt = (desc >> 24) & 7, nt = (desc >> 27) & 1 ? 4 : 2, b = (desc >> 7) & 31;
      const unsigned bm = ((1u << nt) - 1) << b;
      const int np = (bdone & bm) == bm ? mt : mt + nt, l2 = np <= 4 ? 2 : np <= 8 ? 3 : 4;
      const int q = lane & ((1 << l2) - 1), ahead = lane >> l2;
      const unsigned fo = OFF_PF + (q < mt ? (desc & 127) + q : NAP + bfirst + b + (q - mt)) * (NCH * 128);
      unsigned pub = 0, win = 16;
      while (pub < NCH) {
        const unsigned c = pub + ahead;
        const bool act = q < np && c < NCH && ahead < win;
        const unsigned v = __builtin_amdgcn_raw_buffer_load_b32(rl, act ? fo + c * 128 : NOWHERE, 0, AUX_SYS);
        const unsigned tm = __builtin_amdgcn_readfirstlane(ld_sys(myws + OFF_TMO));
        const unsigned long long m = __ballot(act ? (int)(v - call) < 0 : ahead >= win);
        if (tm == call || __builtin_amdgcn_s_memrealtime() - t0 > DEADLINE) {
          st_sys(myws + OFF_TMO, call);
          ctl->bad = 1;
          LDSWAIT();
          ctl->prog = 0xffffffffu;
          return;
        }
        const unsigned n = m ? (unsigned)__builtin_ctzll(m) >> l2 : 64u >> l2;
        win = n >= win || n >= (64u >> l2) ? 16 : n + P_PWIN;
        if (n) {
          pub = min(pub + n, (unsigned)NCH);
          ctl->prog = t * NCH + pub;
        }
        if (t > 0 && __builtin_amdgcn_readfirstlane(ctl->full[(t - 1) & 1]) < 4u * ((t - 1) >> 1) + 1u)
          __builtin_amdgcn_s_sleep(P_PSLOW);
        else if (!n) __builtin_amdgcn_s_sleep(P_PSLEEP);
      }
      bdone |= bm;
      STAMP(8 + t * 8 + 4);
    }
    return;
  }
  unsigned have = 0;
  if (wave > 4) {
    for (int t = wave - 5; t < MAXT && !P_NOGEMM; t += 2) {
      const unsigned desc = tl[t];
      if (!(desc >> 31)) break;
      BY_SHAPE(((desc >> 24) & 7) > 4 ? 11u << 24 : desc, false, finish_tile, p, lds, bfirst, desc, t, have, call, lane)
    }
    return;
  }
  for (int t = 0; t < MAXT && !P_NOGEMM; ++t) {
    const unsigned desc = tl[t];
    if (!(desc >> 31)) break;
    if ((desc >> 28) & 1) continue;
    STAMP(8 + t * 8);
    const int a0 = desc & 127, b0 = bfirst + ((desc >> 7) & 31);
    BY_SHAPE(desc, true, gemm_tile, p, lds, a0, b0, t, have, stg, wave, lane)
  }
  STAMP(1);
}

static const int kClass[2][4] = {{0, 2, 3, 6}, {1, 4, 5, 7}};

static bool build_params(Params& p, int rank, int* roles) {
  int cls = 0, idx = 0;
  for (int c = 0; c < 2; ++c)
    for (int i = 0; i < 4; ++i) if (kClass[c][i] == rank) { cls = c; idx = i; }
  for (int i = 0; i < 4; ++i) { roles[i] = kClass[cls][i]; roles[4 + i] = kClass[cls ^ 1][i]; }
  p.idx = idx;
  for (int a = 0; a < NAP; ++a) {
    int src, rt;
    if (a < NI) { src = rank; rt = a; }
    else if (a < 4 * NI) { rt = (a - NI) / 3; src = roles[(idx + 1 + (a - NI) % 3) & 3]; }
    else { rt = NI + (a - 4 * NI) / 4; src = roles[4 + (a - 4 * NI) % 4]; }
    p.rowbase[a] = (unsigned short)(src * SH + rt * 16);
  }
  struct Seg { int a0, n, mt, nt, staged; double t0, t1; };
  std::vector<Seg> segs;
  const double per = P_PER, t00 = P_T00;
  segs.push_back({0, NI, P_OWN, 4, 1, 11.0, 12.5});
  int f = 0;
  for (int b = 0; b < NBI; ++b) {
    segs.push_back({NI + 3 * f, 3 * kBI[b][0], kBI[b][1], kBI[b][2], kBI[b][3], f ? t00 + f * per : 10.0,
                    t00 + (f + kBI[b][0]) * per - (b == NBI - 1 ? 3.2 : 0.0)});
    f += kBI[b][0];
  }
  if (f != NI) return false;
  for (int b = 0; b < NBX; ++b) {
    segs.push_back({4 * NI + 4 * (f - NI), 4 * kBX[b][0], kBX[b][1], kBX[b][2], kBX[b][3], t00 + (8 + f - NI) * per,
                    t00 + 1.0 + (8 + f - NI + kBX[b][0]) * per - (b == NBX - 1 && P_NRAWX ? 2.7 : 0.0)});
    f += kBX[b][0];
  }
  if (f != 32) return false;
  auto dur = [](int mt, int nt) { return 0.6 + P_K * (mt + nt); };
  std::stable_sort(segs.begin(), segs.end(), [&](const Seg& a, const Seg& b) {
    return std::max(a.t0, a.t1 + 1.8 - dur(a.mt, a.nt)) < std::max(b.t0, b.t1 + 1.8 - dur(b.mt, b.nt));
  });
  for (int h = 0; h < 2; ++h) {
    std::vector<double> fr(XN * NG, 0.0);
    std::vector<int> cnt(XN * NG, 0);
    std::vector<std::pair<unsigned*, int>> direct;
    int ndirect = 0;
    std::vector<std::pair<double, int>> done;
    for (auto& s : segs) {
      if ((s.nt != 2 && s.nt != 4) || s.mt < 1 || s.mt > (s.nt == 4 ? 6 : 3) || (s.staged && s.nt != 4)) return false;
      const double t1 = h ? std::max(s.t1, t00 + 0.5 + 8 * per - (P_NRAWW ? 2.0 : 0.0)) : s.t1;
      const int rows = (s.n + s.mt - 1) / s.mt;
      for (int r = 0, a = s.a0; r < rows; ++r) {
        const int mt = s.n / rows + (r < s.n % rows), ntile = 8 * XN / s.nt;
        auto finish = [&](double free_at) { return std::max(std::max(free_at, s.t0) + dur(mt, s.nt), t1 + 1.8); };
        int home = 0; double bt = 1e9;
        for (int x = 0; x < XN; ++x) {
          std::vector<double> v(fr.begin() + x * NG, fr.begin() + (x + 1) * NG);
          std::sort(v.begin(), v.end());
          double sum = 0;
          for (int q = 0; q < NG; ++q) sum += (q < ntile ? finish(v[q]) : 0.0) + 1e-3 * v[q];
          if (sum < bt) { bt = sum; home = x; }
        }
        const bool st1 = s.staged != 0, st0 = st1 && r >= s.staged - 1;
        double fin = 0;
        for (int b = 0; b < 8 * XN; b += s.nt) {
          int best = -1; double bf = 0;
          for (int g = 0; g < XN * NG; ++g) {
            const double e = finish(fr[g]) + (g / NG == home ? 0.0 : 0.7) + 1e-3 * fr[g];
            if (cnt[g] + (mt > 4) < MAXT && (best < 0 || e < bf)) { best = g; bf = e; }
          }
          if (best < 0) return false;
          fr[best] = finish(fr[best]);
          fin = std::max(fin, fr[best]);
          unsigned* e = &p.tiles[(h * XN * NG + best) * MAXT + cnt[best]++];
          const unsigned common = 0x80000000u | (s.nt == 4 ? 1u << 27 : 0u) | ((unsigned)b << 7);
          *e = common | (st0 ? 1u << 29 : 0u) | ((unsigned)mt << 24) | (unsigned)a;
          if (!st0) direct.push_back({e, ndirect + (b & 7) / s.nt});
          if (mt > 4) {
            e[1] = common | (st1 ? 1u << 29 : 0u) | 1u << 28 | ((unsigned)(mt - 3) << 24) | (unsigned)(a + 3);
            if (!st1) direct.push_back({e + 1, ndirect + 2 + (b & 7) / s.nt});
            ++cnt[best];
          }
        }
        if (!st0) ndirect += (mt > 4 && !st1 ? 16 : 8) / s.nt;
        for (int m = 0; m < mt; ++m)
          if (mt > 4 && m >= 3 ? st1 : st0) done.push_back({fin, a + m});
        a += mt;
      }
    }
    const int nyf = (int)done.size();
    for (auto& d : direct) *d.first |= (unsigned)(nyf + d.second) << 13;
    p.nyf = nyf;
    p.ny = nyf + ndirect;
    if (p.ny > YMAX) return false;
  }
  return true;
}

}

extern "C" void kernel_launch(int rank, int nranks, ncclComm_t comm, hipStream_t stream, void* const* d_in,
                              const int* in_sizes, int n_in, void* d_out, int out_size, const int* in_dtypes,
                              int out_dtype, void* const* d_out_all, void* const* d_ws_all, size_t ws_bytes) {
  using namespace agg;
  if (nranks != NR || n_in != 2 || ws_bytes < WS_USED || out_size != 4096 * SH || in_sizes[0] != SH * KD ||
      in_sizes[1] != KD * SH || (unsigned)rank >= (unsigned)NR) {
    fprintf(stderr, "ag_gemm: unexpected problem shape\n");
    return;
  }
  Params p;
  memset(&p, 0, sizeof(p));
  int roles[NR];
  if (!build_params(p, rank, roles)) { fprintf(stderr, "ag_gemm: bad schedule\n"); return; }
  p.x = (const float*)d_in[0];
  p.w = (const float*)d_in[1];
  for (int r = 0; r < NR; ++r) { p.out[r] = (char*)d_out_all[roles[r]]; p.ws[r] = (char*)d_ws_all[roles[r]]; }
  HIP_CHECK(hipFuncSetAttribute((const void*)ag_gemm, hipFuncAttributeMaxDynamicSharedMemorySize, LDS_BYTES));
  ag_gemm<<<GRID, NTHR, LDS_BYTES, stream>>>(p);
  HIP_CHECK(hipGetLastError());
}
```

```cpp
#include <hip/hip_runtime.h>
#include <rccl/rccl.h>
#include <cstdio>
#include <cstring>
#include <algorithm>
#include <vector>
#include <type_traits>

#define HIP_CHECK(expr) do { hipError_t _e = (expr); if (_e != hipSuccess) { \
    fprintf(stderr, "HIP error %s at %s:%d\n", hipGetErrorString(_e), __FILE__, __LINE__); \
    return; } } while (0)

#define P_NI 23
#define P_NC 4
#define P_NU 3
#define P_BATCH_I {{8, 4, 4, 1}, {12, 0, 2, 1}, {3, 0, 2, 0}}
#define P_BATCH_X {{9, 0, 2, 1}}
#define P_NOGEMM 0
#define P_ND 50
#define P_NDRAW 18
#define P_LEAD 330
#define P_HOLD 40
#define P_WIN 10
#define P_YWIN 6
#define P_OWN 4
#define P_SWAP 4
#define P_NRAWW 16
#define P_NTIN 1
#define P_OWNX 5
#define P_AUXU 17
#define P_AUXUL 17
#define P_AUXLL 17
#define P_AUXP 17
#define P_AUXE 17
#define P_AUXA 17
#define P_ROT 15
#define P_WIN0 16
#define P_R0 12
#define P_SD 16
#define P_SWIN 12
#define P_XO 6
#define P_SPEN 6.0
#define P_W2 -1
#define P_OT0 11.0
#define P_GSTART 950
#define P_OWNHOME 1
#define P_LATE 750
#define P_RATE 503
#define P_SLAST2 2
#define P_SLAST 2
#define P_B1O 6
#define P_OWNO 0
#define P_LEADW 330
#define P_GSO 1
#define P_LWIN 3
#define P_PACE 25
#define P_PACE0 80
#define P_TOUCH 1
#define P_CDELAY 230
#define P_CDELAY0 0
#define P_DS 4
#define P_DL 2
#define P_PSLEEP 2
#define P_PWIN 2
#define P_PSLOW 24
#define P_XS 1
#define P_K 1.65
#define P_T00 8.5
#define P_PRIO 1
#define P_PACK 1
#define P_NRAW 48
#define P_NRAWX 0
#define P_PER 2.03

namespace agg {

#define GAS __attribute__((address_space(1)))
#define LAS __attribute__((address_space(3)))
typedef unsigned v4u __attribute__((ext_vector_type(4)));
typedef unsigned v2u __attribute__((ext_vector_type(2)));
typedef unsigned short us2 __attribute__((ext_vector_type(2)));
typedef float f32x4 __attribute__((ext_vector_type(4)));
typedef float f32x2 __attribute__((ext_vector_type(2)));
typedef _Float16 h8 __attribute__((ext_vector_type(8)));
typedef GAS unsigned gu32;

constexpr int NR = 8;
constexpr int KD = 4096, SH = 512;
constexpr int PANEL = 131072;
constexpr int NAP = 128, NBP = 64, NPAN = NAP + NBP + 16;
constexpr int NCH = 16, CHUNK = 8192;
constexpr int NI = P_NI, NX = 32 - NI;
constexpr int GRID = 256, NWAVE = 7, NTHR = NWAVE * 64;
constexpr int NC = P_NC, NL = NWAVE - 1, WL = NC * NL;
constexpr int NU = P_PACK ? P_NU : 0, WU = NU * NWAVE + NC + NU;
static_assert(NWAVE == 7, "one wave per link in the group without a link");
constexpr int NG = 32 - NC - NU;
constexpr int MAXT = 6;
constexpr int XN = 1 << P_XS;
constexpr int YMAX = 192;
constexpr int NDRAW = P_PACK ? P_NDRAW : P_ND;
constexpr int ND = P_ND;
constexpr int NXY = 128 + NX * NCH;
constexpr int NPI = P_PACK ? NI * NCH - P_NRAW : 0, NPX = P_PACK ? NXY - P_NRAWX : 0;
constexpr int NCV = 5 * (128 - ND) + 384;
static_assert(NG * 8 * 4 >= NCV && NX <= 16 && ND <= 2 * NG && NDRAW <= ND && ND <= 128 - P_NRAWW && P_PACK, "one item per converting wave");
static_assert(NI * NCH <= 63 * WL && NXY + 128 <= 63 * WL, "one lane per forwarded item");
static_assert(!P_PACK || (NI * NCH <= 64 * WU && NXY <= 64 * WU), "one lane per unpacked item");
static_assert((NC + NU) * NWAVE * 64 >= 4 * YMAX, "roles");

constexpr unsigned OFF_CNT = 0;
constexpr unsigned OFF_TMO = 16384;
constexpr unsigned OFF_YQ = 20480;
constexpr unsigned OFF_BAR = 32768;
constexpr unsigned OFF_YF = 65536;
constexpr unsigned OFF_YL = 196608;
constexpr unsigned OFF_PF = 393216;
constexpr unsigned PF_BYTES = NPAN * NCH * 128;
constexpr unsigned OFF_AF = 1u << 20;
constexpr unsigned OFF_YS = 2u << 20;
constexpr unsigned OFF_EXP = OFF_YS + (4u << 20);
constexpr unsigned OFF_STG = OFF_EXP + (8u << 20);
constexpr unsigned STG_BYTES = (unsigned)NPAN * PANEL;
constexpr unsigned OFF_DBG = OFF_STG + 2 * STG_BYTES;
constexpr unsigned WS_USED = OFF_DBG + (1u << 20);
static_assert(OFF_YF + 4 * YMAX * 128 <= OFF_YL && OFF_YL + 1024 * 128 <= OFF_PF && OFF_PF + PF_BYTES <= OFF_AF &&
              OFF_AF + 2 * PF_BYTES <= OFF_YS, "layout");
constexpr unsigned OUT_BYTES = 4096u * SH * 4u;
constexpr unsigned NOWHERE = 0xf0000000u;
constexpr unsigned long long DEADLINE = 50000000ull;
constexpr int AUX_SYS = 17;
constexpr int RSRC3 = 0x00020000;

constexpr int LDS_RED = 65536;
constexpr int LDS_CTL = 2 * LDS_RED;
constexpr int LDS_SIMG = LDS_CTL + 64;
constexpr int LDS_BYTES = LDS_SIMG + 16384;

struct Params {
  const float* x;
  const float* w;
  char* myws;
  char* out[NR];
  char* ws[NR];
  int idx;
  int ny;
  int nyf;
  unsigned short rowbase[256];
  unsigned tiles[2 * XN * NG * MAXT];
};

#define DRAIN() asm volatile("s_waitcnt vmcnt(0)" ::: "memory")
#define LDSWAIT() asm volatile("s_waitcnt lgkmcnt(0)" ::: "memory")
#define CBAR() asm volatile("" ::: "memory")

__device__ __forceinline__ __amdgpu_buffer_rsrc_t mk(const void* p, unsigned bytes) {
  return __builtin_amdgcn_make_buffer_rsrc((void*)p, 0, (int)bytes, RSRC3);
}
__device__ __forceinline__ unsigned long long uniform64(const void* q) {
  const unsigned long long v = (unsigned long long)q;
  return (unsigned long long)(unsigned)__builtin_amdgcn_readfirstlane((unsigned)(v >> 32)) << 32 |
         (unsigned)__builtin_amdgcn_readfirstlane((unsigned)v);
}
__device__ __forceinline__ unsigned pk(float a, float b) {
  typedef _Float16 h2 __attribute__((ext_vector_type(2)));
  h2 v = {(_Float16)a, (_Float16)b};
  return __builtin_bit_cast(unsigned, v);
}
__device__ __forceinline__ unsigned ld_sys(const char* p) {
  return __hip_atomic_load((gu32*)p, __ATOMIC_RELAXED, __HIP_MEMORY_SCOPE_SYSTEM);
}
__device__ __forceinline__ void st_sys(char* p, unsigned v) {
  __hip_atomic_store((gu32*)p, v, __ATOMIC_RELAXED, __HIP_MEMORY_SCOPE_SYSTEM);
}
template <int AUX = AUX_SYS>
__device__ __forceinline__ void st16(__amdgpu_buffer_rsrc_t r, unsigned voff, v4u v) {
  __builtin_amdgcn_raw_buffer_store_b128(v, r, (int)voff, 0, AUX);
}
template <int AUX = AUX_SYS>
__device__ __forceinline__ v4u ld16(__amdgpu_buffer_rsrc_t r, unsigned voff, unsigned soff) {
  return __builtin_amdgcn_raw_buffer_load_b128(r, (int)voff, (int)soff, AUX);
}
__device__ __forceinline__ void flag(bool on, __amdgpu_buffer_rsrc_t r, unsigned off, unsigned call, unsigned hdr = 0) {
  __builtin_amdgcn_raw_buffer_store_b64(v2u{call, hdr}, r, (int)(on ? off : NOWHERE), 0, AUX_SYS);
}
__device__ __forceinline__ v2u ldflag(__amdgpu_buffer_rsrc_t r, unsigned off) {
  return __builtin_amdgcn_raw_buffer_load_b64(r, (int)off, 0, AUX_SYS);
}

#define STAMP(slot) do {} while (0)
#define TREC_DECL() do {} while (0)
#define TREC(i) do {} while (0)
#define TREC_DUMP() do {} while (0)

constexpr unsigned TOP2 = 0xC000C000u;
constexpr int LIST_CAP = 128;
__device__ __forceinline__ unsigned bfi(unsigned m, unsigned a, unsigned b) { return (a & m) | (b & ~m); }
__device__ __forceinline__ void pack8(v4u (&h)[8], unsigned base) {
  const us2 b2 = __builtin_bit_cast(us2, base * 0x08000800u);
#pragma unroll
  for (int d = 0; d < 4; ++d) {
    unsigned t[8], pl = 0;
#pragma unroll
    for (int s = 0; s < 8; ++s) {
      t[s] = __builtin_bit_cast(unsigned, (us2)(__builtin_bit_cast(us2, __builtin_amdgcn_alignbit(h[s][d], h[s][d], 31)) - b2));
      pl |= (t[s] & TOP2) >> (2 * s);
    }
#pragma unroll
    for (int s = 0; s < 7; ++s) h[s][d] = bfi(TOP2, t[7] << (14 - 2 * s), t[s]);
    h[7][d] = pl;
  }
}
__device__ __forceinline__ void unpack8(v4u (&h)[8], unsigned base) {
  const us2 b2 = __builtin_bit_cast(us2, base * 0x08000800u);
#pragma unroll
  for (int d = 0; d < 4; ++d) {
    const unsigned pl = h[7][d];
    unsigned t7 = (pl << 14) & TOP2;
#pragma unroll
    for (int s = 0; s < 7; ++s) {
      t7 |= (h[s][d] & TOP2) >> (14 - 2 * s);
      const unsigned t = __builtin_bit_cast(unsigned, (us2)(__builtin_bit_cast(us2, bfi(TOP2, pl << (2 * s), h[s][d])) + b2));
      h[s][d] = __builtin_amdgcn_alignbit(t, t, 1);
    }
    const unsigned t = __builtin_bit_cast(unsigned, (us2)(__builtin_bit_cast(us2, t7) + b2));
    h[7][d] = __builtin_amdgcn_alignbit(t, t, 1);
  }
}
template <int CTRL> __device__ __forceinline__ unsigned dpp0(unsigned v) {
  return (unsigned)__builtin_amdgcn_update_dpp(0, (int)v, CTRL, 0xf, 0xf, true);
}

__device__ __forceinline__ void item_offsets(bool isw, unsigned stg, unsigned pan, int v, unsigned& o0, unsigned& o1,
                                             unsigned& fl) {
  o0 = stg + pan * PANEL + v * (isw ? 4096 : CHUNK);
  o1 = o0 + (isw ? PANEL : 4096);
  fl = OFF_PF + (isw ? (pan + (v & 1)) * NCH + (v >> 1) : pan * NCH + v) * 128;
}
constexpr int kBI[][4] = P_BATCH_I, kBX[][4] = P_BATCH_X;
constexpr int NBI = sizeof(kBI) / sizeof(kBI[0]), NBX = sizeof(kBX) / sizeof(kBX[0]);
constexpr int GS = kBI[NBI - 1][1] ? 0 : kBI[NBI - 1][0], GS1 = kBI[NBI - 2][1] ? 0 : kBI[NBI - 2][0];
template <int N> __device__ __forceinline__ void batch_item(const int (&b)[N][4], int i, int& u, int& c) {
  int f = 0;
#pragma unroll
  for (int k = 0; k < N; ++k) {
    if (k == 0 || i >= f * NCH) {
      const int r = i - f * NCH;
      if (b[k][1]) { u = f + r % b[k][0]; c = r / b[k][0]; }
      else { u = i >> 4; c = i & 15; }
    }
    f += b[k][0];
  }
}
__device__ __forceinline__ void link_item(bool cross, int i, bool& isw, int& u, int& c) {
  isw = cross && i < 128;
  if (isw) { u = i & 3; c = i >> 2; }
  else if (cross) batch_item(kBX, i - 128, u, c);
  else batch_item(kBI, i, u, c);
}
__device__ __forceinline__ unsigned landing_panel(bool cross, bool isw, int u, int from, int to) {
  return !cross ? NI + 3 * u + ((from - to - 1) & 3) : isw ? NAP + 32 + from * 8 + 2 * u : 4 * NI + from * NX + u;
}
__device__ __forceinline__ int tile_a0(unsigned desc, int bfirst) {
  const int a = desc & 127;
  if ((desc >> 12) & 1) return NAP + NBP + a;
  if (bfirst < 32 || a < 4 * NI) return a;
  const int src = (a - 4 * NI) / NX;
  return a + ((src ^ ((bfirst >> 3) & 3)) - src) * NX;
}

#define LDIN(q) ((P_NTIN & 1) ? __builtin_nontemporal_load(q) : *(q))
#define LDINX(q) ((P_NTIN & 2) ? __builtin_nontemporal_load(q) : *(q))
__device__ __forceinline__ void load_inputs(const Params& p, bool isw, int g, int c, int lane, f32x4 (&v)[16]) {
  if (isw) {
    const GAS f32x2* src =
        (const GAS f32x2*)(p.w + (size_t)((c & 31) * 128 + 4 * (lane >> 4)) * SH + (g & 15) * 32 + 2 * (lane & 15));
#pragma unroll
    for (int q = 0; q < 16; ++q) {
      const f32x2 a = LDIN(src + ((q >> 1) * 16 + (q & 1) * 2) * (SH / 2));
      const f32x2 b = LDIN(src + ((q >> 1) * 16 + (q & 1) * 2 + 1) * (SH / 2));
      v[q] = f32x4{a.x, b.x, a.y, b.y};
    }
  } else {
    const GAS f32x4* src =
        (const GAS f32x4*)(p.x + (size_t)((g & 31) * 16 + (lane >> 2)) * KD + (c & 15) * 256 + 4 * (lane & 3));
#pragma unroll
    for (int q = 0; q < 16; ++q) v[q] = LDINX(src + q * 4);
  }
}
__device__ __forceinline__ void to_fragments(bool isw, const f32x4 (&v)[16], LAS v4u* scr, int lane, v4u (&h)[8]) {
  if (isw) {
#pragma unroll
    for (int s = 0; s < 4; ++s) {
      h[s] = v4u{pk(v[4 * s].x, v[4 * s].y), pk(v[4 * s + 1].x, v[4 * s + 1].y), pk(v[4 * s + 2].x, v[4 * s + 2].y),
                 pk(v[4 * s + 3].x, v[4 * s + 3].y)};
      h[4 + s] = v4u{pk(v[4 * s].z, v[4 * s].w), pk(v[4 * s + 1].z, v[4 * s + 1].w), pk(v[4 * s + 2].z, v[4 * s + 2].w),
                     pk(v[4 * s + 3].z, v[4 * s + 3].w)};
    }
  } else {
#pragma unroll
    for (int s = 0; s < 8; ++s)
      scr[s * 64 + lane] = v4u{pk(v[2 * s].x, v[2 * s].y), pk(v[2 * s].z, v[2 * s].w), pk(v[2 * s + 1].x, v[2 * s + 1].y),
                               pk(v[2 * s + 1].z, v[2 * s + 1].w)};
    LDSWAIT();
#pragma unroll
    for (int s = 0; s < 8; ++s) h[s] = scr[s * 64 + (lane & 15) * 4 + (lane >> 4)];
    LDSWAIT();
  }
}

struct Ctl { unsigned prog, bad, full[2], fre[2], conv, sfull[2], sfre[2]; };

__device__ __forceinline__ void need(volatile LAS unsigned* word, unsigned& have, unsigned n) {
  while (have < n) {
    have = __builtin_amdgcn_readfirstlane(*word);
    if (have < n) __builtin_amdgcn_s_sleep(1);
  }
  CBAR();
}

template <int MT, int NT>
__device__ __forceinline__ void gemm_tile(const Params& p, LAS char* lds, int a0, int b0, int t, unsigned& have,
                                          unsigned stg, int wave, int lane) {
  constexpr int D = (MT + NT <= 5) ? P_DS : P_DL;
  constexpr int NS = 32;
  volatile LAS Ctl* ctl = (volatile LAS Ctl*)(lds + LDS_CTL);
  const __amdgpu_buffer_rsrc_t rs = mk(p.ws[p.idx & 3], WS_USED);
  const unsigned ab = stg + (unsigned)a0 * PANEL + wave * 1024, bb = stg + (unsigned)(NAP + b0) * PANEL + wave * 1024;
  const int voff = lane * 16;
  const unsigned base = (unsigned)t * NCH;
  f32x4 acc[MT][NT];
#pragma unroll
  for (int m = 0; m < MT; ++m)
#pragma unroll
    for (int n = 0; n < NT; ++n) acc[m][n] = f32x4{0.f, 0.f, 0.f, 0.f};
  v4u ra[D][MT], rb[D][NT];
#define LOADSTEP(d, i) do { \
    _Pragma("unroll") for (int m = 0; m < MT; ++m) ra[d][m] = ld16<P_AUXA>(rs, voff, ab + m * PANEL + (i) * 4096); \
    _Pragma("unroll") for (int n = 0; n < NT; ++n) rb[d][n] = ld16(rs, voff, bb + n * PANEL + (i) * 4096); \
    __builtin_amdgcn_sched_barrier(0); } while (0)
#define MMASTEP(d) do { \
    _Pragma("unroll") for (int m = 0; m < MT; ++m) \
    _Pragma("unroll") for (int n = 0; n < NT; ++n) \
      acc[m][n] = __builtin_amdgcn_mfma_f32_16x16x32_f16(__builtin_bit_cast(h8, rb[d][n]), \
                                                         __builtin_bit_cast(h8, ra[d][m]), acc[m][n], 0, 0, 0); \
    __builtin_amdgcn_sched_barrier(0); } while (0)
#pragma unroll
  for (int d = 0; d < D; ++d) {
    if ((d & 1) == 0) need(&ctl->prog, have, base + (d >> 1) + 1);
    LOADSTEP(d, d);
  }
#pragma unroll 1
  for (int s0 = D; s0 < NS; s0 += D) {
#pragma unroll
    for (int d = 0; d < D; ++d) {
      MMASTEP(d);
      if ((d & 1) == 0) need(&ctl->prog, have, base + ((s0 + d) >> 1) + 1);
      LOADSTEP(d, s0 + d);
    }
  }
#pragma unroll
  for (int d = 0; d < D; ++d) MMASTEP(d);
#undef LOADSTEP
#undef MMASTEP
  asm volatile("" : "+v"(lane));
  constexpr int C4 = 4 * NT, M1 = MT > 4 ? (MT + 1) / 2 : MT;
#pragma unroll
  for (int part = 0; part < (MT > 4 ? 2 : 1); ++part) {
    const int tp = t + part, m0 = part ? M1 : 0, mn = part ? MT - M1 : M1;
    unsigned f = 0;
    if (tp >= 2) need(&ctl->fre[tp & 1], f, (unsigned)(tp >> 1));
    else need(&ctl->conv, f, NWAVE);
    LAS f32x4* red = (LAS f32x4*)(lds + (tp & 1) * LDS_RED) + wave * (64 * mn * NT);
#pragma unroll
    for (int m = 0; m < mn; ++m)
#pragma unroll
      for (int n = 0; n < NT; n += 2) {
        const int row = m * 16 + (lane & 15), c4 = n * 4 + 2 * (lane >> 4);
        const f32x4 e = acc[m0 + m][n], o = acc[m0 + m][n + 1];
        red[row * C4 + (c4 ^ (row & (C4 - 1)))] = f32x4{e.x, o.x, e.y, o.y};
        red[row * C4 + ((c4 + 1) ^ (row & (C4 - 1)))] = f32x4{e.z, o.z, e.w, o.w};
      }
    LDSWAIT();
    if (lane == 0)
      __hip_atomic_fetch_add((LAS unsigned*)&ctl->full[tp & 1], 1u, __ATOMIC_RELAXED, __HIP_MEMORY_SCOPE_WORKGROUP);
  }
}

template <int MT, int NT>
__device__ __forceinline__ void finish_tile(const Params& p, LAS char* lds, int bfirst, unsigned desc, int t,
                                            unsigned& have, unsigned call, int lane) {
  constexpr int C4 = 4 * NT, IMG = 64 * MT * NT, RQ = 64 / C4;
  volatile LAS Ctl* ctl = (volatile LAS Ctl*)(lds + LDS_CTL);
  const LAS f32x4* red = (const LAS f32x4*)(lds + (t & 1) * LDS_RED);
  const int idx = p.idx & 3, a0 = tile_a0(desc, bfirst), b = (desc >> 7) & 7;
  const int grp = ((bfirst + ((desc >> 7) & 31)) >> 3) & 7;
  const bool staged = NT == 4 && grp >= 4 && ((desc >> 29) & 1);
  const __amdgpu_buffer_rsrc_t ro = staged ? mk(p.ws[idx], WS_USED) : mk(p.out[grp < 4 ? idx : grp], OUT_BYTES);
  const int r0 = lane / C4, c4 = lane % C4;
  const unsigned slot = ((grp & 3) * NAP + a0) * 2 + (b >> 2);
  unsigned vo[MT];
#pragma unroll
  for (int m = 0; m < MT; ++m)
    vo[m] = staged ? OFF_YS + (slot + 2 * m) * 4096 + lane * 16
                   : ((p.rowbase[(a0 + m) & 255] + r0) * SH + (grp < 4 ? grp : idx) * 128 + b * 16 + c4 * 4) * 4;
  const unsigned qs = staged ? 1024 : RQ * SH * 4;
  const unsigned dst = NT == 4 ? (p.rowbase[(a0 + lane) & 255] * SH + idx * 128 + b * 16) * 4 : 0;
  need(&ctl->full[t & 1], have, 4u * ((t >> 1) + 1));
  STAMP(8 + t * 8 + 2);
  const bool bad = ctl->bad != 0;
  const unsigned ticket = NT != 4 ? 0u : (unsigned)__builtin_amdgcn_raw_ptr_buffer_atomic_add_i32(
      MT, ro, staged && lane == 0 ? OFF_YQ + (grp & 3) * 128 : NOWHERE, 0, 16);
  f32x4 s[MT * NT];
#pragma unroll
  for (int q = 0; q < MT * NT; ++q) {
    const int row = q * RQ + r0;
    const int li = row * C4 + (c4 ^ (row & (C4 - 1)));
    s[q] = (red[li] + red[IMG + li]) + (red[2 * IMG + li] + red[3 * IMG + li]);
    if (bad) s[q] = f32x4{__builtin_nanf(""), __builtin_nanf(""), __builtin_nanf(""), __builtin_nanf("")};
    if (q < P_YWIN) st16<P_AUXE>(ro, vo[q / NT] + (q % NT) * qs, __builtin_bit_cast(v4u, s[q]));
  }
  LDSWAIT();
  if (lane == 0) ctl->fre[t & 1] = (t >> 1) + 1;
  if (grp >= 4 && !staged) {
#pragma unroll
    for (int q = P_YWIN; q < MT * NT; ++q) {
      asm volatile("s_waitcnt vmcnt(%0)" :: "n"(P_YWIN - 1) : "memory");
      st16<P_AUXE>(ro, vo[q / NT] + (q % NT) * qs, __builtin_bit_cast(v4u, s[q]));
    }
  } else {
#pragma unroll
    for (int q = P_YWIN; q < MT * NT; ++q) st16<P_AUXE>(ro, vo[q / NT] + (q % NT) * qs, __builtin_bit_cast(v4u, s[q]));
  }
  if (grp >= 4) {
    DRAIN();
    if (staged) {
      const unsigned pos = __builtin_amdgcn_readfirstlane(ticket) - (call - 1) * 2 * p.nyf + lane;
      flag(lane < MT && pos < 2 * NAP, ro, OFF_YL + ((grp & 3) * 2 * NAP + pos) * 128, call,
           ((slot + 2 * lane) & 255) * 2 | dst >> 7 << 16);
    }
    else flag(lane == 0, mk(p.ws[grp], WS_USED), OFF_YF + (idx * YMAX + ((desc >> 13) & 255)) * 128, call);
  }
  STAMP(8 + t * 8 + 3);
}

constexpr bool XS = kBX[0][1] == 0;
__device__ __forceinline__ int stream_nx(unsigned desc) {
  const int kind = (desc >> 21) & 3;
  return kind == 2 ? NX : kind == 3 ? (4 * NX + 2 - (int)(desc & 3)) / 3 : 0;
}
__device__ __forceinline__ int stream_len(unsigned desc) {
  const int mv = (desc >> 23) & 1 ? P_GSO : 0;
  return (desc >> 21) & 3 ? stream_nx(desc) + GS - mv : GS1 + mv;
}
__device__ __forceinline__ int stream_panel(unsigned desc, int bfirst, int i) {
  const int kind = (desc >> 21) & 3, j = desc & 3, nx = stream_nx(desc), q = 3 * i + j;
  if (kind == 0) return NI + 3 * (NI - GS - GS1) + q;
  if (i >= nx) return NI + 3 * (NI - stream_len(desc)) + q;
  if (kind == 3) return 4 * NI + (q & 3) * NX + (q >> 2);
  return 4 * NI + ((((bfirst + (int)((desc >> 7) & 31)) >> 3) + 1 + j) & 3) * NX + i;
}
__device__ __forceinline__ void gemm_stream(const Params& p, LAS char* lds, unsigned desc, int bfirst, int t, unsigned& have,
                                            unsigned stg, int wave, int lane) {
  constexpr int D = P_SD;
  volatile LAS Ctl* ctl = (volatile LAS Ctl*)(lds + LDS_CTL);
  const __amdgpu_buffer_rsrc_t rs = mk(p.myws, WS_USED);
  const int G = stream_len(desc);
  const unsigned bb = stg + (unsigned)(NAP + bfirst + ((desc >> 7) & 31)) * PANEL + wave * 1024;
  unsigned ab = stg + (unsigned)stream_panel(desc, bfirst, 0) * PANEL + wave * 1024, base = (unsigned)t * NCH;
  const unsigned voff = lane * 16;
  LAS v4u* bl = (LAS v4u*)(lds + wave * 32768) + lane;
  static_assert(D == 16, "the ring doubles as the way into LDS");
  v4u br[32], ra[D];
  unsigned f = 0;
#pragma unroll
  for (int g = 0; g < 32; g += 4) {
    if (g >= 16) {
      if (g == 16) {
        need(&ctl->conv, f, NWAVE);
        f = 0; need(&ctl->fre[0], f, (unsigned)(t + 1) >> 1);
        f = 0; need(&ctl->fre[1], f, (unsigned)t >> 1);
      }
#pragma unroll
      for (int i = 0; i < 4; ++i) bl[(g - 16 + i) * 64] = ra[g - 16 + i];
      __builtin_amdgcn_sched_barrier(0);
    }
#pragma unroll
    for (int i = 0; i < 4; ++i) {
      if ((i & 1) == 0) need(&ctl->prog, have, base + ((g + i) >> 1) + 1);
      ra[(g + i) & 15] = ld16(rs, voff, bb + PANEL + (g + i) * 4096);
      br[g + i] = ld16(rs, voff, bb + (g + i) * 4096);
      __builtin_amdgcn_sched_barrier(0);
    }
  }
  base += NCH;
#pragma unroll
  for (int d = 0; d < D; ++d) {
    bl[(16 + d) * 64] = ra[d];
    if ((d & 1) == 0) need(&ctl->prog, have, base + (d >> 1) + 1);
    ra[d] = ld16(rs, voff, ab + d * 4096);
    __builtin_amdgcn_sched_barrier(0);
  }
#pragma unroll 1
  for (int i = 0; i < G; ++i) {
    const bool more = i + 1 < G;
    const unsigned vnext = more ? voff : NOWHERE;
    const unsigned an = stg + (unsigned)stream_panel(desc, bfirst, i + more) * PANEL + wave * 1024;
    f32x4 e = {0.f, 0.f, 0.f, 0.f}, o = {0.f, 0.f, 0.f, 0.f};
    v4u q[4];
    q[0] = bl[0]; q[1] = bl[64]; q[2] = bl[128];
#pragma unroll
    for (int s = 0; s < 32; ++s) {
      if (s + 3 < 32) q[(s + 3) & 3] = bl[(s + 3) * 64];
      e = __builtin_amdgcn_mfma_f32_16x16x32_f16(__builtin_bit_cast(h8, br[s]), __builtin_bit_cast(h8, ra[s % D]), e, 0, 0, 0);
      o = __builtin_amdgcn_mfma_f32_16x16x32_f16(__builtin_bit_cast(h8, q[s & 3]), __builtin_bit_cast(h8, ra[s % D]), o, 0, 0, 0);
      __builtin_amdgcn_sched_barrier(0);
      const int n = s + D;
      if (n < 32) {
        if ((n & 1) == 0) need(&ctl->prog, have, base + (n >> 1) + 1);
        ra[s % D] = ld16(rs, voff, ab + n * 4096);
      } else {
        if ((n & 1) == 0 && more) need(&ctl->prog, have, base + (n >> 1) + 1);
        ra[s % D] = ld16(rs, vnext, an + (n - 32) * 4096);
      }
      __builtin_amdgcn_sched_barrier(0);
    }
    f = 0; need(&ctl->sfre[i & 1], f, (unsigned)i >> 1);
    LAS f32x4* red = (LAS f32x4*)(lds + LDS_SIMG + (i & 1) * 8192) + wave * 128;
    const int row = lane & 15, c4 = 2 * (lane >> 4);
    red[row * 8 + (c4 ^ (row & 7))] = f32x4{e.x, o.x, e.y, o.y};
    red[row * 8 + ((c4 + 1) ^ (row & 7))] = f32x4{e.z, o.z, e.w, o.w};
    LDSWAIT();
    if (lane == 0)
      __hip_atomic_fetch_add((LAS unsigned*)&ctl->sfull[i & 1], 1u, __ATOMIC_RELAXED, __HIP_MEMORY_SCOPE_WORKGROUP);
    ab = an;
    base += NCH;
  }
}

__device__ __forceinline__ void finish_stream(const Params& p, LAS char* lds, int bfirst, unsigned desc, int par,
                                              unsigned call, int lane) {
  volatile LAS Ctl* ctl = (volatile LAS Ctl*)(lds + LDS_CTL);
  const int idx = p.idx & 3, b = (desc >> 7) & 7, grp = ((bfirst + ((desc >> 7) & 31)) >> 3) & 7;
  const int r0 = lane >> 3, c4 = lane & 7, li = r0 * 8 + (c4 ^ r0);
  const int G = stream_len(desc), nx = stream_nx(desc);
  const int ns = G - ((desc >> 21) & 3 ? P_SLAST2 : P_SLAST);
  static_assert(P_SLAST2 <= 4 && P_SLAST2 <= GS && P_SLAST2 > 0 && P_SLAST <= 4, "unit");
  TREC_DECL();
  for (int i = 0, unit = 0; i < G; ++unit) {
    const bool staged = grp >= 4 && i < ns;
    const int np = grp < 4 ? 1 : staged ? min(2, (i < nx ? nx : ns) - i) : G - i;
    if ((unit & 1) == par) {
      const int a = stream_panel(desc, bfirst, i), a2 = stream_panel(desc, bfirst, i + np - 1);
      const __amdgpu_buffer_rsrc_t ro = staged ? mk(p.myws, WS_USED) : mk(p.out[grp < 4 ? idx : grp], OUT_BYTES);
      const unsigned dst = (p.rowbase[a] * SH + (grp < 4 ? grp : idx) * 128 + b * 16) * 4;
      const unsigned slot = np == 2 ? (b & 4 ? a2 : a) * 4 + (b & 2) : a * 4 + (b >> 1);
      const unsigned vo = staged ? OFF_YS + ((grp & 3) * 4 * NAP + slot) * 2048 + lane * 16 : dst + (r0 * SH + c4 * 4) * 4;
      const unsigned qs = staged ? 1024 : 8 * SH * 4;
      f32x4 s[8];
#pragma unroll
      for (int e = 0; e < 4; ++e) {
        if (e < np) {
          const int h = (i + e) & 1;
          const LAS f32x4* red = (const LAS f32x4*)(lds + LDS_SIMG + h * 8192);
          unsigned have = 0;
          need(&ctl->sfull[h], have, 4u * (((i + e) >> 1) + 1));
          TREC(32 + i + e);
          s[2 * e] = (red[li] + red[128 + li]) + (red[256 + li] + red[384 + li]);
          s[2 * e + 1] = (red[64 + li] + red[192 + li]) + (red[320 + li] + red[448 + li]);
          asm volatile("" : "+v"(s[2 * e]), "+v"(s[2 * e + 1]));
          ctl->sfre[h] = ((i + e) >> 1) + 1;
        } else {
          s[2 * e] = s[2 * e + 1] = f32x4{0.f, 0.f, 0.f, 0.f};
        }
      }
      const unsigned ticket = (unsigned)__builtin_amdgcn_raw_ptr_buffer_atomic_add_i32(
          1, ro, staged && lane == 0 ? OFF_YQ + (grp & 3) * 128 : NOWHERE, 0, 16);
      const bool bad = ctl->bad != 0;
#pragma unroll
      for (int q = 0; q < 8; ++q) {
        if (bad) s[q] = f32x4{__builtin_nanf(""), __builtin_nanf(""), __builtin_nanf(""), __builtin_nanf("")};
        if (q < 2 * np) st16(ro, vo + q * qs, __builtin_bit_cast(v4u, s[q]));
      }
      if (grp >= 4) {
        DRAIN();
        const unsigned pos = __builtin_amdgcn_readfirstlane(ticket) - (call - 1) * 2 * p.nyf;
        if (staged) flag(lane == 0 && pos < 2 * NAP, ro, OFF_YL + ((grp & 3) * 2 * NAP + pos) * 128, call,
                         (slot & 511) | (np == 2 ? 1u << 9 : 3u << 9) | dst >> 7 << 16);
        else flag(lane == 0, mk(p.ws[grp], WS_USED), OFF_YF + (idx * YMAX + ((desc >> 13) & 255)) * 128, call);
      }
    }
    i += np;
  }
}

#define BY_SHAPE(desc, BIG, F, ...) \
  switch (((desc) >> 24) & 15) { \
    case 1: F<1, 2>(__VA_ARGS__); break; \
    case 2: F<2, 2>(__VA_ARGS__); break; \
    case 3: F<3, 2>(__VA_ARGS__); break; \
    case 9: F<1, 4>(__VA_ARGS__); break; \
    case 10: F<2, 4>(__VA_ARGS__); break; \
    case 11: F<3, 4>(__VA_ARGS__); break; \
    case 12: F<4, 4>(__VA_ARGS__); break; \
    case 13: F<BIG ? 5 : 3, 4>(__VA_ARGS__); break; \
    default: F<BIG ? 6 : 3, 4>(__VA_ARGS__); break; \
  }

__global__ __launch_bounds__(NTHR, 2) void ag_gemm(const Params p) {
  extern __shared__ __attribute__((aligned(16))) char smem[];
  LAS char* lds = (LAS char*)smem;
  const int lane = threadIdx.x & 63;
  const int wave = __builtin_amdgcn_readfirstlane(threadIdx.x >> 6);
  const int blk = blockIdx.x, grp = blk & 7, j = blk >> 3;
  asm volatile("" :: "s"(p.x), "s"(p.w), "s"(p.myws), "s"(p.idx), "s"(p.ws[grp]));
  const int idx = p.idx & 3;
  const bool cross = grp >= 4;
  char* myws = p.myws;
  const __amdgpu_buffer_rsrc_t rl = mk(myws, WS_USED), rd = mk(p.ws[grp], WS_USED);
  STAMP(0);
  char* counter = myws + OFF_CNT + (blk * 8 + wave) * 4;
  const unsigned last_call = ld_sys(counter);
  const unsigned long long tw = __builtin_amdgcn_s_memrealtime();
  const int jg = j - NC - NU;
  const bool keeper = cross || grp == ((idx + 1) & 3);
  const bool early = wave == 6 && (unsigned)jg < (unsigned)NDRAW && grp != idx;
  if (early) {
    static_assert(NDRAW <= NG && NDRAW <= kBI[0][0] * NCH, "in the first batch");
    const int u = cross ? jg & 3 : jg % kBI[0][0], c = cross ? jg >> 2 : jg / kBI[0][0];
    f32x4 v[16];
    v4u h[8];
    if (cross) {
      load_inputs(p, true, (grp - 4) * 4 + u, c, lane, v);
      __builtin_amdgcn_s_barrier();
      to_fragments(true, v, (LAS v4u*)(lds + 6 * 8192), lane, h);
    } else {
      load_inputs(p, false, u, c, lane, v);
      __builtin_amdgcn_s_barrier();
      to_fragments(false, v, (LAS v4u*)(lds + 6 * 8192), lane, h);
    }
    const unsigned call = __builtin_amdgcn_readfirstlane(last_call) + 1u;
    unsigned o0, o1, fl;
    item_offsets(cross, OFF_STG + (call & 1u) * STG_BYTES, landing_panel(cross, cross, u, idx, grp & 3), c, o0, o1, fl);
    STAMP(7);
    while ((int)(__builtin_amdgcn_s_memrealtime() - tw) < P_PACE0 + jg * P_PACE / 2) __builtin_amdgcn_s_sleep(1);
#pragma unroll
    for (int s = 0; s < 4; ++s) st16(rd, o0 + lane * 16 + s * 1024, h[s]);
#pragma unroll
    for (int s = 0; s < 4; ++s) st16(rd, o1 + lane * 16 + s * 1024, h[4 + s]);
    STAMP(6);
    unsigned l0, l1, ll;
    item_offsets(cross, OFF_STG + (call & 1u) * STG_BYTES, cross ? NAP + (grp - 4) * 8 + 2 * u : u, c, l0, l1, ll);
    if (keeper) {
#pragma unroll
      for (int s = 0; s < 4; ++s) st16(rl, l0 + lane * 16 + s * 1024, h[s]);
#pragma unroll
      for (int s = 0; s < 4; ++s) st16(rl, l1 + lane * 16 + s * 1024, h[4 + s]);
    }
    DRAIN();
    flag(lane == 0, rd, fl, call);
    flag(lane == 0 && keeper, rl, ll, call);
  } else {
    if (threadIdx.x < 16) ((volatile LAS unsigned*)(lds + LDS_CTL))[threadIdx.x] = 0;
    LDSWAIT();
    __builtin_amdgcn_s_barrier();
  }

  if (j < NC && grp != idx) {
    struct Ring { unsigned full[NL], done, abort; };
    volatile LAS Ring* rg = (volatile LAS Ring*)(lds + LDS_CTL);
    constexpr int SLOT = 10240;
    volatile LAS v2u* meta = (volatile LAS v2u*)(lds + 2 * NL * SLOT);
    const int n = !cross ? NI * NCH : NXY + (P_NOGEMM ? 0 : p.nyf);
    if (wave == 0) {
      const unsigned call = __builtin_amdgcn_readfirstlane(last_call) + 1u;
      if (lane == 0) st_sys(counter, call);
      __builtin_amdgcn_s_setprio(3);
      const unsigned long long peer_out = uniform64(p.out[grp]);
      unsigned pl[3], ph[3];
#pragma unroll
      for (int d = 0; d < 3; ++d) { pl[d] = NOWHERE; ph[d] = 0; }
      const int cnt = (n - ND - j + NC - 1) / NC;
      const __amdgpu_buffer_rsrc_t ro = mk((char*)peer_out, OUT_BYTES);
      const int ry = min(cnt, (max((cross ? NXY : n) - ND - j, 0) + NC - 1) / NC);
      unsigned lane0 = lane == 0 ? ~0u : 0u, lane1 = lane == 1 ? ~0u : 0u;
      asm volatile("" : "+v"(lane0), "+v"(lane1));
      int r = 0, l = 0, k = 0;
      bool dead = false;
      TREC_DECL();
      auto rounds = [&](int end, auto wide, __amdgpu_buffer_rsrc_t rs) {
        constexpr int W = wide() ? P_WIN0 : P_WIN;
        static_assert(W <= (wide() ? 22 : 13), "window");
        for (; r < end; ++r) {
          LAS char* sl = lds + (2 * l + (k & 1)) * SLOT + lane * 16;
          unsigned got;
          v2u m;
          v4u a0, a1, v[8];
#define READ_SLOT() do { \
          const unsigned fu = rg->full[l]; \
          m = meta[2 * l + (k & 1)]; \
          a0 = *(volatile LAS v4u*)(sl + 8192); a1 = *(volatile LAS v4u*)(sl + 9216); \
          _Pragma("unroll") for (int q = 0; q < 8; ++q) v[q] = *(volatile LAS v4u*)(sl + q * 1024); \
          got = __builtin_amdgcn_readfirstlane(fu); } while (0)
          READ_SLOT();
          if (__builtin_expect(got <= (unsigned)k, 0)) {
            DRAIN();
            flag(lane < 3, rd, lane == 0 ? pl[0] : lane == 1 ? pl[1] : pl[2], call,
                 lane == 0 ? ph[0] : lane == 1 ? ph[1] : ph[2]);
            pl[0] = pl[1] = pl[2] = NOWHERE;
            while (__builtin_amdgcn_readfirstlane(rg->full[l]) <= (unsigned)k) __builtin_amdgcn_s_sleep(0);
            READ_SLOT();
          }
#undef READ_SLOT
          if (got == 0xffffffffu) { dead = true; r = cnt; break; }
          const unsigned fo = wide() ? bfi(lane0, pl[0], NOWHERE) : bfi(lane0, pl[1], bfi(lane1, pl[0], NOWHERE));
          v2u fv = {call, wide() || lane ? ph[0] : ph[1]};
#pragma unroll
          for (int q = 0; q < 8; ++q) asm volatile("" : "+v"(v[q]));
          rg->done = r + 1;
          asm volatile("s_waitcnt vmcnt(%0)" :: "n"(W) : "memory");
#pragma unroll
          for (int q = 0; q < 4; ++q) st16<P_AUXP>(rs, a0[q], v[q]);
          asm volatile("s_waitcnt vmcnt(%0)" :: "n"(W) : "memory");
          __builtin_amdgcn_raw_buffer_store_b64(fv, rd, (int)fo, 0, AUX_SYS);
          CBAR();
#pragma unroll
          for (int q = 0; q < 4; ++q) st16<P_AUXP>(rs, a1[q], v[4 + q]);
          CBAR();
          if (!(r & 1)) TREC(r >> 1);
          pl[0] = wide() ? pl[1] : NOWHERE; ph[0] = ph[1]; pl[1] = pl[2]; ph[1] = ph[2];
          pl[2] = __builtin_amdgcn_readfirstlane(m.x); ph[2] = __builtin_amdgcn_readfirstlane(m.y);
          if (++l == NL) { l = 0; ++k; }
        }
      };
      while ((int)(__builtin_amdgcn_s_memrealtime() - tw) < P_PACE0 + ND * P_PACE / 2 - P_HOLD) __builtin_amdgcn_s_sleep(2);
      rounds(min(P_R0, ry), std::true_type(), rd);
      while (r < ry) {
        if (P_LATE && (int)(__builtin_amdgcn_s_memrealtime() - tw) > P_LATE + r * P_RATE / 10)
          rounds(min(r + 2, ry), std::true_type(), rd);
        else rounds(min(r + 2, ry), std::false_type(), rd);
      }
      rounds(cnt, std::false_type(), ro);
      if (dead) { rg->done = 0xffffffffu; return; }
      asm volatile("s_waitcnt vmcnt(18)" ::: "memory");
      flag(lane == 0, rd, pl[0], call, ph[0]);
      asm volatile("s_waitcnt vmcnt(10)" ::: "memory");
      flag(lane == 0, rd, pl[1], call, ph[1]);
      asm volatile("s_waitcnt vmcnt(2)" ::: "memory");
      flag(lane == 0, rd, pl[2], call, ph[2]);
      STAMP(1);
      TREC_DUMP();
      return;
    }
    const int l = wave - 1, first = ND + l * NC + j, cnt = (n - first + WL - 1) / WL;
    bool isw; int u, c;
    const unsigned call = __builtin_amdgcn_readfirstlane(last_call) + 1u;
    if (lane == 0) st_sys(counter, call);
    const unsigned stg = OFF_STG + (call & 1u) * STG_BYTES;
    const unsigned long long t0 = __builtin_amdgcn_s_memrealtime();
    if (P_TOUCH && wave == NL) {
      unsigned t = __builtin_amdgcn_raw_buffer_load_b32(rd, lane < 26 ? stg + (lane << 20) : lane < 28 ? (lane - 26) << 20 : NOWHERE, 0, AUX_SYS);
      t += __builtin_amdgcn_raw_buffer_load_b32(mk(p.out[grp], OUT_BYTES), cross && lane < 8 ? lane << 20 : NOWHERE, 0, AUX_SYS);
      t += __builtin_amdgcn_raw_buffer_load_b32(rl, lane < 26 ? stg + (lane << 20) : lane < 40 ? (lane - 26) << 20 : NOWHERE, 0, AUX_SYS);
      t += __builtin_amdgcn_raw_buffer_load_b32(mk(p.out[idx], OUT_BYTES), lane < 8 ? lane << 20 : NOWHERE, 0, AUX_SYS);
      asm volatile("" :: "v"(t));
    }
    const int i = min(first + lane * WL, n - 1);
    unsigned s0, s1, sf, sg, d0, d1, df, kind;
    if (!cross || i < NXY) {
      link_item(cross, i, isw, u, c);
      const int g = isw ? (grp - 4) * 4 + u : cross ? NI + u : u;
      item_offsets(isw, stg, isw ? NAP + 2 * g : cross ? NAP + NBP + u : u, c, s0, s1, sf);
      item_offsets(isw, stg, landing_panel(cross, isw, u, idx, grp & 3), c, d0, d1, df);
      sg = sf;
      kind = i < (cross ? NPX : NPI) && !(isw && i >= 128 - P_NRAWW);
      if (kind) {
        s0 = OFF_EXP + (isw ? 512 + g * 32 + c : g * 16 + c) * 8192; s1 = s0 + 4096;
        df += OFF_AF + (call & 1u) * PF_BYTES - OFF_PF;
      }
    } else {
      sf = OFF_YL + ((grp & 3) * NAP + ((i - NXY) & 127)) * 256; sg = sf + 128;
      s0 = s1 = d0 = d1 = 0;
      df = OFF_YF + (idx * YMAX + (i - NXY)) * 128;
      kind = 2;
    }
    if (lane == 63) { sf = OFF_TMO; sg = OFF_TMO; }
    unsigned long long ready = 0;
    unsigned hdv = 64u << 24 | kind << 16;
    for (int k = 0; k < cnt; ++k) {
      while (!((ready >> k) & 1)) {
        const bool ask = ((lane >= k && lane < min(cnt, k + (k < 2 ? P_LWIN : 64))) || lane == 63) && !((ready >> lane) & 1);
        const v2u f = ldflag(rl, ask ? sf : NOWHERE);
        const v2u g = ldflag(rl, ask && kind == 2 ? sg : NOWHERE);
        const bool ok = f.x == call && (kind != 2 || g.x == call);
        if (ok && kind == 1) hdv = f.y;
        if (ok && kind == 2) { s0 = f.y; s1 = g.y; }
        ready |= __ballot(ok);
        if ((ready >> 63) || __builtin_amdgcn_s_memrealtime() - t0 > DEADLINE) {
          st_sys(myws + OFF_TMO, call);
          rg->abort = 1; LDSWAIT(); rg->full[l] = 0xffffffffu;
          return;
        }
      }
      CBAR();
      unsigned hd = __builtin_amdgcn_readlane(hdv, k);
      unsigned a0 = __builtin_amdgcn_readlane(s0, k), a1 = __builtin_amdgcn_readlane(s1, k);
      unsigned e0 = __builtin_amdgcn_readlane(d0, k), e1 = __builtin_amdgcn_readlane(d1, k);
      if ((hd >> 17) & 1) {
        e0 = a0 >> 16 << 7; e1 = a1 >> 16 << 7;
        hd |= (a0 & 512) << 9 | (a1 & 512) << 10 | (a0 & 1024) << 10 | (a1 & 1024) << 11;
        a0 = OFF_YS + ((grp & 3) * 4 * NAP + (a0 & 511)) * 2048;
        a1 = OFF_YS + ((grp & 3) * 4 * NAP + (a1 & 511)) * 2048;
      }
      v4u h[8];
#pragma unroll
      for (int q = 0; q < 4; ++q) h[q] = ld16<P_AUXLL>(rl, lane * 16 + q * 1024, a0);
#pragma unroll
      for (int q = 0; q < 3; ++q) h[4 + q] = ld16<P_AUXLL>(rl, lane * 16 + q * 1024, a1);
      h[7] = ld16<P_AUXLL>(rl, lane < (hd >> 24) ? lane * 16 + 3072 : NOWHERE, a1);
      const bool isy = (hd >> 17) & 1;
      const unsigned va = (lane >> 4) * (SH * 4) + (lane & 15) * 16, vc = (lane >> 3) * (SH * 4) + (lane & 7) * 16;
      const unsigned vb = e0 + (!isy ? lane * 16 : (hd >> 18) & 1 ? vc : va), qs = !isy ? 1024 : 8192 << ((hd >> 18) & 1);
      const unsigned vd = e1 + (!isy ? lane * 16 : (hd >> 19) & 1 ? vc : va), qt = !isy ? 1024 : 8192 << ((hd >> 19) & 1);
      if (k >= 2) {
        unsigned f = 0;
        need(&rg->done, f, (unsigned)((k - 2) * NL + l + 1));
        if (__builtin_amdgcn_readfirstlane(rg->abort)) { rg->full[l] = 0xffffffffu; return; }
      }
      LAS v4u* sl = (LAS v4u*)(lds + (2 * l + (k & 1)) * SLOT);
      const bool sh0 = (hd >> 20) & 1, sh1 = (hd >> 21) & 1;
      sl[512 + lane] = v4u{vb, vb + qs, sh0 ? NOWHERE : vb + 2 * qs, sh0 ? NOWHERE : vb + 3 * qs};
      sl[576 + lane] = v4u{vd, vd + qt, sh1 ? NOWHERE : vd + 2 * qt, lane < (hd >> 24) && !sh1 ? vd + 3 * qt : NOWHERE};
      meta[2 * l + (k & 1)] = v2u{(unsigned)__builtin_amdgcn_readlane(df, k), hd};
#pragma unroll
      for (int q = 0; q < 8; ++q) sl[q * 64 + lane] = h[q];
      LDSWAIT();
      rg->full[l] = k + 1;
    }
    STAMP(1);
    return;
  }
  if (j < NC + NU) {
    const unsigned call = __builtin_amdgcn_readfirstlane(last_call) + 1u;
    if (lane == 0) st_sys(counter, call);
    const unsigned stg = OFF_STG + (call & 1u) * STG_BYTES;
    const unsigned aoff = OFF_AF + (call & 1u) * PF_BYTES - OFF_PF;
    const unsigned long long t0 = __builtin_amdgcn_s_memrealtime();
    if (P_PACK) {
      const int lk = grp == idx ? (idx + 1 + wave) & 7 : grp;
      const bool xl = lk >= 4;
      const int n = xl ? NPX - P_NRAWW : NPI, first = NDRAW + (grp == idx ? NU * NWAVE + j : wave * NU + j - NC);
      const int cnt = (n - first + WU - 1) / WU;
      bool isw; int u, c;
      const int ui = min(first + lane * WU, n - 1);
      link_item(xl, xl && ui >= 128 - P_NRAWW ? ui + P_NRAWW : ui, isw, u, c);
      unsigned o0, o1, fl;
      item_offsets(isw, stg, landing_panel(xl, isw, u, lk & 3, idx), c, o0, o1, fl);
      LAS unsigned* plane = (LAS unsigned*)(lds + wave * 1024);
      unsigned pend = NOWHERE, hdr = 0;
      bool have_next = false;
      TREC_DECL();
      for (int r = 0; r < cnt; ++r) {
        const unsigned fr = __builtin_amdgcn_readlane(fl, r);
        *(LAS v4u*)(plane + lane * 4) = v4u{0, 0, 0, 0};
        while (!have_next) {
          const v2u f = ldflag(rl, lane == 0 ? fr + aoff : lane == 1 ? OFF_TMO : NOWHERE);
          if (pend != NOWHERE) {
            DRAIN();
            flag(lane == 0, rl, pend, call);
            TREC(4 * r - 1);
            pend = NOWHERE;
          }
          hdr = __builtin_amdgcn_readfirstlane(f.y);
          if ((int)(__builtin_amdgcn_readfirstlane(f.x) - call) >= 0) break;
          if (__builtin_amdgcn_readlane(f.x, 1) == call || __builtin_amdgcn_s_memrealtime() - t0 > DEADLINE) {
            st_sys(myws + OFF_TMO, call);
            st_sys(p.out[idx] + (size_t)p.rowbase[0] * SH * 4, 0x7fc00000u);
            return;
          }
        }
        CBAR();
        TREC(4 * r);
        const unsigned a0 = __builtin_amdgcn_readlane(o0, r), a1 = __builtin_amdgcn_readlane(o1, r);
        const bool dense = (hdr >> 16) & 1;
        const unsigned e2 = __builtin_amdgcn_raw_buffer_load_b32(rl, lane * 4 + 3072, a1, AUX_SYS);
        CBAR();
        v4u h[8];
#pragma unroll
        for (int q = 0; q < 4; ++q) h[q] = ld16<P_AUXUL>(rl, lane * 16 + q * 1024, a0);
#pragma unroll
        for (int q = 0; q < 3; ++q) h[4 + q] = ld16<P_AUXUL>(rl, lane * 16 + q * 1024, a1);
        h[7] = ld16(rl, dense ? lane * 16 + 3072 : NOWHERE, a1);
        const v2u fn = ldflag(rl, lane == 0 ? __builtin_amdgcn_readlane(fl, min(r + 1, cnt - 1)) + aoff : NOWHERE);
        CBAR();
        const unsigned cn = dense ? 0 : (hdr >> 8) & 255;
        LDSWAIT();
#pragma unroll
        for (int k = 0; k < 2; ++k) {
          const unsigned e = (e2 >> (16 * k)) & 0xffffu;
          __hip_atomic_fetch_or(plane + ((e >> 7) & 255), 2 * lane + k < cn ? (e & 15) << ((e >> 2) & 28) : 0u,
                                __ATOMIC_RELAXED, __HIP_MEMORY_SCOPE_WORKGROUP);
        }
        LDSWAIT();
        const v4u pl = *(volatile LAS v4u*)(plane + lane * 4);
        DRAIN();
        flag(lane == 0, rl, pend, call);
        if (pend != NOWHERE) TREC(4 * r - 1);
        TREC(4 * r + 1);
        h[7] |= pl;
        unpack8(h, hdr & 31);
#pragma unroll
        for (int q = 0; q < 4; ++q) st16<P_AUXU>(rl, a0 + lane * 16 + q * 1024, h[q]);
#pragma unroll
        for (int q = 0; q < 4; ++q) st16<P_AUXU>(rl, a1 + lane * 16 + q * 1024, h[4 + q]);
        pend = fr;
        TREC(4 * r + 2);
        hdr = __builtin_amdgcn_readfirstlane(fn.y);
        have_next = (int)(__builtin_amdgcn_readfirstlane(fn.x) - call) >= 0;
      }
      DRAIN();
      flag(lane == 0, rl, pend, call);
      TREC_DUMP();
      STAMP(2);
    }
    if (grp == idx) {
      const int f = (j * NWAVE + wave) * 64 + lane;
      if (f - lane >= 4 * YMAX) return;
      bool mine = (f % YMAX) < p.ny;
      while (!P_NOGEMM) {
        const unsigned v = __builtin_amdgcn_raw_buffer_load_b32(rl, mine ? OFF_YF + f * 128 : NOWHERE, 0, AUX_SYS);
        const unsigned tm = __builtin_amdgcn_readfirstlane(ld_sys(myws + OFF_TMO));
        mine = mine && (int)(v - call) < 0;
        if (!__any(mine)) break;
        if (tm == call || __builtin_amdgcn_s_memrealtime() - t0 > DEADLINE) {
          st_sys(myws + OFF_TMO, call);
          st_sys(p.out[idx] + (size_t)p.rowbase[0] * SH * 4, 0x7fc00000u);
          break;
        }
        __builtin_amdgcn_s_sleep(1);
      }
      STAMP(1);
    }
    return;
  }

  volatile LAS Ctl* ctl = (volatile LAS Ctl*)(lds + LDS_CTL);
  unsigned call;
#pragma unroll 1
  for (int round = 0; round < (wave == P_W2 ? 2 : 1); ++round) {
    const bool direct = wave >= 5 && round == 0;
    const int di = wave == 6 ? jg : NG + (jg + P_ROT) % NG;
    const int leave = P_PACE0 + di * P_PACE / 2;
    const int it = min(wave, 3) * (NG * 8) + jg * 8 + grp;
    const bool work = direct ? grp != idx && di >= NDRAW && di < ND : (wave < 3 || round || (wave == 3 && P_W2 < 0)) && it < NCV;
    bool isw = false, raw = true, packed = true;
    int g = 0, c = 0, u = 0;
    f32x4 v[16];
    if (P_PRIO) __builtin_amdgcn_s_setprio(3);
    if (work && direct) {
      link_item(cross, di, isw, u, c);
      g = isw ? (grp - 4) * 4 + u : u;
      raw = keeper;
      while ((int)(__builtin_amdgcn_s_memrealtime() - tw) < leave - (cross ? P_LEADW : P_LEAD)) __builtin_amdgcn_s_sleep(2);
    } else if (work) {
      const int k = it / 5 + ND, m = it % 5;
      isw = k < 128 && m < 4;
      if (isw) { g = m * 4 + (k & 3); c = k >> 2; packed = k < 128 - P_NRAWW; }
      else {
        const int xi = k < 128 ? k : it - (NCV - 512);
        const bool xc = xi >= NI * NCH;
        const int li = xc ? xi - NI * NCH + 128 : xi;
        bool dummy;
        link_item(xc, li, dummy, u, c);
        g = xc ? NI + u : u;
        packed = li < (xc ? NPX : NPI);
      }
      while ((int)(__builtin_amdgcn_s_memrealtime() - tw) < P_CDELAY0 + (it * P_CDELAY >> 8)) __builtin_amdgcn_s_sleep(2);
    }
    if (work) load_inputs(p, isw, g, c, lane, v);
    call = __builtin_amdgcn_readfirstlane(last_call) + 1u;
    if (work) {
      const unsigned stg = OFF_STG + (call & 1u) * STG_BYTES;
      LAS v4u* scr = (LAS v4u*)(lds + wave * 8192);
      v4u h[8];
      to_fragments(isw, v, scr, lane, h);
      unsigned o0, o1, fl, hdr = 0, keepfl = NOWHERE;
      item_offsets(isw, stg, isw ? NAP + 2 * g : g < NI ? g : NAP + NBP + u, c, o0, o1, fl);
      const __amdgpu_buffer_rsrc_t rs = mk(direct ? p.ws[grp] : myws, WS_USED);
      STAMP(7);
      if (raw) {
#pragma unroll
        for (int s = 0; s < 4; ++s) st16(rl, o0 + lane * 16 + s * 1024, h[s]);
#pragma unroll
        for (int s = 0; s < 4; ++s) st16(rl, o1 + lane * 16 + s * 1024, h[4 + s]);
      }
      if (P_PACK && packed) {
        float ss = 0.f;
#pragma unroll
        for (int q = 0; q < 16; q += 4) ss += v[q].x * v[q].x + v[q].y * v[q].y + v[q].z * v[q].z + v[q].w * v[q].w;
        ss += __builtin_bit_cast(float, dpp0<0x111>(__builtin_bit_cast(unsigned, ss)));
        ss += __builtin_bit_cast(float, dpp0<0x112>(__builtin_bit_cast(unsigned, ss)));
        ss += __builtin_bit_cast(float, dpp0<0x114>(__builtin_bit_cast(unsigned, ss)));
        ss += __builtin_bit_cast(float, dpp0<0x118>(__builtin_bit_cast(unsigned, ss)));
        const unsigned sb = __builtin_bit_cast(unsigned, ss);
        const float tot = (__builtin_bit_cast(float, __builtin_amdgcn_readlane(sb, 15)) +
                           __builtin_bit_cast(float, __builtin_amdgcn_readlane(sb, 31))) +
                          (__builtin_bit_cast(float, __builtin_amdgcn_readlane(sb, 47)) +
                           __builtin_bit_cast(float, __builtin_amdgcn_readlane(sb, 63)));
        const float top = __builtin_amdgcn_sqrtf(tot * (6.76f / 1024.f));
        const int ex = (int)((__builtin_amdgcn_readfirstlane(__builtin_bit_cast(unsigned, top)) >> 23) & 255) - 127 + 15 - 7;
        const unsigned base = (unsigned)min(max(ex, 0), 24);
        pack8(h, base);
        unsigned nz[4], cnt = 0;
#pragma unroll
        for (int d = 0; d < 4; ++d) {
          unsigned z = h[7][d];
          z |= z >> 1; z |= z >> 2;
          nz[d] = z & 0x11111111u;
          cnt += __builtin_popcount(nz[d]);
        }
        unsigned inc = cnt;
        inc += dpp0<0x111>(inc); inc += dpp0<0x112>(inc); inc += dpp0<0x114>(inc); inc += dpp0<0x118>(inc);
        const unsigned r0 = __builtin_amdgcn_readlane(inc, 15), r1 = r0 + __builtin_amdgcn_readlane(inc, 31),
                       r2 = r1 + __builtin_amdgcn_readlane(inc, 47), total = r2 + __builtin_amdgcn_readlane(inc, 63);
        const bool dense = total > (unsigned)LIST_CAP;
        if (!dense) {
          unsigned pos = inc - cnt + (lane < 16 ? 0 : lane < 32 ? r0 : lane < 48 ? r1 : r2);
          LAS unsigned short* list = (LAS unsigned short*)scr;
#pragma unroll
          for (int d = 0; d < 4; ++d) {
            unsigned m = nz[d];
            while (m) {
              const unsigned b = __builtin_ctz(m);
              list[pos++] = (unsigned short)(((lane * 4 + d) << 7) | (b << 2) | ((h[7][d] >> b) & 15));
              m &= m - 1;
            }
          }
          LDSWAIT();
          h[7] = *(volatile LAS v4u*)(scr + (lane & 15));
        }
        const unsigned nl = dense ? 64 : max((total + 31) >> 5, 1u) * 4;
        hdr = base | (dense ? 1u << 16 : total << 8) | nl << 24;
        if (direct) {
          if (raw) keepfl = fl;
          item_offsets(isw, stg, landing_panel(cross, isw, u, idx, grp & 3), c, o0, o1, fl);
          fl += OFF_AF + (call & 1u) * PF_BYTES - OFF_PF;
          while ((int)(__builtin_amdgcn_s_memrealtime() - tw) < leave) __builtin_amdgcn_s_sleep(1);
        } else {
          o0 = OFF_EXP + (isw ? 512 + g * 32 + c : g * 16 + c) * 8192;
          o1 = o0 + 4096;
        }
#pragma unroll
        for (int s = 0; s < 4; ++s) st16(rs, o0 + lane * 16 + s * 1024, h[s]);
#pragma unroll
        for (int s = 0; s < 3; ++s) st16(rs, o1 + lane * 16 + s * 1024, h[4 + s]);
        st16(rs, lane < nl ? o1 + lane * 16 + 3072 : NOWHERE, h[7]);
      }
      STAMP(6);
      DRAIN();
      flag(lane == 0, rs, fl, call, hdr);
      flag(lane == 0, rl, keepfl, call);
    }
    STAMP(2);
  }
  if (lane == 0) {
    st_sys(counter, call);
    __hip_atomic_fetch_add((LAS unsigned*)&ctl->conv, 1u, __ATOMIC_RELAXED, __HIP_MEMORY_SCOPE_WORKGROUP);
  }
  const unsigned stg = OFF_STG + (call & 1u) * STG_BYTES;
  const int gg = grp ^ P_SWAP;
  const unsigned* tl = p.tiles + (((gg >> 2) * XN + (gg & (XN - 1))) * NG + jg) * MAXT;
  const int bfirst = (gg >> P_XS << P_XS) * 8;
  if (P_PRIO && wave < 4) __builtin_amdgcn_s_setprio(0);
  if (wave == 4) {
    const unsigned long long t0 = __builtin_amdgcn_s_memrealtime();
    unsigned bdone = 0;
    while ((int)(__builtin_amdgcn_s_memrealtime() - tw) < P_GSTART) __builtin_amdgcn_s_sleep(8);
    for (int t = 0; t < MAXT && !P_NOGEMM; ++t) {
      const unsigned desc = tl[t];
      if (!(desc >> 31)) break;
      if ((desc >> 28) & 1) continue;
      const int mt = (desc >> 24) & 7, nt = (desc >> 27) & 1 ? 4 : 2, b = (desc >> 7) & 31;
      const unsigned bm = ((1u << nt) - 1) << b;
      const int np = (bdone & bm) == bm ? mt : mt + nt, l2 = np <= 4 ? 2 : np <= 8 ? 3 : 4;
      const int q = lane & ((1 << l2) - 1), ahead = lane >> l2;
      const unsigned fo = OFF_PF + (q < mt ? tile_a0(desc, bfirst) + q : NAP + bfirst + b + (q - mt)) * (NCH * 128);
      unsigned pub = 0, win = 16;
      while (pub < NCH) {
        const unsigned c = pub + ahead;
        const bool act = q < np && c < NCH && ahead < win;
        const unsigned v = __builtin_amdgcn_raw_buffer_load_b32(rl, act ? fo + c * 128 : NOWHERE, 0, AUX_SYS);
        const unsigned tm = __builtin_amdgcn_readfirstlane(ld_sys(myws + OFF_TMO));
        const unsigned long long m = __ballot(act ? (int)(v - call) < 0 : ahead >= win);
        if (tm == call || __builtin_amdgcn_s_memrealtime() - t0 > DEADLINE) {
          st_sys(myws + OFF_TMO, call);
          ctl->bad = 1;
          LDSWAIT();
          ctl->prog = 0xffffffffu;
          return;
        }
        const unsigned n = m ? (unsigned)__builtin_ctzll(m) >> l2 : 64u >> l2;
        win = n >= win || n >= (64u >> l2) ? 16 : n + P_PWIN;
        if (n) {
          pub = min(pub + n, (unsigned)NCH);
          ctl->prog = t * NCH + pub;
        }
        if (t > 0 && __builtin_amdgcn_readfirstlane(ctl->full[(t - 1) & 1]) < 4u * ((t - 1) >> 1) + 1u)
          __builtin_amdgcn_s_sleep(P_PSLOW);
        else if (!n) __builtin_amdgcn_s_sleep(P_PSLEEP);
      }
      bdone |= bm;
      STAMP(8 + t * 8 + 4);
      if (GS + GS1 && mt == 0) {
        const unsigned total = stream_len(desc) * NCH;
        pub = 0; win = 64;
        while (pub < total) {
          const unsigned c = pub + lane;
          const bool act = c < total && lane < win;
          const unsigned v = __builtin_amdgcn_raw_buffer_load_b32(
              rl, act ? OFF_PF + (stream_panel(desc, bfirst, c >> 4) * NCH + (c & 15)) * 128 : NOWHERE, 0, AUX_SYS);
          const unsigned tm = __builtin_amdgcn_readfirstlane(ld_sys(myws + OFF_TMO));
          const unsigned long long m = __ballot(act ? (int)(v - call) < 0 : lane >= win);
          if (tm == call || __builtin_amdgcn_s_memrealtime() - t0 > DEADLINE) {
            st_sys(myws + OFF_TMO, call);
            ctl->bad = 1;
            LDSWAIT();
            ctl->prog = 0xffffffffu;
            return;
          }
          const unsigned n = m ? (unsigned)__builtin_ctzll(m) : 64u;
          win = n >= win ? 64 : P_SWIN;
          pub = min(pub + n, total);
          ctl->prog = (t + 1) * NCH + pub;
        }
      }
    }
    return;
  }
  unsigned have = 0;
  if (wave > 4) {
    for (int t = 0; t < MAXT && !P_NOGEMM; ++t) {
      const unsigned desc = tl[t];
      if (!(desc >> 31)) break;
      if (GS + GS1 && ((desc >> 24) & 7) == 0) { finish_stream(p, lds, bfirst, desc, wave - 5, call, lane); break; }
      if ((t & 1) != wave - 5) continue;
      BY_SHAPE(((desc >> 24) & 7) > 4 ? 11u << 24 : desc, false, finish_tile, p, lds, bfirst, desc, t, have, call, lane)
    }
    return;
  }
  for (int t = 0; t < MAXT && !P_NOGEMM; ++t) {
    const unsigned desc = tl[t];
    if (!(desc >> 31)) break;
    if ((desc >> 28) & 1) continue;
    STAMP(8 + t * 8);
    const int a0 = tile_a0(desc, bfirst), b0 = bfirst + ((desc >> 7) & 31);
    if (GS + GS1 && ((desc >> 24) & 7) == 0) { gemm_stream(p, lds, desc, bfirst, t, have, stg, wave, lane); break; }
    BY_SHAPE(desc, true, gemm_tile, p, lds, a0, b0, t, have, stg, wave, lane)
  }
  STAMP(1);
}

static const int kClass[2][4] = {{0, 2, 3, 6}, {1, 4, 5, 7}};

static bool build_params(Params& p, int rank, int* roles) {
  int cls = 0, idx = 0;
  for (int c = 0; c < 2; ++c)
    for (int i = 0; i < 4; ++i) if (kClass[c][i] == rank) { cls = c; idx = i; }
  for (int i = 0; i < 4; ++i) { roles[i] = kClass[cls][i]; roles[4 + i] = kClass[cls ^ 1][i]; }
  p.idx = idx;
  for (int a = 0; a < NAP; ++a) {
    int src, rt;
    if (a < NI) { src = rank; rt = a; }
    else if (a < 4 * NI) { rt = (a - NI) / 3; src = roles[(idx + 1 + (a - NI) % 3) & 3]; }
    else { rt = NI + (a - 4 * NI) % NX; src = roles[4 + (a - 4 * NI) / NX]; }
    p.rowbase[a] = (unsigned short)(src * SH + rt * 16);
  }
  for (int u = 0; u < NX; ++u) p.rowbase[NAP + NBP + u] = (unsigned short)(rank * SH + (NI + u) * 16);
  struct Seg { int a0, n, mt, nt, staged; double t0, t1; int src = -1; };
  std::vector<Seg> segs;
  const double per = P_PER, t00 = P_T00;
  segs.push_back({0, NI, P_OWN, 4, 1, P_OT0, P_OT0 + 1.5});
  segs.push_back({0, NX, P_OWNX, 4, 1, 12.0, 13.0, -2});
  int f = 0;
  for (int b = 0; b < NBI; ++b) {
    segs.push_back({NI + 3 * f, 3 * kBI[b][0], kBI[b][1], kBI[b][2], kBI[b][3], f ? t00 + f * per : 10.0,
                    t00 + (f + kBI[b][0]) * per - (b == NBI - 1 ? 3.2 : 0.0)});
    if (XS && b == NBI - 1) segs.back().t0 = t00 + 8 * per + 0.1;
    f += kBI[b][0];
  }
  if (f != NI || (XS && (NBX != 1 || !GS))) return false;
  for (int b = 0; b < NBX; ++b) {
    for (int src = 3; src >= 0 && !XS; --src)
      segs.push_back({4 * NI + src * NX + f - NI, kBX[b][0], kBX[b][1], kBX[b][2], kBX[b][3], t00 + (8 + f - NI) * per,
                      t00 + 1.0 + (8 + f - NI + kBX[b][0]) * per - (b == NBX - 1 && P_NRAWX ? 2.7 : 0.0), src});
    f += kBX[b][0];
  }
  if (f != 32) return false;
  auto dur = [](int mt, int nt) { return 0.6 + P_K * (mt + nt); };
  std::stable_sort(segs.begin(), segs.end(), [&](const Seg& a, const Seg& b) {
    auto key = [&](const Seg& s) { return s.mt ? std::max(s.t0, s.t1 + 1.8 - dur(s.mt, s.nt)) : s.t0 - dur(0, 2); };
    return key(a) < key(b);
  });
  for (int h = 0; h < 2; ++h) {
    std::vector<double> fr(XN * NG, 0.0);
    std::vector<int> cnt(XN * NG, 0);
    std::vector<std::pair<unsigned*, int>> direct;
    int ndirect[XN] = {}, nstaged[XN] = {};
    if (XS && GS1) {
      const size_t z = segs.size();
      if (segs[z - 2].mt || segs[z - 1].mt) return false;
      if ((segs[z - 2].a0 > segs[z - 1].a0) == (h == 1)) std::swap(segs[z - 2], segs[z - 1]);
    }
    for (auto s : segs) {
      if (!h && P_B1O && s.a0 == NI && s.src == -1) s.mt = P_B1O;
      if (!h && P_OWNO && s.a0 == 0 && s.src == -1) s.mt = P_OWNO;
      if (s.mt == 0) {
        const int sg = (GS1 && s.a0 + s.n == 4 * NI) == (h || !XS) ? 0 : XN - 1;
        for (int peer = 0; peer < 3; ++peer)
          for (int b = 0; b < 8 * XN; b += 2) {
            int best = -1; double bf = 0;
            for (int g = 0; g < XN * NG; ++g) {
              const double e = std::max(fr[g], s.t0 - dur(0, 2)) + (g / NG == sg ? 0.0 : P_SPEN) + 1e-3 * fr[g];
              if (cnt[g] < MAXT && (best < 0 || e < bf)) { best = g; bf = e; }
            }
            if (best < 0) return false;
            unsigned* e = &p.tiles[(h * XN * NG + best) * MAXT + cnt[best]];
            const bool last = s.a0 + s.n == 4 * NI;
            *e = 0x80000000u | (!h && XS && GS1 ? 1u << 23 : 0u) | (!last ? 0u : !XS ? 1u : h ? 2u : 3u) << 21 | ((unsigned)b << 7) | (unsigned)peer;
            nstaged[b >> 3] += last ? (XS ? (NX + 1) / 2 : 0) + (GS - P_SLAST2 + 1) / 2 : (GS1 - P_SLAST + 1) / 2;
            if (last || P_SLAST) direct.push_back({e, ndirect[b >> 3]++});
            cnt[best] = MAXT; fr[best] = 1e3;
          }
        continue;
      }
      if (!h && P_XO && NBX == 1 && s.src >= 0) {
        if (s.src) continue;
        s.n = 4 * NX; s.mt = P_XO;
      }
      if ((s.nt != 2 && s.nt != 4) || s.mt < 1 || s.mt > (s.nt == 4 ? 6 : 3) || (s.staged && s.nt != 4)) return false;
      if (h && s.src == -2) continue;
      const double t1 = h ? std::max(s.t1, t00 + 0.5 + 8 * per - (P_NRAWW ? 2.0 : 0.0)) : s.t1;
      const int rows = (s.n + s.mt - 1) / s.mt;
      for (int r = 0, a = s.a0; r < rows; ++r) {
        const int mt = s.n / rows + (r < s.n % rows), ntile = 8 * (XN - (h && s.src >= 0 && s.src < XN)) / s.nt;
        auto finish = [&](double free_at) { return std::max(std::max(free_at, s.t0) + dur(mt, s.nt), t1 + 1.8); };
        int home = 0; double bt = 1e9;
        for (int x = 0; x < XN; ++x) {
          std::vector<double> v(fr.begin() + x * NG, fr.begin() + (x + 1) * NG);
          std::sort(v.begin(), v.end());
          double sum = 0;
          for (int q = 0; q < NG; ++q) sum += (q < ntile ? finish(v[q]) : 0.0) + 1e-3 * v[q];
          if (GS + GS1 && x == XN - 1) sum += P_OWNHOME && s.a0 == 0 && s.src == -1 ? -0.5 : 0.5;
          if (sum < bt) { bt = sum; home = x; }
        }
        const bool st1 = s.staged != 0, st0 = st1 && r >= s.staged - 1;
        for (int b = 0; b < 8 * XN; b += s.nt) {
          if (h && s.src == b >> 3) continue;
          int best = -1; double bf = 0;
          for (int g = 0; g < XN * NG; ++g) {
            const double e = finish(fr[g]) + (g / NG == home ? 0.0 : 0.7) + 1e-3 * fr[g];
            if (cnt[g] + (mt > 4) < MAXT && (best < 0 || e < bf)) { best = g; bf = e; }
          }
          if (best < 0) return false;
          fr[best] = finish(fr[best]);
          unsigned* e = &p.tiles[(h * XN * NG + best) * MAXT + cnt[best]++];
          const unsigned common =
              0x80000000u | (s.nt == 4 ? 1u << 27 : 0u) | (s.src == -2 ? 1u << 12 : 0u) | ((unsigned)b << 7);
          *e = common | (st0 ? 1u << 29 : 0u) | ((unsigned)mt << 24) | (unsigned)a;
          if (st0) nstaged[b >> 3] += mt > 4 ? 3 : mt;
          else direct.push_back({e, ndirect[b >> 3]++});
          if (mt > 4) {
            e[1] = common | (st1 ? 1u << 29 : 0u) | 1u << 28 | ((unsigned)(mt - 3) << 24) | (unsigned)(a + 3);
            if (st1) nstaged[b >> 3] += mt - 3;
            else direct.push_back({e + 1, ndirect[b >> 3]++});
            ++cnt[best];
          }
        }
        a += mt;
      }
    }
    const int nyf = nstaged[0] / 2;
    for (int x = 0; x < XN; ++x) if (nstaged[x] != 2 * nyf || ndirect[x] != ndirect[0]) return false;
    for (auto& d : direct) *d.first |= (unsigned)(nyf + d.second) << 13;
    p.nyf = nyf;
    p.ny = nyf + ndirect[0];
    if (h && (p.ny > YMAX || nyf > NAP)) return false;
  }
  return true;
}

}

extern "C" void kernel_launch(int rank, int nranks, ncclComm_t comm, hipStream_t stream, void* const* d_in,
                              const int* in_sizes, int n_in, void* d_out, int out_size, const int* in_dtypes,
                              int out_dtype, void* const* d_out_all, void* const* d_ws_all, size_t ws_bytes) {
  using namespace agg;
  if (nranks != NR || n_in != 2 || ws_bytes < WS_USED || out_size != 4096 * SH || in_sizes[0] != SH * KD ||
      in_sizes[1] != KD * SH || (unsigned)rank >= (unsigned)NR) {
    fprintf(stderr, "ag_gemm: unexpected problem shape\n");
    return;
  }
  Params p;
  memset(&p, 0, sizeof(p));
  int roles[NR];
  if (!build_params(p, rank, roles)) { fprintf(stderr, "ag_gemm: bad schedule\n"); return; }
  p.x = (const float*)d_in[0];
  p.w = (const float*)d_in[1];
  p.myws = (char*)d_ws_all[rank];
  for (int r = 0; r < NR; ++r) { p.out[r] = (char*)d_out_all[roles[r]]; p.ws[r] = (char*)d_ws_all[roles[r]]; }
  HIP_CHECK(hipFuncSetAttribute((const void*)ag_gemm, hipFuncAttributeMaxDynamicSharedMemorySize, LDS_BYTES));
  ag_gemm<<<GRID, NTHR, LDS_BYTES, stream>>>(p);
  HIP_CHECK(hipGetLastError());
}
```

```cpp
#include <hip/hip_runtime.h>
#include <rccl/rccl.h>
#include <cstdio>
#include <cstring>
#include <algorithm>
#include <vector>
#include <type_traits>

#define HIP_CHECK(expr) do { hipError_t _e = (expr); if (_e != hipSuccess) { \
    fprintf(stderr, "HIP error %s at %s:%d\n", hipGetErrorString(_e), __FILE__, __LINE__); \
    return; } } while (0)

#define P_NI 23
#define P_NC 4
#define P_NU 3
#define P_BATCH_I {{8, 4, 4, 1}, {12, 0, 2, 1}, {3, 0, 2, 0}}
#define P_BATCH_X {{9, 0, 2, 1}}
#define P_NOGEMM 0
#define P_ND 66
#define P_NDRAW 16
#define P_LEAD 330
#define P_HOLD -40
#define P_WIN 10
#define P_YWIN 6
#define P_OWN 4
#define P_SWAP 4
#define P_NRAWW 0
#define P_NTIN 1
#define P_OWNX 5
#define P_AUXU 17
#define P_AUXUL 17
#define P_AUXLL 17
#define P_AUXP 17
#define P_AUXE 17
#define P_AUXA 17
#define P_ROT 15
#define P_WIN0 16
#define P_R0 0
#define P_SD 16
#define P_SWIN 8
#define P_XO 6
#define P_SPEN 6.0
#define P_W2 -1
#define P_OT0 11.0
#define P_GSTART 950
#define P_OWNHOME 1
#define P_LATE 750
#define P_RATE 503
#define P_SLAST2 1
#define P_SLAST 3
#define P_B1O 6
#define P_OWNO 0
#define P_LEADW 330
#define P_GSO 1
#define P_LSTART 0
#define P_LSTEP 0
#define P_LSLEEP 0
#define P_USTART 200
#define P_ROT2 5
#define P_PACKY 1
#define P_NRAWY 22
#define P_ROTA 0
#define P_ROTB 0
#define P_ALT 1
#define P_NOWIDE 0
#define P_SSLEEP 0
#define P_SAHEAD 40
#define P_SWMAX 32
#define P_UGAP 240
#define P_UBACK 60
#define P_TMOMASK 15
#define P_USLEEP 0
#define P_ULOOK 1
#define P_YSLEEP 16
#define P_FSTART 5000
#define P_SWIDLE 2
#define P_LWIN 3
#define P_PACE 25
#define P_PACE0 80
#define P_TOUCH 0
#define P_CDELAY 230
#define P_CDELAY0 0
#define P_DS 4
#define P_DL 2
#define P_PSLEEP 8
#define P_PWIN 1
#define P_PSLOW 24
#define P_XS 1
#define P_K 1.65
#define P_T00 8.5
#define P_PRIO 1
#define P_PACK 1
#define P_NRAW 48
#define P_NRAWX 0
#define P_PER 2.03

namespace agg {

#define GAS __attribute__((address_space(1)))
#define LAS __attribute__((address_space(3)))
typedef unsigned v4u __attribute__((ext_vector_type(4)));
typedef unsigned v2u __attribute__((ext_vector_type(2)));
typedef unsigned short us2 __attribute__((ext_vector_type(2)));
typedef float f32x4 __attribute__((ext_vector_type(4)));
typedef float f32x2 __attribute__((ext_vector_type(2)));
typedef _Float16 h8 __attribute__((ext_vector_type(8)));
typedef GAS unsigned gu32;

constexpr int NR = 8;
constexpr int KD = 4096, SH = 512;
constexpr int PANEL = 131072;
constexpr int NAP = 128, NBP = 64, NPAN = NAP + NBP + 16;
constexpr int NCH = 16, CHUNK = 8192;
constexpr int NI = P_NI, NX = 32 - NI;
constexpr int GRID = 256, NWAVE = 7, NTHR = NWAVE * 64;
constexpr int NC = P_NC, NL = NWAVE - 1, WL = NC * NL;
constexpr int NU = P_PACK ? P_NU : 0, WU = NU * NWAVE + NC + NU;
static_assert(NWAVE == 7, "one wave per link in the group without a link");
constexpr int NG = 32 - NC - NU;
constexpr int MAXT = 6;
constexpr int XN = 1 << P_XS;
constexpr int YMAX = 192;
constexpr int NDRAW = P_PACK ? P_NDRAW : P_ND;
constexpr int ND = P_ND;
constexpr int NXY = 128 + NX * NCH;
constexpr int NPI = P_PACK ? NI * NCH - P_NRAW : 0, NPX = P_PACK ? NXY - P_NRAWX : 0;
constexpr int NCV = 5 * (128 - ND) + 384;
static_assert(NG * 8 * 4 >= NCV && NX <= 16 && ND <= 3 * NG && NDRAW <= ND && ND <= 128 - P_NRAWW && P_PACK, "one item per converting wave");
static_assert(NI * NCH <= 63 * WL && NXY + 128 <= 63 * WL, "one lane per forwarded item");
static_assert(!P_PACK || (NI * NCH <= 64 * WU && NXY <= 64 * WU), "one lane per unpacked item");
static_assert((NC + NU) * NWAVE * 64 >= 4 * YMAX, "roles");

constexpr unsigned OFF_CNT = 0;
constexpr unsigned OFF_TMO = 16384;
constexpr unsigned OFF_YQ = 20480;
constexpr unsigned OFF_BAR = 32768;
constexpr unsigned OFF_YF = 65536;
constexpr unsigned OFF_YL = 196608;
constexpr unsigned OFF_PF = 393216;
constexpr unsigned PF_BYTES = NPAN * NCH * 128;
constexpr unsigned OFF_AF = 1u << 20;
constexpr unsigned OFF_YS = 2u << 20;
constexpr unsigned OFF_YP = OFF_YS + (4u << 20);
constexpr unsigned OFF_EXP = OFF_YP + (4u << 20);
constexpr unsigned OFF_STG = OFF_EXP + (8u << 20);
constexpr unsigned STG_BYTES = (unsigned)NPAN * PANEL;
constexpr unsigned OFF_DBG = OFF_STG + 2 * STG_BYTES;
constexpr unsigned WS_USED = OFF_DBG + (1u << 20);
static_assert(OFF_YF + 4 * YMAX * 128 <= OFF_YL && OFF_YL + 1024 * 128 <= OFF_PF && OFF_PF + PF_BYTES <= OFF_AF &&
              OFF_AF + 2 * PF_BYTES <= OFF_YS, "layout");
constexpr unsigned OUT_BYTES = 4096u * SH * 4u;
constexpr unsigned NOWHERE = 0xf0000000u;
constexpr unsigned long long DEADLINE = 50000000ull;
constexpr int AUX_SYS = 17;
constexpr int RSRC3 = 0x00020000;

constexpr int LDS_RED = 65536;
constexpr int LDS_CTL = 2 * LDS_RED;
constexpr int LDS_SIMG = LDS_CTL + 64;
constexpr int LDS_BYTES = LDS_SIMG + 16384;

struct Params {
  const float* x;
  const float* w;
  char* myws;
  char* out[NR];
  char* ws[NR];
  int idx;
  int ny;
  int nyf;
  unsigned short rowbase[256];
  unsigned tiles[2 * XN * NG * MAXT];
};

#define DRAIN() asm volatile("s_waitcnt vmcnt(0)" ::: "memory")
#define LDSWAIT() asm volatile("s_waitcnt lgkmcnt(0)" ::: "memory")
#define CBAR() asm volatile("" ::: "memory")

__device__ __forceinline__ __amdgpu_buffer_rsrc_t mk(const void* p, unsigned bytes) {
  return __builtin_amdgcn_make_buffer_rsrc((void*)p, 0, (int)bytes, RSRC3);
}
__device__ __forceinline__ unsigned long long uniform64(const void* q) {
  const unsigned long long v = (unsigned long long)q;
  return (unsigned long long)(unsigned)__builtin_amdgcn_readfirstlane((unsigned)(v >> 32)) << 32 |
         (unsigned)__builtin_amdgcn_readfirstlane((unsigned)v);
}
__device__ __forceinline__ unsigned pk(float a, float b) {
  typedef _Float16 h2 __attribute__((ext_vector_type(2)));
  h2 v = {(_Float16)a, (_Float16)b};
  return __builtin_bit_cast(unsigned, v);
}
__device__ __forceinline__ unsigned ld_sys(const char* p) {
  return __hip_atomic_load((gu32*)p, __ATOMIC_RELAXED, __HIP_MEMORY_SCOPE_SYSTEM);
}
__device__ __forceinline__ void st_sys(char* p, unsigned v) {
  __hip_atomic_store((gu32*)p, v, __ATOMIC_RELAXED, __HIP_MEMORY_SCOPE_SYSTEM);
}
template <int AUX = AUX_SYS>
__device__ __forceinline__ void st16(__amdgpu_buffer_rsrc_t r, unsigned voff, v4u v) {
  __builtin_amdgcn_raw_buffer_store_b128(v, r, (int)voff, 0, AUX);
}
template <int AUX = AUX_SYS>
__device__ __forceinline__ v4u ld16(__amdgpu_buffer_rsrc_t r, unsigned voff, unsigned soff) {
  return __builtin_amdgcn_raw_buffer_load_b128(r, (int)voff, (int)soff, AUX);
}
__device__ __forceinline__ void flag(bool on, __amdgpu_buffer_rsrc_t r, unsigned off, unsigned call, unsigned hdr = 0) {
  __builtin_amdgcn_raw_buffer_store_b64(v2u{call, hdr}, r, (int)(on ? off : NOWHERE), 0, AUX_SYS);
}
__device__ __forceinline__ v2u ldflag(__amdgpu_buffer_rsrc_t r, unsigned off) {
  return __builtin_amdgcn_raw_buffer_load_b64(r, (int)off, 0, AUX_SYS);
}

#define STAMP(slot) do {} while (0)
#define TREC_DECL() do {} while (0)
#define TREC(i) do {} while (0)
#define TREC_DUMP() do {} while (0)

constexpr unsigned TOP2 = 0xC000C000u, TOP4 = 0xF0000000u;
constexpr int LIST_CAP = 128;
__device__ __forceinline__ unsigned bfi(unsigned m, unsigned a, unsigned b) { return (a & m) | (b & ~m); }
__device__ __forceinline__ void pack8(v4u (&h)[8], unsigned base) {
  const us2 b2 = __builtin_bit_cast(us2, base * 0x08000800u);
#pragma unroll
  for (int d = 0; d < 4; ++d) {
    unsigned t[8], pl = 0;
#pragma unroll
    for (int s = 0; s < 8; ++s) {
      t[s] = __builtin_bit_cast(unsigned, (us2)(__builtin_bit_cast(us2, __builtin_amdgcn_alignbit(h[s][d], h[s][d], 31)) - b2));
      pl |= (t[s] & TOP2) >> (2 * s);
    }
#pragma unroll
    for (int s = 0; s < 7; ++s) h[s][d] = bfi(TOP2, t[7] << (14 - 2 * s), t[s]);
    h[7][d] = pl;
  }
}
__device__ __forceinline__ void unpack8(v4u (&h)[8], unsigned base) {
  const us2 b2 = __builtin_bit_cast(us2, base * 0x08000800u);
#pragma unroll
  for (int d = 0; d < 4; ++d) {
    const unsigned pl = h[7][d];
    unsigned t7 = (pl << 14) & TOP2;
#pragma unroll
    for (int s = 0; s < 7; ++s) {
      t7 |= (h[s][d] & TOP2) >> (14 - 2 * s);
      const unsigned t = __builtin_bit_cast(unsigned, (us2)(__builtin_bit_cast(us2, bfi(TOP2, pl << (2 * s), h[s][d])) + b2));
      h[s][d] = __builtin_amdgcn_alignbit(t, t, 1);
    }
    const unsigned t = __builtin_bit_cast(unsigned, (us2)(__builtin_bit_cast(us2, t7) + b2));
    h[7][d] = __builtin_amdgcn_alignbit(t, t, 1);
  }
}
__device__ __forceinline__ void pack8f(v4u (&h)[8], unsigned base) {
#pragma unroll
  for (int d = 0; d < 4; ++d) {
    unsigned t[8], pl = 0;
#pragma unroll
    for (int s = 0; s < 8; ++s) {
      t[s] = __builtin_amdgcn_alignbit(h[s][d], h[s][d], 31) - (base << 24);
      pl |= (t[s] & TOP4) >> (4 * s);
    }
#pragma unroll
    for (int s = 0; s < 7; ++s) h[s][d] = bfi(TOP4, t[7] << (28 - 4 * s), t[s]);
    h[7][d] = pl;
  }
}
__device__ __forceinline__ void unpack8f(v4u (&h)[8], unsigned base) {
#pragma unroll
  for (int d = 0; d < 4; ++d) {
    const unsigned pl = h[7][d];
    unsigned t7 = pl << 28;
#pragma unroll
    for (int s = 0; s < 7; ++s) {
      t7 |= (h[s][d] & TOP4) >> (28 - 4 * s);
      const unsigned t = bfi(TOP4, pl << (4 * s), h[s][d]) + (base << 24);
      h[s][d] = __builtin_amdgcn_alignbit(t, t, 1);
    }
    t7 += base << 24;
    h[7][d] = __builtin_amdgcn_alignbit(t7, t7, 1);
  }
}
template <int CTRL> __device__ __forceinline__ unsigned dpp0(unsigned v) {
  return (unsigned)__builtin_amdgcn_update_dpp(0, (int)v, CTRL, 0xf, 0xf, true);
}
__device__ __forceinline__ int top_exponent(float ss, float scale) {
  ss += __builtin_bit_cast(float, dpp0<0x111>(__builtin_bit_cast(unsigned, ss)));
  ss += __builtin_bit_cast(float, dpp0<0x112>(__builtin_bit_cast(unsigned, ss)));
  ss += __builtin_bit_cast(float, dpp0<0x114>(__builtin_bit_cast(unsigned, ss)));
  ss += __builtin_bit_cast(float, dpp0<0x118>(__builtin_bit_cast(unsigned, ss)));
  const unsigned sb = __builtin_bit_cast(unsigned, ss);
  const float tot = (__builtin_bit_cast(float, __builtin_amdgcn_readlane(sb, 15)) +
                     __builtin_bit_cast(float, __builtin_amdgcn_readlane(sb, 31))) +
                    (__builtin_bit_cast(float, __builtin_amdgcn_readlane(sb, 47)) +
                     __builtin_bit_cast(float, __builtin_amdgcn_readlane(sb, 63)));
  const float top = __builtin_amdgcn_sqrtf(tot * scale);
  return (int)((__builtin_amdgcn_readfirstlane(__builtin_bit_cast(unsigned, top)) >> 23) & 255);
}
__device__ __forceinline__ unsigned list_plane(const v4u& pl, LAS unsigned short* list, unsigned cap, int lane) {
  unsigned nz[4], cnt = 0;
#pragma unroll
  for (int d = 0; d < 4; ++d) {
    unsigned z = pl[d];
    z |= z >> 1; z |= z >> 2;
    nz[d] = z & 0x11111111u;
    cnt += __builtin_popcount(nz[d]);
  }
  unsigned inc = cnt;
  inc += dpp0<0x111>(inc); inc += dpp0<0x112>(inc); inc += dpp0<0x114>(inc); inc += dpp0<0x118>(inc);
  const unsigned r0 = __builtin_amdgcn_readlane(inc, 15), r1 = r0 + __builtin_amdgcn_readlane(inc, 31),
                 r2 = r1 + __builtin_amdgcn_readlane(inc, 47), total = r2 + __builtin_amdgcn_readlane(inc, 63);
  if (total <= cap) {
    unsigned pos = inc - cnt + (lane < 16 ? 0 : lane < 32 ? r0 : lane < 48 ? r1 : r2);
#pragma unroll
    for (int d = 0; d < 4; ++d) {
      unsigned m = nz[d];
      while (m) {
        const unsigned b = __builtin_ctz(m);
        list[pos++] = (unsigned short)(((lane * 4 + d) << 7) | (b << 2) | ((pl[d] >> b) & 15));
        m &= m - 1;
      }
    }
  }
  return total;
}

__device__ __forceinline__ void item_offsets(bool isw, unsigned stg, unsigned pan, int v, unsigned& o0, unsigned& o1,
                                             unsigned& fl) {
  o0 = stg + pan * PANEL + v * (isw ? 4096 : CHUNK);
  o1 = o0 + (isw ? PANEL : 4096);
  fl = OFF_PF + (isw ? (pan + (v & 1)) * NCH + (v >> 1) : pan * NCH + v) * 128;
}
constexpr int kBI[][4] = P_BATCH_I, kBX[][4] = P_BATCH_X;
constexpr int NBI = sizeof(kBI) / sizeof(kBI[0]), NBX = sizeof(kBX) / sizeof(kBX[0]);
constexpr int GS = kBI[NBI - 1][1] ? 0 : kBI[NBI - 1][0], GS1 = kBI[NBI - 2][1] ? 0 : kBI[NBI - 2][0];
template <int N> __device__ __forceinline__ void batch_item(const int (&b)[N][4], int i, int& u, int& c) {
  int f = 0;
#pragma unroll
  for (int k = 0; k < N; ++k) {
    if (k == 0 || i >= f * NCH) {
      const int r = i - f * NCH;
      if (b[k][1]) { u = f + r % b[k][0]; c = r / b[k][0]; }
      else { u = i >> 4; c = i & 15; }
    }
    f += b[k][0];
  }
}
__device__ __forceinline__ void link_item(bool cross, int i, bool& isw, int& u, int& c) {
  isw = cross && i < 128;
  if (isw) { u = i & 3; c = i >> 2; }
  else if (cross) batch_item(kBX, i - 128, u, c);
  else batch_item(kBI, i, u, c);
}
__device__ __forceinline__ unsigned landing_panel(bool cross, bool isw, int u, int from, int to) {
  return !cross ? NI + 3 * u + ((from - to - 1) & 3) : isw ? NAP + 32 + from * 8 + 2 * u : 4 * NI + from * NX + u;
}
__device__ __forceinline__ int tile_a0(unsigned desc, int bfirst) {
  const int a = desc & 127;
  if ((desc >> 12) & 1) return NAP + NBP + a;
  if (bfirst < 32 || a < 4 * NI) return a;
  const int src = (a - 4 * NI) / NX;
  return a + ((src ^ ((bfirst >> 3) & 3)) - src) * NX;
}

#define LDIN(q) ((P_NTIN & 1) ? __builtin_nontemporal_load(q) : *(q))
#define LDINX(q) ((P_NTIN & 2) ? __builtin_nontemporal_load(q) : *(q))
__device__ __forceinline__ void load_inputs(const Params& p, bool isw, int g, int c, int lane, f32x4 (&v)[16]) {
  if (isw) {
    const GAS f32x2* src =
        (const GAS f32x2*)(p.w + (size_t)((c & 31) * 128 + 4 * (lane >> 4)) * SH + (g & 15) * 32 + 2 * (lane & 15));
#pragma unroll
    for (int q = 0; q < 16; ++q) {
      const f32x2 a = LDIN(src + ((q >> 1) * 16 + (q & 1) * 2) * (SH / 2));
      const f32x2 b = LDIN(src + ((q >> 1) * 16 + (q & 1) * 2 + 1) * (SH / 2));
      v[q] = f32x4{a.x, b.x, a.y, b.y};
    }
  } else {
    const GAS f32x4* src =
        (const GAS f32x4*)(p.x + (size_t)((g & 31) * 16 + (lane >> 2)) * KD + (c & 15) * 256 + 4 * (lane & 3));
#pragma unroll
    for (int q = 0; q < 16; ++q) v[q] = LDINX(src + q * 4);
  }
}
__device__ __forceinline__ void to_fragments(bool isw, const f32x4 (&v)[16], LAS v4u* scr, int lane, v4u (&h)[8]) {
  if (isw) {
#pragma unroll
    for (int s = 0; s < 4; ++s) {
      h[s] = v4u{pk(v[4 * s].x, v[4 * s].y), pk(v[4 * s + 1].x, v[4 * s + 1].y), pk(v[4 * s + 2].x, v[4 * s + 2].y),
                 pk(v[4 * s + 3].x, v[4 * s + 3].y)};
      h[4 + s] = v4u{pk(v[4 * s].z, v[4 * s].w), pk(v[4 * s + 1].z, v[4 * s + 1].w), pk(v[4 * s + 2].z, v[4 * s + 2].w),
                     pk(v[4 * s + 3].z, v[4 * s + 3].w)};
    }
  } else {
#pragma unroll
    for (int s = 0; s < 8; ++s)
      scr[s * 64 + lane] = v4u{pk(v[2 * s].x, v[2 * s].y), pk(v[2 * s].z, v[2 * s].w), pk(v[2 * s + 1].x, v[2 * s + 1].y),
                               pk(v[2 * s + 1].z, v[2 * s + 1].w)};
    LDSWAIT();
#pragma unroll
    for (int s = 0; s < 8; ++s) h[s] = scr[s * 64 + (lane & 15) * 4 + (lane >> 4)];
    LDSWAIT();
  }
}

struct Ctl { unsigned prog, bad, full[2], fre[2], conv, sfull[2], sfre[2]; };

__device__ __forceinline__ void need(volatile LAS unsigned* word, unsigned& have, unsigned n) {
  while (have < n) {
    have = __builtin_amdgcn_readfirstlane(*word);
    if (have < n) __builtin_amdgcn_s_sleep(1);
  }
  CBAR();
}

template <int MT, int NT>
__device__ __forceinline__ void gemm_tile(const Params& p, LAS char* lds, int a0, int b0, int t, unsigned& have,
                                          unsigned stg, int wave, int lane) {
  constexpr int D = (MT + NT <= 5) ? P_DS : P_DL;
  constexpr int NS = 32;
  volatile LAS Ctl* ctl = (volatile LAS Ctl*)(lds + LDS_CTL);
  const __amdgpu_buffer_rsrc_t rs = mk(p.ws[p.idx & 3], WS_USED);
  const unsigned ab = stg + (unsigned)a0 * PANEL + wave * 1024, bb = stg + (unsigned)(NAP + b0) * PANEL + wave * 1024;
  const int voff = lane * 16;
  const unsigned base = (unsigned)t * NCH;
  have = __builtin_amdgcn_readfirstlane(have);
  f32x4 acc[MT][NT];
#pragma unroll
  for (int m = 0; m < MT; ++m)
#pragma unroll
    for (int n = 0; n < NT; ++n) acc[m][n] = f32x4{0.f, 0.f, 0.f, 0.f};
  v4u ra[D][MT], rb[D][NT];
#define LOADSTEP(d, i) do { \
    _Pragma("unroll") for (int m = 0; m < MT; ++m) ra[d][m] = ld16<P_AUXA>(rs, voff, ab + m * PANEL + (i) * 4096); \
    _Pragma("unroll") for (int n = 0; n < NT; ++n) rb[d][n] = ld16(rs, voff, bb + n * PANEL + (i) * 4096); \
    __builtin_amdgcn_sched_barrier(0); } while (0)
#define MMASTEP(d) do { \
    _Pragma("unroll") for (int m = 0; m < MT; ++m) \
    _Pragma("unroll") for (int n = 0; n < NT; ++n) \
      acc[m][n] = __builtin_amdgcn_mfma_f32_16x16x32_f16(__builtin_bit_cast(h8, rb[d][n]), \
                                                         __builtin_bit_cast(h8, ra[d][m]), acc[m][n], 0, 0, 0); \
    __builtin_amdgcn_sched_barrier(0); } while (0)
#pragma unroll
  for (int d = 0; d < D; ++d) {
    if ((d & 1) == 0) need(&ctl->prog, have, base + (d >> 1) + 1);
    LOADSTEP(d, d);
  }
#pragma unroll 1
  for (int s0 = D; s0 < NS; s0 += D) {
#pragma unroll
    for (int d = 0; d < D; ++d) {
      MMASTEP(d);
      if ((d & 1) == 0) need(&ctl->prog, have, base + ((s0 + d) >> 1) + 1);
      LOADSTEP(d, s0 + d);
    }
  }
#pragma unroll
  for (int d = 0; d < D; ++d) MMASTEP(d);
#undef LOADSTEP
#undef MMASTEP
  asm volatile("" : "+v"(lane));
  constexpr int C4 = 4 * NT, M1 = MT > 4 ? (MT + 1) / 2 : MT;
#pragma unroll
  for (int part = 0; part < (MT > 4 ? 2 : 1); ++part) {
    const int tp = t + part, m0 = part ? M1 : 0, mn = part ? MT - M1 : M1;
    unsigned f = 0;
    if (tp >= 2) need(&ctl->fre[tp & 1], f, (unsigned)(tp >> 1));
    else need(&ctl->conv, f, NWAVE);
    LAS f32x4* red = (LAS f32x4*)(lds + (tp & 1) * LDS_RED) + wave * (64 * mn * NT);
#pragma unroll
    for (int m = 0; m < mn; ++m)
#pragma unroll
      for (int n = 0; n < NT; n += 2) {
        const int row = m * 16 + (lane & 15), c4 = n * 4 + 2 * (lane >> 4);
        const f32x4 e = acc[m0 + m][n], o = acc[m0 + m][n + 1];
        red[row * C4 + (c4 ^ (row & (C4 - 1)))] = f32x4{e.x, o.x, e.y, o.y};
        red[row * C4 + ((c4 + 1) ^ (row & (C4 - 1)))] = f32x4{e.z, o.z, e.w, o.w};
      }
    LDSWAIT();
    if (lane == 0)
      __hip_atomic_fetch_add((LAS unsigned*)&ctl->full[tp & 1], 1u, __ATOMIC_RELAXED, __HIP_MEMORY_SCOPE_WORKGROUP);
  }
}

template <int MT, int NT>
__device__ __forceinline__ void finish_tile(const Params& p, LAS char* lds, int bfirst, unsigned desc, int t,
                                            unsigned& have, unsigned call, int lane) {
  constexpr int C4 = 4 * NT, IMG = 64 * MT * NT, RQ = 64 / C4;
  volatile LAS Ctl* ctl = (volatile LAS Ctl*)(lds + LDS_CTL);
  const LAS f32x4* red = (const LAS f32x4*)(lds + (t & 1) * LDS_RED);
  const int idx = p.idx & 3, a0 = tile_a0(desc, bfirst), b = (desc >> 7) & 7;
  const int grp = ((bfirst + ((desc >> 7) & 31)) >> 3) & 7;
  const bool staged = NT == 4 && grp >= 4 && ((desc >> 29) & 1);
  const __amdgpu_buffer_rsrc_t ro = staged ? mk(p.ws[idx], WS_USED) : mk(p.out[grp < 4 ? idx : grp], OUT_BYTES);
  const int r0 = lane / C4, c4 = lane % C4;
  const unsigned slot = ((grp & 3) * NAP + a0) * 2 + (b >> 2);
  unsigned vo[MT];
#pragma unroll
  for (int m = 0; m < MT; ++m)
    vo[m] = staged ? OFF_YS + (slot + 2 * m) * 4096 + lane * 16
                   : ((p.rowbase[(a0 + m) & 255] + r0) * SH + (grp < 4 ? grp : idx) * 128 + b * 16 + c4 * 4) * 4;
  const unsigned qs = staged ? 1024 : RQ * SH * 4;
  const unsigned dst = NT == 4 ? (p.rowbase[(a0 + lane) & 255] * SH + idx * 128 + b * 16) * 4 : 0;
  need(&ctl->full[t & 1], have, 4u * ((t >> 1) + 1));
  STAMP(8 + t * 8 + 2);
  const bool bad = ctl->bad != 0;
  const unsigned ticket = NT != 4 ? 0u : (unsigned)__builtin_amdgcn_raw_ptr_buffer_atomic_add_i32(
      MT, ro, staged && lane == 0 ? OFF_YQ + (grp & 3) * 128 : NOWHERE, 0, 16);
  f32x4 s[MT * NT];
#pragma unroll
  for (int q = 0; q < MT * NT; ++q) {
    const int row = q * RQ + r0;
    const int li = row * C4 + (c4 ^ (row & (C4 - 1)));
    s[q] = (red[li] + red[IMG + li]) + (red[2 * IMG + li] + red[3 * IMG + li]);
    if (bad) s[q] = f32x4{__builtin_nanf(""), __builtin_nanf(""), __builtin_nanf(""), __builtin_nanf("")};
    if (q < P_YWIN) st16<P_AUXE>(ro, vo[q / NT] + (q % NT) * qs, __builtin_bit_cast(v4u, s[q]));
  }
  LDSWAIT();
  if (lane == 0) ctl->fre[t & 1] = (t >> 1) + 1;
  if (grp >= 4 && !staged) {
#pragma unroll
    for (int q = P_YWIN; q < MT * NT; ++q) {
      asm volatile("s_waitcnt vmcnt(%0)" :: "n"(P_YWIN - 1) : "memory");
      st16<P_AUXE>(ro, vo[q / NT] + (q % NT) * qs, __builtin_bit_cast(v4u, s[q]));
    }
  } else {
#pragma unroll
    for (int q = P_YWIN; q < MT * NT; ++q) st16<P_AUXE>(ro, vo[q / NT] + (q % NT) * qs, __builtin_bit_cast(v4u, s[q]));
  }
  if (grp >= 4) {
    DRAIN();
    if (staged) {
      const unsigned pos = __builtin_amdgcn_readfirstlane(ticket) - (call - 1) * 2 * p.nyf + lane;
      flag(lane < MT && pos < 2 * NAP, ro, OFF_YL + ((grp & 3) * 2 * NAP + pos) * 128, call,
           ((slot + 2 * lane) & 255) * 2 | dst >> 7 << 16);
    }
    else flag(lane == 0, mk(p.ws[grp], WS_USED), OFF_YF + (idx * YMAX + ((desc >> 13) & 255)) * 128, call);
  }
  STAMP(8 + t * 8 + 3);
}

constexpr bool XS = kBX[0][1] == 0;
__device__ __forceinline__ int stream_nx(unsigned desc) {
  const int kind = (desc >> 21) & 3;
  return kind == 2 ? NX : kind == 3 ? (4 * NX + 2 - (int)(desc & 3)) / 3 : 0;
}
__device__ __forceinline__ int stream_len(unsigned desc) {
  const int mv = ((desc >> 23) & 1 ? P_GSO : 0) + P_ALT;
  return (desc >> 21) & 3 ? stream_nx(desc) + GS - mv : GS1 + mv;
}
__device__ __forceinline__ int stream_panel(unsigned desc, int bfirst, int i) {
  const int kind = (desc >> 21) & 3, j = desc & 3, nx = stream_nx(desc), q = 3 * i + j;
  const bool last = i == stream_len(desc) - 1;
  if (kind == 0) return P_ALT && last ? NI + 3 * (NI - 2) + j : NI + 3 * (NI - GS - GS1) + q;
  if (i >= nx) return NI + 3 * (NI - stream_len(desc) - (last ? 0 : P_ALT)) + q;
  if (kind == 3) return 4 * NI + (q & 3) * NX + (q >> 2);
  return 4 * NI + ((((bfirst + (int)((desc >> 7) & 31)) >> 3) + 1 + j) & 3) * NX + i;
}
__device__ __forceinline__ void gemm_stream(const Params& p, LAS char* lds, unsigned desc, int bfirst, int t, unsigned& have,
                                            unsigned stg, int wave, int lane) {
  constexpr int D = P_SD;
  volatile LAS Ctl* ctl = (volatile LAS Ctl*)(lds + LDS_CTL);
  const __amdgpu_buffer_rsrc_t rs = mk(p.myws, WS_USED);
  const int G = stream_len(desc);
  const unsigned bb = stg + (unsigned)(NAP + bfirst + ((desc >> 7) & 31)) * PANEL + wave * 1024;
  unsigned ab = stg + (unsigned)stream_panel(desc, bfirst, 0) * PANEL + wave * 1024, base = (unsigned)t * NCH;
  const unsigned voff = lane * 16;
  LAS v4u* bl = (LAS v4u*)(lds + wave * 32768) + lane;
  static_assert(D == 16, "the ring doubles as the way into LDS");
  const unsigned rot = (((desc >> 8) & 7) * 4) & ((desc >> 21) & 3 ? P_ROTB : P_ROTA);
#define KS(n) (((n) + rot) & 31)
  have = __builtin_amdgcn_readfirstlane(have);
  v4u br[32], ra[D];
  unsigned f = 0;
#pragma unroll
  for (int s = 0; s < 32; ++s) {
    if ((s & 1) == 0) need(&ctl->prog, have, base + (KS(s) >> 1) + 1);
    br[s] = ld16(rs, voff, bb + KS(s) * 4096);
    if (s >= 16) ra[s - 16] = ld16(rs, voff, bb + PANEL + KS(s - 16) * 4096);
    __builtin_amdgcn_sched_barrier(0);
  }
  need(&ctl->conv, f, NWAVE);
  f = 0; need(&ctl->fre[0], f, (unsigned)(t + 1) >> 1);
  f = 0; need(&ctl->fre[1], f, (unsigned)t >> 1);
#pragma unroll
  for (int d = 0; d < D; ++d) {
    bl[d * 64] = ra[d];
    ra[d] = ld16(rs, voff, bb + PANEL + KS(16 + d) * 4096);
    __builtin_amdgcn_sched_barrier(0);
  }
  base += NCH;
#pragma unroll
  for (int d = 0; d < D; ++d) {
    bl[(16 + d) * 64] = ra[d];
    if ((d & 1) == 0) need(&ctl->prog, have, base + (KS(d) >> 1) + 1);
    ra[d] = ld16(rs, voff, ab + KS(d) * 4096);
    __builtin_amdgcn_sched_barrier(0);
  }
  const unsigned span = rot > 16 ? NCH : (rot >> 1) + 8;
  bool primed = true;
#pragma unroll 1
  for (int i = 0; i < G; ++i) {
    if (!primed) {
#pragma unroll
      for (int d = 0; d < D; ++d) {
        if ((d & 1) == 0) need(&ctl->prog, have, base + (KS(d) >> 1) + 1);
        ra[d] = ld16(rs, voff, ab + KS(d) * 4096);
        __builtin_amdgcn_sched_barrier(0);
      }
    }
    const bool more = i + 1 < G;
    const unsigned an = stg + (unsigned)stream_panel(desc, bfirst, i + more) * PANEL + wave * 1024;
    unsigned vnext = NOWHERE;
    f32x4 e = {0.f, 0.f, 0.f, 0.f}, o = {0.f, 0.f, 0.f, 0.f};
    v4u q[4];
    q[0] = bl[0]; q[1] = bl[64]; q[2] = bl[128];
#pragma unroll
    for (int s = 0; s < 32; ++s) {
      if (s + 3 < 32) q[(s + 3) & 3] = bl[(s + 3) * 64];
      e = __builtin_amdgcn_mfma_f32_16x16x32_f16(__builtin_bit_cast(h8, br[s]), __builtin_bit_cast(h8, ra[s % D]), e, 0, 0, 0);
      o = __builtin_amdgcn_mfma_f32_16x16x32_f16(__builtin_bit_cast(h8, q[s & 3]), __builtin_bit_cast(h8, ra[s % D]), o, 0, 0, 0);
      __builtin_amdgcn_sched_barrier(0);
      const int n = s + D;
      if (n < 32 && (n & 1) == 0) need(&ctl->prog, have, base + (KS(n) >> 1) + 1);
      if (n == 32) {
        if (have < base + NCH + span) have = __builtin_amdgcn_readfirstlane(ctl->prog);
        CBAR();
        primed = more && have >= base + NCH + span;
        if (primed) vnext = voff;
      }
      ra[s % D] = ld16(rs, n < 32 ? voff : vnext, (n < 32 ? ab : an) + KS(n) * 4096);
      __builtin_amdgcn_sched_barrier(0);
    }
#undef KS
    f = 0; need(&ctl->sfre[i & 1], f, (unsigned)i >> 1);
    LAS f32x4* red = (LAS f32x4*)(lds + LDS_SIMG + (i & 1) * 8192) + wave * 128;
    const int row = lane & 15, c4 = 2 * (lane >> 4);
    red[row * 8 + (c4 ^ (row & 7))] = f32x4{e.x, o.x, e.y, o.y};
    red[row * 8 + ((c4 + 1) ^ (row & 7))] = f32x4{e.z, o.z, e.w, o.w};
    LDSWAIT();
    if (lane == 0)
      __hip_atomic_fetch_add((LAS unsigned*)&ctl->sfull[i & 1], 1u, __ATOMIC_RELAXED, __HIP_MEMORY_SCOPE_WORKGROUP);
    ab = an;
    base += NCH;
  }
}

__device__ __forceinline__ void finish_stream(const Params& p, LAS char* lds, int bfirst, unsigned desc, int par,
                                              unsigned call, int lane) {
  volatile LAS Ctl* ctl = (volatile LAS Ctl*)(lds + LDS_CTL);
  const int idx = p.idx & 3, b = (desc >> 7) & 7, grp = ((bfirst + ((desc >> 7) & 31)) >> 3) & 7;
  const int r0 = lane >> 3, c4 = lane & 7, li = r0 * 8 + (c4 ^ r0);
  const int G = stream_len(desc), nx = stream_nx(desc);
  const int ns = G - ((desc >> 21) & 3 ? P_SLAST2 : P_SLAST);
  static_assert(P_SLAST2 <= GS - P_ALT && P_SLAST >= 1 && P_SLAST2 >= 1, "a half holds rows in a row");
  static_assert(!P_ALT || (GS >= 2 + P_GSO && GS1), "");
  TREC_DECL();
  int unit = 0;
  for (int i = 0; i < (grp >= 4 ? ns : G); ++unit) {
    const bool staged = grp >= 4;
    const int np = staged ? min(2, (i < nx ? nx : ns) - i) : 1;
    if ((unit & 1) == par) {
      const int a = stream_panel(desc, bfirst, i), a2 = stream_panel(desc, bfirst, i + np - 1);
      const __amdgpu_buffer_rsrc_t ro = staged ? mk(p.myws, WS_USED) : mk(p.out[idx], OUT_BYTES);
      const unsigned dst = (p.rowbase[a] * SH + (grp < 4 ? grp : idx) * 128 + b * 16) * 4;
      const unsigned slot = np == 2 ? (b & 4 ? a2 : a) * 4 + (b & 2) : a * 4 + (b >> 1);
      const unsigned vo = staged ? OFF_YS + ((grp & 3) * 4 * NAP + slot) * 2048 + lane * 16 : dst + (r0 * SH + c4 * 4) * 4;
      const unsigned qs = staged ? 1024 : 8 * SH * 4;
      unsigned ticket = 0;
      for (int e = 0; e < np; ++e) {
        const int h = (i + e) & 1;
        const LAS f32x4* red = (const LAS f32x4*)(lds + LDS_SIMG + h * 8192);
        unsigned have = 0;
        need(&ctl->sfull[h], have, 4u * (((i + e) >> 1) + 1));
        TREC(32 + i + e);
        ticket = (unsigned)__builtin_amdgcn_raw_ptr_buffer_atomic_add_i32(
            1, ro, staged && e == np - 1 && lane == 0 ? OFF_YQ + (grp & 3) * 128 : NOWHERE, 0, 16);
        f32x4 s0 = (red[li] + red[128 + li]) + (red[256 + li] + red[384 + li]);
        f32x4 s1 = (red[64 + li] + red[192 + li]) + (red[320 + li] + red[448 + li]);
        if (ctl->bad) s0 = s1 = f32x4{__builtin_nanf(""), __builtin_nanf(""), __builtin_nanf(""), __builtin_nanf("")};
        st16(ro, vo + 2 * e * qs, __builtin_bit_cast(v4u, s0));
        st16(ro, vo + (2 * e + 1) * qs, __builtin_bit_cast(v4u, s1));
        LDSWAIT();
        ctl->sfre[h] = ((i + e) >> 1) + 1;
      }
      if (staged) {
        DRAIN();
        const unsigned pos = __builtin_amdgcn_readfirstlane(ticket) - (call - 1) * 2 * p.nyf;
        flag(lane == 0 && pos < 2 * NAP, ro, OFF_YL + ((grp & 3) * 2 * NAP + pos) * 128, call,
             (slot & 511) | (np == 2 ? 1u << 9 : 3u << 9) | dst >> 7 << 16);
      }
    }
    i += np;
  }
  if (grp >= 4 && (unit & 1) == par) {
    constexpr int NL_ = P_SLAST > P_SLAST2 ? P_SLAST : P_SLAST2;
    const __amdgpu_buffer_rsrc_t ro = mk(p.out[grp], OUT_BYTES);
    f32x4 s[2 * NL_];
    unsigned vo[NL_];
    const __amdgpu_buffer_rsrc_t rf = mk(p.ws[grp], WS_USED);
#pragma unroll
    for (int e = 0; e < NL_; ++e) {
      const int i = G - NL_ + e;
      vo[e] = i >= ns ? ((p.rowbase[stream_panel(desc, bfirst, i)] + r0) * SH + idx * 128 + b * 16 + c4 * 4) * 4 : NOWHERE;
      asm volatile("" : "+v"(vo[e]));
    }
#pragma unroll
    for (int e = 0; e < NL_; ++e) {
      const int i = G - NL_ + e, h = i & 1;
      s[2 * e] = s[2 * e + 1] = f32x4{0.f, 0.f, 0.f, 0.f};
      if (i < ns) continue;
      const LAS f32x4* red = (const LAS f32x4*)(lds + LDS_SIMG + h * 8192);
      unsigned have = 0;
      need(&ctl->sfull[h], have, 4u * ((i >> 1) + 1));
      TREC(32 + i);
      s[2 * e] = (red[li] + red[128 + li]) + (red[256 + li] + red[384 + li]);
      s[2 * e + 1] = (red[64 + li] + red[192 + li]) + (red[320 + li] + red[448 + li]);
      asm volatile("" : "+v"(s[2 * e]), "+v"(s[2 * e + 1]));
      ctl->sfre[h] = (i >> 1) + 1;
    }
    const bool bad = ctl->bad != 0;
#pragma unroll
    for (int q = 0; q < 2 * NL_; ++q) {
      if (bad) s[q] = f32x4{__builtin_nanf(""), __builtin_nanf(""), __builtin_nanf(""), __builtin_nanf("")};
      st16(ro, vo[q >> 1] + (q & 1) * (8 * SH * 4), __builtin_bit_cast(v4u, s[q]));
    }
    DRAIN();
    flag(lane == 0, rf, OFF_YF + (idx * YMAX + ((desc >> 13) & 255)) * 128, call);
  }
}

#define BY_SHAPE(desc, BIG, F, ...) \
  switch (((desc) >> 24) & 15) { \
    case 1: F<1, 2>(__VA_ARGS__); break; \
    case 2: F<2, 2>(__VA_ARGS__); break; \
    case 3: F<3, 2>(__VA_ARGS__); break; \
    case 9: F<1, 4>(__VA_ARGS__); break; \
    case 10: F<2, 4>(__VA_ARGS__); break; \
    case 11: F<3, 4>(__VA_ARGS__); break; \
    case 12: F<4, 4>(__VA_ARGS__); break; \
    case 13: F<BIG ? 5 : 3, 4>(__VA_ARGS__); break; \
    default: F<BIG ? 6 : 3, 4>(__VA_ARGS__); break; \
  }

__global__ __launch_bounds__(NTHR, 2) void ag_gemm(const Params p) {
  extern __shared__ __attribute__((aligned(16))) char smem[];
  LAS char* lds = (LAS char*)smem;
  const int lane = threadIdx.x & 63;
  const int wave = __builtin_amdgcn_readfirstlane(threadIdx.x >> 6);
  const int blk = blockIdx.x, grp = blk & 7, j = blk >> 3;
  asm volatile("" :: "s"(p.x), "s"(p.w), "s"(p.myws), "s"(p.idx), "s"(p.ws[grp]));
  const int idx = p.idx & 3;
  const bool cross = grp >= 4;
  char* myws = p.myws;
  const __amdgpu_buffer_rsrc_t rl = mk(myws, WS_USED), rd = mk(p.ws[grp], WS_USED);
  STAMP(0);
  char* counter = myws + OFF_CNT + (blk * 8 + wave) * 4;
  const unsigned last_call = ld_sys(counter);
  const unsigned long long tw = __builtin_amdgcn_s_memrealtime();
  const int jg = j - NC - NU;
  const bool keeper = cross || grp == ((idx + 1) & 3);
  const bool early = wave == 6 && (unsigned)jg < (unsigned)NDRAW && grp != idx;
  if (early) {
    static_assert(NDRAW <= NG && NDRAW <= kBI[0][0] * NCH, "in the first batch");
    const int u = cross ? jg & 3 : jg % kBI[0][0], c = cross ? jg >> 2 : jg / kBI[0][0];
    f32x4 v[16];
    v4u h[8];
    if (cross) {
      load_inputs(p, true, (grp - 4) * 4 + u, c, lane, v);
      __builtin_amdgcn_s_barrier();
      to_fragments(true, v, (LAS v4u*)(lds + 6 * 8192), lane, h);
    } else {
      load_inputs(p, false, u, c, lane, v);
      __builtin_amdgcn_s_barrier();
      to_fragments(false, v, (LAS v4u*)(lds + 6 * 8192), lane, h);
    }
    const unsigned call = __builtin_amdgcn_readfirstlane(last_call) + 1u;
    unsigned o0, o1, fl;
    item_offsets(cross, OFF_STG + (call & 1u) * STG_BYTES, landing_panel(cross, cross, u, idx, grp & 3), c, o0, o1, fl);
    STAMP(7);
    while ((int)(__builtin_amdgcn_s_memrealtime() - tw) < P_PACE0 + jg * P_PACE / 2) __builtin_amdgcn_s_sleep(1);
#pragma unroll
    for (int s = 0; s < 4; ++s) st16(rd, o0 + lane * 16 + s * 1024, h[s]);
#pragma unroll
    for (int s = 0; s < 4; ++s) st16(rd, o1 + lane * 16 + s * 1024, h[4 + s]);
    STAMP(6);
    unsigned l0, l1, ll;
    item_offsets(cross, OFF_STG + (call & 1u) * STG_BYTES, cross ? NAP + (grp - 4) * 8 + 2 * u : u, c, l0, l1, ll);
    if (keeper) {
#pragma unroll
      for (int s = 0; s < 4; ++s) st16(rl, l0 + lane * 16 + s * 1024, h[s]);
#pragma unroll
      for (int s = 0; s < 4; ++s) st16(rl, l1 + lane * 16 + s * 1024, h[4 + s]);
    }
    DRAIN();
    flag(lane == 0, rd, fl, call);
    flag(lane == 0 && keeper, rl, ll, call);
  } else {
    if (threadIdx.x < 16) ((volatile LAS unsigned*)(lds + LDS_CTL))[threadIdx.x] = 0;
    LDSWAIT();
    __builtin_amdgcn_s_barrier();
  }

  if (j < NC && grp != idx) {
    struct Ring { unsigned full[NL], done, abort; };
    volatile LAS Ring* rg = (volatile LAS Ring*)(lds + LDS_CTL);
    constexpr int SLOT = 10240;
    volatile LAS v2u* meta = (volatile LAS v2u*)(lds + 2 * NL * SLOT);
    const int n = !cross ? NI * NCH : NXY + (P_NOGEMM ? 0 : p.nyf);
    if (wave == 0) {
      const unsigned call = __builtin_amdgcn_readfirstlane(last_call) + 1u;
      if (lane == 0) st_sys(counter, call);
      __builtin_amdgcn_s_setprio(3);
      const unsigned long long peer_out = uniform64(p.out[grp]), peer_ws = uniform64(p.ws[grp]);
      unsigned pl[3], ph[3];
#pragma unroll
      for (int d = 0; d < 3; ++d) { pl[d] = NOWHERE; ph[d] = 0; }
      const int cnt = (n - ND - j + NC - 1) / NC;
      const int ry = min(cnt, (max((cross ? NXY : n) - ND - j, 0) + NC - 1) / NC);
      unsigned lane0 = lane == 0 ? ~0u : 0u, lane1 = lane == 1 ? ~0u : 0u;
      asm volatile("" : "+v"(lane0), "+v"(lane1));
      int r = 0, l = 0, k = 0;
      bool dead = false;
      TREC_DECL();
      auto rounds = [&](int end, auto wide, auto yphase) {
        constexpr int W = wide() ? P_WIN0 : P_WIN;
        static_assert(W <= (wide() ? 22 : 13), "window");
        for (; r < end; ++r) {
          LAS char* sl = lds + (2 * l + (k & 1)) * SLOT + lane * 16;
          unsigned got;
          v2u m;
          v4u a0, a1, v[8];
#define READ_SLOT() do { \
          const unsigned fu = rg->full[l]; \
          m = meta[2 * l + (k & 1)]; \
          a0 = *(volatile LAS v4u*)(sl + 8192); a1 = *(volatile LAS v4u*)(sl + 9216); \
          _Pragma("unroll") for (int q = 0; q < 8; ++q) v[q] = *(volatile LAS v4u*)(sl + q * 1024); \
          got = __builtin_amdgcn_readfirstlane(fu); } while (0)
          READ_SLOT();
          if (__builtin_expect(got <= (unsigned)k, 0)) {
            DRAIN();
            flag(lane < 3, rd, lane == 0 ? pl[0] : lane == 1 ? pl[1] : pl[2], call,
                 lane == 0 ? ph[0] : lane == 1 ? ph[1] : ph[2]);
            pl[0] = pl[1] = pl[2] = NOWHERE;
            while (__builtin_amdgcn_readfirstlane(rg->full[l]) <= (unsigned)k) __builtin_amdgcn_s_sleep(0);
            READ_SLOT();
          }
#undef READ_SLOT
          if (got == 0xffffffffu) { dead = true; r = cnt; break; }
          const bool to_out = yphase() && ((__builtin_amdgcn_readfirstlane(m.y) >> 17) & 1);
          const __amdgpu_buffer_rsrc_t rs = mk((char*)(to_out ? peer_out : peer_ws), to_out ? OUT_BYTES : WS_USED);
          const unsigned fo = wide() ? bfi(lane0, pl[0], NOWHERE) : bfi(lane0, pl[1], bfi(lane1, pl[0], NOWHERE));
          v2u fv = {call, wide() || lane ? ph[0] : ph[1]};
#pragma unroll
          for (int q = 0; q < 8; ++q) asm volatile("" : "+v"(v[q]));
          rg->done = r + 1;
          asm volatile("s_waitcnt vmcnt(%0)" :: "n"(W) : "memory");
#pragma unroll
          for (int q = 0; q < 4; ++q) st16<P_AUXP>(rs, a0[q], v[q]);
          asm volatile("s_waitcnt vmcnt(%0)" :: "n"(W) : "memory");
          __builtin_amdgcn_raw_buffer_store_b64(fv, rd, (int)fo, 0, AUX_SYS);
          CBAR();
#pragma unroll
          for (int q = 0; q < 4; ++q) st16<P_AUXP>(rs, a1[q], v[4 + q]);
          CBAR();
          if (!(r & 1)) TREC(r >> 1);
          pl[0] = wide() ? pl[1] : NOWHERE; ph[0] = ph[1]; pl[1] = pl[2]; ph[1] = ph[2];
          pl[2] = __builtin_amdgcn_readfirstlane(m.x); ph[2] = __builtin_amdgcn_readfirstlane(m.y);
          if (++l == NL) { l = 0; ++k; }
        }
      };
      while ((int)(__builtin_amdgcn_s_memrealtime() - tw) < P_PACE0 + ND * P_PACE / 2 - P_HOLD) __builtin_amdgcn_s_sleep(2);
      rounds(min(P_R0, ry), std::true_type(), std::false_type());
      while (r < ry) {
        const int raw = cross ? 0 : max(r - (NPI - ND) / NC, 0);
        if (P_LATE && r < ry - P_NOWIDE &&
            (int)(__builtin_amdgcn_s_memrealtime() - tw) > P_LATE + (ND - 50) * P_PACE / 2 + r * P_RATE / 10 + raw * 7)
          rounds(min(r + 2, ry), std::true_type(), std::false_type());
        else rounds(min(r + 2, ry), std::false_type(), std::false_type());
      }
      rounds(cnt, std::false_type(), std::true_type());
      if (dead) { rg->done = 0xffffffffu; return; }
      asm volatile("s_waitcnt vmcnt(18)" ::: "memory");
      flag(lane == 0, rd, pl[0], call, ph[0]);
      asm volatile("s_waitcnt vmcnt(10)" ::: "memory");
      flag(lane == 0, rd, pl[1], call, ph[1]);
      asm volatile("s_waitcnt vmcnt(2)" ::: "memory");
      flag(lane == 0, rd, pl[2], call, ph[2]);
      STAMP(1);
      TREC_DUMP();
      return;
    }
    const int l = wave - 1, first = ND + l * NC + j, cnt = (n - first + WL - 1) / WL;
    bool isw; int u, c;
    const unsigned call = __builtin_amdgcn_readfirstlane(last_call) + 1u;
    if (lane == 0) st_sys(counter, call);
    const unsigned stg = OFF_STG + (call & 1u) * STG_BYTES;
    const unsigned long long t0 = __builtin_amdgcn_s_memrealtime();
    if (P_TOUCH && wave == NL) {
      unsigned t = __builtin_amdgcn_raw_buffer_load_b32(rd, lane < 26 ? stg + (lane << 20) : lane < 28 ? (lane - 26) << 20 : NOWHERE, 0, AUX_SYS);
      t += __builtin_amdgcn_raw_buffer_load_b32(mk(p.out[grp], OUT_BYTES), cross && lane < 8 ? lane << 20 : NOWHERE, 0, AUX_SYS);
      t += __builtin_amdgcn_raw_buffer_load_b32(rl, lane < 26 ? stg + (lane << 20) : lane < 40 ? (lane - 26) << 20 : NOWHERE, 0, AUX_SYS);
      t += __builtin_amdgcn_raw_buffer_load_b32(mk(p.out[idx], OUT_BYTES), lane < 8 ? lane << 20 : NOWHERE, 0, AUX_SYS);
      asm volatile("" :: "v"(t));
    }
    const int i = min(first + lane * WL, n - 1);
    unsigned s0, s1, sf, sg, d0, d1, df, kind;
    if (!cross || i < NXY) {
      link_item(cross, i, isw, u, c);
      const int g = isw ? (grp - 4) * 4 + u : cross ? NI + u : u;
      item_offsets(isw, stg, isw ? NAP + 2 * g : cross ? NAP + NBP + u : u, c, s0, s1, sf);
      item_offsets(isw, stg, landing_panel(cross, isw, u, idx, grp & 3), c, d0, d1, df);
      sg = sf;
      kind = i < (cross ? NPX : NPI) && !(isw && i >= 128 - P_NRAWW);
      if (kind) {
        s0 = OFF_EXP + (isw ? 512 + g * 32 + c : g * 16 + c) * 8192; s1 = s0 + 4096;
        df += OFF_AF + (call & 1u) * PF_BYTES - OFF_PF;
      }
    } else {
      sf = OFF_YL + ((grp & 3) * NAP + ((i - NXY) & 127)) * 256; sg = sf + 128;
      s0 = s1 = d0 = d1 = 0;
      df = OFF_YF + (idx * YMAX + (i - NXY)) * 128;
      kind = 2;
    }
    if (lane == 63) { sf = OFF_TMO; sg = OFF_TMO; }
    unsigned long long ready = 0;
    unsigned hdv = 64u << 24 | kind << 16;
    while ((int)(__builtin_amdgcn_s_memrealtime() - tw) < P_LSTART + (first - ND) * P_LSTEP) __builtin_amdgcn_s_sleep(4);
    for (int k = 0; k < cnt; ++k) {
      while (!((ready >> k) & 1)) {
        const bool ask = ((lane >= k && lane < min(cnt, k + (k < 2 ? P_LWIN : 64))) || lane == 63) && !((ready >> lane) & 1);
        const v2u f = ldflag(rl, ask ? sf : NOWHERE);
        const v2u g = ldflag(rl, ask && kind == 2 ? sg : NOWHERE);
        const bool ok = f.x == call && (kind != 2 || g.x == call);
        if (ok && kind == 1) hdv = f.y;
        if (ok && kind == 2) { s0 = f.y; s1 = g.y; }
        ready |= __ballot(ok);
        if ((ready >> 63) || __builtin_amdgcn_s_memrealtime() - t0 > DEADLINE) {
          st_sys(myws + OFF_TMO, call);
          rg->abort = 1; LDSWAIT(); rg->full[l] = 0xffffffffu;
          return;
        }
        if (P_LSLEEP && k < 3 && !((ready >> k) & 1)) __builtin_amdgcn_s_sleep(P_LSLEEP);
      }
      CBAR();
      unsigned hd = __builtin_amdgcn_readlane(hdv, k);
      unsigned a0 = __builtin_amdgcn_readlane(s0, k), a1 = __builtin_amdgcn_readlane(s1, k);
      unsigned e0 = __builtin_amdgcn_readlane(d0, k), e1 = __builtin_amdgcn_readlane(d1, k);
      const unsigned q0 = a0, q1 = a1;
      if ((hd >> 17) & 1) {
        e0 = a0 >> 16 << 7; e1 = a1 >> 16 << 7;
        hd |= (a0 & 512) << 9 | (a1 & 512) << 10 | (a0 & 1024) << 10 | (a1 & 1024) << 11;
        a0 = OFF_YS + ((grp & 3) * 4 * NAP + (a0 & 511)) * 2048;
        a1 = OFF_YS + ((grp & 3) * 4 * NAP + (a1 & 511)) * 2048;
      }
      v4u h[8];
#pragma unroll
      for (int q = 0; q < 4; ++q) h[q] = ld16<P_AUXLL>(rl, lane * 16 + q * 1024, a0);
#pragma unroll
      for (int q = 0; q < 3; ++q) h[4 + q] = ld16<P_AUXLL>(rl, lane * 16 + q * 1024, a1);
      h[7] = ld16<P_AUXLL>(rl, lane < (hd >> 24) ? lane * 16 + 3072 : NOWHERE, a1);
      const int yi = first + k * WL - NXY;
      if (P_PACKY && ((hd >> 17) & 1) && !((hd >> 20) & 3) && yi < p.nyf - P_NRAWY) {
        float ss = 0.f;
#pragma unroll
        for (int q = 0; q < 8; q += 2) {
          const f32x4 z = __builtin_bit_cast(f32x4, h[q]);
          ss += z.x * z.x + z.y * z.y + z.z * z.z + z.w * z.w;
        }
        const unsigned base = (unsigned)min(max(top_exponent(ss, 20.f / 1024.f) - 15, 0), 240);
        pack8f(h, base);
        LAS unsigned* scr = (LAS unsigned*)(lds + 2 * NL * SLOT + 128 + l * 256);
        const unsigned total = list_plane(h[7], (LAS unsigned short*)(scr + 2), LIST_CAP - 4, lane);
        if (total <= LIST_CAP - 4) {
          scr[lane & 1] = lane & 1 ? q1 : q0;
          LDSWAIT();
          h[7] = *(volatile LAS v4u*)(scr + 4 * (lane & 15));
          hd = base | total << 8 | 1u << 22 | ((total + 35) >> 5) * 4 << 24;
          e0 = OFF_YP + (idx * NAP + yi) * 8192; e1 = e0 + 4096;
        } else {
          unpack8f(h, base);
        }
      }
      const bool isy = (hd >> 17) & 1;
      const unsigned va = (lane >> 4) * (SH * 4) + (lane & 15) * 16, vc = (lane >> 3) * (SH * 4) + (lane & 7) * 16;
      const unsigned vb = e0 + (!isy ? lane * 16 : (hd >> 18) & 1 ? vc : va), qs = !isy ? 1024 : 8192 << ((hd >> 18) & 1);
      const unsigned vd = e1 + (!isy ? lane * 16 : (hd >> 19) & 1 ? vc : va), qt = !isy ? 1024 : 8192 << ((hd >> 19) & 1);
      if (k >= 2) {
        unsigned f = 0;
        need(&rg->done, f, (unsigned)((k - 2) * NL + l + 1));
        if (__builtin_amdgcn_readfirstlane(rg->abort)) { rg->full[l] = 0xffffffffu; return; }
      }
      LAS v4u* sl = (LAS v4u*)(lds + (2 * l + (k & 1)) * SLOT);
      const bool sh0 = (hd >> 20) & 1, sh1 = (hd >> 21) & 1;
      sl[512 + lane] = v4u{vb, vb + qs, sh0 ? NOWHERE : vb + 2 * qs, sh0 ? NOWHERE : vb + 3 * qs};
      sl[576 + lane] = v4u{vd, vd + qt, sh1 ? NOWHERE : vd + 2 * qt, lane < (hd >> 24) && !sh1 ? vd + 3 * qt : NOWHERE};
      meta[2 * l + (k & 1)] = v2u{(unsigned)__builtin_amdgcn_readlane(df, k), hd};
#pragma unroll
      for (int q = 0; q < 8; ++q) sl[q * 64 + lane] = h[q];
      LDSWAIT();
      rg->full[l] = k + 1;
    }
    STAMP(1);
    return;
  }
  if (j < NC + NU) {
    const unsigned call = __builtin_amdgcn_readfirstlane(last_call) + 1u;
    if (lane == 0) st_sys(counter, call);
    const unsigned stg = OFF_STG + (call & 1u) * STG_BYTES;
    const unsigned aoff = OFF_AF + (call & 1u) * PF_BYTES - OFF_PF;
    const unsigned long long t0 = __builtin_amdgcn_s_memrealtime();
    if (P_PACK) {
      const int lk = grp == idx ? (idx + 1 + wave) & 7 : grp;
      const bool xl = lk >= 4;
      const int n = xl ? NPX - P_NRAWW : NPI, first = NDRAW + (grp == idx ? NU * NWAVE + j : wave * NU + j - NC);
      const int cnt = (n - first + WU - 1) / WU;
      bool isw; int u, c;
      const int ui = min(first + lane * WU, n - 1);
      link_item(xl, xl && ui >= 128 - P_NRAWW ? ui + P_NRAWW : ui, isw, u, c);
      unsigned o0, o1, fl;
      item_offsets(isw, stg, landing_panel(xl, isw, u, lk & 3, idx), c, o0, o1, fl);
      LAS unsigned* plane = (LAS unsigned*)(lds + wave * 1024);
      unsigned pend = NOWHERE, hdr = 0;
      bool have_next = false;
      unsigned long long tseen = tw;
      if (P_USTART)
        while ((int)(__builtin_amdgcn_s_memrealtime() - tw) < P_USTART + first * P_PACE / 2) __builtin_amdgcn_s_sleep(4);
      TREC_DECL();
      for (int r = 0; r < cnt; ++r) {
        const unsigned fr = __builtin_amdgcn_readlane(fl, r);
        *(LAS v4u*)(plane + lane * 4) = v4u{0, 0, 0, 0};
        if (!have_next && pend != NOWHERE) {
          DRAIN();
          flag(lane == 0, rl, pend, call);
          TREC(4 * r - 1);
          pend = NOWHERE;
          while ((int)(__builtin_amdgcn_s_memrealtime() - tseen) < P_UGAP) __builtin_amdgcn_s_sleep(2);
        }
        for (unsigned spin = 1; !have_next; ++spin) {
          const v2u f = ldflag(rl, lane == 0 ? fr + aoff : lane == 1 && !(spin & P_TMOMASK) ? OFF_TMO : NOWHERE);
          hdr = __builtin_amdgcn_readfirstlane(f.y);
          if ((int)(__builtin_amdgcn_readfirstlane(f.x) - call) >= 0) {
            tseen = __builtin_amdgcn_s_memrealtime() - (spin == 1 ? P_UBACK : 0);
            break;
          }
          if (__builtin_amdgcn_readlane(f.x, 1) == call || __builtin_amdgcn_s_memrealtime() - t0 > DEADLINE) {
            st_sys(myws + OFF_TMO, call);
            st_sys(p.out[idx] + (size_t)p.rowbase[0] * SH * 4, 0x7fc00000u);
            return;
          }
          if (P_USLEEP) __builtin_amdgcn_s_sleep(P_USLEEP);
        }
        CBAR();
        TREC(4 * r);
        const unsigned a0 = __builtin_amdgcn_readlane(o0, r), a1 = __builtin_amdgcn_readlane(o1, r);
        const bool dense = (hdr >> 16) & 1;
        const unsigned e2 = __builtin_amdgcn_raw_buffer_load_b32(rl, lane * 4 + 3072, a1, AUX_SYS);
        CBAR();
        v4u h[8];
#pragma unroll
        for (int q = 0; q < 4; ++q) h[q] = ld16<P_AUXUL>(rl, lane * 16 + q * 1024, a0);
#pragma unroll
        for (int q = 0; q < 3; ++q) h[4 + q] = ld16<P_AUXUL>(rl, lane * 16 + q * 1024, a1);
        h[7] = ld16(rl, dense ? lane * 16 + 3072 : NOWHERE, a1);
        const v2u fn = ldflag(rl, P_ULOOK && lane == 0 ? __builtin_amdgcn_readlane(fl, min(r + 1, cnt - 1)) + aoff : NOWHERE);
        CBAR();
        const unsigned cn = dense ? 0 : (hdr >> 8) & 255;
        LDSWAIT();
#pragma unroll
        for (int k = 0; k < 2; ++k) {
          const unsigned e = (e2 >> (16 * k)) & 0xffffu;
          __hip_atomic_fetch_or(plane + ((e >> 7) & 255), 2 * lane + k < cn ? (e & 15) << ((e >> 2) & 28) : 0u,
                                __ATOMIC_RELAXED, __HIP_MEMORY_SCOPE_WORKGROUP);
        }
        LDSWAIT();
        const v4u pl = *(volatile LAS v4u*)(plane + lane * 4);
        DRAIN();
        flag(lane == 0, rl, pend, call);
        if (pend != NOWHERE) TREC(4 * r - 1);
        TREC(4 * r + 1);
        h[7] |= pl;
        unpack8(h, hdr & 31);
#pragma unroll
        for (int q = 0; q < 4; ++q) st16<P_AUXU>(rl, a0 + lane * 16 + q * 1024, h[q]);
#pragma unroll
        for (int q = 0; q < 4; ++q) st16<P_AUXU>(rl, a1 + lane * 16 + q * 1024, h[4 + q]);
        pend = fr;
        TREC(4 * r + 2);
        hdr = __builtin_amdgcn_readfirstlane(fn.y);
        have_next = (int)(__builtin_amdgcn_readfirstlane(fn.x) - call) >= 0;
      }
      DRAIN();
      flag(lane == 0, rl, pend, call);
      TREC_DUMP();
      STAMP(2);
      const __amdgpu_buffer_rsrc_t ro = mk(p.out[idx], OUT_BYTES);
      for (int yi = first - NDRAW; xl && !P_NOGEMM && yi < p.nyf; yi += WU) {
        *(LAS v4u*)(plane + lane * 4) = v4u{0, 0, 0, 0};
        for (unsigned spin = 1;; ++spin) {
          if (P_YSLEEP && spin > 1 && yi + WU < p.nyf) __builtin_amdgcn_s_sleep(P_YSLEEP);
          const v2u f = ldflag(rl, lane == 0 ? OFF_YF + ((lk & 3) * YMAX + yi) * 128
                                   : lane == 1 && !(spin & P_TMOMASK) ? OFF_TMO : NOWHERE);
          hdr = __builtin_amdgcn_readfirstlane(f.y);
          if ((int)(__builtin_amdgcn_readfirstlane(f.x) - call) >= 0) break;
          if (__builtin_amdgcn_readlane(f.x, 1) == call || __builtin_amdgcn_s_memrealtime() - t0 > DEADLINE) {
            st_sys(myws + OFF_TMO, call);
            st_sys(p.out[idx] + (size_t)p.rowbase[0] * SH * 4, 0x7fc00000u);
            return;
          }
        }
        if (!P_PACKY || !((hdr >> 22) & 1)) continue;
        CBAR();
        const unsigned a0 = OFF_YP + ((lk & 3) * NAP + yi) * 8192;
        const unsigned e2 = __builtin_amdgcn_raw_buffer_load_b32(rl, lane * 4 + 7168, a0, AUX_SYS);
        CBAR();
        v4u h[8];
#pragma unroll
        for (int q = 0; q < 7; ++q) h[q] = ld16(rl, lane * 16 + (q & 3) * 1024, a0 + (q >> 2) * 4096);
        CBAR();
        LDSWAIT();
#pragma unroll
        for (int k = 0; k < 2; ++k) {
          const unsigned e = (e2 >> (16 * k)) & 0xffffu;
          __hip_atomic_fetch_or(plane + ((e >> 7) & 255),
                                2u * lane + k - 4u < ((hdr >> 8) & 255) ? (e & 15) << ((e >> 2) & 28) : 0u,
                                __ATOMIC_RELAXED, __HIP_MEMORY_SCOPE_WORKGROUP);
        }
        LDSWAIT();
        h[7] = *(volatile LAS v4u*)(plane + lane * 4);
        unpack8f(h, hdr & 255);
        const unsigned va = (lane >> 4) * (SH * 4) + (lane & 15) * 16, vc = (lane >> 3) * (SH * 4) + (lane & 7) * 16;
#pragma unroll
        for (int e = 0; e < 2; ++e) {
          const unsigned qe = __builtin_amdgcn_readlane(e2, e);
          const unsigned vb = (qe >> 16 << 7) + (qe & 512 ? vc : va), qs = qe & 512 ? 16384 : 8192;
#pragma unroll
          for (int q = 0; q < 4; ++q) st16(ro, vb + q * qs, h[4 * e + q]);
        }
      }
      STAMP(1);
    }
    if (grp == idx) {
      const int f = (j * NWAVE + wave) * 64 + lane;
      if (f - lane >= 4 * YMAX) return;
      bool mine = (f % YMAX) >= p.nyf && (f % YMAX) < p.ny;
      while ((int)(__builtin_amdgcn_s_memrealtime() - tw) < P_FSTART) __builtin_amdgcn_s_sleep(8);
      while (!P_NOGEMM) {
        const unsigned v = __builtin_amdgcn_raw_buffer_load_b32(rl, mine ? OFF_YF + f * 128 : NOWHERE, 0, AUX_SYS);
        const unsigned tm = __builtin_amdgcn_readfirstlane(ld_sys(myws + OFF_TMO));
        mine = mine && (int)(v - call) < 0;
        if (!__any(mine)) break;
        if (tm == call || __builtin_amdgcn_s_memrealtime() - t0 > DEADLINE) {
          st_sys(myws + OFF_TMO, call);
          st_sys(p.out[idx] + (size_t)p.rowbase[0] * SH * 4, 0x7fc00000u);
          break;
        }
        __builtin_amdgcn_s_sleep(1);
      }
      STAMP(1);
    }
    return;
  }

  volatile LAS Ctl* ctl = (volatile LAS Ctl*)(lds + LDS_CTL);
  unsigned call;
#pragma unroll 1
  for (int round = 0; round < (wave == P_W2 ? 2 : 1); ++round) {
    const bool direct = wave >= 4 && round == 0;
    const int di = wave == 6 ? jg : wave == 5 ? NG + (jg + P_ROT) % NG : 2 * NG + (jg + P_ROT2) % NG;
    const int leave = P_PACE0 + di * P_PACE / 2;
    const int it = min(wave, 3) * (NG * 8) + jg * 8 + grp;
    const bool work = direct ? grp != idx && di >= NDRAW && di < ND : (wave < 3 || round || (wave == 3 && P_W2 < 0)) && it < NCV;
    bool isw = false, raw = true, packed = true;
    int g = 0, c = 0, u = 0;
    f32x4 v[16];
    if (P_PRIO) __builtin_amdgcn_s_setprio(3);
    if (work && direct) {
      link_item(cross, di, isw, u, c);
      g = isw ? (grp - 4) * 4 + u : u;
      raw = keeper;
      while ((int)(__builtin_amdgcn_s_memrealtime() - tw) < leave - (cross ? P_LEADW : P_LEAD)) __builtin_amdgcn_s_sleep(2);
    } else if (work) {
      const int k = it / 5 + ND, m = it % 5;
      isw = k < 128 && m < 4;
      if (isw) { g = m * 4 + (k & 3); c = k >> 2; packed = k < 128 - P_NRAWW; }
      else {
        const int xi = k < 128 ? k : it - (NCV - 512);
        const bool xc = xi >= NI * NCH;
        const int li = xc ? xi - NI * NCH + 128 : xi;
        bool dummy;
        link_item(xc, li, dummy, u, c);
        g = xc ? NI + u : u;
        packed = li < (xc ? NPX : NPI);
      }
      while ((int)(__builtin_amdgcn_s_memrealtime() - tw) < P_CDELAY0 + (it * P_CDELAY >> 8)) __builtin_amdgcn_s_sleep(2);
    }
    if (work) load_inputs(p, isw, g, c, lane, v);
    call = __builtin_amdgcn_readfirstlane(last_call) + 1u;
    if (work) {
      const unsigned stg = OFF_STG + (call & 1u) * STG_BYTES;
      LAS v4u* scr = (LAS v4u*)(lds + wave * 8192);
      v4u h[8];
      to_fragments(isw, v, scr, lane, h);
      unsigned o0, o1, fl, hdr = 0, keepfl = NOWHERE;
      item_offsets(isw, stg, isw ? NAP + 2 * g : g < NI ? g : NAP + NBP + u, c, o0, o1, fl);
      const __amdgpu_buffer_rsrc_t rs = mk(direct ? p.ws[grp] : myws, WS_USED);
      STAMP(7);
      if (raw) {
#pragma unroll
        for (int s = 0; s < 4; ++s) st16(rl, o0 + lane * 16 + s * 1024, h[s]);
#pragma unroll
        for (int s = 0; s < 4; ++s) st16(rl, o1 + lane * 16 + s * 1024, h[4 + s]);
      }
      if (P_PACK && packed) {
        float ss = 0.f;
#pragma unroll
        for (int q = 0; q < 16; q += 4) ss += v[q].x * v[q].x + v[q].y * v[q].y + v[q].z * v[q].z + v[q].w * v[q].w;
        const unsigned base = (unsigned)min(max(top_exponent(ss, 6.76f / 1024.f) - 127 + 15 - 7, 0), 24);
        pack8(h, base);
        const unsigned total = list_plane(h[7], (LAS unsigned short*)scr, LIST_CAP, lane);
        const bool dense = total > (unsigned)LIST_CAP;
        if (!dense) {
          LDSWAIT();
          h[7] = *(volatile LAS v4u*)(scr + (lane & 15));
        }
        const unsigned nl = dense ? 64 : max((total + 31) >> 5, 1u) * 4;
        hdr = base | (dense ? 1u << 16 : total << 8) | nl << 24;
        if (direct) {
          if (raw) keepfl = fl;
          item_offsets(isw, stg, landing_panel(cross, isw, u, idx, grp & 3), c, o0, o1, fl);
          fl += OFF_AF + (call & 1u) * PF_BYTES - OFF_PF;
          while ((int)(__builtin_amdgcn_s_memrealtime() - tw) < leave) __builtin_amdgcn_s_sleep(1);
        } else {
          o0 = OFF_EXP + (isw ? 512 + g * 32 + c : g * 16 + c) * 8192;
          o1 = o0 + 4096;
        }
#pragma unroll
        for (int s = 0; s < 4; ++s) st16(rs, o0 + lane * 16 + s * 1024, h[s]);
#pragma unroll
        for (int s = 0; s < 3; ++s) st16(rs, o1 + lane * 16 + s * 1024, h[4 + s]);
        st16(rs, lane < nl ? o1 + lane * 16 + 3072 : NOWHERE, h[7]);
      }
      STAMP(6);
      DRAIN();
      flag(lane == 0, rs, fl, call, hdr);
      flag(lane == 0, rl, keepfl, call);
    }
    STAMP(2);
  }
  if (lane == 0) {
    st_sys(counter, call);
    __hip_atomic_fetch_add((LAS unsigned*)&ctl->conv, 1u, __ATOMIC_RELAXED, __HIP_MEMORY_SCOPE_WORKGROUP);
  }
  const unsigned stg = OFF_STG + (call & 1u) * STG_BYTES;
  const int gg = grp ^ P_SWAP;
  const unsigned* tl = p.tiles + (((gg >> 2) * XN + (gg & (XN - 1))) * NG + jg) * MAXT;
  const int bfirst = (gg >> P_XS << P_XS) * 8;
  if (P_PRIO && wave < 4) __builtin_amdgcn_s_setprio(0);
  if (wave == 4) {
    const unsigned long long t0 = __builtin_amdgcn_s_memrealtime();
    unsigned bdone = 0;
    while ((int)(__builtin_amdgcn_s_memrealtime() - tw) < P_GSTART) __builtin_amdgcn_s_sleep(8);
    for (int t = 0; t < MAXT && !P_NOGEMM; ++t) {
      const unsigned desc = tl[t];
      if (!(desc >> 31)) break;
      if ((desc >> 28) & 1) continue;
      const int mt = (desc >> 24) & 7, nt = (desc >> 27) & 1 ? 4 : 2, b = (desc >> 7) & 31;
      const unsigned bm = ((1u << nt) - 1) << b;
      const int np = (bdone & bm) == bm ? mt : mt + nt, l2 = np <= 4 ? 2 : np <= 8 ? 3 : 4;
      const int q = lane & ((1 << l2) - 1), ahead = lane >> l2;
      const unsigned fo = OFF_PF + (q < mt ? tile_a0(desc, bfirst) + q : NAP + bfirst + b + (q - mt)) * (NCH * 128);
      unsigned pub = 0, win = 16;
      while (pub < NCH) {
        const unsigned c = pub + ahead;
        const bool act = q < np && c < NCH && ahead < win;
        const unsigned v = __builtin_amdgcn_raw_buffer_load_b32(rl, act ? fo + c * 128 : NOWHERE, 0, AUX_SYS);
        const unsigned tm = __builtin_amdgcn_readfirstlane(ld_sys(myws + OFF_TMO));
        const unsigned long long m = __ballot(act ? (int)(v - call) < 0 : ahead >= win);
        if (tm == call || __builtin_amdgcn_s_memrealtime() - t0 > DEADLINE) {
          st_sys(myws + OFF_TMO, call);
          ctl->bad = 1;
          LDSWAIT();
          ctl->prog = 0xffffffffu;
          return;
        }
        const unsigned n = m ? (unsigned)__builtin_ctzll(m) >> l2 : 64u >> l2;
        win = n >= win || n >= (64u >> l2) ? 16 : n + P_PWIN;
        if (n) {
          pub = min(pub + n, (unsigned)NCH);
          ctl->prog = t * NCH + pub;
        }
        if (t > 0 && __builtin_amdgcn_readfirstlane(ctl->full[(t - 1) & 1]) < 4u * ((t - 1) >> 1) + 1u)
          __builtin_amdgcn_s_sleep(P_PSLOW);
        else if (!n) __builtin_amdgcn_s_sleep(P_PSLEEP);
      }
      bdone |= bm;
      STAMP(8 + t * 8 + 4);
      if (GS + GS1 && mt == 0) {
        const unsigned total = stream_len(desc) * NCH;
        pub = 0; win = 63;
        unsigned idle = 0;
        while (pub < total) {
          const unsigned c = pub + lane;
          const bool act = c < total && lane < win;
          const unsigned v = __builtin_amdgcn_raw_buffer_load_b32(
              rl, act ? OFF_PF + (stream_panel(desc, bfirst, c >> 4) * NCH + (c & 15)) * 128 : lane == 63 ? OFF_TMO : NOWHERE,
              0, AUX_SYS);
          const unsigned tm = __builtin_amdgcn_readlane(v, 63);
          const unsigned long long m = __ballot(act ? (int)(v - call) < 0 : lane >= win);
          if (tm == call || __builtin_amdgcn_s_memrealtime() - t0 > DEADLINE) {
            st_sys(myws + OFF_TMO, call);
            ctl->bad = 1;
            LDSWAIT();
            ctl->prog = 0xffffffffu;
            return;
          }
          const unsigned n = m ? (unsigned)__builtin_ctzll(m) : 64u;
          idle = n ? 0 : idle + 1;
          win = n >= win ? P_SWMAX : idle > 2 ? P_SWIDLE : P_SWIN;
          pub = min(pub + n, total);
          ctl->prog = (t + 1) * NCH + pub;
          if (P_SSLEEP) __builtin_amdgcn_s_sleep(P_SSLEEP);
          while (P_SAHEAD && pub < total &&
                 pub >= 4 * __builtin_amdgcn_readfirstlane(ctl->sfull[0] + ctl->sfull[1]) + P_SAHEAD)
            __builtin_amdgcn_s_sleep(4);
        }
      }
    }
    return;
  }
  unsigned have = 0;
  if (wave > 4) {
    for (int t = 0; t < MAXT && !P_NOGEMM; ++t) {
      const unsigned desc = tl[t];
      if (!(desc >> 31)) break;
      if (GS + GS1 && ((desc >> 24) & 7) == 0) { finish_stream(p, lds, bfirst, desc, wave - 5, call, lane); break; }
      if ((t & 1) != wave - 5) continue;
      BY_SHAPE(((desc >> 24) & 7) > 4 ? 11u << 24 : desc, false, finish_tile, p, lds, bfirst, desc, t, have, call, lane)
    }
    return;
  }
  for (int t = 0; t < MAXT && !P_NOGEMM; ++t) {
    const unsigned desc = tl[t];
    if (!(desc >> 31)) break;
    if ((desc >> 28) & 1) continue;
    STAMP(8 + t * 8);
    const int a0 = tile_a0(desc, bfirst), b0 = bfirst + ((desc >> 7) & 31);
    if (GS + GS1 && ((desc >> 24) & 7) == 0) { gemm_stream(p, lds, desc, bfirst, t, have, stg, wave, lane); break; }
    BY_SHAPE(desc, true, gemm_tile, p, lds, a0, b0, t, have, stg, wave, lane)
  }
  STAMP(1);
}

static const int kClass[2][4] = {{0, 2, 3, 6}, {1, 4, 5, 7}};

static bool build_params(Params& p, int rank, int* roles) {
  int cls = 0, idx = 0;
  for (int c = 0; c < 2; ++c)
    for (int i = 0; i < 4; ++i) if (kClass[c][i] == rank) { cls = c; idx = i; }
  for (int i = 0; i < 4; ++i) { roles[i] = kClass[cls][i]; roles[4 + i] = kClass[cls ^ 1][i]; }
  p.idx = idx;
  for (int a = 0; a < NAP; ++a) {
    int src, rt;
    if (a < NI) { src = rank; rt = a; }
    else if (a < 4 * NI) { rt = (a - NI) / 3; src = roles[(idx + 1 + (a - NI) % 3) & 3]; }
    else { rt = NI + (a - 4 * NI) % NX; src = roles[4 + (a - 4 * NI) / NX]; }
    p.rowbase[a] = (unsigned short)(src * SH + rt * 16);
  }
  for (int u = 0; u < NX; ++u) p.rowbase[NAP + NBP + u] = (unsigned short)(rank * SH + (NI + u) * 16);
  struct Seg { int a0, n, mt, nt, staged; double t0, t1; int src = -1; };
  std::vector<Seg> segs;
  const double per = P_PER, t00 = P_T00;
  segs.push_back({0, NI, P_OWN, 4, 1, P_OT0, P_OT0 + 1.5});
  segs.push_back({0, NX, P_OWNX, 4, 1, 12.0, 13.0, -2});
  int f = 0;
  for (int b = 0; b < NBI; ++b) {
    segs.push_back({NI + 3 * f, 3 * kBI[b][0], kBI[b][1], kBI[b][2], kBI[b][3], f ? t00 + f * per : 10.0,
                    t00 + (f + kBI[b][0]) * per - (b == NBI - 1 ? 3.2 : 0.0)});
    if (XS && b == NBI - 1) segs.back().t0 = t00 + 8 * per + 0.1;
    f += kBI[b][0];
  }
  if (f != NI || (XS && (NBX != 1 || !GS))) return false;
  for (int b = 0; b < NBX; ++b) {
    for (int src = 3; src >= 0 && !XS; --src)
      segs.push_back({4 * NI + src * NX + f - NI, kBX[b][0], kBX[b][1], kBX[b][2], kBX[b][3], t00 + (8 + f - NI) * per,
                      t00 + 1.0 + (8 + f - NI + kBX[b][0]) * per - (b == NBX - 1 && P_NRAWX ? 2.7 : 0.0), src});
    f += kBX[b][0];
  }
  if (f != 32) return false;
  auto dur = [](int mt, int nt) { return 0.6 + P_K * (mt + nt); };
  std::stable_sort(segs.begin(), segs.end(), [&](const Seg& a, const Seg& b) {
    auto key = [&](const Seg& s) { return s.mt ? std::max(s.t0, s.t1 + 1.8 - dur(s.mt, s.nt)) : s.t0 - dur(0, 2); };
    return key(a) < key(b);
  });
  for (int h = 0; h < 2; ++h) {
    std::vector<double> fr(XN * NG, 0.0);
    std::vector<int> cnt(XN * NG, 0);
    std::vector<std::pair<unsigned*, int>> direct;
    int ndirect[XN] = {}, nstaged[XN] = {};
    if (XS && GS1) {
      const size_t z = segs.size();
      if (segs[z - 2].mt || segs[z - 1].mt) return false;
      if ((segs[z - 2].a0 > segs[z - 1].a0) == (h == 1)) std::swap(segs[z - 2], segs[z - 1]);
    }
    for (auto s : segs) {
      if (!h && P_B1O && s.a0 == NI && s.src == -1) s.mt = P_B1O;
      if (!h && P_OWNO && s.a0 == 0 && s.src == -1) s.mt = P_OWNO;
      if (s.mt == 0) {
        const int sg = (GS1 && s.a0 + s.n == 4 * NI) == (h || !XS) ? 0 : XN - 1;
        for (int peer = 0; peer < 3; ++peer)
          for (int b = 0; b < 8 * XN; b += 2) {
            int best = -1; double bf = 0;
            for (int g = 0; g < XN * NG; ++g) {
              const double e = std::max(fr[g], s.t0 - dur(0, 2)) + (g / NG == sg ? 0.0 : P_SPEN) + 1e-3 * fr[g];
              if (cnt[g] < MAXT && (best < 0 || e < bf)) { best = g; bf = e; }
            }
            if (best < 0) return false;
            unsigned* e = &p.tiles[(h * XN * NG + best) * MAXT + cnt[best]];
            const bool last = s.a0 + s.n == 4 * NI;
            *e = 0x80000000u | (!h && XS && GS1 ? 1u << 23 : 0u) | (!last ? 0u : !XS ? 1u : h ? 2u : 3u) << 21 | ((unsigned)b << 7) | (unsigned)peer;
            nstaged[b >> 3] += last ? (XS ? (NX + 1) / 2 : 0) + (GS - P_ALT - P_SLAST2 + 1) / 2
                                    : (GS1 + P_ALT - P_SLAST + 1) / 2;
            direct.push_back({e, ndirect[b >> 3]++});
            cnt[best] = MAXT; fr[best] = 1e3;
          }
        continue;
      }
      if (!h && P_XO && NBX == 1 && s.src >= 0) {
        if (s.src) continue;
        s.n = 4 * NX; s.mt = P_XO;
      }
      if ((s.nt != 2 && s.nt != 4) || s.mt < 1 || s.mt > (s.nt == 4 ? 6 : 3) || (s.staged && s.nt != 4)) return false;
      if (h && s.src == -2) continue;
      const double t1 = h ? std::max(s.t1, t00 + 0.5 + 8 * per - (P_NRAWW ? 2.0 : 0.0)) : s.t1;
      const int rows = (s.n + s.mt - 1) / s.mt;
      for (int r = 0, a = s.a0; r < rows; ++r) {
        const int mt = s.n / rows + (r < s.n % rows), ntile = 8 * (XN - (h && s.src >= 0 && s.src < XN)) / s.nt;
        auto finish = [&](double free_at) { return std::max(std::max(free_at, s.t0) + dur(mt, s.nt), t1 + 1.8); };
        int home = 0; double bt = 1e9;
        for (int x = 0; x < XN; ++x) {
          std::vector<double> v(fr.begin() + x * NG, fr.begin() + (x + 1) * NG);
          std::sort(v.begin(), v.end());
          double sum = 0;
          for (int q = 0; q < NG; ++q) sum += (q < ntile ? finish(v[q]) : 0.0) + 1e-3 * v[q];
          if (GS + GS1 && x == XN - 1) sum += P_OWNHOME && s.a0 == 0 && s.src == -1 ? -0.5 : 0.5;
          if (sum < bt) { bt = sum; home = x; }
        }
        const bool st1 = s.staged != 0, st0 = st1 && r >= s.staged - 1;
        for (int b = 0; b < 8 * XN; b += s.nt) {
          if (h && s.src == b >> 3) continue;
          int best = -1; double bf = 0;
          for (int g = 0; g < XN * NG; ++g) {
            const double e = finish(fr[g]) + (g / NG == home ? 0.0 : 0.7) + 1e-3 * fr[g];
            if (cnt[g] + (mt > 4) < MAXT && (best < 0 || e < bf)) { best = g; bf = e; }
          }
          if (best < 0) return false;
          fr[best] = finish(fr[best]);
          unsigned* e = &p.tiles[(h * XN * NG + best) * MAXT + cnt[best]++];
          const unsigned common =
              0x80000000u | (s.nt == 4 ? 1u << 27 : 0u) | (s.src == -2 ? 1u << 12 : 0u) | ((unsigned)b << 7);
          *e = common | (st0 ? 1u << 29 : 0u) | ((unsigned)mt << 24) | (unsigned)a;
          if (st0) nstaged[b >> 3] += mt > 4 ? 3 : mt;
          else direct.push_back({e, ndirect[b >> 3]++});
          if (mt > 4) {
            e[1] = common | (st1 ? 1u << 29 : 0u) | 1u << 28 | ((unsigned)(mt - 3) << 24) | (unsigned)(a + 3);
            if (st1) nstaged[b >> 3] += mt - 3;
            else direct.push_back({e + 1, ndirect[b >> 3]++});
            ++cnt[best];
          }
        }
        a += mt;
      }
    }
    const int nyf = nstaged[0] / 2;
    for (int x = 0; x < XN; ++x) if (nstaged[x] != 2 * nyf || ndirect[x] != ndirect[0]) return false;
    for (auto& d : direct) *d.first |= (unsigned)(nyf + d.second) << 13;
    p.nyf = nyf;
    p.ny = nyf + ndirect[0];
    if (h && (p.ny > YMAX || nyf > NAP)) return false;
  }
  return true;
}

}

extern "C" void kernel_launch(int rank, int nranks, ncclComm_t comm, hipStream_t stream, void* const* d_in,
                              const int* in_sizes, int n_in, void* d_out, int out_size, const int* in_dtypes,
                              int out_dtype, void* const* d_out_all, void* const* d_ws_all, size_t ws_bytes) {
  using namespace agg;
  if (nranks != NR || n_in != 2 || ws_bytes < WS_USED || out_size != 4096 * SH || in_sizes[0] != SH * KD ||
      in_sizes[1] != KD * SH || (unsigned)rank >= (unsigned)NR) {
    fprintf(stderr, "ag_gemm: unexpected problem shape\n");
    return;
  }
  Params p;
  memset(&p, 0, sizeof(p));
  int roles[NR];
  if (!build_params(p, rank, roles)) { fprintf(stderr, "ag_gemm: bad schedule\n"); return; }
  p.x = (const float*)d_in[0];
  p.w = (const float*)d_in[1];
  p.myws = (char*)d_ws_all[rank];
  for (int r = 0; r < NR; ++r) { p.out[r] = (char*)d_out_all[roles[r]]; p.ws[r] = (char*)d_ws_all[roles[r]]; }
  HIP_CHECK(hipFuncSetAttribute((const void*)ag_gemm, hipFuncAttributeMaxDynamicSharedMemorySize, LDS_BYTES));
  ag_gemm<<<GRID, NTHR, LDS_BYTES, stream>>>(p);
  HIP_CHECK(hipGetLastError());
}
```

```cpp
#include <hip/hip_runtime.h>
#include <rccl/rccl.h>
#include <cstdio>
#include <cstring>
#include <type_traits>

#define HIP_CHECK(expr) do { hipError_t _e = (expr); if (_e != hipSuccess) { \
    fprintf(stderr, "HIP error %s at %s:%d\n", hipGetErrorString(_e), __FILE__, __LINE__); \
    return; } } while (0)

#define P_ND 66
#define P_NDRAW 16
#define P_PACE0 80
#define P_PACE 25
#define P_LEAD 330
#define P_HOLD 40
#define P_CSTEP 250
#define P_GSTART 950
#define P_FAST 16
#define P_FNAP 3
#define P_WIN 11
#define P_WIDE 16
#define P_LATE 105
#define P_RATE 503
#define P_NARROW 4
#define P_NUI 3
#define P_SI 8
#define P_NRAW 48
#define P_NRAWY 22
#define P_ROT1 15
#define P_ROT2 5
#define P_UGAP 140
#define P_LASTA 3
#define P_LASTB 1

namespace agg {

#define GAS __attribute__((address_space(1)))
#define LAS __attribute__((address_space(3)))
typedef unsigned v4u __attribute__((ext_vector_type(4)));
typedef unsigned v2u __attribute__((ext_vector_type(2)));
typedef unsigned short us2 __attribute__((ext_vector_type(2)));
typedef float f32x4 __attribute__((ext_vector_type(4)));
typedef float f32x2 __attribute__((ext_vector_type(2)));
typedef _Float16 h8 __attribute__((ext_vector_type(8)));
typedef GAS unsigned gu32;
typedef __amdgpu_buffer_rsrc_t rsrc_t;

constexpr int NR = 8;
constexpr int KD = 4096, SH = 512;
constexpr int PANEL = 131072;
constexpr int NCH = 16, CHUNK = 8192;
constexpr int NI = 23, NX = 32 - NI;
constexpr int NB1 = 8;
constexpr int NAP = 128, NBP = 64, PSENT = NAP + NBP, NPAN = PSENT + 16;
constexpr int GRID = 256, NWAVE = 7, NTHR = NWAVE * 64;
constexpr int NC = 4, NL = NWAVE - 1, WL = NC * NL;
constexpr int NU = 4;
constexpr int SX = ((NC + NU) * NWAVE - 3 * P_SI) / 4;
constexpr int WUI = P_NUI * NWAVE + P_SI, WUX = NU * NWAVE + SX;
constexpr int NG = 32 - NC - NU;
constexpr int ND = P_ND, NDRAW = P_NDRAW;
constexpr int NIN = NI * NCH, NXY = 128 + NX * NCH;
constexpr int NPI = NIN - P_NRAW;
constexpr int NCV = 5 * (128 - ND) + 384;
constexpr int LEN_A = NI - NB1 - 2, LEN_B = NX + 2;
constexpr int NHALF = 2 * (NI + 3 * NB1) + 12 * ((LEN_A - P_LASTA + 1) / 2) + 12 * ((NX + 1) / 2 + (3 - P_LASTB) / 2);
constexpr int NYF = NHALF / 2, NYD = 24;
static_assert(NG == 24 && NI == 23 && NHALF % 2 == 0 && NYF <= NAP, "the deal of tiles and streams");
static_assert(NG * 32 >= NCV && ND <= 3 * NG && NDRAW <= NG && NDRAW <= ND, "one item per converting wave");
static_assert(NIN <= 63 * WL && NXY + NYF <= 63 * WL && NIN <= 64 * WUI && SX >= 0, "one lane per item");
static_assert(NXY <= NPI && P_LASTA >= 1 && P_LASTA <= 4 && P_LASTB >= 1 && P_LASTB <= 2, "");

constexpr unsigned OFF_CNT = 0;
constexpr unsigned OFF_TMO = 16384;
constexpr unsigned OFF_YQ = 20480;
constexpr unsigned OFF_BAR = 32768;
constexpr unsigned OFF_YF = 65536;
constexpr unsigned OFF_YL = 196608;
constexpr unsigned OFF_PF = 393216;
constexpr unsigned PF_BYTES = NPAN * NCH * 128;
constexpr unsigned OFF_AF = 1u << 20;
constexpr unsigned OFF_YS = 2u << 20;
constexpr unsigned OFF_YP = OFF_YS + (4u << 20);
constexpr unsigned OFF_EXP = OFF_YP + (4u << 20);
constexpr unsigned OFF_STG = OFF_EXP + (8u << 20);
constexpr unsigned STG_BYTES = (unsigned)NPAN * PANEL;
constexpr unsigned OFF_DBG = OFF_STG + 2 * STG_BYTES;
constexpr unsigned WS_USED = OFF_DBG + (1u << 20);
static_assert(OFF_YF + 1024 * 128 <= OFF_YL && OFF_YL + 1024 * 128 <= OFF_PF && OFF_PF + PF_BYTES <= OFF_AF &&
              OFF_AF + 2 * PF_BYTES <= OFF_YS, "layout");
constexpr unsigned OUT_BYTES = 4096u * SH * 4u;
constexpr unsigned NOWHERE = 0xf0000000u;
constexpr unsigned long long DEADLINE = 50000000ull;
constexpr int AUX_SYS = 17;
constexpr unsigned QNAN = 0x7fc00000u;

constexpr int LDS_RED = 65536;
constexpr int LDS_CTL = 2 * LDS_RED;
constexpr int LDS_SIMG = LDS_CTL + 64;
constexpr int LDS_BYTES = LDS_SIMG + 16384;

struct Params {
  const float* x;
  const float* w;
  char* myws;
  int idx;
  unsigned ranks;
  char* out[NR];
  char* ws[NR];
};

#define DRAIN() asm volatile("s_waitcnt vmcnt(0)" ::: "memory")
#define LDSWAIT() asm volatile("s_waitcnt lgkmcnt(0)" ::: "memory")
#define CBAR() asm volatile("" ::: "memory")
#define NOW() __builtin_amdgcn_s_memrealtime()
#define UNI(v) ((unsigned)__builtin_amdgcn_readfirstlane(v))
#define LANE(v, l) ((unsigned)__builtin_amdgcn_readlane(v, l))

__device__ __forceinline__ rsrc_t mk(const void* p, unsigned bytes) {
  return __builtin_amdgcn_make_buffer_rsrc((void*)p, 0, (int)bytes, 0x00020000);
}
__device__ __forceinline__ unsigned long long uniform64(const void* q) {
  const unsigned long long v = (unsigned long long)q;
  return (unsigned long long)UNI((unsigned)(v >> 32)) << 32 | UNI((unsigned)v);
}
__device__ __forceinline__ unsigned pk(float a, float b) {
  typedef _Float16 h2 __attribute__((ext_vector_type(2)));
  h2 v = {(_Float16)a, (_Float16)b};
  return __builtin_bit_cast(unsigned, v);
}
__device__ __forceinline__ void st_sys(char* p, unsigned v) {
  __hip_atomic_store((gu32*)p, v, __ATOMIC_RELAXED, __HIP_MEMORY_SCOPE_SYSTEM);
}
__device__ __forceinline__ void st16(rsrc_t r, unsigned voff, v4u v) {
  __builtin_amdgcn_raw_buffer_store_b128(v, r, (int)voff, 0, AUX_SYS);
}
__device__ __forceinline__ v4u ld16(rsrc_t r, unsigned voff, unsigned soff) {
  return __builtin_amdgcn_raw_buffer_load_b128(r, (int)voff, (int)soff, AUX_SYS);
}
__device__ __forceinline__ unsigned ld4(rsrc_t r, unsigned voff, unsigned soff = 0) {
  return __builtin_amdgcn_raw_buffer_load_b32(r, (int)voff, (int)soff, AUX_SYS);
}
__device__ __forceinline__ void flag(bool on, rsrc_t r, unsigned off, unsigned call, unsigned hdr = 0) {
  __builtin_amdgcn_raw_buffer_store_b64(v2u{call, hdr}, r, (int)(on ? off : NOWHERE), 0, AUX_SYS);
}
__device__ __forceinline__ v2u ldflag(rsrc_t r, unsigned off) {
  return __builtin_amdgcn_raw_buffer_load_b64(r, (int)off, 0, AUX_SYS);
}
__device__ __forceinline__ void store_item(rsrc_t r, unsigned o0, unsigned o1, const v4u (&h)[8], int lane, unsigned nl = 64) {
#pragma unroll
  for (int s = 0; s < 4; ++s) st16(r, o0 + lane * 16 + s * 1024, h[s]);
#pragma unroll
  for (int s = 0; s < 3; ++s) st16(r, o1 + lane * 16 + s * 1024, h[4 + s]);
  st16(r, lane < nl ? o1 + lane * 16 + 3072 : NOWHERE, h[7]);
}
template <int S> __device__ __forceinline__ void sleep_until(unsigned long long t0, int ticks) {
  while ((int)(NOW() - t0) < ticks) __builtin_amdgcn_s_sleep(S);
}

#define STAMP(slot) do {} while (0)
#define TREC_DECL() do {} while (0)
#define TREC(i) do {} while (0)
#define TREC_DUMP(from) do {} while (0)
#define DIAG_DRAIN() do {} while (0)

constexpr int LIST_CAP = 128;
__device__ __forceinline__ unsigned bfi(unsigned m, unsigned a, unsigned b) { return (a & m) | (b & ~m); }
template <bool F32> __device__ __forceinline__ unsigned add_base(unsigned v, unsigned b) {
  return F32 ? v + b : __builtin_bit_cast(unsigned, (us2)(__builtin_bit_cast(us2, v) + __builtin_bit_cast(us2, b)));
}
template <bool F32> __device__ __forceinline__ void pack8(v4u (&h)[8], unsigned base) {
  constexpr unsigned TOP = F32 ? 0xF0000000u : 0xC000C000u;
  constexpr int B = F32 ? 4 : 2;
  const unsigned nb = F32 ? (0u - base) << 24 : ((32u - base) & 31u) * 0x08000800u;
#pragma unroll
  for (int d = 0; d < 4; ++d) {
    unsigned t[8], pl = 0;
#pragma unroll
    for (int s = 0; s < 8; ++s) {
      t[s] = add_base<F32>(__builtin_amdgcn_alignbit(h[s][d], h[s][d], 31), nb);
      pl |= (t[s] & TOP) >> (B * s);
    }
#pragma unroll
    for (int s = 0; s < 7; ++s) h[s][d] = bfi(TOP, t[7] << (B * (7 - s)), t[s]);
    h[7][d] = pl;
  }
}
template <bool F32> __device__ __forceinline__ void unpack8(v4u (&h)[8], unsigned base) {
  constexpr unsigned TOP = F32 ? 0xF0000000u : 0xC000C000u;
  constexpr int B = F32 ? 4 : 2;
  const unsigned pb = F32 ? base << 24 : base * 0x08000800u;
#pragma unroll
  for (int d = 0; d < 4; ++d) {
    const unsigned pl = h[7][d];
    unsigned t7 = (pl << (B * 7)) & TOP;
#pragma unroll
    for (int s = 0; s < 7; ++s) {
      t7 |= (h[s][d] & TOP) >> (B * (7 - s));
      const unsigned t = add_base<F32>(bfi(TOP, pl << (B * s), h[s][d]), pb);
      h[s][d] = __builtin_amdgcn_alignbit(t, t, 1);
    }
    t7 = add_base<F32>(t7, pb);
    h[7][d] = __builtin_amdgcn_alignbit(t7, t7, 1);
  }
}
template <int CTRL> __device__ __forceinline__ unsigned dpp0(unsigned v) {
  return (unsigned)__builtin_amdgcn_update_dpp(0, (int)v, CTRL, 0xf, 0xf, true);
}
__device__ __forceinline__ int top_exponent(float ss, float scale) {
  ss += __builtin_bit_cast(float, dpp0<0x111>(__builtin_bit_cast(unsigned, ss)));
  ss += __builtin_bit_cast(float, dpp0<0x112>(__builtin_bit_cast(unsigned, ss)));
  ss += __builtin_bit_cast(float, dpp0<0x114>(__builtin_bit_cast(unsigned, ss)));
  ss += __builtin_bit_cast(float, dpp0<0x118>(__builtin_bit_cast(unsigned, ss)));
  const unsigned sb = __builtin_bit_cast(unsigned, ss);
  const float tot = (__builtin_bit_cast(float, LANE(sb, 15)) + __builtin_bit_cast(float, LANE(sb, 31))) +
                    (__builtin_bit_cast(float, LANE(sb, 47)) + __builtin_bit_cast(float, LANE(sb, 63)));
  return (int)((UNI(__builtin_bit_cast(unsigned, __builtin_amdgcn_sqrtf(tot * scale))) >> 23) & 255);
}
__device__ __forceinline__ float sumsq(f32x4 z) { return z.x * z.x + z.y * z.y + z.z * z.z + z.w * z.w; }
__device__ __forceinline__ unsigned list_plane(const v4u& pl, LAS unsigned short* list, unsigned cap, int lane) {
  unsigned nz[4], cnt = 0;
#pragma unroll
  for (int d = 0; d < 4; ++d) {
    unsigned z = pl[d];
    z |= z >> 1; z |= z >> 2;
    nz[d] = z & 0x11111111u;
    cnt += __builtin_popcount(nz[d]);
  }
  unsigned inc = cnt;
  inc += dpp0<0x111>(inc); inc += dpp0<0x112>(inc); inc += dpp0<0x114>(inc); inc += dpp0<0x118>(inc);
  const unsigned r0 = LANE(inc, 15), r1 = r0 + LANE(inc, 31), r2 = r1 + LANE(inc, 47), total = r2 + LANE(inc, 63);
  if (total <= cap) {
    unsigned pos = inc - cnt + (lane < 16 ? 0 : lane < 32 ? r0 : lane < 48 ? r1 : r2);
#pragma unroll
    for (int d = 0; d < 4; ++d) {
      unsigned m = nz[d];
      while (m) {
        const unsigned b = __builtin_ctz(m);
        list[pos++] = (unsigned short)(((lane * 4 + d) << 7) | (b << 2) | ((pl[d] >> b) & 15));
        m &= m - 1;
      }
    }
  }
  return total;
}
__device__ __forceinline__ v4u unlist_plane(unsigned e2, unsigned skip, unsigned count, LAS unsigned* plane, int lane) {
  LDSWAIT();
#pragma unroll
  for (int k = 0; k < 2; ++k) {
    const unsigned e = (e2 >> (16 * k)) & 0xffffu;
    __hip_atomic_fetch_or(plane + ((e >> 7) & 255), 2u * lane + k - skip < count ? (e & 15) << ((e >> 2) & 28) : 0u,
                          __ATOMIC_RELAXED, __HIP_MEMORY_SCOPE_WORKGROUP);
  }
  LDSWAIT();
  return *(volatile LAS v4u*)(plane + lane * 4);
}

__device__ __forceinline__ void item_offsets(bool isw, unsigned stg, unsigned pan, int c, unsigned& o0, unsigned& o1,
                                             unsigned& fl) {
  o0 = stg + pan * PANEL + c * (isw ? 4096 : CHUNK);
  o1 = o0 + (isw ? PANEL : 4096);
  fl = OFF_PF + (isw ? (pan + (c & 1)) * NCH + (c >> 1) : pan * NCH + c) * 128;
}
__device__ __forceinline__ void link_item(bool cross, int i, bool& isw, int& u, int& c) {
  static_assert(NB1 == 8, "as many items as w");
  isw = cross && i < 128;
  u = i >= 128 ? (i >> 4) - (cross ? 8 : 0) : cross ? i & 3 : i & 7;
  c = i >= 128 ? i & 15 : cross ? i >> 2 : i >> 3;
}
__device__ __forceinline__ unsigned landing_panel(bool cross, bool isw, int u, int from, int to) {
  return !cross ? NI + 3 * u + ((from - to - 1) & 3) : isw ? NAP + 32 + from * 8 + 2 * u : 4 * NI + from * NX + u;
}
__device__ __forceinline__ unsigned panel_row(const Params& p, int a) {
  const int idx = p.idx & 3, q = a - NI, z = a - 4 * NI;
  const int role = a < NI || a >= PSENT ? idx : z < 0 ? (idx + 1 + q % 3) & 3 : 4 + z / NX;
  const int g = a < NI ? a : a >= PSENT ? NI + a - PSENT : z < 0 ? q / 3 : NI + z % NX;
  return ((p.ranks >> (4 * role)) & 7) * SH + g * 16;
}
__device__ __forceinline__ void load_inputs(const Params& p, bool isw, int g, int c, int lane, f32x4 (&v)[16]) {
  if (isw) {
    const GAS f32x2* src =
        (const GAS f32x2*)(p.w + (size_t)((c & 31) * 128 + 4 * (lane >> 4)) * SH + (g & 15) * 32 + 2 * (lane & 15));
#pragma unroll
    for (int q = 0; q < 16; ++q) {
      const f32x2 a = __builtin_nontemporal_load(src + ((q >> 1) * 16 + (q & 1) * 2) * (SH / 2));
      const f32x2 b = __builtin_nontemporal_load(src + ((q >> 1) * 16 + (q & 1) * 2 + 1) * (SH / 2));
      v[q] = f32x4{a.x, b.x, a.y, b.y};
    }
  } else {
    const GAS f32x4* src =
        (const GAS f32x4*)(p.x + (size_t)((g & 31) * 16 + (lane >> 2)) * KD + (c & 15) * 256 + 4 * (lane & 3));
#pragma unroll
    for (int q = 0; q < 16; ++q) v[q] = src[q * 4];
  }
}
__device__ __forceinline__ void to_fragments(bool isw, const f32x4 (&v)[16], LAS v4u* scr, int lane, v4u (&h)[8]) {
  if (isw) {
#pragma unroll
    for (int s = 0; s < 4; ++s) {
      h[s] = v4u{pk(v[4 * s].x, v[4 * s].y), pk(v[4 * s + 1].x, v[4 * s + 1].y), pk(v[4 * s + 2].x, v[4 * s + 2].y),
                 pk(v[4 * s + 3].x, v[4 * s + 3].y)};
      h[4 + s] = v4u{pk(v[4 * s].z, v[4 * s].w), pk(v[4 * s + 1].z, v[4 * s + 1].w), pk(v[4 * s + 2].z, v[4 * s + 2].w),
                     pk(v[4 * s + 3].z, v[4 * s + 3].w)};
    }
  } else {
#pragma unroll
    for (int s = 0; s < 8; ++s)
      scr[s * 64 + lane] = v4u{pk(v[2 * s].x, v[2 * s].y), pk(v[2 * s].z, v[2 * s].w), pk(v[2 * s + 1].x, v[2 * s + 1].y),
                               pk(v[2 * s + 1].z, v[2 * s + 1].w)};
    LDSWAIT();
#pragma unroll
    for (int s = 0; s < 8; ++s) h[s] = scr[s * 64 + (lane & 15) * 4 + (lane >> 4)];
    LDSWAIT();
  }
}
__device__ __forceinline__ unsigned half_offset(unsigned e, int q, int lane) {
  const unsigned o = e & 512 ? (q * 8 + (lane >> 3)) * (SH * 4) + (lane & 7) * 16
                             : (q * 4 + (lane >> 4)) * (SH * 4) + (lane & 15) * 16;
  return (e & 1024) && q >= 2 ? NOWHERE : (e >> 16 << 7) + o;
}
__device__ __forceinline__ unsigned queue_places(rsrc_t rl, bool on, int owner, int n) {
  return (unsigned)__builtin_amdgcn_raw_ptr_buffer_atomic_add_i32(n, rl, on ? OFF_YQ + owner * 128 : NOWHERE, 0, 16);
}

struct Ctl { unsigned prog, bad, full[2], fre[2], conv, sfull[2], sfre[2]; };
struct Job {
  int a0, mt;
  int b0, bs;
  int kind, j;
};
__device__ __forceinline__ Job make_job(int gg, int jg) {
  const int r = jg >> 2;
  Job q;
  q.b0 = (gg & 6) * 8 + (jg & 3) * 4; q.bs = q.b0 + (r & 1) * 2; q.j = r >> 1;
  if (gg & 1) { q.a0 = 4 * r; q.mt = r == 5 ? 3 : 4; q.kind = gg & 4 ? 0 : 3; }
  else if (gg & 4) { q.a0 = NI + 4 * r; q.mt = 4; q.kind = 2; }
  else { q.a0 = r < 4 ? NI + 6 * r : PSENT + 5 * (r - 4); q.mt = r < 4 ? 6 : 9 - r; q.kind = 1; }
  return q;
}
__device__ __forceinline__ int stream_len(const Job& q) {
  return q.kind == 0 ? LEN_A : q.kind == 1 ? LEN_A + 1 : q.kind == 2 ? LEN_B : 4 * NX / 3 + 1;
}
__device__ __forceinline__ int stream_panel(const Job& q, int i) {
  const int z = 3 * i + q.j;
  if (i == stream_len(q) - 1) return NI + 3 * (NI - 2 + (q.kind >> 1)) + q.j;
  if (q.kind < 2) return NI + 3 * NB1 + z;
  if (q.kind == 3) return 4 * NI + (z & 3) * NX + (z >> 2);
  return i < NX ? 4 * NI + (((q.bs >> 3) + 1 + q.j) & 3) * NX + i : NI + 3 * (NI - 3) + q.j;
}

__device__ __forceinline__ void need(volatile LAS unsigned* word, unsigned& have, unsigned n) {
  while (have < n) {
    have = UNI(*word);
    if (have < n) __builtin_amdgcn_s_sleep(1);
  }
  CBAR();
}
__device__ __forceinline__ void need(volatile LAS unsigned* word, unsigned n) { unsigned h = 0; need(word, h, n); }
__device__ __forceinline__ void arrive(volatile LAS unsigned* word, int lane) {
  LDSWAIT();
  if (lane == 0) __hip_atomic_fetch_add((LAS unsigned*)word, 1u, __ATOMIC_RELAXED, __HIP_MEMORY_SCOPE_WORKGROUP);
}
__device__ __forceinline__ f32x4 mfma(v4u b, v4u a, f32x4 c) {
  return __builtin_amdgcn_mfma_f32_16x16x32_f16(__builtin_bit_cast(h8, b), __builtin_bit_cast(h8, a), c, 0, 0, 0);
}

template <int MT>
__device__ __forceinline__ void gemm_tile(rsrc_t rs, LAS char* lds, const Job& job, unsigned& have, unsigned stg,
                                          int wave, int lane) {
  volatile LAS Ctl* ctl = (volatile LAS Ctl*)(lds + LDS_CTL);
  const unsigned ab = stg + job.a0 * PANEL + wave * 1024, bb = stg + (NAP + job.b0) * PANEL + wave * 1024;
  const int voff = lane * 16;
  f32x4 acc[MT][4];
#pragma unroll
  for (int m = 0; m < MT; ++m)
#pragma unroll
    for (int n = 0; n < 4; ++n) acc[m][n] = f32x4{0.f, 0.f, 0.f, 0.f};
  v4u ra[2][MT], rb[2][4];
#define LOADSTEP(d, i) do { \
    _Pragma("unroll") for (int m = 0; m < MT; ++m) ra[d][m] = ld16(rs, voff, ab + m * PANEL + (i) * 4096); \
    _Pragma("unroll") for (int n = 0; n < 4; ++n) rb[d][n] = ld16(rs, voff, bb + n * PANEL + (i) * 4096); \
    __builtin_amdgcn_sched_barrier(0); } while (0)
#define MMASTEP(d) do { \
    _Pragma("unroll") for (int m = 0; m < MT; ++m) \
    _Pragma("unroll") for (int n = 0; n < 4; ++n) acc[m][n] = mfma(rb[d][n], ra[d][m], acc[m][n]); \
    __builtin_amdgcn_sched_barrier(0); } while (0)
  need(&ctl->prog, have, 1);
  LOADSTEP(0, 0);
  LOADSTEP(1, 1);
#pragma unroll 1
  for (int c = 1; c < NCH; ++c) {
    MMASTEP(0);
    need(&ctl->prog, have, c + 1);
    LOADSTEP(0, 2 * c);
    MMASTEP(1);
    LOADSTEP(1, 2 * c + 1);
  }
  MMASTEP(0);
  MMASTEP(1);
#undef LOADSTEP
#undef MMASTEP
  asm volatile("" : "+v"(lane));
  need(&ctl->conv, NWAVE);
  constexpr int M1 = (MT + 1) / 2;
#pragma unroll
  for (int part = 0; part < 2; ++part) {
    const int m0 = part ? M1 : 0, mn = part ? MT - M1 : M1;
    LAS f32x4* red = (LAS f32x4*)(lds + part * LDS_RED) + wave * 1024;
#pragma unroll
    for (int m = 0; m < mn; ++m)
#pragma unroll
      for (int n = 0; n < 4; n += 2) {
        const int r = lane & 15, c4 = n * 4 + 2 * (lane >> 4);
        const f32x4 e = acc[m0 + m][n], o = acc[m0 + m][n + 1];
        red[(m * 16 + r) * 16 + (c4 ^ r)] = f32x4{e.x, o.x, e.y, o.y};
        red[(m * 16 + r) * 16 + ((c4 + 1) ^ r)] = f32x4{e.z, o.z, e.w, o.w};
      }
    arrive(&ctl->full[part], lane);
  }
}

__device__ __forceinline__ void finish_tile(const Params& p, rsrc_t rl, LAS char* lds, const Job& job, int part,
                                            unsigned call, int lane) {
  volatile LAS Ctl* ctl = (volatile LAS Ctl*)(lds + LDS_CTL);
  const LAS f32x4* red = (const LAS f32x4*)(lds + part * LDS_RED);
  const int idx = p.idx & 3, grp = job.b0 >> 3, m1 = (job.mt + 1) / 2;
  const int a0 = job.a0 + (part ? m1 : 0), mn = part ? job.mt - m1 : m1;
  const bool staged = grp >= 4;
  const rsrc_t ro = staged ? rl : mk(p.out[idx], OUT_BYTES);
  const unsigned ent = ((a0 + lane) * 4 + ((job.b0 >> 1) & 2)) % 512 |
                       (panel_row(p, a0 + min(lane, 2)) * SH + (staged ? idx : grp) * 128 + (job.b0 & 7) * 16) * 4 >> 7 << 16;
  need(&ctl->full[part], 4);
  STAMP(10);
  const bool bad = ctl->bad != 0;
  const unsigned ticket = queue_places(rl, staged && lane == 0, grp & 3, mn);
  for (int m = 0; m < mn; ++m) {
    const unsigned e = LANE(ent, m);
#pragma unroll
    for (int q = 0; q < 4; ++q) {
      const int r = q * 4 + (lane >> 4), li = (m * 16 + r) * 16 + ((lane & 15) ^ r);
      f32x4 s = (red[li] + red[1024 + li]) + (red[2048 + li] + red[3072 + li]);
      if (bad) s = __builtin_bit_cast(f32x4, v4u{QNAN, QNAN, QNAN, QNAN});
      st16(ro, staged ? OFF_YS + ((grp & 3) * 512 + (e & 511)) * 2048 + q * 1024 + lane * 16 : half_offset(e, q, lane),
           __builtin_bit_cast(v4u, s));
    }
  }
  LDSWAIT();
  ctl->fre[part] = 1;
  if (staged) {
    DRAIN();
    const unsigned pos = UNI(ticket) - (call - 1) * NHALF + lane;
    flag(lane < mn && pos < 256, rl, OFF_YL + ((grp & 3) * 256 + pos) * 128, call, ent);
  }
  STAMP(11);
}

__device__ __forceinline__ void gemm_stream(rsrc_t rs, LAS char* lds, const Job& job, unsigned& have, unsigned stg,
                                            int wave, int lane) {
  constexpr int D = 16;
  volatile LAS Ctl* ctl = (volatile LAS Ctl*)(lds + LDS_CTL);
  const int G = stream_len(job);
  const unsigned bb = stg + (NAP + job.bs) * PANEL + wave * 1024, voff = lane * 16;
  unsigned ab = stg + stream_panel(job, 0) * PANEL + wave * 1024, base = NCH;
  LAS v4u* bl = (LAS v4u*)(lds + wave * 32768) + lane;
  v4u br[32], ra[D];
#pragma unroll
  for (int s = 0; s < 32; ++s) {
    br[s] = ld16(rs, voff, bb + s * 4096);
    if (s >= 16) ra[s - 16] = ld16(rs, voff, bb + PANEL + (s - 16) * 4096);
    __builtin_amdgcn_sched_barrier(0);
  }
  need(&ctl->fre[0], 1);
  need(&ctl->fre[1], 1);
#pragma unroll
  for (int d = 0; d < D; ++d) {
    bl[d * 64] = ra[d];
    ra[d] = ld16(rs, voff, bb + PANEL + (16 + d) * 4096);
    __builtin_amdgcn_sched_barrier(0);
  }
#pragma unroll
  for (int d = 0; d < D; ++d) {
    bl[(16 + d) * 64] = ra[d];
    if ((d & 1) == 0) need(&ctl->prog, have, base + (d >> 1) + 1);
    ra[d] = ld16(rs, voff, ab + d * 4096);
    __builtin_amdgcn_sched_barrier(0);
  }
  bool primed = true;
#pragma unroll 1
  for (int i = 0; i < G; ++i) {
    if (!primed) {
#pragma unroll
      for (int d = 0; d < D; ++d) {
        if ((d & 1) == 0) need(&ctl->prog, have, base + (d >> 1) + 1);
        ra[d] = ld16(rs, voff, ab + d * 4096);
        __builtin_amdgcn_sched_barrier(0);
      }
    }
    const bool more = i + 1 < G;
    const unsigned an = stg + stream_panel(job, i + more) * PANEL + wave * 1024;
    unsigned vnext = NOWHERE;
    f32x4 e = {0.f, 0.f, 0.f, 0.f}, o = {0.f, 0.f, 0.f, 0.f};
    v4u q[4];
    q[0] = bl[0]; q[1] = bl[64]; q[2] = bl[128];
#pragma unroll
    for (int s = 0; s < 32; ++s) {
      if (s + 3 < 32) q[(s + 3) & 3] = bl[(s + 3) * 64];
      e = mfma(br[s], ra[s % D], e);
      o = mfma(q[s & 3], ra[s % D], o);
      __builtin_amdgcn_sched_barrier(0);
      const int n = s + D;
      if (n < 32 && (n & 1) == 0) need(&ctl->prog, have, base + (n >> 1) + 1);
      if (n == 32) {
        if (have < base + NCH + D / 2) have = UNI(ctl->prog);
        CBAR();
        primed = more && have >= base + NCH + D / 2;
        if (primed) vnext = voff;
      }
      ra[s % D] = ld16(rs, n < 32 ? voff : vnext, (n < 32 ? ab : an) + (n & 31) * 4096);
      __builtin_amdgcn_sched_barrier(0);
    }
    need(&ctl->sfre[i & 1], (unsigned)i >> 1);
    LAS f32x4* red = (LAS f32x4*)(lds + LDS_SIMG + (i & 1) * 8192) + wave * 128;
    const int row = lane & 15, c4 = 2 * (lane >> 4);
    red[row * 8 + (c4 ^ (row & 7))] = f32x4{e.x, o.x, e.y, o.y};
    red[row * 8 + ((c4 + 1) ^ (row & 7))] = f32x4{e.z, o.z, e.w, o.w};
    arrive(&ctl->sfull[i & 1], lane);
    ab = an;
    base += NCH;
  }
}

__device__ __forceinline__ void take_panel(LAS char* lds, int i, int lane, f32x4& s0, f32x4& s1) {
  volatile LAS Ctl* ctl = (volatile LAS Ctl*)(lds + LDS_CTL);
  const LAS f32x4* red = (const LAS f32x4*)(lds + LDS_SIMG + (i & 1) * 8192) + ((lane & 56) + ((lane ^ (lane >> 3)) & 7));
  need(&ctl->sfull[i & 1], 4u * ((i >> 1) + 1));
  s0 = (red[0] + red[128]) + (red[256] + red[384]);
  s1 = (red[64] + red[192]) + (red[320] + red[448]);
  if (ctl->bad) s0 = s1 = __builtin_bit_cast(f32x4, v4u{QNAN, QNAN, QNAN, QNAN});
  asm volatile("" : "+v"(s0), "+v"(s1));
  ctl->sfre[i & 1] = (i >> 1) + 1;
}
__device__ __forceinline__ void finish_stream(const Params& p, rsrc_t rl, LAS char* lds, const Job& job, int par,
                                              unsigned call, int lane) {
  const int idx = p.idx & 3, grp = job.bs >> 3, b = job.bs & 7, G = stream_len(job);
  const bool staged = grp >= 4;
  const int ns = !staged ? G : G - (job.kind ? P_LASTB : P_LASTA), nx = job.kind == 2 ? NX : 0;
  const unsigned col = ((staged ? idx : grp) * 128 + b * 16) * 4;
  const rsrc_t ro = staged ? rl : mk(p.out[idx], OUT_BYTES);
  TREC_DECL();
  int unit = 0;
  for (int i = 0; i < ns; ++unit) {
    const int np = staged ? min(2, (i < nx ? nx : ns) - i) : 1;
    if ((unit & 1) == par) {
      const int a = stream_panel(job, i), a2 = stream_panel(job, i + np - 1);
      const unsigned slot = np == 2 ? (b & 4 ? a2 : a) * 4 + (b & 2) : a * 4 + (b >> 1);
      const unsigned ent = slot % 512 | (np == 2 ? 512 : 1536) | (panel_row(p, a) * SH * 4 + col) >> 7 << 16;
      const unsigned vo = staged ? OFF_YS + ((grp & 3) * 512 + slot) * 2048 + lane * 16 : half_offset(ent, 0, lane);
      const unsigned qs = staged ? 1024 : 8 * SH * 4;
      unsigned ticket = 0;
      for (int e = 0; e < np; ++e) {
        f32x4 s0, s1;
        take_panel(lds, i + e, lane, s0, s1);
        TREC(32 + i + e);
        ticket = queue_places(rl, staged && e == np - 1 && lane == 0, grp & 3, 1);
        st16(ro, vo + 2 * e * qs, __builtin_bit_cast(v4u, s0));
        st16(ro, vo + (2 * e + 1) * qs, __builtin_bit_cast(v4u, s1));
      }
      if (staged) {
        DRAIN();
        const unsigned pos = UNI(ticket) - (call - 1) * NHALF;
        flag(lane == 0 && pos < 256, rl, OFF_YL + ((grp & 3) * 256 + pos) * 128, call, ent);
      }
    }
    i += np;
  }
  if (staged && (unit & 1) == par) {
    constexpr int NE = P_LASTA > P_LASTB ? P_LASTA : P_LASTB;
    const rsrc_t rp = mk(p.out[grp], OUT_BYTES), rf = mk(p.ws[grp], WS_USED);
    f32x4 s[2 * NE];
    unsigned vo[NE];
#pragma unroll
    for (int e = 0; e < NE; ++e) {
      const int i = G - NE + e;
      vo[e] = i >= ns ? (panel_row(p, stream_panel(job, i)) + (lane >> 3)) * SH * 4 + col + (lane & 7) * 16 : NOWHERE;
      asm volatile("" : "+v"(vo[e]));
    }
#pragma unroll
    for (int e = 0; e < NE; ++e) {
      s[2 * e] = s[2 * e + 1] = f32x4{0.f, 0.f, 0.f, 0.f};
      if (G - NE + e < ns) continue;
      take_panel(lds, G - NE + e, lane, s[2 * e], s[2 * e + 1]);
      TREC(32 + G - NE + e);
    }
#pragma unroll
    for (int q = 0; q < 2 * NE; ++q) st16(rp, vo[q >> 1] + (q & 1) * (8 * SH * 4), __builtin_bit_cast(v4u, s[q]));
    DRAIN();
    flag(lane == 0, rf, OFF_YF + (idx * 256 + NYF + (job.kind >> 1) * 12 + job.j * 4 + (b >> 1)) * 128, call);
  }
  DIAG_DRAIN();
  STAMP(1);
  TREC(55);
  TREC_DUMP(40);
}

__global__ __launch_bounds__(NTHR, 2) void ag_gemm(const Params p) {
  extern __shared__ __attribute__((aligned(16))) char smem[];
  LAS char* lds = (LAS char*)smem;
  const int lane = threadIdx.x & 63;
  const int wave = UNI(threadIdx.x >> 6);
  const int blk = blockIdx.x, grp = blk & 7, j = blk >> 3;
  asm volatile("" :: "s"(p.x), "s"(p.w), "s"(p.myws), "s"(p.idx), "s"(p.ws[grp]));
  const int idx = p.idx & 3;
  const bool cross = grp >= 4;
  char* myws = p.myws;
  const rsrc_t rl = mk(myws, WS_USED), rd = mk(p.ws[grp], WS_USED);
  STAMP(0);
  char* counter = myws + OFF_CNT + (blk * 8 + wave) * 4;
  const unsigned last_call = __hip_atomic_load((gu32*)counter, __ATOMIC_RELAXED, __HIP_MEMORY_SCOPE_SYSTEM);
  const unsigned long long tw = NOW();
  const int jg = j - NC - NU;
  const bool keeper = cross || grp == idx;
  if (wave == 6 && (unsigned)jg < (unsigned)NDRAW && grp != idx) {
    const int u = cross ? jg & 3 : jg & 7, c = cross ? jg >> 2 : jg >> 3;
    f32x4 v[16];
    v4u h[8];
    if (cross) {
      load_inputs(p, true, (grp - 4) * 4 + u, c, lane, v);
      __builtin_amdgcn_s_barrier();
      to_fragments(true, v, (LAS v4u*)(lds + 6 * 8192), lane, h);
    } else {
      load_inputs(p, false, u, c, lane, v);
      __builtin_amdgcn_s_barrier();
      to_fragments(false, v, (LAS v4u*)(lds + 6 * 8192), lane, h);
    }
    const unsigned call = UNI(last_call) + 1u, stg = OFF_STG + (call & 1u) * STG_BYTES;
    unsigned o0, o1, fl, kl;
    item_offsets(cross, stg, landing_panel(cross, cross, u, idx, grp & 3), c, o0, o1, fl);
    STAMP(7);
    sleep_until<1>(tw, P_PACE0 + jg * P_PACE / 2);
    store_item(rd, o0, o1, h, lane);
    STAMP(6);
    item_offsets(cross, stg, cross ? NAP + (grp - 4) * 8 + 2 * u : u, c, o0, o1, kl);
    if (keeper) store_item(rl, o0, o1, h, lane);
    DRAIN();
    flag(lane == 0, rd, fl, call);
    flag(lane == 0 && keeper, rl, kl, call);
  } else {
    if (threadIdx.x < 16) ((volatile LAS unsigned*)(lds + LDS_CTL))[threadIdx.x] = 0;
    LDSWAIT();
    __builtin_amdgcn_s_barrier();
  }

  if (jg >= 0) {
    volatile LAS Ctl* ctl = (volatile LAS Ctl*)(lds + LDS_CTL);
    const bool direct = wave >= 4;
    const int di = wave == 6 ? jg : wave == 5 ? NG + (jg + P_ROT1) % NG : 2 * NG + (jg + P_ROT2) % NG;
    const int leave = P_PACE0 + di * P_PACE / 2, it = wave * (NG * 8) + jg * 8 + grp;
    const bool work = direct ? (grp == idx || di >= NDRAW) && di < ND : it < NCV;
    bool isw = false;
    int li = 0, g = 0, c = 0, u = 0;
    f32x4 v[16];
    __builtin_amdgcn_s_setprio(3);
    if (work) {
      const int k = it / 5 + ND, m = direct ? grp - 4 : it % 5, xi = k < 128 ? k : it - (NCV - 512);
      const bool xc = direct ? cross : k < 128 ? m < 4 : xi >= NIN;
      li = direct ? di : xi >= NIN ? xi - NIN + 128 : xi;
      link_item(xc, li, isw, u, c);
      g = isw ? m * 4 + u : xc ? NI + u : u;
      sleep_until<2>(tw, direct ? leave - P_LEAD : it * P_CSTEP >> 8);
      load_inputs(p, isw, g, c, lane, v);
    }
    const unsigned call = UNI(last_call) + 1u, stg = OFF_STG + (call & 1u) * STG_BYTES;
    if (work) {
      LAS v4u* scr = (LAS v4u*)(lds + wave * 8192);
      v4u h[8];
      to_fragments(isw, v, scr, lane, h);
      unsigned o0, o1, fl, hdr = 0, kl = NOWHERE;
      item_offsets(isw, stg, isw ? NAP + 2 * g : g < NI ? g : PSENT + u, c, o0, o1, fl);
      const rsrc_t rs = mk(direct ? p.ws[grp] : myws, WS_USED);
      STAMP(7);
      if (!direct || keeper) store_item(rl, o0, o1, h, lane);
      if (li < NPI && !(direct && grp == idx)) {
        const float ss = sumsq(v[0]) + sumsq(v[4]) + sumsq(v[8]) + sumsq(v[12]);
        const unsigned base = (unsigned)min(max(top_exponent(ss, 6.76f / 1024.f) - 127 + 15 - 7, 0), 24);
        pack8<false>(h, base);
        const unsigned total = list_plane(h[7], (LAS unsigned short*)scr, LIST_CAP, lane);
        const bool whole = total > (unsigned)LIST_CAP;
        if (!whole) {
          LDSWAIT();
          h[7] = *(volatile LAS v4u*)(scr + (lane & 15));
        }
        const unsigned nl = whole ? 64 : max((total + 31) >> 5, 1u) * 4;
        hdr = base | (whole ? 1u << 16 : total << 8) | nl << 24;
        if (direct) {
          if (keeper) kl = fl;
          item_offsets(isw, stg, landing_panel(cross, isw, u, idx, grp & 3), c, o0, o1, fl);
          fl += OFF_AF + (call & 1u) * PF_BYTES - OFF_PF;
          sleep_until<1>(tw, leave);
        } else {
          o0 = OFF_EXP + (isw ? 512 + g * 32 + c : g * 16 + c) * 8192;
          o1 = o0 + 4096;
        }
        store_item(rs, o0, o1, h, lane, nl);
      }
      STAMP(6);
      DRAIN();
      flag(lane == 0, rs, fl, call, hdr);
      flag(lane == 0, rl, kl, call);
    }
    STAMP(2);
    if (lane == 0) st_sys(counter, call);
    arrive(&ctl->conv, lane);
    const Job job = make_job(grp ^ 4, jg);
    if (wave < 4) {
      __builtin_amdgcn_s_setprio(0);
      unsigned have = 0;
      STAMP(8);
      switch (job.mt) {
        case 3: gemm_tile<3>(rl, lds, job, have, stg, wave, lane); break;
        case 4: gemm_tile<4>(rl, lds, job, have, stg, wave, lane); break;
        case 5: gemm_tile<5>(rl, lds, job, have, stg, wave, lane); break;
        default: gemm_tile<6>(rl, lds, job, have, stg, wave, lane); break;
      }
      STAMP(16);
      have = UNI(have);
      gemm_stream(rl, lds, job, have, stg, wave, lane);
      STAMP(1);
    } else if (wave > 4) {
      finish_tile(p, rl, lds, job, wave - 5, call, lane);
      finish_stream(p, rl, lds, job, wave - 5, call, lane);
    } else {
      const int np = job.mt + 4, l2 = np <= 8 ? 3 : 4, q = lane & ((1 << l2) - 1);
      const unsigned fo = OFF_PF + (q < job.mt ? job.a0 + q : NAP + job.b0 + q - job.mt) * (NCH * 128);
      const unsigned total = (stream_len(job) + 1) * NCH;
      unsigned pub = 0, win = 64, idle = 0;
      sleep_until<8>(tw, P_GSTART);
      TREC_DECL();
      auto give_up = [&]() {
        st_sys(myws + OFF_TMO, call);
        ctl->bad = 1;
        LDSWAIT();
        ctl->prog = 0xffffffffu;
      };
      while (pub <= total - P_FAST) {
        CBAR();
        const bool tile = pub < NCH;
        const unsigned most = tile ? 63 >> l2 : 32, ahead = tile ? lane >> l2 : lane, c = pub + ahead;
        win = min(win, most);
        const bool act = ahead < win && (tile ? q < np && c < NCH : c < total);
        const unsigned v = ld4(rl, !act ? (lane == 63 ? OFF_TMO : NOWHERE) : tile ? fo + c * 128
                                   : OFF_PF + (stream_panel(job, (c >> 4) - 1) * NCH + (c & 15)) * 128);
        if (LANE(v, 63) == call || NOW() - tw > DEADLINE) { give_up(); return; }
        const unsigned n = (unsigned)__builtin_ctzll(__ballot(act ? (int)(v - call) < 0 : ahead >= win)) >> (tile ? l2 : 0);
        idle = n ? 0 : idle + 1;
        win = n >= win ? 64 : tile ? n + 1 : idle > 2 ? 2 : 8;
        pub = min(pub + n, tile ? (unsigned)NCH : total);
        ctl->prog = pub;
        if (n && pub + NCH > total) TREC(32 + pub + NCH - total);
        if (tile) {
          if (pub == NCH) { win = 64; need(&ctl->full[0], 1); }
          else if (!n) __builtin_amdgcn_s_sleep(8);
        }
        while (pub < total && pub >= NCH + 40 + 4 * UNI(ctl->sfull[0] + ctl->sfull[1])) __builtin_amdgcn_s_sleep(4);
      }
      const unsigned fl = lane < NCH ? OFF_PF + (stream_panel(job, stream_len(job) - 1) * NCH + lane) * 128
                                     : lane == 63 ? OFF_TMO : NOWHERE;
      unsigned v[4];
#pragma unroll
      for (int d = 0; d < 4; ++d) {
        v[d] = ld4(rl, pub < total ? fl : NOWHERE);
        CBAR();
        __builtin_amdgcn_s_sleep(P_FNAP);
      }
      while (pub < total) {
        bool late = NOW() - tw > DEADLINE;
#pragma unroll
        for (int d = 0; d < 4; ++d) {
          const unsigned was = pub;
          pub = total - NCH + __builtin_ctzll(__ballot(lane >= NCH || (lane + total >= pub + NCH && (int)(v[d] - call) < 0)));
          late |= LANE(v[d], 63) == call;
          ctl->prog = pub;
          if (pub > was) TREC(32 + pub + NCH - total);
          v[d] = ld4(rl, lane + total >= pub + NCH ? fl : NOWHERE);
          CBAR();
          __builtin_amdgcn_s_sleep(P_FNAP);
        }
        if (late) { give_up(); return; }
      }
      TREC_DUMP(40);
    }
    return;
  }

  const unsigned call = UNI(last_call) + 1u, stg = OFF_STG + (call & 1u) * STG_BYTES;
  const unsigned aoff = OFF_AF + (call & 1u) * PF_BYTES - OFF_PF;
  if (lane == 0) st_sys(counter, call);
  if (j < NC && grp != idx) {
    struct Ring { unsigned full[NL], done; };
    volatile LAS Ring* rg = (volatile LAS Ring*)(lds + LDS_CTL);
    constexpr int SLOT = 10240;
    volatile LAS v2u* meta = (volatile LAS v2u*)(lds + 2 * NL * SLOT);
    const int n = cross ? NXY + NYF : NIN;
    if (wave == 0) {
      __builtin_amdgcn_s_setprio(3);
      const unsigned long long peer_out = uniform64(p.out[grp]), peer_ws = uniform64(p.ws[grp]);
      unsigned pl[3] = {NOWHERE, NOWHERE, NOWHERE}, ph[3] = {0, 0, 0};
      const int cnt = (n - ND - j + NC - 1) / NC, ry = ((cross ? NXY : n) - ND - j + NC - 1) / NC;
      unsigned lane0 = lane == 0 ? ~0u : 0u, lane1 = lane == 1 ? ~0u : 0u;
      asm volatile("" : "+v"(lane0), "+v"(lane1));
      int r = 0, l = 0, k = 0;
      TREC_DECL();
      auto flush = [&]() {
        DRAIN();
        flag(lane < 3, rd, bfi(lane0, pl[0], bfi(lane1, pl[1], pl[2])), call, bfi(lane0, ph[0], bfi(lane1, ph[1], ph[2])));
        pl[0] = pl[1] = pl[2] = NOWHERE;
      };
      auto rounds = [&](int end, auto wide, auto yphase) {
        constexpr int W = wide() ? P_WIDE : P_WIN;
        static_assert(W <= (wide() ? 20 : 12) && !(wide() && yphase()), "window");
        for (; r < end; ++r) {
          LAS char* sl = lds + (2 * l + (k & 1)) * SLOT + lane * 16;
          unsigned got;
          v2u m;
          v4u a0, a1, v[8];
#define READ_SLOT() do { \
          const unsigned fu = rg->full[l]; \
          m = meta[2 * l + (k & 1)]; \
          a0 = *(volatile LAS v4u*)(sl + 8192); a1 = *(volatile LAS v4u*)(sl + 9216); \
          _Pragma("unroll") for (int q = 0; q < 8; ++q) v[q] = *(volatile LAS v4u*)(sl + q * 1024); \
          got = UNI(fu); } while (0)
          READ_SLOT();
          if (__builtin_expect(got <= (unsigned)k, 0)) {
            flush();
            while (UNI(rg->full[l]) <= (unsigned)k) __builtin_amdgcn_s_sleep(0);
            READ_SLOT();
          }
#undef READ_SLOT
          if (got == 0xffffffffu) { r = cnt; rg->done = 0xffffffffu; break; }
          const unsigned hd = UNI(m.y);
          const bool to_out = yphase() && ((hd >> 17) & 1);
          const rsrc_t rs = mk((char*)(to_out ? peer_out : peer_ws), to_out ? OUT_BYTES : WS_USED);
          const unsigned fo = wide() ? bfi(lane0, pl[0], NOWHERE) : bfi(lane0, pl[1], bfi(lane1, pl[0], NOWHERE));
          const v2u fv = {call, wide() ? ph[0] : bfi(lane0, ph[1], ph[0])};
#pragma unroll
          for (int q = 0; q < 8; ++q) asm volatile("" : "+v"(v[q]));
          rg->done = r + 1;
          asm volatile("s_waitcnt vmcnt(%0)" :: "n"(W) : "memory");
#pragma unroll
          for (int q = 0; q < 4; ++q) st16(rs, a0[q], v[q]);
          asm volatile("s_waitcnt vmcnt(%0)" :: "n"(W) : "memory");
          if (yphase() && (((hd | ph[2]) >> 18) & 1)) asm volatile("s_waitcnt vmcnt(6)" ::: "memory");
          __builtin_amdgcn_raw_buffer_store_b64(fv, rd, (int)fo, 0, AUX_SYS);
          CBAR();
#pragma unroll
          for (int q = 0; q < 4; ++q) st16(rs, a1[q], v[4 + q]);
          CBAR();
          if (!(r & 1)) TREC(r >> 1);
          if (r >= cnt - 6) TREC(48 + r - cnt + 6);
          pl[0] = wide() ? pl[1] : NOWHERE; ph[0] = ph[1]; pl[1] = pl[2]; ph[1] = ph[2];
          pl[2] = UNI(m.x); ph[2] = hd;
          if (++l == NL) { l = 0; ++k; }
        }
      };
      sleep_until<2>(tw, P_PACE0 + ND * P_PACE / 2 + P_HOLD);
      while (r < ry) {
        const int raw = cross ? 0 : max(r - (NPI - ND) / NC, 0);
        if (r < ry - P_NARROW && (int)(NOW() - tw) > P_LATE + ND * P_PACE / 2 + r * P_RATE / 10 + raw * 7)
          rounds(min(r + 2, ry), std::true_type(), std::false_type());
        else rounds(min(r + 2, ry), std::false_type(), std::false_type());
      }
      rounds(cnt, std::false_type(), std::true_type());
      if (!cross && pl[0] == NOWHERE && pl[1] != NOWHERE) {
        asm volatile("s_waitcnt vmcnt(8)" ::: "memory");
        flag(lane == 0, rd, pl[1], call, ph[1]);
        TREC(54);
        asm volatile("s_waitcnt vmcnt(1)" ::: "memory");
        flag(lane == 0, rd, pl[2], call, ph[2]);
      } else {
        flush();
      }
      STAMP(1);
      TREC_DUMP(8);
      return;
    }
    const int l = wave - 1, first = ND + l * NC + j, cnt = (n - first + WL - 1) / WL;
    const int i = min(first + lane * WL, n - 1);
    const bool isy = cross && i >= NXY, packed = !isy && i < NPI;
    unsigned s0, s1, sf, d0, d1, df, hdv = 64u << 24;
    {
      bool isw; int u, c;
      link_item(cross, i, isw, u, c);
      const int g = isw ? (grp - 4) * 4 + u : cross ? NI + u : u;
      item_offsets(isw, stg, isw ? NAP + 2 * g : cross ? PSENT + u : u, c, s0, s1, sf);
      item_offsets(isw, stg, landing_panel(cross, isw, u, idx, grp & 3), c, d0, d1, df);
      if (packed) {
        s0 = OFF_EXP + (isw ? 512 + g * 32 + c : g * 16 + c) * 8192; s1 = s0 + 4096;
        df += aoff;
      }
      if (isy) {
        sf = OFF_YL + ((grp & 3) * 128 + i - NXY) * 256;
        df = OFF_YF + (idx * 256 + i - NXY) * 128;
      }
      if (lane == 63) sf = OFF_TMO;
    }
    unsigned long long ready = 0;
    for (int k = 0; k < cnt; ++k) {
      while (!((ready >> k) & 1)) {
        CBAR();
        const bool ask = ((lane >= k && lane < min(cnt, k < 2 ? k + 3 : 63)) || lane == 63) && !((ready >> lane) & 1);
        const bool two = isy && lane < 63;
        const v2u f = ldflag(rl, ask ? sf : NOWHERE);
        const v2u g = ldflag(rl, ask && two ? sf + 128 : NOWHERE);
        const bool ok = f.x == call && (!two || g.x == call);
        if (ok && packed) hdv = f.y;
        if (ok && isy) { d0 = f.y; d1 = g.y; }
        ready |= __ballot(ok);
        if ((ready >> 63) || NOW() - tw > DEADLINE) {
          st_sys(myws + OFF_TMO, call);
          rg->full[l] = 0xffffffffu;
          return;
        }
      }
      CBAR();
      const int yi = cross ? first + k * WL - NXY : -1;
      unsigned hd = LANE(hdv, k), a0 = LANE(s0, k), a1 = LANE(s1, k), e0 = LANE(d0, k), e1 = LANE(d1, k);
      if (yi >= 0) {
        a0 = OFF_YS + ((grp & 3) * 512 + (e0 & 511)) * 2048;
        a1 = OFF_YS + ((grp & 3) * 512 + (e1 & 511)) * 2048;
        hd |= 1u << 17 | ((e0 | e1) & 1024) << 8;
      }
      v4u h[8];
#pragma unroll
      for (int q = 0; q < 4; ++q) h[q] = ld16(rl, lane * 16 + q * 1024, a0);
#pragma unroll
      for (int q = 0; q < 3; ++q) h[4 + q] = ld16(rl, lane * 16 + q * 1024, a1);
      h[7] = ld16(rl, lane < (hd >> 24) ? lane * 16 + 3072 : NOWHERE, a1);
      if (yi >= 0 && yi < NYF - P_NRAWY && !((hd >> 18) & 1)) {
        const float ss = sumsq(__builtin_bit_cast(f32x4, h[0])) + sumsq(__builtin_bit_cast(f32x4, h[2])) +
                         sumsq(__builtin_bit_cast(f32x4, h[4])) + sumsq(__builtin_bit_cast(f32x4, h[6]));
        const unsigned base = (unsigned)min(max(top_exponent(ss, 20.f / 1024.f) - 15, 0), 240);
        pack8<true>(h, base);
        LAS unsigned* scr = (LAS unsigned*)(lds + 2 * NL * SLOT + 128 + l * 256);
        const unsigned total = list_plane(h[7], (LAS unsigned short*)(scr + 2), LIST_CAP - 4, lane);
        if (total <= LIST_CAP - 4) {
          scr[lane & 1] = lane & 1 ? e1 : e0;
          LDSWAIT();
          h[7] = *(volatile LAS v4u*)(scr + 4 * (lane & 15));
          hd = base | total << 8 | ((total + 35) >> 5) * 4 << 24;
          e0 = OFF_YP + (idx * NAP + yi) * 8192; e1 = e0 + 4096;
        } else {
          unpack8<true>(h, base);
        }
      }
      v4u t0, t1;
#pragma unroll
      for (int q = 0; q < 4; ++q) {
        t0[q] = (hd >> 17) & 1 ? half_offset(e0, q, lane) : e0 + q * 1024 + lane * 16;
        t1[q] = (hd >> 17) & 1 ? half_offset(e1, q, lane) : e1 + q * 1024 + lane * 16;
      }
      if (lane >= (hd >> 24)) t1[3] = NOWHERE;
      if (k >= 2) need(&rg->done, (k - 2) * NL + l + 1);
      LAS v4u* sl = (LAS v4u*)(lds + (2 * l + (k & 1)) * SLOT);
      sl[512 + lane] = t0;
      sl[576 + lane] = t1;
      meta[2 * l + (k & 1)] = v2u{LANE(df, k), hd};
#pragma unroll
      for (int q = 0; q < 8; ++q) sl[q * 64 + lane] = h[q];
      LDSWAIT();
      rg->full[l] = k + 1;
    }
    STAMP(1);
    return;
  }

  char* marker = p.out[idx] + (size_t)panel_row(p, 0) * SH * 4;
  auto await = [&](unsigned off, unsigned& hdr, unsigned& spins, int nap) {
    for (spins = 1;; ++spins) {
      const v2u f = ldflag(rl, lane == 0 ? off : lane == 1 && !(spins & 15) ? OFF_TMO : NOWHERE);
      hdr = UNI(f.y);
      if ((int)(UNI(f.x) - call) >= 0) return true;
      if (LANE(f.x, 1) == call || NOW() - tw > DEADLINE) {
        st_sys(myws + OFF_TMO, call);
        st_sys(marker, QNAN);
        return false;
      }
      if (nap) __builtin_amdgcn_s_sleep(16);
      CBAR();
    }
  };
  const int sw = wave * (NC + NU) + j, sx = sw - 3 * P_SI;
  const int lk = grp != idx ? grp : sx < 0 ? (idx + 1 + sw % 3) & 3 : 4 + (sx & 3);
  const bool xl = lk >= 4;
  const int nu = xl ? NU : P_NUI, WU = xl ? WUX : WUI;
  const int me = grp != idx ? wave * nu + j - NC : nu * NWAVE + (xl ? sx >> 2 : sw / 3);
  if ((grp != idx && j - NC >= nu) || me >= WU) return;
  LAS unsigned* plane = (LAS unsigned*)(lds + wave * 1024);
  {
    const int n = xl ? NXY : NPI, first = NDRAW + me, cnt = (n - first + WU - 1) / WU;
    bool isw; int u, c;
    link_item(xl, min(first + lane * WU, n - 1), isw, u, c);
    unsigned o0, o1, fl;
    item_offsets(isw, stg, landing_panel(xl, isw, u, lk & 3, idx), c, o0, o1, fl);
    unsigned pend = NOWHERE, hdr = 0;
    bool have_next = false;
    unsigned long long tseen = tw;
    sleep_until<4>(tw, 200 + first * P_PACE / 2);
    TREC_DECL();
    for (int r = 0; r < cnt; ++r) {
      const unsigned fr = LANE(fl, r);
      *(LAS v4u*)(plane + lane * 4) = v4u{0, 0, 0, 0};
      if (!have_next) {
        DRAIN();
        flag(lane == 0, rl, pend, call);
        pend = NOWHERE;
        sleep_until<2>(tseen, WU * 25 / 2 - P_UGAP);
        unsigned spins;
        if (!await(fr + aoff, hdr, spins, 0)) return;
        tseen = NOW() - (spins == 1 ? 60 : 0);
      }
      CBAR();
      TREC(4 * r);
      const unsigned a0 = LANE(o0, r), a1 = LANE(o1, r);
      const bool whole = (hdr >> 16) & 1;
      const unsigned e2 = ld4(rl, lane * 4 + 3072, a1);
      CBAR();
      v4u h[8];
#pragma unroll
      for (int q = 0; q < 4; ++q) h[q] = ld16(rl, lane * 16 + q * 1024, a0);
#pragma unroll
      for (int q = 0; q < 3; ++q) h[4 + q] = ld16(rl, lane * 16 + q * 1024, a1);
      h[7] = ld16(rl, whole ? lane * 16 + 3072 : NOWHERE, a1);
      const v2u fn = ldflag(rl, lane == 0 ? LANE(fl, min(r + 1, cnt - 1)) + aoff : NOWHERE);
      CBAR();
      const v4u pl = unlist_plane(e2, 0, (hdr >> 8) & 255, plane, lane);
      DRAIN();
      flag(lane == 0, rl, pend, call);
      h[7] |= pl;
      unpack8<false>(h, hdr & 31);
      store_item(rl, a0, a1, h, lane);
      pend = fr;
      TREC(4 * r + 2);
      hdr = UNI(fn.y);
      have_next = (int)(UNI(fn.x) - call) >= 0;
    }
    DRAIN();
    flag(lane == 0, rl, pend, call);
    TREC_DUMP(8);
    STAMP(2);
  }
  const rsrc_t ro = mk(p.out[idx], OUT_BYTES);
  for (int yi = me; xl && yi < NYF + NYD; yi += WU) {
    *(LAS v4u*)(plane + lane * 4) = v4u{0, 0, 0, 0};
    unsigned hdr, spins;
    if (yi >= NYF) sleep_until<8>(tw, 5000);
    if (!await(OFF_YF + ((lk & 3) * 256 + yi) * 128, hdr, spins, yi < NYF - P_NRAWY - 16)) return;
    if (yi >= NYF || ((hdr >> 17) & 1)) continue;
    CBAR();
    const unsigned a0 = OFF_YP + ((lk & 3) * NAP + yi) * 8192;
    const unsigned e2 = ld4(rl, lane * 4 + 7168, a0);
    CBAR();
    v4u h[8];
#pragma unroll
    for (int q = 0; q < 7; ++q) h[q] = ld16(rl, lane * 16 + (q & 3) * 1024, a0 + (q >> 2) * 4096);
    CBAR();
    h[7] = unlist_plane(e2, 4, (hdr >> 8) & 255, plane, lane);
    unpack8<true>(h, hdr & 255);
#pragma unroll
    for (int q = 0; q < 8; ++q) st16(ro, half_offset(LANE(e2, q >> 2), q & 3, lane), h[q]);
  }
  DIAG_DRAIN();
  STAMP(1);
}

}

extern "C" void kernel_launch(int rank, int nranks, ncclComm_t comm, hipStream_t stream, void* const* d_in,
                              const int* in_sizes, int n_in, void* d_out, int out_size, const int* in_dtypes,
                              int out_dtype, void* const* d_out_all, void* const* d_ws_all, size_t ws_bytes) {
  using namespace agg;
  if (nranks != NR || n_in != 2 || ws_bytes < WS_USED || out_size != 4096 * SH || in_sizes[0] != SH * KD ||
      in_sizes[1] != KD * SH || (unsigned)rank >= (unsigned)NR) {
    fprintf(stderr, "ag_gemm: unexpected problem shape\n");
    return;
  }
  static const int kClass[2][4] = {{0, 2, 3, 6}, {1, 4, 5, 7}};
  Params p;
  memset(&p, 0, sizeof(p));
  int cls = 0;
  for (int c = 0; c < 2; ++c)
    for (int i = 0; i < 4; ++i) if (kClass[c][i] == rank) { cls = c; p.idx = i; }
  for (int r = 0; r < NR; ++r) {
    const int q = kClass[cls ^ (r >> 2)][r & 3];
    p.ranks |= (unsigned)q << (4 * r);
    p.out[r] = (char*)d_out_all[q];
    p.ws[r] = (char*)d_ws_all[q];
  }
  p.x = (const float*)d_in[0];
  p.w = (const float*)d_in[1];
  p.myws = (char*)d_ws_all[rank];
  HIP_CHECK(hipFuncSetAttribute((const void*)ag_gemm, hipFuncAttributeMaxDynamicSharedMemorySize, LDS_BYTES));
  ag_gemm<<<GRID, NTHR, LDS_BYTES, stream>>>(p);
  HIP_CHECK(hipGetLastError());
}
```

```cpp
#include <hip/hip_runtime.h>
#include <rccl/rccl.h>
#include <cstdio>
#include <cstring>
#include <type_traits>

#define HIP_CHECK(expr) do { hipError_t _e = (expr); if (_e != hipSuccess) { \
    fprintf(stderr, "HIP error %s at %s:%d\n", hipGetErrorString(_e), __FILE__, __LINE__); \
    return; } } while (0)

#define P_ND 66
#define P_NDRAW 16
#define P_PACE0 80
#define P_PACE 25
#define P_LEAD 330
#define P_HOLD 40
#define P_CSTEP 250
#define P_GSTART 950
#define P_FAST 16
#define P_FNAP 3
#define P_WIN 11
#define P_WIDE 16
#define P_LATE 105
#define P_RATE 503
#define P_NARROW 4
#define P_NUI 3
#define P_SI 8
#define P_NRAW 48
#define P_NRAWY 22
#define P_ROT1 15
#define P_ROT2 5
#define P_UGAP 140
#define P_LASTA 3
#define P_LASTB 1

namespace agg {

#define GAS __attribute__((address_space(1)))
#define LAS __attribute__((address_space(3)))
typedef unsigned v4u __attribute__((ext_vector_type(4)));
typedef unsigned v2u __attribute__((ext_vector_type(2)));
typedef unsigned short us2 __attribute__((ext_vector_type(2)));
typedef float f32x4 __attribute__((ext_vector_type(4)));
typedef float f32x2 __attribute__((ext_vector_type(2)));
typedef _Float16 h8 __attribute__((ext_vector_type(8)));
typedef GAS unsigned gu32;
typedef __amdgpu_buffer_rsrc_t rsrc_t;

constexpr int NR = 8;
constexpr int KD = 4096, SH = 512;
constexpr int PANEL = 131072;
constexpr int NCH = 16, CHUNK = 8192;
constexpr int NI = 23, NX = 32 - NI;
constexpr int NB1 = 8;
constexpr int NAP = 128, NBP = 64, PSENT = NAP + NBP, NPAN = PSENT + 16;
constexpr int GRID = 256, NWAVE = 7, NTHR = NWAVE * 64;
constexpr int NC = 4, NL = NWAVE - 1, WL = NC * NL;
constexpr int NU = 4;
constexpr int SX = ((NC + NU) * NWAVE - 3 * P_SI) / 4;
constexpr int WUI = P_NUI * NWAVE + P_SI, WUX = NU * NWAVE + SX;
constexpr int NG = 32 - NC - NU;
constexpr int ND = P_ND, NDRAW = P_NDRAW;
constexpr int NIN = NI * NCH, NXY = 128 + NX * NCH;
constexpr int NPI = NIN - P_NRAW;
constexpr int NCV = 5 * (128 - ND) + 384;
constexpr int LEN_A = NI - NB1 - 2, LEN_B = NX + 2;
constexpr int NHALF = 2 * (NI + 3 * NB1) + 12 * ((LEN_A - P_LASTA + 1) / 2) + 12 * ((NX + 1) / 2 + (3 - P_LASTB) / 2);
constexpr int NYF = NHALF / 2, NYD = 24;
static_assert(NG == 24 && NI == 23 && NHALF % 2 == 0 && NYF <= NAP, "the deal of tiles and streams");
static_assert(NG * 32 >= NCV && ND <= 3 * NG && NDRAW <= NG && NDRAW <= ND, "one item per converting wave");
static_assert(NIN <= 63 * WL && NXY + NYF <= 63 * WL && NIN <= 64 * WUI && SX >= 0, "one lane per item");
static_assert(NXY <= NPI && P_LASTA >= 1 && P_LASTA <= 4 && P_LASTB >= 1 && P_LASTB <= 2, "");

constexpr unsigned OFF_CNT = 0;
constexpr unsigned OFF_TMO = 16384;
constexpr unsigned OFF_XCC = 18432;
constexpr unsigned OFF_YQ = 20480;
constexpr unsigned OFF_BAR = 32768;
constexpr unsigned OFF_YF = 65536;
constexpr unsigned OFF_YL = 196608;
constexpr unsigned OFF_PF = 393216;
constexpr unsigned PF_BYTES = NPAN * NCH * 128;
constexpr unsigned OFF_AF = 1u << 20;
constexpr unsigned OFF_YS = 2u << 20;
constexpr unsigned OFF_YP = OFF_YS + (4u << 20);
constexpr unsigned OFF_EXP = OFF_YP + (4u << 20);
constexpr unsigned OFF_STG = OFF_EXP + (8u << 20);
constexpr unsigned STG_BYTES = (unsigned)NPAN * PANEL;
constexpr unsigned OFF_DBG = OFF_STG + 2 * STG_BYTES;
constexpr unsigned WS_USED = OFF_DBG + (1u << 20);
static_assert(OFF_YF + 1024 * 128 <= OFF_YL && OFF_YL + 1024 * 128 <= OFF_PF && OFF_PF + PF_BYTES <= OFF_AF &&
              OFF_AF + 2 * PF_BYTES <= OFF_YS, "layout");
constexpr unsigned OUT_BYTES = 4096u * SH * 4u;
constexpr unsigned NOWHERE = 0xf0000000u;
constexpr unsigned long long DEADLINE = 50000000ull;
constexpr int AUX_SYS = 17;
constexpr unsigned QNAN = 0x7fc00000u;

constexpr int LDS_RED = 65536;
constexpr int LDS_CTL = 2 * LDS_RED;
constexpr int LDS_SIMG = LDS_CTL + 64;
constexpr int LDS_BYTES = LDS_SIMG + 16384;

struct Params {
  const float* x;
  const float* w;
  char* myws;
  int idx;
  unsigned ranks;
  int rot, pad;
  char* wsb[NR];
  char* out[NR];
  char* ws[NR];
};

#define DRAIN() asm volatile("s_waitcnt vmcnt(0)" ::: "memory")
#define LDSWAIT() asm volatile("s_waitcnt lgkmcnt(0)" ::: "memory")
#define CBAR() asm volatile("" ::: "memory")
#define NOW() __builtin_amdgcn_s_memrealtime()
#define UNI(v) ((unsigned)__builtin_amdgcn_readfirstlane(v))
#define LANE(v, l) ((unsigned)__builtin_amdgcn_readlane(v, l))

__device__ __forceinline__ rsrc_t mk(const void* p, unsigned bytes) {
  return __builtin_amdgcn_make_buffer_rsrc((void*)p, 0, (int)bytes, 0x00020000);
}
__device__ __forceinline__ unsigned long long uniform64(const void* q) {
  const unsigned long long v = (unsigned long long)q;
  return (unsigned long long)UNI((unsigned)(v >> 32)) << 32 | UNI((unsigned)v);
}
__device__ __forceinline__ unsigned pk(float a, float b) {
  typedef _Float16 h2 __attribute__((ext_vector_type(2)));
  h2 v = {(_Float16)a, (_Float16)b};
  return __builtin_bit_cast(unsigned, v);
}
__device__ __forceinline__ void st_sys(char* p, unsigned v) {
  __hip_atomic_store((gu32*)p, v, __ATOMIC_RELAXED, __HIP_MEMORY_SCOPE_SYSTEM);
}
__device__ __forceinline__ void st16(rsrc_t r, unsigned voff, v4u v) {
  __builtin_amdgcn_raw_buffer_store_b128(v, r, (int)voff, 0, AUX_SYS);
}
__device__ __forceinline__ v4u ld16(rsrc_t r, unsigned voff, unsigned soff) {
  return __builtin_amdgcn_raw_buffer_load_b128(r, (int)voff, (int)soff, AUX_SYS);
}
__device__ __forceinline__ unsigned ld4(rsrc_t r, unsigned voff, unsigned soff = 0) {
  return __builtin_amdgcn_raw_buffer_load_b32(r, (int)voff, (int)soff, AUX_SYS);
}
__device__ __forceinline__ void flag(bool on, rsrc_t r, unsigned off, unsigned call, unsigned hdr = 0) {
  __builtin_amdgcn_raw_buffer_store_b64(v2u{call, hdr}, r, (int)(on ? off : NOWHERE), 0, AUX_SYS);
}
__device__ __forceinline__ v2u ldflag(rsrc_t r, unsigned off) {
  return __builtin_amdgcn_raw_buffer_load_b64(r, (int)off, 0, AUX_SYS);
}
__device__ __forceinline__ void store_item(rsrc_t r, unsigned o0, unsigned o1, const v4u (&h)[8], int lane, unsigned nl = 64) {
#pragma unroll
  for (int s = 0; s < 4; ++s) st16(r, o0 + lane * 16 + s * 1024, h[s]);
#pragma unroll
  for (int s = 0; s < 3; ++s) st16(r, o1 + lane * 16 + s * 1024, h[4 + s]);
  st16(r, lane < nl ? o1 + lane * 16 + 3072 : NOWHERE, h[7]);
}
template <int S> __device__ __forceinline__ void sleep_until(unsigned long long t0, int ticks) {
  while ((int)(NOW() - t0) < ticks) __builtin_amdgcn_s_sleep(S);
}

#define STAMP(slot) do {} while (0)
#define TREC_DECL() do {} while (0)
#define TREC(i) do {} while (0)
#define TREC_DUMP(from) do {} while (0)
#define DIAG_DRAIN() do {} while (0)

constexpr int LIST_CAP = 128;
__device__ __forceinline__ unsigned bfi(unsigned m, unsigned a, unsigned b) { return (a & m) | (b & ~m); }
template <bool F32> __device__ __forceinline__ unsigned add_base(unsigned v, unsigned b) {
  return F32 ? v + b : __builtin_bit_cast(unsigned, (us2)(__builtin_bit_cast(us2, v) + __builtin_bit_cast(us2, b)));
}
template <bool F32> __device__ __forceinline__ void pack8(v4u (&h)[8], unsigned base) {
  constexpr unsigned TOP = F32 ? 0xF0000000u : 0xC000C000u;
  constexpr int B = F32 ? 4 : 2;
  const unsigned nb = F32 ? (0u - base) << 24 : ((32u - base) & 31u) * 0x08000800u;
#pragma unroll
  for (int d = 0; d < 4; ++d) {
    unsigned t[8], pl = 0;
#pragma unroll
    for (int s = 0; s < 8; ++s) {
      t[s] = add_base<F32>(__builtin_amdgcn_alignbit(h[s][d], h[s][d], 31), nb);
      pl |= (t[s] & TOP) >> (B * s);
    }
#pragma unroll
    for (int s = 0; s < 7; ++s) h[s][d] = bfi(TOP, t[7] << (B * (7 - s)), t[s]);
    h[7][d] = pl;
  }
}
template <bool F32> __device__ __forceinline__ void unpack8(v4u (&h)[8], unsigned base) {
  constexpr unsigned TOP = F32 ? 0xF0000000u : 0xC000C000u;
  constexpr int B = F32 ? 4 : 2;
  const unsigned pb = F32 ? base << 24 : base * 0x08000800u;
#pragma unroll
  for (int d = 0; d < 4; ++d) {
    const unsigned pl = h[7][d];
    unsigned t7 = (pl << (B * 7)) & TOP;
#pragma unroll
    for (int s = 0; s < 7; ++s) {
      t7 |= (h[s][d] & TOP) >> (B * (7 - s));
      const unsigned t = add_base<F32>(bfi(TOP, pl << (B * s), h[s][d]), pb);
      h[s][d] = __builtin_amdgcn_alignbit(t, t, 1);
    }
    t7 = add_base<F32>(t7, pb);
    h[7][d] = __builtin_amdgcn_alignbit(t7, t7, 1);
  }
}
template <int CTRL> __device__ __forceinline__ unsigned dpp0(unsigned v) {
  return (unsigned)__builtin_amdgcn_update_dpp(0, (int)v, CTRL, 0xf, 0xf, true);
}
__device__ __forceinline__ int top_exponent(float ss, float scale) {
  ss += __builtin_bit_cast(float, dpp0<0x111>(__builtin_bit_cast(unsigned, ss)));
  ss += __builtin_bit_cast(float, dpp0<0x112>(__builtin_bit_cast(unsigned, ss)));
  ss += __builtin_bit_cast(float, dpp0<0x114>(__builtin_bit_cast(unsigned, ss)));
  ss += __builtin_bit_cast(float, dpp0<0x118>(__builtin_bit_cast(unsigned, ss)));
  const unsigned sb = __builtin_bit_cast(unsigned, ss);
  const float tot = (__builtin_bit_cast(float, LANE(sb, 15)) + __builtin_bit_cast(float, LANE(sb, 31))) +
                    (__builtin_bit_cast(float, LANE(sb, 47)) + __builtin_bit_cast(float, LANE(sb, 63)));
  return (int)((UNI(__builtin_bit_cast(unsigned, __builtin_amdgcn_sqrtf(tot * scale))) >> 23) & 255);
}
__device__ __forceinline__ float sumsq(f32x4 z) { return z.x * z.x + z.y * z.y + z.z * z.z + z.w * z.w; }
__device__ __forceinline__ unsigned list_plane(const v4u& pl, LAS unsigned short* list, unsigned cap, int lane) {
  unsigned nz[4], cnt = 0;
#pragma unroll
  for (int d = 0; d < 4; ++d) {
    unsigned z = pl[d];
    z |= z >> 1; z |= z >> 2;
    nz[d] = z & 0x11111111u;
    cnt += __builtin_popcount(nz[d]);
  }
  unsigned inc = cnt;
  inc += dpp0<0x111>(inc); inc += dpp0<0x112>(inc); inc += dpp0<0x114>(inc); inc += dpp0<0x118>(inc);
  const unsigned r0 = LANE(inc, 15), r1 = r0 + LANE(inc, 31), r2 = r1 + LANE(inc, 47), total = r2 + LANE(inc, 63);
  if (total <= cap) {
    unsigned pos = inc - cnt + (lane < 16 ? 0 : lane < 32 ? r0 : lane < 48 ? r1 : r2);
#pragma unroll
    for (int d = 0; d < 4; ++d) {
      unsigned m = nz[d];
      while (m) {
        const unsigned b = __builtin_ctz(m);
        list[pos++] = (unsigned short)(((lane * 4 + d) << 7) | (b << 2) | ((pl[d] >> b) & 15));
        m &= m - 1;
      }
    }
  }
  return total;
}
__device__ __forceinline__ v4u unlist_plane(unsigned e2, unsigned skip, unsigned count, LAS unsigned* plane, int lane) {
  LDSWAIT();
#pragma unroll
  for (int k = 0; k < 2; ++k) {
    const unsigned e = (e2 >> (16 * k)) & 0xffffu;
    __hip_atomic_fetch_or(plane + ((e >> 7) & 255), 2u * lane + k - skip < count ? (e & 15) << ((e >> 2) & 28) : 0u,
                          __ATOMIC_RELAXED, __HIP_MEMORY_SCOPE_WORKGROUP);
  }
  LDSWAIT();
  return *(volatile LAS v4u*)(plane + lane * 4);
}

__device__ __forceinline__ void item_offsets(bool isw, unsigned stg, unsigned pan, int c, unsigned& o0, unsigned& o1,
                                             unsigned& fl) {
  o0 = stg + pan * PANEL + c * (isw ? 4096 : CHUNK);
  o1 = o0 + (isw ? PANEL : 4096);
  fl = OFF_PF + (isw ? (pan + (c & 1)) * NCH + (c >> 1) : pan * NCH + c) * 128;
}
__device__ __forceinline__ void link_item(bool cross, int i, bool& isw, int& u, int& c) {
  static_assert(NB1 == 8, "as many items as w");
  isw = cross && i < 128;
  u = i >= 128 ? (i >> 4) - (cross ? 8 : 0) : cross ? i & 3 : i & 7;
  c = i >= 128 ? i & 15 : cross ? i >> 2 : i >> 3;
}
__device__ __forceinline__ unsigned landing_panel(bool cross, bool isw, int u, int from, int to) {
  return !cross ? NI + 3 * u + ((from - to - 1) & 3) : isw ? NAP + 32 + from * 8 + 2 * u : 4 * NI + from * NX + u;
}
__device__ __forceinline__ unsigned panel_row(const Params& p, int a) {
  const int idx = p.idx & 3, q = a - NI, z = a - 4 * NI;
  const int role = a < NI || a >= PSENT ? idx : z < 0 ? (idx + 1 + q % 3) & 3 : 4 + z / NX;
  const int g = a < NI ? a : a >= PSENT ? NI + a - PSENT : z < 0 ? q / 3 : NI + z % NX;
  return ((p.ranks >> (4 * role)) & 7) * SH + g * 16;
}
__device__ __forceinline__ void load_inputs(const Params& p, bool isw, int g, int c, int lane, f32x4 (&v)[16]) {
  if (isw) {
    const GAS f32x2* src =
        (const GAS f32x2*)(p.w + (size_t)((c & 31) * 128 + 4 * (lane >> 4)) * SH + (g & 15) * 32 + 2 * (lane & 15));
#pragma unroll
    for (int q = 0; q < 16; ++q) {
      const f32x2 a = __builtin_nontemporal_load(src + ((q >> 1) * 16 + (q & 1) * 2) * (SH / 2));
      const f32x2 b = __builtin_nontemporal_load(src + ((q >> 1) * 16 + (q & 1) * 2 + 1) * (SH / 2));
      v[q] = f32x4{a.x, b.x, a.y, b.y};
    }
  } else {
    const GAS f32x4* src =
        (const GAS f32x4*)(p.x + (size_t)((g & 31) * 16 + (lane >> 2)) * KD + (c & 15) * 256 + 4 * (lane & 3));
#pragma unroll
    for (int q = 0; q < 16; ++q) v[q] = src[q * 4];
  }
}
__device__ __forceinline__ void to_fragments(bool isw, const f32x4 (&v)[16], LAS v4u* scr, int lane, v4u (&h)[8]) {
  if (isw) {
#pragma unroll
    for (int s = 0; s < 4; ++s) {
      h[s] = v4u{pk(v[4 * s].x, v[4 * s].y), pk(v[4 * s + 1].x, v[4 * s + 1].y), pk(v[4 * s + 2].x, v[4 * s + 2].y),
                 pk(v[4 * s + 3].x, v[4 * s + 3].y)};
      h[4 + s] = v4u{pk(v[4 * s].z, v[4 * s].w), pk(v[4 * s + 1].z, v[4 * s + 1].w), pk(v[4 * s + 2].z, v[4 * s + 2].w),
                     pk(v[4 * s + 3].z, v[4 * s + 3].w)};
    }
  } else {
#pragma unroll
    for (int s = 0; s < 8; ++s)
      scr[s * 64 + lane] = v4u{pk(v[2 * s].x, v[2 * s].y), pk(v[2 * s].z, v[2 * s].w), pk(v[2 * s + 1].x, v[2 * s + 1].y),
                               pk(v[2 * s + 1].z, v[2 * s + 1].w)};
    LDSWAIT();
#pragma unroll
    for (int s = 0; s < 8; ++s) h[s] = scr[s * 64 + (lane & 15) * 4 + (lane >> 4)];
    LDSWAIT();
  }
}
__device__ __forceinline__ unsigned half_offset(unsigned e, int q, int lane) {
  const unsigned o = e & 512 ? (q * 8 + (lane >> 3)) * (SH * 4) + (lane & 7) * 16
                             : (q * 4 + (lane >> 4)) * (SH * 4) + (lane & 15) * 16;
  return (e & 1024) && q >= 2 ? NOWHERE : (e >> 16 << 7) + o;
}
__device__ __forceinline__ unsigned queue_places(rsrc_t rl, bool on, int owner, int n) {
  return (unsigned)__builtin_amdgcn_raw_ptr_buffer_atomic_add_i32(n, rl, on ? OFF_YQ + owner * 128 : NOWHERE, 0, 16);
}

struct Ctl { unsigned prog, bad, full[2], fre[2], conv, sfull[2], sfre[2]; };
struct Job {
  int a0, mt;
  int b0, bs;
  int kind, j;
};
__device__ __forceinline__ Job make_job(int gg, int jg) {
  const int r = jg >> 2;
  Job q;
  q.b0 = (gg & 6) * 8 + (jg & 3) * 4; q.bs = q.b0 + (r & 1) * 2; q.j = r >> 1;
  if (gg & 1) { q.a0 = 4 * r; q.mt = r == 5 ? 3 : 4; q.kind = gg & 4 ? 0 : 3; }
  else if (gg & 4) { q.a0 = NI + 4 * r; q.mt = 4; q.kind = 2; }
  else { q.a0 = r < 4 ? NI + 6 * r : PSENT + 5 * (r - 4); q.mt = r < 4 ? 6 : 9 - r; q.kind = 1; }
  return q;
}
__device__ __forceinline__ int stream_len(const Job& q) {
  return q.kind == 0 ? LEN_A : q.kind == 1 ? LEN_A + 1 : q.kind == 2 ? LEN_B : 4 * NX / 3 + 1;
}
__device__ __forceinline__ int stream_panel(const Job& q, int i) {
  const int z = 3 * i + q.j;
  if (i == stream_len(q) - 1) return NI + 3 * (NI - 2 + (q.kind >> 1)) + q.j;
  if (q.kind < 2) return NI + 3 * NB1 + z;
  if (q.kind == 3) return 4 * NI + (z & 3) * NX + (z >> 2);
  return i < NX ? 4 * NI + (((q.bs >> 3) + 1 + q.j) & 3) * NX + i : NI + 3 * (NI - 3) + q.j;
}

__device__ __forceinline__ void need(volatile LAS unsigned* word, unsigned& have, unsigned n) {
  while (have < n) {
    have = UNI(*word);
    if (have < n) __builtin_amdgcn_s_sleep(1);
  }
  CBAR();
}
__device__ __forceinline__ void need(volatile LAS unsigned* word, unsigned n) { unsigned h = 0; need(word, h, n); }
__device__ __forceinline__ void arrive(volatile LAS unsigned* word, int lane) {
  LDSWAIT();
  if (lane == 0) __hip_atomic_fetch_add((LAS unsigned*)word, 1u, __ATOMIC_RELAXED, __HIP_MEMORY_SCOPE_WORKGROUP);
}
__device__ __forceinline__ f32x4 mfma(v4u b, v4u a, f32x4 c) {
  return __builtin_amdgcn_mfma_f32_16x16x32_f16(__builtin_bit_cast(h8, b), __builtin_bit_cast(h8, a), c, 0, 0, 0);
}

template <int MT>
__device__ __forceinline__ void gemm_tile(rsrc_t rs, LAS char* lds, const Job& job, unsigned& have, unsigned stg,
                                          int wave, int lane) {
  volatile LAS Ctl* ctl = (volatile LAS Ctl*)(lds + LDS_CTL);
  const unsigned ab = stg + job.a0 * PANEL + wave * 1024, bb = stg + (NAP + job.b0) * PANEL + wave * 1024;
  const int voff = lane * 16;
  f32x4 acc[MT][4];
#pragma unroll
  for (int m = 0; m < MT; ++m)
#pragma unroll
    for (int n = 0; n < 4; ++n) acc[m][n] = f32x4{0.f, 0.f, 0.f, 0.f};
  v4u ra[2][MT], rb[2][4];
#define LOADSTEP(d, i) do { \
    _Pragma("unroll") for (int m = 0; m < MT; ++m) ra[d][m] = ld16(rs, voff, ab + m * PANEL + (i) * 4096); \
    _Pragma("unroll") for (int n = 0; n < 4; ++n) rb[d][n] = ld16(rs, voff, bb + n * PANEL + (i) * 4096); \
    __builtin_amdgcn_sched_barrier(0); } while (0)
#define MMASTEP(d) do { \
    _Pragma("unroll") for (int m = 0; m < MT; ++m) \
    _Pragma("unroll") for (int n = 0; n < 4; ++n) acc[m][n] = mfma(rb[d][n], ra[d][m], acc[m][n]); \
    __builtin_amdgcn_sched_barrier(0); } while (0)
  need(&ctl->prog, have, 1);
  LOADSTEP(0, 0);
  LOADSTEP(1, 1);
#pragma unroll 1
  for (int c = 1; c < NCH; ++c) {
    MMASTEP(0);
    need(&ctl->prog, have, c + 1);
    LOADSTEP(0, 2 * c);
    MMASTEP(1);
    LOADSTEP(1, 2 * c + 1);
  }
  MMASTEP(0);
  MMASTEP(1);
#undef LOADSTEP
#undef MMASTEP
  asm volatile("" : "+v"(lane));
  need(&ctl->conv, NWAVE);
  constexpr int M1 = (MT + 1) / 2;
#pragma unroll
  for (int part = 0; part < 2; ++part) {
    const int m0 = part ? M1 : 0, mn = part ? MT - M1 : M1;
    LAS f32x4* red = (LAS f32x4*)(lds + part * LDS_RED) + wave * 1024;
#pragma unroll
    for (int m = 0; m < mn; ++m)
#pragma unroll
      for (int n = 0; n < 4; n += 2) {
        const int r = lane & 15, c4 = n * 4 + 2 * (lane >> 4);
        const f32x4 e = acc[m0 + m][n], o = acc[m0 + m][n + 1];
        red[(m * 16 + r) * 16 + (c4 ^ r)] = f32x4{e.x, o.x, e.y, o.y};
        red[(m * 16 + r) * 16 + ((c4 + 1) ^ r)] = f32x4{e.z, o.z, e.w, o.w};
      }
    arrive(&ctl->full[part], lane);
  }
}

__device__ __forceinline__ void finish_tile(const Params& p, rsrc_t rl, LAS char* lds, const Job& job, int part,
                                            unsigned call, int lane) {
  volatile LAS Ctl* ctl = (volatile LAS Ctl*)(lds + LDS_CTL);
  const LAS f32x4* red = (const LAS f32x4*)(lds + part * LDS_RED);
  const int idx = p.idx & 3, grp = job.b0 >> 3, m1 = (job.mt + 1) / 2;
  const int a0 = job.a0 + (part ? m1 : 0), mn = part ? job.mt - m1 : m1;
  const bool staged = grp >= 4;
  const rsrc_t ro = staged ? rl : mk(p.out[idx], OUT_BYTES);
  const unsigned ent = ((a0 + lane) * 4 + ((job.b0 >> 1) & 2)) % 512 |
                       (panel_row(p, a0 + min(lane, 2)) * SH + (staged ? idx : grp) * 128 + (job.b0 & 7) * 16) * 4 >> 7 << 16;
  need(&ctl->full[part], 4);
  STAMP(10);
  const bool bad = ctl->bad != 0;
  const unsigned ticket = queue_places(rl, staged && lane == 0, grp & 3, mn);
  for (int m = 0; m < mn; ++m) {
    const unsigned e = LANE(ent, m);
#pragma unroll
    for (int q = 0; q < 4; ++q) {
      const int r = q * 4 + (lane >> 4), li = (m * 16 + r) * 16 + ((lane & 15) ^ r);
      f32x4 s = (red[li] + red[1024 + li]) + (red[2048 + li] + red[3072 + li]);
      if (bad) s = __builtin_bit_cast(f32x4, v4u{QNAN, QNAN, QNAN, QNAN});
      st16(ro, staged ? OFF_YS + ((grp & 3) * 512 + (e & 511)) * 2048 + q * 1024 + lane * 16 : half_offset(e, q, lane),
           __builtin_bit_cast(v4u, s));
    }
  }
  LDSWAIT();
  ctl->fre[part] = 1;
  if (staged) {
    DRAIN();
    const unsigned pos = UNI(ticket) - (call - 1) * NHALF + lane;
    flag(lane < mn && pos < 256, rl, OFF_YL + ((grp & 3) * 256 + pos) * 128, call, ent);
  }
  STAMP(11);
}

__device__ __forceinline__ void gemm_stream(rsrc_t rs, LAS char* lds, const Job& job, unsigned& have, unsigned stg,
                                            int wave, int lane) {
  constexpr int D = 16;
  volatile LAS Ctl* ctl = (volatile LAS Ctl*)(lds + LDS_CTL);
  const int G = stream_len(job);
  const unsigned bb = stg + (NAP + job.bs) * PANEL + wave * 1024, voff = lane * 16;
  unsigned ab = stg + stream_panel(job, 0) * PANEL + wave * 1024, base = NCH;
  LAS v4u* bl = (LAS v4u*)(lds + wave * 32768) + lane;
  v4u br[32], ra[D];
#pragma unroll
  for (int s = 0; s < 32; ++s) {
    br[s] = ld16(rs, voff, bb + s * 4096);
    if (s >= 16) ra[s - 16] = ld16(rs, voff, bb + PANEL + (s - 16) * 4096);
    __builtin_amdgcn_sched_barrier(0);
  }
  need(&ctl->fre[0], 1);
  need(&ctl->fre[1], 1);
#pragma unroll
  for (int d = 0; d < D; ++d) {
    bl[d * 64] = ra[d];
    ra[d] = ld16(rs, voff, bb + PANEL + (16 + d) * 4096);
    __builtin_amdgcn_sched_barrier(0);
  }
#pragma unroll
  for (int d = 0; d < D; ++d) {
    bl[(16 + d) * 64] = ra[d];
    if ((d & 1) == 0) need(&ctl->prog, have, base + (d >> 1) + 1);
    ra[d] = ld16(rs, voff, ab + d * 4096);
    __builtin_amdgcn_sched_barrier(0);
  }
  bool primed = true;
#pragma unroll 1
  for (int i = 0; i < G; ++i) {
    if (!primed) {
#pragma unroll
      for (int d = 0; d < D; ++d) {
        if ((d & 1) == 0) need(&ctl->prog, have, base + (d >> 1) + 1);
        ra[d] = ld16(rs, voff, ab + d * 4096);
        __builtin_amdgcn_sched_barrier(0);
      }
    }
    const bool more = i + 1 < G;
    const unsigned an = stg + stream_panel(job, i + more) * PANEL + wave * 1024;
    unsigned vnext = NOWHERE;
    f32x4 e = {0.f, 0.f, 0.f, 0.f}, o = {0.f, 0.f, 0.f, 0.f};
    v4u q[4];
    q[0] = bl[0]; q[1] = bl[64]; q[2] = bl[128];
#pragma unroll
    for (int s = 0; s < 32; ++s) {
      if (s + 3 < 32) q[(s + 3) & 3] = bl[(s + 3) * 64];
      e = mfma(br[s], ra[s % D], e);
      o = mfma(q[s & 3], ra[s % D], o);
      __builtin_amdgcn_sched_barrier(0);
      const int n = s + D;
      if (n < 32 && (n & 1) == 0) need(&ctl->prog, have, base + (n >> 1) + 1);
      if (n == 32) {
        if (have < base + NCH + D / 2) have = UNI(ctl->prog);
        CBAR();
        primed = more && have >= base + NCH + D / 2;
        if (primed) vnext = voff;
      }
      ra[s % D] = ld16(rs, n < 32 ? voff : vnext, (n < 32 ? ab : an) + (n & 31) * 4096);
      __builtin_amdgcn_sched_barrier(0);
    }
    need(&ctl->sfre[i & 1], (unsigned)i >> 1);
    LAS f32x4* red = (LAS f32x4*)(lds + LDS_SIMG + (i & 1) * 8192) + wave * 128;
    const int row = lane & 15, c4 = 2 * (lane >> 4);
    red[row * 8 + (c4 ^ (row & 7))] = f32x4{e.x, o.x, e.y, o.y};
    red[row * 8 + ((c4 + 1) ^ (row & 7))] = f32x4{e.z, o.z, e.w, o.w};
    arrive(&ctl->sfull[i & 1], lane);
    ab = an;
    base += NCH;
  }
}

__device__ __forceinline__ void take_panel(LAS char* lds, int i, int lane, f32x4& s0, f32x4& s1) {
  volatile LAS Ctl* ctl = (volatile LAS Ctl*)(lds + LDS_CTL);
  const LAS f32x4* red = (const LAS f32x4*)(lds + LDS_SIMG + (i & 1) * 8192) + ((lane & 56) + ((lane ^ (lane >> 3)) & 7));
  need(&ctl->sfull[i & 1], 4u * ((i >> 1) + 1));
  s0 = (red[0] + red[128]) + (red[256] + red[384]);
  s1 = (red[64] + red[192]) + (red[320] + red[448]);
  if (ctl->bad) s0 = s1 = __builtin_bit_cast(f32x4, v4u{QNAN, QNAN, QNAN, QNAN});
  asm volatile("" : "+v"(s0), "+v"(s1));
  ctl->sfre[i & 1] = (i >> 1) + 1;
}
__device__ __forceinline__ void finish_stream(const Params& p, rsrc_t rl, LAS char* lds, const Job& job, int par,
                                              unsigned call, int lane) {
  const int idx = p.idx & 3, grp = job.bs >> 3, b = job.bs & 7, G = stream_len(job);
  const bool staged = grp >= 4;
  const int ns = !staged ? G : G - (job.kind ? P_LASTB : P_LASTA), nx = job.kind == 2 ? NX : 0;
  const unsigned col = ((staged ? idx : grp) * 128 + b * 16) * 4;
  const rsrc_t ro = staged ? rl : mk(p.out[idx], OUT_BYTES);
  TREC_DECL();
  int unit = 0;
  for (int i = 0; i < ns; ++unit) {
    const int np = staged ? min(2, (i < nx ? nx : ns) - i) : 1;
    if ((unit & 1) == par) {
      const int a = stream_panel(job, i), a2 = stream_panel(job, i + np - 1);
      const unsigned slot = np == 2 ? (b & 4 ? a2 : a) * 4 + (b & 2) : a * 4 + (b >> 1);
      const unsigned ent = slot % 512 | (np == 2 ? 512 : 1536) | (panel_row(p, a) * SH * 4 + col) >> 7 << 16;
      const unsigned vo = staged ? OFF_YS + ((grp & 3) * 512 + slot) * 2048 + lane * 16 : half_offset(ent, 0, lane);
      const unsigned qs = staged ? 1024 : 8 * SH * 4;
      unsigned ticket = 0;
      for (int e = 0; e < np; ++e) {
        f32x4 s0, s1;
        take_panel(lds, i + e, lane, s0, s1);
        TREC(32 + i + e);
        ticket = queue_places(rl, staged && e == np - 1 && lane == 0, grp & 3, 1);
        st16(ro, vo + 2 * e * qs, __builtin_bit_cast(v4u, s0));
        st16(ro, vo + (2 * e + 1) * qs, __builtin_bit_cast(v4u, s1));
      }
      if (staged) {
        DRAIN();
        const unsigned pos = UNI(ticket) - (call - 1) * NHALF;
        flag(lane == 0 && pos < 256, rl, OFF_YL + ((grp & 3) * 256 + pos) * 128, call, ent);
      }
    }
    i += np;
  }
  if (staged && (unit & 1) == par) {
    constexpr int NE = P_LASTA > P_LASTB ? P_LASTA : P_LASTB;
    const rsrc_t rp = mk(p.out[grp], OUT_BYTES), rf = mk(p.ws[grp], WS_USED);
    f32x4 s[2 * NE];
    unsigned vo[NE];
#pragma unroll
    for (int e = 0; e < NE; ++e) {
      const int i = G - NE + e;
      vo[e] = i >= ns ? (panel_row(p, stream_panel(job, i)) + (lane >> 3)) * SH * 4 + col + (lane & 7) * 16 : NOWHERE;
      asm volatile("" : "+v"(vo[e]));
    }
#pragma unroll
    for (int e = 0; e < NE; ++e) {
      s[2 * e] = s[2 * e + 1] = f32x4{0.f, 0.f, 0.f, 0.f};
      if (G - NE + e < ns) continue;
      take_panel(lds, G - NE + e, lane, s[2 * e], s[2 * e + 1]);
      TREC(32 + G - NE + e);
    }
#pragma unroll
    for (int q = 0; q < 2 * NE; ++q) st16(rp, vo[q >> 1] + (q & 1) * (8 * SH * 4), __builtin_bit_cast(v4u, s[q]));
    DRAIN();
    flag(lane == 0, rf, OFF_YF + (idx * 256 + NYF + (job.kind >> 1) * 12 + job.j * 4 + (b >> 1)) * 128, call);
  }
  DIAG_DRAIN();
  STAMP(1);
  TREC(55);
  TREC_DUMP(40);
}

__global__ __launch_bounds__(NTHR, 2) void ag_gemm(const Params p) {
  extern __shared__ __attribute__((aligned(16))) char smem[];
  LAS char* lds = (LAS char*)smem;
  const int lane = threadIdx.x & 63;
  const int wave = UNI(threadIdx.x >> 6);
  const int blk = blockIdx.x, j = blk >> 3;
  char* peerws = p.wsb[blk & 7];
  asm volatile("" :: "s"(p.x), "s"(p.w), "s"(p.myws), "s"(p.idx), "s"(p.rot), "s"(peerws));
  const int grp = (blk + p.rot) & 7;
  const int idx = p.idx & 3;
  const bool cross = grp >= 4;
  char* myws = p.myws;
  const rsrc_t rl = mk(myws, WS_USED), rd = mk(peerws, WS_USED);
  STAMP(0);
  char* counter = myws + OFF_CNT + (blk * 8 + wave) * 4;
  const unsigned last_call = __hip_atomic_load((gu32*)counter, __ATOMIC_RELAXED, __HIP_MEMORY_SCOPE_SYSTEM);
  const unsigned long long tw = NOW();
  const int jg = j - NC - NU;
  const bool keeper = cross || grp == idx;
  if (wave == 6 && (unsigned)jg < (unsigned)NDRAW && grp != idx) {
    const int u = cross ? jg & 3 : jg & 7, c = cross ? jg >> 2 : jg >> 3;
    f32x4 v[16];
    v4u h[8];
    if (cross) {
      load_inputs(p, true, (grp - 4) * 4 + u, c, lane, v);
      __builtin_amdgcn_s_barrier();
      to_fragments(true, v, (LAS v4u*)(lds + 6 * 8192), lane, h);
    } else {
      load_inputs(p, false, u, c, lane, v);
      __builtin_amdgcn_s_barrier();
      to_fragments(false, v, (LAS v4u*)(lds + 6 * 8192), lane, h);
    }
    const unsigned call = UNI(last_call) + 1u, stg = OFF_STG + (call & 1u) * STG_BYTES;
    unsigned o0, o1, fl, kl;
    item_offsets(cross, stg, landing_panel(cross, cross, u, idx, grp & 3), c, o0, o1, fl);
    STAMP(7);
    sleep_until<1>(tw, P_PACE0 + jg * P_PACE / 2);
    store_item(rd, o0, o1, h, lane);
    STAMP(6);
    item_offsets(cross, stg, cross ? NAP + (grp - 4) * 8 + 2 * u : u, c, o0, o1, kl);
    if (keeper) store_item(rl, o0, o1, h, lane);
    DRAIN();
    flag(lane == 0, rd, fl, call);
    flag(lane == 0 && keeper, rl, kl, call);
  } else {
    if (threadIdx.x < 16) ((volatile LAS unsigned*)(lds + LDS_CTL))[threadIdx.x] = 0;
    LDSWAIT();
    __builtin_amdgcn_s_barrier();
  }

  if (jg >= 0) {
    volatile LAS Ctl* ctl = (volatile LAS Ctl*)(lds + LDS_CTL);
    const bool direct = wave >= 4;
    const int di = wave == 6 ? jg : wave == 5 ? NG + (jg + P_ROT1) % NG : 2 * NG + (jg + P_ROT2) % NG;
    const int leave = P_PACE0 + di * P_PACE / 2, it = wave * (NG * 8) + jg * 8 + grp;
    const bool work = direct ? (grp == idx || di >= NDRAW) && di < ND : it < NCV;
    bool isw = false;
    int li = 0, g = 0, c = 0, u = 0;
    f32x4 v[16];
    __builtin_amdgcn_s_setprio(3);
    if (work) {
      const int k = it / 5 + ND, m = direct ? grp - 4 : it % 5, xi = k < 128 ? k : it - (NCV - 512);
      const bool xc = direct ? cross : k < 128 ? m < 4 : xi >= NIN;
      li = direct ? di : xi >= NIN ? xi - NIN + 128 : xi;
      link_item(xc, li, isw, u, c);
      g = isw ? m * 4 + u : xc ? NI + u : u;
      sleep_until<2>(tw, direct ? leave - P_LEAD : it * P_CSTEP >> 8);
      load_inputs(p, isw, g, c, lane, v);
    }
    const unsigned call = UNI(last_call) + 1u, stg = OFF_STG + (call & 1u) * STG_BYTES;
    if (work) {
      LAS v4u* scr = (LAS v4u*)(lds + wave * 8192);
      v4u h[8];
      to_fragments(isw, v, scr, lane, h);
      unsigned o0, o1, fl, hdr = 0, kl = NOWHERE;
      item_offsets(isw, stg, isw ? NAP + 2 * g : g < NI ? g : PSENT + u, c, o0, o1, fl);
      const rsrc_t rs = mk(direct ? p.ws[grp] : myws, WS_USED);
      STAMP(7);
      if (!direct || keeper) store_item(rl, o0, o1, h, lane);
      if (li < NPI && !(direct && grp == idx)) {
        const float ss = sumsq(v[0]) + sumsq(v[4]) + sumsq(v[8]) + sumsq(v[12]);
        const unsigned base = (unsigned)min(max(top_exponent(ss, 6.76f / 1024.f) - 127 + 15 - 7, 0), 24);
        pack8<false>(h, base);
        const unsigned total = list_plane(h[7], (LAS unsigned short*)scr, LIST_CAP, lane);
        const bool whole = total > (unsigned)LIST_CAP;
        if (!whole) {
          LDSWAIT();
          h[7] = *(volatile LAS v4u*)(scr + (lane & 15));
        }
        const unsigned nl = whole ? 64 : max((total + 31) >> 5, 1u) * 4;
        hdr = base | (whole ? 1u << 16 : total << 8) | nl << 24;
        if (direct) {
          if (keeper) kl = fl;
          item_offsets(isw, stg, landing_panel(cross, isw, u, idx, grp & 3), c, o0, o1, fl);
          fl += OFF_AF + (call & 1u) * PF_BYTES - OFF_PF;
          sleep_until<1>(tw, leave);
        } else {
          o0 = OFF_EXP + (isw ? 512 + g * 32 + c : g * 16 + c) * 8192;
          o1 = o0 + 4096;
        }
        store_item(rs, o0, o1, h, lane, nl);
      }
      STAMP(6);
      DRAIN();
      flag(lane == 0, rs, fl, call, hdr);
      flag(lane == 0, rl, kl, call);
    }
    STAMP(2);
    if (lane == 0) st_sys(counter, call);
    arrive(&ctl->conv, lane);
    const Job job = make_job(grp ^ 4, jg);
    if (wave < 4) {
      __builtin_amdgcn_s_setprio(0);
      unsigned have = 0;
      STAMP(8);
      switch (job.mt) {
        case 3: gemm_tile<3>(rl, lds, job, have, stg, wave, lane); break;
        case 4: gemm_tile<4>(rl, lds, job, have, stg, wave, lane); break;
        case 5: gemm_tile<5>(rl, lds, job, have, stg, wave, lane); break;
        default: gemm_tile<6>(rl, lds, job, have, stg, wave, lane); break;
      }
      STAMP(16);
      have = UNI(have);
      gemm_stream(rl, lds, job, have, stg, wave, lane);
      STAMP(1);
    } else if (wave > 4) {
      finish_tile(p, rl, lds, job, wave - 5, call, lane);
      finish_stream(p, rl, lds, job, wave - 5, call, lane);
    } else {
      const int np = job.mt + 4, l2 = np <= 8 ? 3 : 4, q = lane & ((1 << l2) - 1);
      const unsigned fo = OFF_PF + (q < job.mt ? job.a0 + q : NAP + job.b0 + q - job.mt) * (NCH * 128);
      const unsigned total = (stream_len(job) + 1) * NCH;
      unsigned pub = 0, win = 64, idle = 0;
      sleep_until<8>(tw, P_GSTART);
      TREC_DECL();
      auto give_up = [&]() {
        st_sys(myws + OFF_TMO, call);
        ctl->bad = 1;
        LDSWAIT();
        ctl->prog = 0xffffffffu;
      };
      while (pub <= total - P_FAST) {
        CBAR();
        const bool tile = pub < NCH;
        const unsigned most = tile ? 63 >> l2 : 32, ahead = tile ? lane >> l2 : lane, c = pub + ahead;
        win = min(win, most);
        const bool act = ahead < win && (tile ? q < np && c < NCH : c < total);
        const unsigned v = ld4(rl, !act ? (lane == 63 ? OFF_TMO : NOWHERE) : tile ? fo + c * 128
                                   : OFF_PF + (stream_panel(job, (c >> 4) - 1) * NCH + (c & 15)) * 128);
        if (LANE(v, 63) == call || NOW() - tw > DEADLINE) { give_up(); return; }
        const unsigned n = (unsigned)__builtin_ctzll(__ballot(act ? (int)(v - call) < 0 : ahead >= win)) >> (tile ? l2 : 0);
        idle = n ? 0 : idle + 1;
        win = n >= win ? 64 : tile ? n + 1 : idle > 2 ? 2 : 8;
        pub = min(pub + n, tile ? (unsigned)NCH : total);
        ctl->prog = pub;
        if (n && pub + NCH > total) TREC(32 + pub + NCH - total);
        if (tile) {
          if (pub == NCH) { win = 64; need(&ctl->full[0], 1); }
          else if (!n) __builtin_amdgcn_s_sleep(8);
        }
        while (pub < total && pub >= NCH + 40 + 4 * UNI(ctl->sfull[0] + ctl->sfull[1])) __builtin_amdgcn_s_sleep(4);
      }
      const unsigned fl = lane < NCH ? OFF_PF + (stream_panel(job, stream_len(job) - 1) * NCH + lane) * 128
                                     : lane == 63 ? OFF_TMO : NOWHERE;
      unsigned v[4];
#pragma unroll
      for (int d = 0; d < 4; ++d) {
        v[d] = ld4(rl, pub < total ? fl : NOWHERE);
        CBAR();
        __builtin_amdgcn_s_sleep(P_FNAP);
      }
      while (pub < total) {
        bool late = NOW() - tw > DEADLINE;
#pragma unroll
        for (int d = 0; d < 4; ++d) {
          const unsigned was = pub;
          pub = total - NCH + __builtin_ctzll(__ballot(lane >= NCH || (lane + total >= pub + NCH && (int)(v[d] - call) < 0)));
          late |= LANE(v[d], 63) == call;
          ctl->prog = pub;
          if (pub > was) TREC(32 + pub + NCH - total);
          v[d] = ld4(rl, lane + total >= pub + NCH ? fl : NOWHERE);
          CBAR();
          __builtin_amdgcn_s_sleep(P_FNAP);
        }
        if (late) { give_up(); return; }
      }
      TREC_DUMP(40);
    }
    return;
  }

  const unsigned call = UNI(last_call) + 1u, stg = OFF_STG + (call & 1u) * STG_BYTES;
  const unsigned aoff = OFF_AF + (call & 1u) * PF_BYTES - OFF_PF;
  if (lane == 0) st_sys(counter, call);
  if (blk == 0 && wave == 1) st_sys(myws + OFF_XCC, 8 + (__builtin_amdgcn_s_getreg(20 | 31 << 11) & 7));
  if (j < NC && grp != idx) {
    struct Ring { unsigned full[NL], done; };
    volatile LAS Ring* rg = (volatile LAS Ring*)(lds + LDS_CTL);
    constexpr int SLOT = 10240;
    volatile LAS v2u* meta = (volatile LAS v2u*)(lds + 2 * NL * SLOT);
    const int n = cross ? NXY + NYF : NIN;
    if (wave == 0) {
      __builtin_amdgcn_s_setprio(3);
      const unsigned long long peer_out = uniform64(p.out[grp]), peer_ws = uniform64(p.ws[grp]);
      unsigned pl[3] = {NOWHERE, NOWHERE, NOWHERE}, ph[3] = {0, 0, 0};
      const int cnt = (n - ND - j + NC - 1) / NC, ry = ((cross ? NXY : n) - ND - j + NC - 1) / NC;
      unsigned lane0 = lane == 0 ? ~0u : 0u, lane1 = lane == 1 ? ~0u : 0u;
      asm volatile("" : "+v"(lane0), "+v"(lane1));
      int r = 0, l = 0, k = 0;
      TREC_DECL();
      auto flush = [&]() {
        DRAIN();
        flag(lane < 3, rd, bfi(lane0, pl[0], bfi(lane1, pl[1], pl[2])), call, bfi(lane0, ph[0], bfi(lane1, ph[1], ph[2])));
        pl[0] = pl[1] = pl[2] = NOWHERE;
      };
      auto rounds = [&](int end, auto wide, auto yphase) {
        constexpr int W = wide() ? P_WIDE : P_WIN;
        static_assert(W <= (wide() ? 20 : 12) && !(wide() && yphase()), "window");
        for (; r < end; ++r) {
          LAS char* sl = lds + (2 * l + (k & 1)) * SLOT + lane * 16;
          unsigned got;
          v2u m;
          v4u a0, a1, v[8];
#define READ_SLOT() do { \
          const unsigned fu = rg->full[l]; \
          m = meta[2 * l + (k & 1)]; \
          a0 = *(volatile LAS v4u*)(sl + 8192); a1 = *(volatile LAS v4u*)(sl + 9216); \
          _Pragma("unroll") for (int q = 0; q < 8; ++q) v[q] = *(volatile LAS v4u*)(sl + q * 1024); \
          got = UNI(fu); } while (0)
          READ_SLOT();
          if (__builtin_expect(got <= (unsigned)k, 0)) {
            flush();
            while (UNI(rg->full[l]) <= (unsigned)k) __builtin_amdgcn_s_sleep(0);
            READ_SLOT();
          }
#undef READ_SLOT
          if (got == 0xffffffffu) { r = cnt; rg->done = 0xffffffffu; break; }
          const unsigned hd = UNI(m.y);
          const bool to_out = yphase() && ((hd >> 17) & 1);
          const rsrc_t rs = mk((char*)(to_out ? peer_out : peer_ws), to_out ? OUT_BYTES : WS_USED);
          const unsigned fo = wide() ? bfi(lane0, pl[0], NOWHERE) : bfi(lane0, pl[1], bfi(lane1, pl[0], NOWHERE));
          const v2u fv = {call, wide() ? ph[0] : bfi(lane0, ph[1], ph[0])};
#pragma unroll
          for (int q = 0; q < 8; ++q) asm volatile("" : "+v"(v[q]));
          rg->done = r + 1;
          asm volatile("s_waitcnt vmcnt(%0)" :: "n"(W) : "memory");
#pragma unroll
          for (int q = 0; q < 4; ++q) st16(rs, a0[q], v[q]);
          asm volatile("s_waitcnt vmcnt(%0)" :: "n"(W) : "memory");
          if (yphase() && (((hd | ph[2]) >> 18) & 1)) asm volatile("s_waitcnt vmcnt(6)" ::: "memory");
          __builtin_amdgcn_raw_buffer_store_b64(fv, rd, (int)fo, 0, AUX_SYS);
          CBAR();
#pragma unroll
          for (int q = 0; q < 4; ++q) st16(rs, a1[q], v[4 + q]);
          CBAR();
          if (!(r & 1)) TREC(r >> 1);
          if (r >= cnt - 6) TREC(48 + r - cnt + 6);
          pl[0] = wide() ? pl[1] : NOWHERE; ph[0] = ph[1]; pl[1] = pl[2]; ph[1] = ph[2];
          pl[2] = UNI(m.x); ph[2] = hd;
          if (++l == NL) { l = 0; ++k; }
        }
      };
      sleep_until<2>(tw, P_PACE0 + ND * P_PACE / 2 + P_HOLD);
      while (r < ry) {
        const int raw = cross ? 0 : max(r - (NPI - ND) / NC, 0);
        if (r < ry - P_NARROW && (int)(NOW() - tw) > P_LATE + ND * P_PACE / 2 + r * P_RATE / 10 + raw * 7)
          rounds(min(r + 2, ry), std::true_type(), std::false_type());
        else rounds(min(r + 2, ry), std::false_type(), std::false_type());
      }
      rounds(cnt, std::false_type(), std::true_type());
      if (!cross && pl[0] == NOWHERE && pl[1] != NOWHERE) {
        asm volatile("s_waitcnt vmcnt(8)" ::: "memory");
        flag(lane == 0, rd, pl[1], call, ph[1]);
        TREC(54);
        asm volatile("s_waitcnt vmcnt(1)" ::: "memory");
        flag(lane == 0, rd, pl[2], call, ph[2]);
      } else {
        flush();
      }
      STAMP(1);
      TREC_DUMP(8);
      return;
    }
    const int l = wave - 1, first = ND + l * NC + j, cnt = (n - first + WL - 1) / WL;
    const int i = min(first + lane * WL, n - 1);
    const bool isy = cross && i >= NXY, packed = !isy && i < NPI;
    unsigned s0, s1, sf, d0, d1, df, hdv = 64u << 24;
    {
      bool isw; int u, c;
      link_item(cross, i, isw, u, c);
      const int g = isw ? (grp - 4) * 4 + u : cross ? NI + u : u;
      item_offsets(isw, stg, isw ? NAP + 2 * g : cross ? PSENT + u : u, c, s0, s1, sf);
      item_offsets(isw, stg, landing_panel(cross, isw, u, idx, grp & 3), c, d0, d1, df);
      if (packed) {
        s0 = OFF_EXP + (isw ? 512 + g * 32 + c : g * 16 + c) * 8192; s1 = s0 + 4096;
        df += aoff;
      }
      if (isy) {
        sf = OFF_YL + ((grp & 3) * 128 + i - NXY) * 256;
        df = OFF_YF + (idx * 256 + i - NXY) * 128;
      }
      if (lane == 63) sf = OFF_TMO;
    }
    unsigned long long ready = 0;
    for (int k = 0; k < cnt; ++k) {
      while (!((ready >> k) & 1)) {
        CBAR();
        const bool ask = ((lane >= k && lane < min(cnt, k < 2 ? k + 3 : 63)) || lane == 63) && !((ready >> lane) & 1);
        const bool two = isy && lane < 63;
        const v2u f = ldflag(rl, ask ? sf : NOWHERE);
        const v2u g = ldflag(rl, ask && two ? sf + 128 : NOWHERE);
        const bool ok = f.x == call && (!two || g.x == call);
        if (ok && packed) hdv = f.y;
        if (ok && isy) { d0 = f.y; d1 = g.y; }
        ready |= __ballot(ok);
        if ((ready >> 63) || NOW() - tw > DEADLINE) {
          st_sys(myws + OFF_TMO, call);
          rg->full[l] = 0xffffffffu;
          return;
        }
      }
      CBAR();
      const int yi = cross ? first + k * WL - NXY : -1;
      unsigned hd = LANE(hdv, k), a0 = LANE(s0, k), a1 = LANE(s1, k), e0 = LANE(d0, k), e1 = LANE(d1, k);
      if (yi >= 0) {
        a0 = OFF_YS + ((grp & 3) * 512 + (e0 & 511)) * 2048;
        a1 = OFF_YS + ((grp & 3) * 512 + (e1 & 511)) * 2048;
        hd |= 1u << 17 | ((e0 | e1) & 1024) << 8;
      }
      v4u h[8];
#pragma unroll
      for (int q = 0; q < 4; ++q) h[q] = ld16(rl, lane * 16 + q * 1024, a0);
#pragma unroll
      for (int q = 0; q < 3; ++q) h[4 + q] = ld16(rl, lane * 16 + q * 1024, a1);
      h[7] = ld16(rl, lane < (hd >> 24) ? lane * 16 + 3072 : NOWHERE, a1);
      if (yi >= 0 && yi < NYF - P_NRAWY && !((hd >> 18) & 1)) {
        const float ss = sumsq(__builtin_bit_cast(f32x4, h[0])) + sumsq(__builtin_bit_cast(f32x4, h[2])) +
                         sumsq(__builtin_bit_cast(f32x4, h[4])) + sumsq(__builtin_bit_cast(f32x4, h[6]));
        const unsigned base = (unsigned)min(max(top_exponent(ss, 20.f / 1024.f) - 15, 0), 240);
        pack8<true>(h, base);
        LAS unsigned* scr = (LAS unsigned*)(lds + 2 * NL * SLOT + 128 + l * 256);
        const unsigned total = list_plane(h[7], (LAS unsigned short*)(scr + 2), LIST_CAP - 4, lane);
        if (total <= LIST_CAP - 4) {
          scr[lane & 1] = lane & 1 ? e1 : e0;
          LDSWAIT();
          h[7] = *(volatile LAS v4u*)(scr + 4 * (lane & 15));
          hd = base | total << 8 | ((total + 35) >> 5) * 4 << 24;
          e0 = OFF_YP + (idx * NAP + yi) * 8192; e1 = e0 + 4096;
        } else {
          unpack8<true>(h, base);
        }
      }
      v4u t0, t1;
#pragma unroll
      for (int q = 0; q < 4; ++q) {
        t0[q] = (hd >> 17) & 1 ? half_offset(e0, q, lane) : e0 + q * 1024 + lane * 16;
        t1[q] = (hd >> 17) & 1 ? half_offset(e1, q, lane) : e1 + q * 1024 + lane * 16;
      }
      if (lane >= (hd >> 24)) t1[3] = NOWHERE;
      if (k >= 2) need(&rg->done, (k - 2) * NL + l + 1);
      LAS v4u* sl = (LAS v4u*)(lds + (2 * l + (k & 1)) * SLOT);
      sl[512 + lane] = t0;
      sl[576 + lane] = t1;
      meta[2 * l + (k & 1)] = v2u{LANE(df, k), hd};
#pragma unroll
      for (int q = 0; q < 8; ++q) sl[q * 64 + lane] = h[q];
      LDSWAIT();
      rg->full[l] = k + 1;
    }
    STAMP(1);
    return;
  }

  char* marker = p.out[idx] + (size_t)panel_row(p, 0) * SH * 4;
  auto await = [&](unsigned off, unsigned& hdr, unsigned& spins, int nap) {
    for (spins = 1;; ++spins) {
      const v2u f = ldflag(rl, lane == 0 ? off : lane == 1 && !(spins & 15) ? OFF_TMO : NOWHERE);
      hdr = UNI(f.y);
      if ((int)(UNI(f.x) - call) >= 0) return true;
      if (LANE(f.x, 1) == call || NOW() - tw > DEADLINE) {
        st_sys(myws + OFF_TMO, call);
        st_sys(marker, QNAN);
        return false;
      }
      if (nap) __builtin_amdgcn_s_sleep(16);
      CBAR();
    }
  };
  const int sw = wave * (NC + NU) + j, sx = sw - 3 * P_SI;
  const int lk = grp != idx ? grp : sx < 0 ? (idx + 1 + sw % 3) & 3 : 4 + (sx & 3);
  const bool xl = lk >= 4;
  const int nu = xl ? NU : P_NUI, WU = xl ? WUX : WUI;
  const int me = grp != idx ? wave * nu + j - NC : nu * NWAVE + (xl ? sx >> 2 : sw / 3);
  if ((grp != idx && j - NC >= nu) || me >= WU) return;
  LAS unsigned* plane = (LAS unsigned*)(lds + wave * 1024);
  {
    const int n = xl ? NXY : NPI, first = NDRAW + me, cnt = (n - first + WU - 1) / WU;
    bool isw; int u, c;
    link_item(xl, min(first + lane * WU, n - 1), isw, u, c);
    unsigned o0, o1, fl;
    item_offsets(isw, stg, landing_panel(xl, isw, u, lk & 3, idx), c, o0, o1, fl);
    unsigned pend = NOWHERE, hdr = 0;
    bool have_next = false;
    unsigned long long tseen = tw;
    sleep_until<4>(tw, 200 + first * P_PACE / 2);
    TREC_DECL();
    for (int r = 0; r < cnt; ++r) {
      const unsigned fr = LANE(fl, r);
      *(LAS v4u*)(plane + lane * 4) = v4u{0, 0, 0, 0};
      if (!have_next) {
        DRAIN();
        flag(lane == 0, rl, pend, call);
        pend = NOWHERE;
        sleep_until<2>(tseen, WU * 25 / 2 - P_UGAP);
        unsigned spins;
        if (!await(fr + aoff, hdr, spins, 0)) return;
        tseen = NOW() - (spins == 1 ? 60 : 0);
      }
      CBAR();
      TREC(4 * r);
      const unsigned a0 = LANE(o0, r), a1 = LANE(o1, r);
      const bool whole = (hdr >> 16) & 1;
      const unsigned e2 = ld4(rl, lane * 4 + 3072, a1);
      CBAR();
      v4u h[8];
#pragma unroll
      for (int q = 0; q < 4; ++q) h[q] = ld16(rl, lane * 16 + q * 1024, a0);
#pragma unroll
      for (int q = 0; q < 3; ++q) h[4 + q] = ld16(rl, lane * 16 + q * 1024, a1);
      h[7] = ld16(rl, whole ? lane * 16 + 3072 : NOWHERE, a1);
      const v2u fn = ldflag(rl, lane == 0 ? LANE(fl, min(r + 1, cnt - 1)) + aoff : NOWHERE);
      CBAR();
      const v4u pl = unlist_plane(e2, 0, (hdr >> 8) & 255, plane, lane);
      DRAIN();
      flag(lane == 0, rl, pend, call);
      h[7] |= pl;
      unpack8<false>(h, hdr & 31);
      store_item(rl, a0, a1, h, lane);
      pend = fr;
      TREC(4 * r + 2);
      hdr = UNI(fn.y);
      have_next = (int)(UNI(fn.x) - call) >= 0;
    }
    DRAIN();
    flag(lane == 0, rl, pend, call);
    TREC_DUMP(8);
    STAMP(2);
  }
  const rsrc_t ro = mk(p.out[idx], OUT_BYTES);
  for (int yi = me; xl && yi < NYF + NYD; yi += WU) {
    *(LAS v4u*)(plane + lane * 4) = v4u{0, 0, 0, 0};
    unsigned hdr, spins;
    if (yi >= NYF) sleep_until<8>(tw, 5000);
    if (!await(OFF_YF + ((lk & 3) * 256 + yi) * 128, hdr, spins, yi < NYF - P_NRAWY - 16)) return;
    if (yi >= NYF || ((hdr >> 17) & 1)) continue;
    CBAR();
    const unsigned a0 = OFF_YP + ((lk & 3) * NAP + yi) * 8192;
    const unsigned e2 = ld4(rl, lane * 4 + 7168, a0);
    CBAR();
    v4u h[8];
#pragma unroll
    for (int q = 0; q < 7; ++q) h[q] = ld16(rl, lane * 16 + (q & 3) * 1024, a0 + (q >> 2) * 4096);
    CBAR();
    h[7] = unlist_plane(e2, 4, (hdr >> 8) & 255, plane, lane);
    unpack8<true>(h, hdr & 255);
#pragma unroll
    for (int q = 0; q < 8; ++q) st16(ro, half_offset(LANE(e2, q >> 2), q & 3, lane), h[q]);
  }
  DIAG_DRAIN();
  STAMP(1);
}

}

extern "C" void kernel_launch(int rank, int nranks, ncclComm_t comm, hipStream_t stream, void* const* d_in,
                              const int* in_sizes, int n_in, void* d_out, int out_size, const int* in_dtypes,
                              int out_dtype, void* const* d_out_all, void* const* d_ws_all, size_t ws_bytes) {
  using namespace agg;
  if (nranks != NR || n_in != 2 || ws_bytes < WS_USED || out_size != 4096 * SH || in_sizes[0] != SH * KD ||
      in_sizes[1] != KD * SH || (unsigned)rank >= (unsigned)NR) {
    fprintf(stderr, "ag_gemm: unexpected problem shape\n");
    return;
  }
  static const int kClass[2][4] = {{0, 2, 3, 6}, {1, 4, 5, 7}};
  Params p;
  memset(&p, 0, sizeof(p));
  int cls = 0;
  for (int c = 0; c < 2; ++c)
    for (int i = 0; i < 4; ++i) if (kClass[c][i] == rank) { cls = c; p.idx = i; }
  for (int r = 0; r < NR; ++r) {
    const int q = kClass[cls ^ (r >> 2)][r & 3];
    p.ranks |= (unsigned)q << (4 * r);
    p.out[r] = (char*)d_out_all[q];
    p.ws[r] = (char*)d_ws_all[q];
  }
  const unsigned seen = *(volatile unsigned*)((char*)d_ws_all[rank] + OFF_XCC);
  p.rot = seen >> 3 == 1 ? seen & 7 : 0;
  for (int r = 0; r < NR; ++r) p.wsb[r] = p.ws[(r + p.rot) & 7];
  p.x = (const float*)d_in[0];
  p.w = (const float*)d_in[1];
  p.myws = (char*)d_ws_all[rank];
  HIP_CHECK(hipFuncSetAttribute((const void*)ag_gemm, hipFuncAttributeMaxDynamicSharedMemorySize, LDS_BYTES));
  ag_gemm<<<GRID, NTHR, LDS_BYTES, stream>>>(p);
  HIP_CHECK(hipGetLastError());
}
```

```cpp
#include <hip/hip_runtime.h>
#include <rccl/rccl.h>
#include <cstdio>
#include <cstring>
#include <type_traits>

#define HIP_CHECK(expr) do { hipError_t _e = (expr); if (_e != hipSuccess) { \
    fprintf(stderr, "HIP error %s at %s:%d\n", hipGetErrorString(_e), __FILE__, __LINE__); \
    return; } } while (0)

#define P_ND 66
#define P_NDRAW 16
#define P_PACE0 80
#define P_PACE 25
#define P_LEAD 330
#define P_HOLD 40
#define P_CSTEP 250
#define P_GSTART 950
#define P_FAST 16
#define P_FNAP 3
#define P_WIN 11
#define P_WIDE 16
#define P_LATE 105
#define P_RATE 503
#define P_NARROW 4
#define P_NUI 3
#define P_SI 8
#define P_NRAW 48
#define P_NRAWY 22
#define P_ROT1 15
#define P_ROT2 5
#define P_UGAP 140
#define P_LASTA 3
#define P_LASTB 1

namespace agg {

#define GAS __attribute__((address_space(1)))
#define LAS __attribute__((address_space(3)))
typedef unsigned v4u __attribute__((ext_vector_type(4)));
typedef unsigned v2u __attribute__((ext_vector_type(2)));
typedef unsigned short us2 __attribute__((ext_vector_type(2)));
typedef float f32x4 __attribute__((ext_vector_type(4)));
typedef float f32x2 __attribute__((ext_vector_type(2)));
typedef _Float16 h8 __attribute__((ext_vector_type(8)));
typedef GAS unsigned gu32;
typedef __amdgpu_buffer_rsrc_t rsrc_t;

constexpr int NR = 8;
constexpr int KD = 4096, SH = 512;
constexpr int PANEL = 131072;
constexpr int NCH = 16, CHUNK = 8192;
constexpr int NI = 23, NX = 32 - NI;
constexpr int NB1 = 8;
constexpr int NAP = 128, NBP = 64, PSENT = NAP + NBP, NPAN = PSENT + 16;
constexpr int GRID = 256, NWAVE = 7, NTHR = NWAVE * 64;
constexpr int NC = 4, NL = NWAVE - 1, WL = NC * NL;
constexpr int NU = 4;
constexpr int SX = ((NC + NU) * NWAVE - 3 * P_SI) / 4;
constexpr int WUI = P_NUI * NWAVE + P_SI, WUX = NU * NWAVE + SX;
constexpr int NG = 32 - NC - NU;
constexpr int ND = P_ND, NDRAW = P_NDRAW;
constexpr int NIN = NI * NCH, NXY = 128 + NX * NCH;
constexpr int NPI = NIN - P_NRAW;
constexpr int NCV = 5 * (128 - ND) + 384;
constexpr int LEN_A = NI - NB1 - 2, LEN_B = NX + 2;
constexpr int NHALF = 2 * (NI + 3 * NB1) + 12 * ((LEN_A - P_LASTA + 1) / 2) + 12 * ((NX + 1) / 2 + (3 - P_LASTB) / 2);
constexpr int NYF = NHALF / 2, NYD = 24;
static_assert(NG == 24 && NI == 23 && NHALF % 2 == 0 && NYF <= NAP, "the deal of tiles and streams");
static_assert(NG * 32 >= NCV && ND <= 3 * NG && NDRAW <= NG && NDRAW <= ND, "one item per converting wave");
static_assert(NIN <= 63 * WL && NXY + NYF <= 63 * WL && NIN <= 64 * WUI && SX >= 0, "one lane per item");
static_assert(NXY <= NPI && P_LASTA >= 1 && P_LASTA <= 4 && P_LASTB >= 1 && P_LASTB <= 2, "");

constexpr unsigned OFF_CNT = 0;
constexpr unsigned OFF_TMO = 16384;
constexpr unsigned OFF_DEAL = 18432;
constexpr unsigned OFF_YQ = 20480;
constexpr unsigned OFF_BAR = 24576;
constexpr unsigned OFF_ROLE = 32768;
constexpr unsigned OFF_YF = 65536;
constexpr unsigned OFF_YL = 196608;
constexpr unsigned OFF_PF = 393216;
constexpr unsigned PF_BYTES = NPAN * NCH * 128;
constexpr unsigned OFF_AF = 1u << 20;
constexpr unsigned OFF_YS = 2u << 20;
constexpr unsigned OFF_YP = OFF_YS + (4u << 20);
constexpr unsigned OFF_EXP = OFF_YP + (4u << 20);
constexpr unsigned OFF_STG = OFF_EXP + (8u << 20);
constexpr unsigned STG_BYTES = (unsigned)NPAN * PANEL;
constexpr unsigned OFF_DBG = OFF_STG + 2 * STG_BYTES;
constexpr unsigned WS_USED = OFF_DBG + (1u << 20);
static_assert(OFF_YF + 1024 * 128 <= OFF_YL && OFF_YL + 1024 * 128 <= OFF_PF && OFF_PF + PF_BYTES <= OFF_AF &&
              OFF_AF + 2 * PF_BYTES <= OFF_YS, "layout");
constexpr unsigned OUT_BYTES = 4096u * SH * 4u;
constexpr unsigned NOWHERE = 0xf0000000u;
constexpr unsigned long long DEADLINE = 50000000ull;
constexpr int AUX_SYS = 17;
constexpr unsigned QNAN = 0x7fc00000u;

constexpr int LDS_RED = 65536;
constexpr int LDS_CTL = 2 * LDS_RED;
constexpr int LDS_SIMG = LDS_CTL + 64;
constexpr int LDS_BYTES = LDS_SIMG + 16384;

struct Params {
  const float* x;
  const float* w;
  char* myws;
  int idx;
  unsigned ranks;
  char* out[NR];
  char* ws[NR];
};

#define DRAIN() asm volatile("s_waitcnt vmcnt(0)" ::: "memory")
#define LDSWAIT() asm volatile("s_waitcnt lgkmcnt(0)" ::: "memory")
#define CBAR() asm volatile("" ::: "memory")
#define NOW() __builtin_amdgcn_s_memrealtime()
#define UNI(v) ((unsigned)__builtin_amdgcn_readfirstlane(v))
#define LANE(v, l) ((unsigned)__builtin_amdgcn_readlane(v, l))

__device__ __forceinline__ rsrc_t mk(const void* p, unsigned bytes) {
  return __builtin_amdgcn_make_buffer_rsrc((void*)p, 0, (int)bytes, 0x00020000);
}
__device__ __forceinline__ unsigned long long uniform64(const void* q) {
  const unsigned long long v = (unsigned long long)q;
  return (unsigned long long)UNI((unsigned)(v >> 32)) << 32 | UNI((unsigned)v);
}
__device__ __forceinline__ unsigned pk(float a, float b) {
  typedef _Float16 h2 __attribute__((ext_vector_type(2)));
  h2 v = {(_Float16)a, (_Float16)b};
  return __builtin_bit_cast(unsigned, v);
}
__device__ __forceinline__ void st_sys(char* p, unsigned v) {
  __hip_atomic_store((gu32*)p, v, __ATOMIC_RELAXED, __HIP_MEMORY_SCOPE_SYSTEM);
}
__device__ __forceinline__ void st16(rsrc_t r, unsigned voff, v4u v) {
  __builtin_amdgcn_raw_buffer_store_b128(v, r, (int)voff, 0, AUX_SYS);
}
__device__ __forceinline__ v4u ld16(rsrc_t r, unsigned voff, unsigned soff) {
  return __builtin_amdgcn_raw_buffer_load_b128(r, (int)voff, (int)soff, AUX_SYS);
}
__device__ __forceinline__ unsigned ld4(rsrc_t r, unsigned voff, unsigned soff = 0) {
  return __builtin_amdgcn_raw_buffer_load_b32(r, (int)voff, (int)soff, AUX_SYS);
}
__device__ __forceinline__ void flag(bool on, rsrc_t r, unsigned off, unsigned call, unsigned hdr = 0) {
  __builtin_amdgcn_raw_buffer_store_b64(v2u{call, hdr}, r, (int)(on ? off : NOWHERE), 0, AUX_SYS);
}
__device__ __forceinline__ v2u ldflag(rsrc_t r, unsigned off) {
  return __builtin_amdgcn_raw_buffer_load_b64(r, (int)off, 0, AUX_SYS);
}
__device__ __forceinline__ void store_item(rsrc_t r, unsigned o0, unsigned o1, const v4u (&h)[8], int lane, unsigned nl = 64) {
#pragma unroll
  for (int s = 0; s < 4; ++s) st16(r, o0 + lane * 16 + s * 1024, h[s]);
#pragma unroll
  for (int s = 0; s < 3; ++s) st16(r, o1 + lane * 16 + s * 1024, h[4 + s]);
  st16(r, lane < nl ? o1 + lane * 16 + 3072 : NOWHERE, h[7]);
}
template <int S> __device__ __forceinline__ void sleep_until(unsigned long long t0, int ticks) {
  while ((int)(NOW() - t0) < ticks) __builtin_amdgcn_s_sleep(S);
}

#define STAMP(slot) do {} while (0)
#define TREC_DECL() do {} while (0)
#define TREC(i) do {} while (0)
#define TREC_DUMP(from) do {} while (0)
#define DIAG_DRAIN() do {} while (0)

constexpr int LIST_CAP = 128;
__device__ __forceinline__ unsigned bfi(unsigned m, unsigned a, unsigned b) { return (a & m) | (b & ~m); }
template <bool F32> __device__ __forceinline__ unsigned add_base(unsigned v, unsigned b) {
  return F32 ? v + b : __builtin_bit_cast(unsigned, (us2)(__builtin_bit_cast(us2, v) + __builtin_bit_cast(us2, b)));
}
template <bool F32> __device__ __forceinline__ void pack8(v4u (&h)[8], unsigned base) {
  constexpr unsigned TOP = F32 ? 0xF0000000u : 0xC000C000u;
  constexpr int B = F32 ? 4 : 2;
  const unsigned nb = F32 ? (0u - base) << 24 : ((32u - base) & 31u) * 0x08000800u;
#pragma unroll
  for (int d = 0; d < 4; ++d) {
    unsigned t[8], pl = 0;
#pragma unroll
    for (int s = 0; s < 8; ++s) {
      t[s] = add_base<F32>(__builtin_amdgcn_alignbit(h[s][d], h[s][d], 31), nb);
      pl |= (t[s] & TOP) >> (B * s);
    }
#pragma unroll
    for (int s = 0; s < 7; ++s) h[s][d] = bfi(TOP, t[7] << (B * (7 - s)), t[s]);
    h[7][d] = pl;
  }
}
template <bool F32> __device__ __forceinline__ void unpack8(v4u (&h)[8], unsigned base) {
  constexpr unsigned TOP = F32 ? 0xF0000000u : 0xC000C000u;
  constexpr int B = F32 ? 4 : 2;
  const unsigned pb = F32 ? base << 24 : base * 0x08000800u;
#pragma unroll
  for (int d = 0; d < 4; ++d) {
    const unsigned pl = h[7][d];
    unsigned t7 = (pl << (B * 7)) & TOP;
#pragma unroll
    for (int s = 0; s < 7; ++s) {
      t7 |= (h[s][d] & TOP) >> (B * (7 - s));
      const unsigned t = add_base<F32>(bfi(TOP, pl << (B * s), h[s][d]), pb);
      h[s][d] = __builtin_amdgcn_alignbit(t, t, 1);
    }
    t7 = add_base<F32>(t7, pb);
    h[7][d] = __builtin_amdgcn_alignbit(t7, t7, 1);
  }
}
template <int CTRL> __device__ __forceinline__ unsigned dpp0(unsigned v) {
  return (unsigned)__builtin_amdgcn_update_dpp(0, (int)v, CTRL, 0xf, 0xf, true);
}
__device__ __forceinline__ int top_exponent(float ss, float scale) {
  ss += __builtin_bit_cast(float, dpp0<0x111>(__builtin_bit_cast(unsigned, ss)));
  ss += __builtin_bit_cast(float, dpp0<0x112>(__builtin_bit_cast(unsigned, ss)));
  ss += __builtin_bit_cast(float, dpp0<0x114>(__builtin_bit_cast(unsigned, ss)));
  ss += __builtin_bit_cast(float, dpp0<0x118>(__builtin_bit_cast(unsigned, ss)));
  const unsigned sb = __builtin_bit_cast(unsigned, ss);
  const float tot = (__builtin_bit_cast(float, LANE(sb, 15)) + __builtin_bit_cast(float, LANE(sb, 31))) +
                    (__builtin_bit_cast(float, LANE(sb, 47)) + __builtin_bit_cast(float, LANE(sb, 63)));
  return (int)((UNI(__builtin_bit_cast(unsigned, __builtin_amdgcn_sqrtf(tot * scale))) >> 23) & 255);
}
__device__ __forceinline__ float sumsq(f32x4 z) { return z.x * z.x + z.y * z.y + z.z * z.z + z.w * z.w; }
__device__ __forceinline__ unsigned list_plane(const v4u& pl, LAS unsigned short* list, unsigned cap, int lane) {
  unsigned nz[4], cnt = 0;
#pragma unroll
  for (int d = 0; d < 4; ++d) {
    unsigned z = pl[d];
    z |= z >> 1; z |= z >> 2;
    nz[d] = z & 0x11111111u;
    cnt += __builtin_popcount(nz[d]);
  }
  unsigned inc = cnt;
  inc += dpp0<0x111>(inc); inc += dpp0<0x112>(inc); inc += dpp0<0x114>(inc); inc += dpp0<0x118>(inc);
  const unsigned r0 = LANE(inc, 15), r1 = r0 + LANE(inc, 31), r2 = r1 + LANE(inc, 47), total = r2 + LANE(inc, 63);
  if (total <= cap) {
    unsigned pos = inc - cnt + (lane < 16 ? 0 : lane < 32 ? r0 : lane < 48 ? r1 : r2);
#pragma unroll
    for (int d = 0; d < 4; ++d) {
      unsigned m = nz[d];
      while (m) {
        const unsigned b = __builtin_ctz(m);
        list[pos++] = (unsigned short)(((lane * 4 + d) << 7) | (b << 2) | ((pl[d] >> b) & 15));
        m &= m - 1;
      }
    }
  }
  return total;
}
__device__ __forceinline__ v4u unlist_plane(unsigned e2, unsigned skip, unsigned count, LAS unsigned* plane, int lane) {
  LDSWAIT();
#pragma unroll
  for (int k = 0; k < 2; ++k) {
    const unsigned e = (e2 >> (16 * k)) & 0xffffu;
    __hip_atomic_fetch_or(plane + ((e >> 7) & 255), 2u * lane + k - skip < count ? (e & 15) << ((e >> 2) & 28) : 0u,
                          __ATOMIC_RELAXED, __HIP_MEMORY_SCOPE_WORKGROUP);
  }
  LDSWAIT();
  return *(volatile LAS v4u*)(plane + lane * 4);
}

__device__ __forceinline__ void item_offsets(bool isw, unsigned stg, unsigned pan, int c, unsigned& o0, unsigned& o1,
                                             unsigned& fl) {
  o0 = stg + pan * PANEL + c * (isw ? 4096 : CHUNK);
  o1 = o0 + (isw ? PANEL : 4096);
  fl = OFF_PF + (isw ? (pan + (c & 1)) * NCH + (c >> 1) : pan * NCH + c) * 128;
}
__device__ __forceinline__ void link_item(bool cross, int i, bool& isw, int& u, int& c) {
  static_assert(NB1 == 8, "as many items as w");
  isw = cross && i < 128;
  u = i >= 128 ? (i >> 4) - (cross ? 8 : 0) : cross ? i & 3 : i & 7;
  c = i >= 128 ? i & 15 : cross ? i >> 2 : i >> 3;
}
__device__ __forceinline__ unsigned landing_panel(bool cross, bool isw, int u, int from, int to) {
  return !cross ? NI + 3 * u + ((from - to - 1) & 3) : isw ? NAP + 32 + from * 8 + 2 * u : 4 * NI + from * NX + u;
}
__device__ __forceinline__ unsigned panel_row(const Params& p, int a) {
  const int idx = p.idx & 3, q = a - NI, z = a - 4 * NI;
  const int role = a < NI || a >= PSENT ? idx : z < 0 ? (idx + 1 + q % 3) & 3 : 4 + z / NX;
  const int g = a < NI ? a : a >= PSENT ? NI + a - PSENT : z < 0 ? q / 3 : NI + z % NX;
  return ((p.ranks >> (4 * role)) & 7) * SH + g * 16;
}
__device__ __forceinline__ void load_inputs(const Params& p, bool isw, int g, int c, int lane, f32x4 (&v)[16]) {
  if (isw) {
    const GAS f32x2* src =
        (const GAS f32x2*)(p.w + (size_t)((c & 31) * 128 + 4 * (lane >> 4)) * SH + (g & 15) * 32 + 2 * (lane & 15));
#pragma unroll
    for (int q = 0; q < 16; ++q) {
      const f32x2 a = __builtin_nontemporal_load(src + ((q >> 1) * 16 + (q & 1) * 2) * (SH / 2));
      const f32x2 b = __builtin_nontemporal_load(src + ((q >> 1) * 16 + (q & 1) * 2 + 1) * (SH / 2));
      v[q] = f32x4{a.x, b.x, a.y, b.y};
    }
  } else {
    const GAS f32x4* src =
        (const GAS f32x4*)(p.x + (size_t)((g & 31) * 16 + (lane >> 2)) * KD + (c & 15) * 256 + 4 * (lane & 3));
#pragma unroll
    for (int q = 0; q < 16; ++q) v[q] = src[q * 4];
  }
}
__device__ __forceinline__ void to_fragments(bool isw, const f32x4 (&v)[16], LAS v4u* scr, int lane, v4u (&h)[8]) {
  if (isw) {
#pragma unroll
    for (int s = 0; s < 4; ++s) {
      h[s] = v4u{pk(v[4 * s].x, v[4 * s].y), pk(v[4 * s + 1].x, v[4 * s + 1].y), pk(v[4 * s + 2].x, v[4 * s + 2].y),
                 pk(v[4 * s + 3].x, v[4 * s + 3].y)};
      h[4 + s] = v4u{pk(v[4 * s].z, v[4 * s].w), pk(v[4 * s + 1].z, v[4 * s + 1].w), pk(v[4 * s + 2].z, v[4 * s + 2].w),
                     pk(v[4 * s + 3].z, v[4 * s + 3].w)};
    }
  } else {
#pragma unroll
    for (int s = 0; s < 8; ++s)
      scr[s * 64 + lane] = v4u{pk(v[2 * s].x, v[2 * s].y), pk(v[2 * s].z, v[2 * s].w), pk(v[2 * s + 1].x, v[2 * s + 1].y),
                               pk(v[2 * s + 1].z, v[2 * s + 1].w)};
    LDSWAIT();
#pragma unroll
    for (int s = 0; s < 8; ++s) h[s] = scr[s * 64 + (lane & 15) * 4 + (lane >> 4)];
    LDSWAIT();
  }
}
__device__ __forceinline__ unsigned half_offset(unsigned e, int q, int lane) {
  const unsigned o = e & 512 ? (q * 8 + (lane >> 3)) * (SH * 4) + (lane & 7) * 16
                             : (q * 4 + (lane >> 4)) * (SH * 4) + (lane & 15) * 16;
  return (e & 1024) && q >= 2 ? NOWHERE : (e >> 16 << 7) + o;
}
__device__ __forceinline__ unsigned queue_places(rsrc_t rl, bool on, int owner, int n) {
  return (unsigned)__builtin_amdgcn_raw_ptr_buffer_atomic_add_i32(n, rl, on ? OFF_YQ + owner * 128 : NOWHERE, 0, 16);
}

struct Ctl { unsigned prog, bad, full[2], fre[2], conv, sfull[2], sfre[2]; };
struct Job {
  int a0, mt;
  int b0, bs;
  int kind, j;
};
__device__ __forceinline__ Job make_job(int gg, int jg) {
  const int r = jg >> 2;
  Job q;
  q.b0 = (gg & 6) * 8 + (jg & 3) * 4; q.bs = q.b0 + (r & 1) * 2; q.j = r >> 1;
  if (gg & 1) { q.a0 = 4 * r; q.mt = r == 5 ? 3 : 4; q.kind = gg & 4 ? 0 : 3; }
  else if (gg & 4) { q.a0 = NI + 4 * r; q.mt = 4; q.kind = 2; }
  else { q.a0 = r < 4 ? NI + 6 * r : PSENT + 5 * (r - 4); q.mt = r < 4 ? 6 : 9 - r; q.kind = 1; }
  return q;
}
__device__ __forceinline__ int stream_len(const Job& q) {
  return q.kind == 0 ? LEN_A : q.kind == 1 ? LEN_A + 1 : q.kind == 2 ? LEN_B : 4 * NX / 3 + 1;
}
__device__ __forceinline__ int stream_panel(const Job& q, int i) {
  const int z = 3 * i + q.j;
  if (i == stream_len(q) - 1) return NI + 3 * (NI - 2 + (q.kind >> 1)) + q.j;
  if (q.kind < 2) return NI + 3 * NB1 + z;
  if (q.kind == 3) return 4 * NI + (z & 3) * NX + (z >> 2);
  return i < NX ? 4 * NI + (((q.bs >> 3) + 1 + q.j) & 3) * NX + i : NI + 3 * (NI - 3) + q.j;
}

__device__ __forceinline__ void need(volatile LAS unsigned* word, unsigned& have, unsigned n) {
  while (have < n) {
    have = UNI(*word);
    if (have < n) __builtin_amdgcn_s_sleep(1);
  }
  CBAR();
}
__device__ __forceinline__ void need(volatile LAS unsigned* word, unsigned n) { unsigned h = 0; need(word, h, n); }
__device__ __forceinline__ void arrive(volatile LAS unsigned* word, int lane) {
  LDSWAIT();
  if (lane == 0) __hip_atomic_fetch_add((LAS unsigned*)word, 1u, __ATOMIC_RELAXED, __HIP_MEMORY_SCOPE_WORKGROUP);
}
__device__ __forceinline__ f32x4 mfma(v4u b, v4u a, f32x4 c) {
  return __builtin_amdgcn_mfma_f32_16x16x32_f16(__builtin_bit_cast(h8, b), __builtin_bit_cast(h8, a), c, 0, 0, 0);
}

template <int MT>
__device__ __forceinline__ void gemm_tile(rsrc_t rs, LAS char* lds, const Job& job, unsigned& have, unsigned stg,
                                          int wave, int lane) {
  volatile LAS Ctl* ctl = (volatile LAS Ctl*)(lds + LDS_CTL);
  const unsigned ab = stg + job.a0 * PANEL + wave * 1024, bb = stg + (NAP + job.b0) * PANEL + wave * 1024;
  const int voff = lane * 16;
  f32x4 acc[MT][4];
#pragma unroll
  for (int m = 0; m < MT; ++m)
#pragma unroll
    for (int n = 0; n < 4; ++n) acc[m][n] = f32x4{0.f, 0.f, 0.f, 0.f};
  v4u ra[2][MT], rb[2][4];
#define LOADSTEP(d, i) do { \
    _Pragma("unroll") for (int m = 0; m < MT; ++m) ra[d][m] = ld16(rs, voff, ab + m * PANEL + (i) * 4096); \
    _Pragma("unroll") for (int n = 0; n < 4; ++n) rb[d][n] = ld16(rs, voff, bb + n * PANEL + (i) * 4096); \
    __builtin_amdgcn_sched_barrier(0); } while (0)
#define MMASTEP(d) do { \
    _Pragma("unroll") for (int m = 0; m < MT; ++m) \
    _Pragma("unroll") for (int n = 0; n < 4; ++n) acc[m][n] = mfma(rb[d][n], ra[d][m], acc[m][n]); \
    __builtin_amdgcn_sched_barrier(0); } while (0)
  need(&ctl->prog, have, 1);
  LOADSTEP(0, 0);
  LOADSTEP(1, 1);
#pragma unroll 1
  for (int c = 1; c < NCH; ++c) {
    MMASTEP(0);
    need(&ctl->prog, have, c + 1);
    LOADSTEP(0, 2 * c);
    MMASTEP(1);
    LOADSTEP(1, 2 * c + 1);
  }
  MMASTEP(0);
  MMASTEP(1);
#undef LOADSTEP
#undef MMASTEP
  asm volatile("" : "+v"(lane));
  need(&ctl->conv, NWAVE);
  constexpr int M1 = (MT + 1) / 2;
#pragma unroll
  for (int part = 0; part < 2; ++part) {
    const int m0 = part ? M1 : 0, mn = part ? MT - M1 : M1;
    LAS f32x4* red = (LAS f32x4*)(lds + part * LDS_RED) + wave * 1024;
#pragma unroll
    for (int m = 0; m < mn; ++m)
#pragma unroll
      for (int n = 0; n < 4; n += 2) {
        const int r = lane & 15, c4 = n * 4 + 2 * (lane >> 4);
        const f32x4 e = acc[m0 + m][n], o = acc[m0 + m][n + 1];
        red[(m * 16 + r) * 16 + (c4 ^ r)] = f32x4{e.x, o.x, e.y, o.y};
        red[(m * 16 + r) * 16 + ((c4 + 1) ^ r)] = f32x4{e.z, o.z, e.w, o.w};
      }
    arrive(&ctl->full[part], lane);
  }
}

__device__ __forceinline__ void finish_tile(const Params& p, rsrc_t rl, LAS char* lds, const Job& job, int part,
                                            unsigned call, int lane) {
  volatile LAS Ctl* ctl = (volatile LAS Ctl*)(lds + LDS_CTL);
  const LAS f32x4* red = (const LAS f32x4*)(lds + part * LDS_RED);
  const int idx = p.idx & 3, grp = job.b0 >> 3, m1 = (job.mt + 1) / 2;
  const int a0 = job.a0 + (part ? m1 : 0), mn = part ? job.mt - m1 : m1;
  const bool staged = grp >= 4;
  const rsrc_t ro = staged ? rl : mk(p.out[idx], OUT_BYTES);
  const unsigned ent = ((a0 + lane) * 4 + ((job.b0 >> 1) & 2)) % 512 |
                       (panel_row(p, a0 + min(lane, 2)) * SH + (staged ? idx : grp) * 128 + (job.b0 & 7) * 16) * 4 >> 7 << 16;
  need(&ctl->full[part], 4);
  STAMP(10);
  const bool bad = ctl->bad != 0;
  const unsigned ticket = queue_places(rl, staged && lane == 0, grp & 3, mn);
  for (int m = 0; m < mn; ++m) {
    const unsigned e = LANE(ent, m);
#pragma unroll
    for (int q = 0; q < 4; ++q) {
      const int r = q * 4 + (lane >> 4), li = (m * 16 + r) * 16 + ((lane & 15) ^ r);
      f32x4 s = (red[li] + red[1024 + li]) + (red[2048 + li] + red[3072 + li]);
      if (bad) s = __builtin_bit_cast(f32x4, v4u{QNAN, QNAN, QNAN, QNAN});
      st16(ro, staged ? OFF_YS + ((grp & 3) * 512 + (e & 511)) * 2048 + q * 1024 + lane * 16 : half_offset(e, q, lane),
           __builtin_bit_cast(v4u, s));
    }
  }
  LDSWAIT();
  ctl->fre[part] = 1;
  if (staged) {
    DRAIN();
    const unsigned pos = UNI(ticket) - (call - 1) * NHALF + lane;
    flag(lane < mn && pos < 256, rl, OFF_YL + ((grp & 3) * 256 + pos) * 128, call, ent);
  }
  STAMP(11);
}

__device__ __forceinline__ void gemm_stream(rsrc_t rs, LAS char* lds, const Job& job, unsigned& have, unsigned stg,
                                            int wave, int lane) {
  constexpr int D = 16;
  volatile LAS Ctl* ctl = (volatile LAS Ctl*)(lds + LDS_CTL);
  const int G = stream_len(job);
  const unsigned bb = stg + (NAP + job.bs) * PANEL + wave * 1024, voff = lane * 16;
  unsigned ab = stg + stream_panel(job, 0) * PANEL + wave * 1024, base = NCH;
  LAS v4u* bl = (LAS v4u*)(lds + wave * 32768) + lane;
  v4u br[32], ra[D];
#pragma unroll
  for (int s = 0; s < 32; ++s) {
    br[s] = ld16(rs, voff, bb + s * 4096);
    if (s >= 16) ra[s - 16] = ld16(rs, voff, bb + PANEL + (s - 16) * 4096);
    __builtin_amdgcn_sched_barrier(0);
  }
  need(&ctl->fre[0], 1);
  need(&ctl->fre[1], 1);
#pragma unroll
  for (int d = 0; d < D; ++d) {
    bl[d * 64] = ra[d];
    ra[d] = ld16(rs, voff, bb + PANEL + (16 + d) * 4096);
    __builtin_amdgcn_sched_barrier(0);
  }
#pragma unroll
  for (int d = 0; d < D; ++d) {
    bl[(16 + d) * 64] = ra[d];
    if ((d & 1) == 0) need(&ctl->prog, have, base + (d >> 1) + 1);
    ra[d] = ld16(rs, voff, ab + d * 4096);
    __builtin_amdgcn_sched_barrier(0);
  }
  bool primed = true;
#pragma unroll 1
  for (int i = 0; i < G; ++i) {
    if (!primed) {
#pragma unroll
      for (int d = 0; d < D; ++d) {
        if ((d & 1) == 0) need(&ctl->prog, have, base + (d >> 1) + 1);
        ra[d] = ld16(rs, voff, ab + d * 4096);
        __builtin_amdgcn_sched_barrier(0);
      }
    }
    const bool more = i + 1 < G;
    const unsigned an = stg + stream_panel(job, i + more) * PANEL + wave * 1024;
    unsigned vnext = NOWHERE;
    f32x4 e = {0.f, 0.f, 0.f, 0.f}, o = {0.f, 0.f, 0.f, 0.f};
    v4u q[4];
    q[0] = bl[0]; q[1] = bl[64]; q[2] = bl[128];
#pragma unroll
    for (int s = 0; s < 32; ++s) {
      if (s + 3 < 32) q[(s + 3) & 3] = bl[(s + 3) * 64];
      e = mfma(br[s], ra[s % D], e);
      o = mfma(q[s & 3], ra[s % D], o);
      __builtin_amdgcn_sched_barrier(0);
      const int n = s + D;
      if (n < 32 && (n & 1) == 0) need(&ctl->prog, have, base + (n >> 1) + 1);
      if (n == 32) {
        if (have < base + NCH + D / 2) have = UNI(ctl->prog);
        CBAR();
        primed = more && have >= base + NCH + D / 2;
        if (primed) vnext = voff;
      }
      ra[s % D] = ld16(rs, n < 32 ? voff : vnext, (n < 32 ? ab : an) + (n & 31) * 4096);
      __builtin_amdgcn_sched_barrier(0);
    }
    need(&ctl->sfre[i & 1], (unsigned)i >> 1);
    LAS f32x4* red = (LAS f32x4*)(lds + LDS_SIMG + (i & 1) * 8192) + wave * 128;
    const int row = lane & 15, c4 = 2 * (lane >> 4);
    red[row * 8 + (c4 ^ (row & 7))] = f32x4{e.x, o.x, e.y, o.y};
    red[row * 8 + ((c4 + 1) ^ (row & 7))] = f32x4{e.z, o.z, e.w, o.w};
    arrive(&ctl->sfull[i & 1], lane);
    ab = an;
    base += NCH;
  }
}

__device__ __forceinline__ void take_panel(LAS char* lds, int i, int lane, f32x4& s0, f32x4& s1) {
  volatile LAS Ctl* ctl = (volatile LAS Ctl*)(lds + LDS_CTL);
  const LAS f32x4* red = (const LAS f32x4*)(lds + LDS_SIMG + (i & 1) * 8192) + ((lane & 56) + ((lane ^ (lane >> 3)) & 7));
  need(&ctl->sfull[i & 1], 4u * ((i >> 1) + 1));
  s0 = (red[0] + red[128]) + (red[256] + red[384]);
  s1 = (red[64] + red[192]) + (red[320] + red[448]);
  if (ctl->bad) s0 = s1 = __builtin_bit_cast(f32x4, v4u{QNAN, QNAN, QNAN, QNAN});
  asm volatile("" : "+v"(s0), "+v"(s1));
  ctl->sfre[i & 1] = (i >> 1) + 1;
}
__device__ __forceinline__ void finish_stream(const Params& p, rsrc_t rl, LAS char* lds, const Job& job, int par,
                                              unsigned call, int lane) {
  const int idx = p.idx & 3, grp = job.bs >> 3, b = job.bs & 7, G = stream_len(job);
  const bool staged = grp >= 4;
  const int ns = !staged ? G : G - (job.kind ? P_LASTB : P_LASTA), nx = job.kind == 2 ? NX : 0;
  const unsigned col = ((staged ? idx : grp) * 128 + b * 16) * 4;
  const rsrc_t ro = staged ? rl : mk(p.out[idx], OUT_BYTES);
  TREC_DECL();
  int unit = 0;
  for (int i = 0; i < ns; ++unit) {
    const int np = staged ? min(2, (i < nx ? nx : ns) - i) : 1;
    if ((unit & 1) == par) {
      const int a = stream_panel(job, i), a2 = stream_panel(job, i + np - 1);
      const unsigned slot = np == 2 ? (b & 4 ? a2 : a) * 4 + (b & 2) : a * 4 + (b >> 1);
      const unsigned ent = slot % 512 | (np == 2 ? 512 : 1536) | (panel_row(p, a) * SH * 4 + col) >> 7 << 16;
      const unsigned vo = staged ? OFF_YS + ((grp & 3) * 512 + slot) * 2048 + lane * 16 : half_offset(ent, 0, lane);
      const unsigned qs = staged ? 1024 : 8 * SH * 4;
      unsigned ticket = 0;
      for (int e = 0; e < np; ++e) {
        f32x4 s0, s1;
        take_panel(lds, i + e, lane, s0, s1);
        TREC(32 + i + e);
        ticket = queue_places(rl, staged && e == np - 1 && lane == 0, grp & 3, 1);
        st16(ro, vo + 2 * e * qs, __builtin_bit_cast(v4u, s0));
        st16(ro, vo + (2 * e + 1) * qs, __builtin_bit_cast(v4u, s1));
      }
      if (staged) {
        DRAIN();
        const unsigned pos = UNI(ticket) - (call - 1) * NHALF;
        flag(lane == 0 && pos < 256, rl, OFF_YL + ((grp & 3) * 256 + pos) * 128, call, ent);
      }
    }
    i += np;
  }
  if (staged && (unit & 1) == par) {
    constexpr int NE = P_LASTA > P_LASTB ? P_LASTA : P_LASTB;
    const rsrc_t rp = mk(p.out[grp], OUT_BYTES), rf = mk(p.ws[grp], WS_USED);
    f32x4 s[2 * NE];
    unsigned vo[NE];
#pragma unroll
    for (int e = 0; e < NE; ++e) {
      const int i = G - NE + e;
      vo[e] = i >= ns ? (panel_row(p, stream_panel(job, i)) + (lane >> 3)) * SH * 4 + col + (lane & 7) * 16 : NOWHERE;
      asm volatile("" : "+v"(vo[e]));
    }
#pragma unroll
    for (int e = 0; e < NE; ++e) {
      s[2 * e] = s[2 * e + 1] = f32x4{0.f, 0.f, 0.f, 0.f};
      if (G - NE + e < ns) continue;
      take_panel(lds, G - NE + e, lane, s[2 * e], s[2 * e + 1]);
      TREC(32 + G - NE + e);
    }
#pragma unroll
    for (int q = 0; q < 2 * NE; ++q) st16(rp, vo[q >> 1] + (q & 1) * (8 * SH * 4), __builtin_bit_cast(v4u, s[q]));
    DRAIN();
    flag(lane == 0, rf, OFF_YF + (idx * 256 + NYF + (job.kind >> 1) * 12 + job.j * 4 + (b >> 1)) * 128, call);
  }
  DIAG_DRAIN();
  STAMP(1);
  TREC(55);
  TREC_DUMP(40);
}

__device__ __forceinline__ unsigned atomic_add(rsrc_t r, unsigned off, int v, int lane) {
  return (unsigned)__builtin_amdgcn_raw_ptr_buffer_atomic_add_i32(v, r, lane == 0 ? off : NOWHERE, 0, 16);
}
__device__ __forceinline__ void redeal(char* myws, volatile LAS unsigned* verdict, unsigned want, unsigned call, int lane) {
  const rsrc_t rl = mk(myws, WS_USED);
  const unsigned long long t0 = NOW();
  gu32* deal = (gu32*)(myws + OFF_DEAL);
  unsigned turn = 0;
  if (lane == 0) {
    __hip_atomic_fetch_max(deal, call << 8, __ATOMIC_RELAXED, __HIP_MEMORY_SCOPE_AGENT);
    __hip_atomic_fetch_max(deal + 32, call << 8, __ATOMIC_RELAXED, __HIP_MEMORY_SCOPE_AGENT);
    DRAIN();
    turn = __hip_atomic_fetch_add(deal, 1u, __ATOMIC_RELAXED, __HIP_MEMORY_SCOPE_AGENT) & 255;
  }
  turn = UNI(turn);
  bool late = false;
  unsigned n[4];
  while (true) {
    CBAR();
    unsigned sum = 0;
    for (int q = 0; q < 4; ++q) sum += n[q] = ld4(rl, OFF_ROLE + (q * 64 + lane) * 128) - (call - 1);
    for (int d = 32; d; d >>= 1) sum += __shfl_xor(sum, d);
    if (UNI(sum) == GRID || (late = NOW() - t0 > DEADLINE)) break;
  }
  unsigned role = want, left = 0;
  for (int q = 0; q < 4; ++q)
    for (unsigned long long m = __ballot(n[q] == 0); m; m &= m - 1, ++left)
      if (left == turn) role = q * 64 + __builtin_ctzll(m);
  *verdict = role + 1;
  atomic_add(rl, OFF_DEAL + 128, 1, lane);
  while (UNI(ld4(rl, OFF_DEAL + 128)) != (call << 8) + left && !(late = late || NOW() - t0 > DEADLINE)) CBAR();
  atomic_add(rl, OFF_ROLE + want * 128, -1, lane);
  atomic_add(rl, OFF_ROLE + role * 128, 1, lane);
  if (late) st_sys(myws + OFF_TMO, call);
}

template <bool TRIAL>
__device__ __forceinline__ void first_item(const Params& p, LAS char* lds, rsrc_t rl, rsrc_t rd, int role, unsigned before,
                                           unsigned last_call, unsigned long long tw, int lane) {
  const int grp = role & 7, jg = (role >> 3) - NC - NU, idx = p.idx & 3;
  const bool cross = grp >= 4;
  const int u = cross ? jg & 3 : jg & 7, c = cross ? jg >> 2 : jg >> 3;
  f32x4 v[16];
  v4u h[8];
  if (cross) {
    load_inputs(p, true, (grp - 4) * 4 + u, c, lane, v);
    if (TRIAL) __builtin_amdgcn_s_barrier();
    to_fragments(true, v, (LAS v4u*)(lds + LDS_SIMG), lane, h);
  } else {
    load_inputs(p, false, u, c, lane, v);
    if (TRIAL) __builtin_amdgcn_s_barrier();
    to_fragments(false, v, (LAS v4u*)(lds + LDS_SIMG), lane, h);
  }
  const unsigned call = UNI(last_call) + 1u, stg = OFF_STG + (call & 1u) * STG_BYTES;
  unsigned o0, o1, fl, kl;
  item_offsets(cross, stg, landing_panel(cross, cross, u, idx, grp & 3), c, o0, o1, fl);
  STAMP(7);
  if (TRIAL) {
    if (UNI(before) != call - 1) return;
    *(volatile LAS unsigned*)(lds + LDS_CTL + 60) = role + 1;
  }
  sleep_until<1>(tw, P_PACE0 + jg * P_PACE / 2);
  store_item(rd, o0, o1, h, lane);
  STAMP(6);
  item_offsets(cross, stg, NAP + (grp - 4) * 8 + 2 * u, c, o0, o1, kl);
  if (cross) store_item(rl, o0, o1, h, lane);
  DRAIN();
  flag(lane == 0, rd, fl, call);
  flag(lane == 0 && cross, rl, kl, call);
}

__global__ __launch_bounds__(NTHR, 2) void ag_gemm(const Params p) {
  extern __shared__ __attribute__((aligned(16))) char smem[];
  LAS char* lds = (LAS char*)smem;
  const int lane = threadIdx.x & 63;
  const int wave = UNI(threadIdx.x >> 6);
#define FAKE_XCC(x) (x)
  const int blk = blockIdx.x, want = (blk & ~7) | (FAKE_XCC(__builtin_amdgcn_s_getreg(20 | 31 << 11)) & 7);
  asm volatile("" :: "s"(p.x), "s"(p.w), "s"(p.myws), "s"(p.idx), "s"(p.ws[want & 7]));
  const int idx = p.idx & 3;
  char* myws = p.myws;
  const rsrc_t rl = mk(myws, WS_USED);
  STAMP(0);
  char* counter = myws + OFF_CNT + (blk * 8 + wave) * 4;
  const unsigned last_call = __hip_atomic_load((gu32*)counter, __ATOMIC_RELAXED, __HIP_MEMORY_SCOPE_SYSTEM);
  const unsigned long long tw = NOW();
  volatile LAS unsigned* verdict = (volatile LAS unsigned*)(lds + LDS_CTL + 60);
  auto opens = [&](int role) { return (unsigned)((role >> 3) - NC - NU) < (unsigned)NDRAW && (role & 7) != idx; };
  const bool claims = wave == ((unsigned)((blk >> 3) - NC - NU) < (unsigned)NDRAW ? 6 : 3);
  unsigned before = 0;
  if (claims) before = atomic_add(rl, OFF_ROLE + want * 128, 1, lane);
  if (wave == 6 && opens(want)) {
    first_item<true>(p, lds, rl, mk(p.ws[want & 7], WS_USED), want, before, last_call, tw, lane);
  } else {
    if (threadIdx.x < 16) ((volatile LAS unsigned*)(lds + LDS_CTL))[threadIdx.x] = 0;
    LDSWAIT();
    __builtin_amdgcn_s_barrier();
  }
  if (claims) {
    if (__builtin_expect(UNI(before) == UNI(last_call), 1)) *verdict = want + 1;
    else redeal(myws, verdict, want, UNI(last_call) + 1u, lane);
  }
  const bool direct = wave >= 4;
  LAS v4u* scr = (LAS v4u*)(lds + wave * 8192);
  unsigned role = want;
  bool work, isw;
  int leave, li, g, c, u;
  f32x4 v[16];
  __builtin_amdgcn_s_setprio(3);
  while (true) {
    const int grp = role & 7, jg = (role >> 3) - NC - NU;
    const int di = wave == 6 ? jg : wave == 5 ? NG + (jg + P_ROT1) % NG : 2 * NG + (jg + P_ROT2) % NG;
    const int it = wave * (NG * 8) + jg * 8 + grp;
    leave = P_PACE0 + di * P_PACE / 2;
    work = jg >= 0 && (direct ? (grp == idx || di >= NDRAW) && di < ND : it < NCV);
    if (work) {
      const int k = it / 5 + ND, m = direct ? grp - 4 : it % 5, xi = k < 128 ? k : it - (NCV - 512);
      const bool xc = direct ? grp >= 4 : k < 128 ? m < 4 : xi >= NIN;
      li = direct ? di : xi >= NIN ? xi - NIN + 128 : xi;
      link_item(xc, li, isw, u, c);
      g = isw ? m * 4 + u : xc ? NI + u : u;
      sleep_until<2>(tw, direct ? leave - P_LEAD : it * P_CSTEP >> 8);
      load_inputs(p, isw, g, c, lane, v);
    }
    unsigned got = 0;
    need(verdict, got, 1);
    got = (got - 1) & 255;
    if (__builtin_expect(got == role, 1)) break;
    role = got;
    if (wave == 6 && opens(role)) first_item<false>(p, lds, rl, mk(p.ws[role & 7], WS_USED), role, 0, last_call, tw, lane);
  }
  const int grp = role & 7, j = role >> 3, jg = j - NC - NU;
  const bool cross = grp >= 4;
  const rsrc_t rd = mk(p.ws[grp], WS_USED);
  const unsigned call = UNI(last_call) + 1u, stg = OFF_STG + (call & 1u) * STG_BYTES;

  if (jg >= 0) {
    volatile LAS Ctl* ctl = (volatile LAS Ctl*)(lds + LDS_CTL);
    if (work) {
      const bool keeper = cross || grp == idx;
      v4u h[8];
      to_fragments(isw, v, scr, lane, h);
      unsigned o0, o1, fl, hdr = 0, kl = NOWHERE;
      item_offsets(isw, stg, isw ? NAP + 2 * g : g < NI ? g : PSENT + u, c, o0, o1, fl);
      const rsrc_t rs = mk(direct ? p.ws[grp] : myws, WS_USED);
      STAMP(7);
      if (!direct || keeper) store_item(rl, o0, o1, h, lane);
      if (li < NPI && !(direct && grp == idx)) {
        const float ss = sumsq(v[0]) + sumsq(v[4]) + sumsq(v[8]) + sumsq(v[12]);
        const unsigned base = (unsigned)min(max(top_exponent(ss, 6.76f / 1024.f) - 127 + 15 - 7, 0), 24);
        pack8<false>(h, base);
        const unsigned total = list_plane(h[7], (LAS unsigned short*)scr, LIST_CAP, lane);
        const bool whole = total > (unsigned)LIST_CAP;
        if (!whole) {
          LDSWAIT();
          h[7] = *(volatile LAS v4u*)(scr + (lane & 15));
        }
        const unsigned nl = whole ? 64 : max((total + 31) >> 5, 1u) * 4;
        hdr = base | (whole ? 1u << 16 : total << 8) | nl << 24;
        if (direct) {
          if (keeper) kl = fl;
          item_offsets(isw, stg, landing_panel(cross, isw, u, idx, grp & 3), c, o0, o1, fl);
          fl += OFF_AF + (call & 1u) * PF_BYTES - OFF_PF;
          sleep_until<1>(tw, leave);
        } else {
          o0 = OFF_EXP + (isw ? 512 + g * 32 + c : g * 16 + c) * 8192;
          o1 = o0 + 4096;
        }
        store_item(rs, o0, o1, h, lane, nl);
      }
      STAMP(6);
      DRAIN();
      flag(lane == 0, rs, fl, call, hdr);
      flag(lane == 0, rl, kl, call);
    }
    STAMP(2);
    if (lane == 0) st_sys(counter, call);
    arrive(&ctl->conv, lane);
    const Job job = make_job(grp ^ 4, jg);
    if (wave < 4) {
      __builtin_amdgcn_s_setprio(0);
      unsigned have = 0;
      STAMP(8);
      switch (job.mt) {
        case 3: gemm_tile<3>(rl, lds, job, have, stg, wave, lane); break;
        case 4: gemm_tile<4>(rl, lds, job, have, stg, wave, lane); break;
        case 5: gemm_tile<5>(rl, lds, job, have, stg, wave, lane); break;
        default: gemm_tile<6>(rl, lds, job, have, stg, wave, lane); break;
      }
      STAMP(16);
      have = UNI(have);
      gemm_stream(rl, lds, job, have, stg, wave, lane);
      STAMP(1);
    } else if (wave > 4) {
      finish_tile(p, rl, lds, job, wave - 5, call, lane);
      finish_stream(p, rl, lds, job, wave - 5, call, lane);
    } else {
      const int np = job.mt + 4, l2 = np <= 8 ? 3 : 4, q = lane & ((1 << l2) - 1);
      const unsigned fo = OFF_PF + (q < job.mt ? job.a0 + q : NAP + job.b0 + q - job.mt) * (NCH * 128);
      const unsigned total = (stream_len(job) + 1) * NCH;
      unsigned pub = 0, win = 64, idle = 0;
      sleep_until<8>(tw, P_GSTART);
      TREC_DECL();
      auto give_up = [&]() {
        st_sys(myws + OFF_TMO, call);
        ctl->bad = 1;
        LDSWAIT();
        ctl->prog = 0xffffffffu;
      };
      while (pub <= total - P_FAST) {
        CBAR();
        const bool tile = pub < NCH;
        const unsigned most = tile ? 63 >> l2 : 32, ahead = tile ? lane >> l2 : lane, c = pub + ahead;
        win = min(win, most);
        const bool act = ahead < win && (tile ? q < np && c < NCH : c < total);
        const unsigned v = ld4(rl, !act ? (lane == 63 ? OFF_TMO : NOWHERE) : tile ? fo + c * 128
                                   : OFF_PF + (stream_panel(job, (c >> 4) - 1) * NCH + (c & 15)) * 128);
        if (LANE(v, 63) == call || NOW() - tw > DEADLINE) { give_up(); return; }
        const unsigned n = (unsigned)__builtin_ctzll(__ballot(act ? (int)(v - call) < 0 : ahead >= win)) >> (tile ? l2 : 0);
        idle = n ? 0 : idle + 1;
        win = n >= win ? 64 : tile ? n + 1 : idle > 2 ? 2 : 8;
        pub = min(pub + n, tile ? (unsigned)NCH : total);
        ctl->prog = pub;
        if (n && pub + NCH > total) TREC(32 + pub + NCH - total);
        if (tile) {
          if (pub == NCH) { win = 64; need(&ctl->full[0], 1); }
          else if (!n) __builtin_amdgcn_s_sleep(8);
        }
        while (pub < total && pub >= NCH + 40 + 4 * UNI(ctl->sfull[0] + ctl->sfull[1])) __builtin_amdgcn_s_sleep(4);
      }
      const unsigned fl = lane < NCH ? OFF_PF + (stream_panel(job, stream_len(job) - 1) * NCH + lane) * 128
                                     : lane == 63 ? OFF_TMO : NOWHERE;
      unsigned v[4];
#pragma unroll
      for (int d = 0; d < 4; ++d) {
        v[d] = ld4(rl, pub < total ? fl : NOWHERE);
        CBAR();
        __builtin_amdgcn_s_sleep(P_FNAP);
      }
      while (pub < total) {
        bool late = NOW() - tw > DEADLINE;
#pragma unroll
        for (int d = 0; d < 4; ++d) {
          const unsigned was = pub;
          pub = total - NCH + __builtin_ctzll(__ballot(lane >= NCH || (lane + total >= pub + NCH && (int)(v[d] - call) < 0)));
          late |= LANE(v[d], 63) == call;
          ctl->prog = pub;
          if (pub > was) TREC(32 + pub + NCH - total);
          v[d] = ld4(rl, lane + total >= pub + NCH ? fl : NOWHERE);
          CBAR();
          __builtin_amdgcn_s_sleep(P_FNAP);
        }
        if (late) { give_up(); return; }
      }
      TREC_DUMP(40);
    }
    return;
  }

  __builtin_amdgcn_s_setprio(0);
  const unsigned aoff = OFF_AF + (call & 1u) * PF_BYTES - OFF_PF;
  if (lane == 0) st_sys(counter, call);
  if (j < NC && grp != idx) {
    struct Ring { unsigned full[NL], done; };
    volatile LAS Ring* rg = (volatile LAS Ring*)(lds + LDS_CTL);
    constexpr int SLOT = 10240;
    volatile LAS v2u* meta = (volatile LAS v2u*)(lds + 2 * NL * SLOT);
    const int n = cross ? NXY + NYF : NIN;
    if (wave == 0) {
      __builtin_amdgcn_s_setprio(3);
      const unsigned long long peer_out = uniform64(p.out[grp]), peer_ws = uniform64(p.ws[grp]);
      unsigned pl[3] = {NOWHERE, NOWHERE, NOWHERE}, ph[3] = {0, 0, 0};
      const int cnt = (n - ND - j + NC - 1) / NC, ry = ((cross ? NXY : n) - ND - j + NC - 1) / NC;
      unsigned lane0 = lane == 0 ? ~0u : 0u, lane1 = lane == 1 ? ~0u : 0u;
      asm volatile("" : "+v"(lane0), "+v"(lane1));
      int r = 0, l = 0, k = 0;
      TREC_DECL();
      auto flush = [&]() {
        DRAIN();
        flag(lane < 3, rd, bfi(lane0, pl[0], bfi(lane1, pl[1], pl[2])), call, bfi(lane0, ph[0], bfi(lane1, ph[1], ph[2])));
        pl[0] = pl[1] = pl[2] = NOWHERE;
      };
      auto rounds = [&](int end, auto wide, auto yphase) {
        constexpr int W = wide() ? P_WIDE : P_WIN;
        static_assert(W <= (wide() ? 20 : 12) && !(wide() && yphase()), "window");
        for (; r < end; ++r) {
          LAS char* sl = lds + (2 * l + (k & 1)) * SLOT + lane * 16;
          unsigned got;
          v2u m;
          v4u a0, a1, v[8];
#define READ_SLOT() do { \
          const unsigned fu = rg->full[l]; \
          m = meta[2 * l + (k & 1)]; \
          a0 = *(volatile LAS v4u*)(sl + 8192); a1 = *(volatile LAS v4u*)(sl + 9216); \
          _Pragma("unroll") for (int q = 0; q < 8; ++q) v[q] = *(volatile LAS v4u*)(sl + q * 1024); \
          got = UNI(fu); } while (0)
          READ_SLOT();
          if (__builtin_expect(got <= (unsigned)k, 0)) {
            flush();
            while (UNI(rg->full[l]) <= (unsigned)k) __builtin_amdgcn_s_sleep(0);
            READ_SLOT();
          }
#undef READ_SLOT
          if (got == 0xffffffffu) { r = cnt; rg->done = 0xffffffffu; break; }
          const unsigned hd = UNI(m.y);
          const bool to_out = yphase() && ((hd >> 17) & 1);
          const rsrc_t rs = mk((char*)(to_out ? peer_out : peer_ws), to_out ? OUT_BYTES : WS_USED);
          const unsigned fo = wide() ? bfi(lane0, pl[0], NOWHERE) : bfi(lane0, pl[1], bfi(lane1, pl[0], NOWHERE));
          const v2u fv = {call, wide() ? ph[0] : bfi(lane0, ph[1], ph[0])};
#pragma unroll
          for (int q = 0; q < 8; ++q) asm volatile("" : "+v"(v[q]));
          rg->done = r + 1;
          asm volatile("s_waitcnt vmcnt(%0)" :: "n"(W) : "memory");
#pragma unroll
          for (int q = 0; q < 4; ++q) st16(rs, a0[q], v[q]);
          asm volatile("s_waitcnt vmcnt(%0)" :: "n"(W) : "memory");
          if (yphase() && (((hd | ph[2]) >> 18) & 1)) asm volatile("s_waitcnt vmcnt(6)" ::: "memory");
          __builtin_amdgcn_raw_buffer_store_b64(fv, rd, (int)fo, 0, AUX_SYS);
          CBAR();
#pragma unroll
          for (int q = 0; q < 4; ++q) st16(rs, a1[q], v[4 + q]);
          CBAR();
          if (!(r & 1)) TREC(r >> 1);
          if (r >= cnt - 6) TREC(48 + r - cnt + 6);
          pl[0] = wide() ? pl[1] : NOWHERE; ph[0] = ph[1]; pl[1] = pl[2]; ph[1] = ph[2];
          pl[2] = UNI(m.x); ph[2] = hd;
          if (++l == NL) { l = 0; ++k; }
        }
      };
      sleep_until<2>(tw, P_PACE0 + ND * P_PACE / 2 + P_HOLD);
      while (r < ry) {
        const int raw = cross ? 0 : max(r - (NPI - ND) / NC, 0);
        if (r < ry - P_NARROW && (int)(NOW() - tw) > P_LATE + ND * P_PACE / 2 + r * P_RATE / 10 + raw * 7)
          rounds(min(r + 2, ry), std::true_type(), std::false_type());
        else rounds(min(r + 2, ry), std::false_type(), std::false_type());
      }
      rounds(cnt, std::false_type(), std::true_type());
      if (!cross && pl[0] == NOWHERE && pl[1] != NOWHERE) {
        asm volatile("s_waitcnt vmcnt(8)" ::: "memory");
        flag(lane == 0, rd, pl[1], call, ph[1]);
        TREC(54);
        asm volatile("s_waitcnt vmcnt(1)" ::: "memory");
        flag(lane == 0, rd, pl[2], call, ph[2]);
      } else {
        flush();
      }
      STAMP(1);
      TREC_DUMP(8);
      return;
    }
    const int l = wave - 1, first = ND + l * NC + j, cnt = (n - first + WL - 1) / WL;
    const int i = min(first + lane * WL, n - 1);
    const bool isy = cross && i >= NXY, packed = !isy && i < NPI;
    unsigned s0, s1, sf, d0, d1, df, hdv = 64u << 24;
    {
      bool isw; int u, c;
      link_item(cross, i, isw, u, c);
      const int g = isw ? (grp - 4) * 4 + u : cross ? NI + u : u;
      item_offsets(isw, stg, isw ? NAP + 2 * g : cross ? PSENT + u : u, c, s0, s1, sf);
      item_offsets(isw, stg, landing_panel(cross, isw, u, idx, grp & 3), c, d0, d1, df);
      if (packed) {
        s0 = OFF_EXP + (isw ? 512 + g * 32 + c : g * 16 + c) * 8192; s1 = s0 + 4096;
        df += aoff;
      }
      if (isy) {
        sf = OFF_YL + ((grp & 3) * 128 + i - NXY) * 256;
        df = OFF_YF + (idx * 256 + i - NXY) * 128;
      }
      if (lane == 63) sf = OFF_TMO;
    }
    unsigned long long ready = 0;
    for (int k = 0; k < cnt; ++k) {
      while (!((ready >> k) & 1)) {
        CBAR();
        const bool ask = ((lane >= k && lane < min(cnt, k < 2 ? k + 3 : 63)) || lane == 63) && !((ready >> lane) & 1);
        const bool two = isy && lane < 63;
        const v2u f = ldflag(rl, ask ? sf : NOWHERE);
        const v2u g = ldflag(rl, ask && two ? sf + 128 : NOWHERE);
        const bool ok = f.x == call && (!two || g.x == call);
        if (ok && packed) hdv = f.y;
        if (ok && isy) { d0 = f.y; d1 = g.y; }
        ready |= __ballot(ok);
        if ((ready >> 63) || NOW() - tw > DEADLINE) {
          st_sys(myws + OFF_TMO, call);
          rg->full[l] = 0xffffffffu;
          return;
        }
      }
      CBAR();
      const int yi = cross ? first + k * WL - NXY : -1;
      unsigned hd = LANE(hdv, k), a0 = LANE(s0, k), a1 = LANE(s1, k), e0 = LANE(d0, k), e1 = LANE(d1, k);
      if (yi >= 0) {
        a0 = OFF_YS + ((grp & 3) * 512 + (e0 & 511)) * 2048;
        a1 = OFF_YS + ((grp & 3) * 512 + (e1 & 511)) * 2048;
        hd |= 1u << 17 | ((e0 | e1) & 1024) << 8;
      }
      v4u h[8];
#pragma unroll
      for (int q = 0; q < 4; ++q) h[q] = ld16(rl, lane * 16 + q * 1024, a0);
#pragma unroll
      for (int q = 0; q < 3; ++q) h[4 + q] = ld16(rl, lane * 16 + q * 1024, a1);
      h[7] = ld16(rl, lane < (hd >> 24) ? lane * 16 + 3072 : NOWHERE, a1);
      if (yi >= 0 && yi < NYF - P_NRAWY && !((hd >> 18) & 1)) {
        const float ss = sumsq(__builtin_bit_cast(f32x4, h[0])) + sumsq(__builtin_bit_cast(f32x4, h[2])) +
                         sumsq(__builtin_bit_cast(f32x4, h[4])) + sumsq(__builtin_bit_cast(f32x4, h[6]));
        const unsigned base = (unsigned)min(max(top_exponent(ss, 20.f / 1024.f) - 15, 0), 240);
        pack8<true>(h, base);
        LAS unsigned* scr = (LAS unsigned*)(lds + 2 * NL * SLOT + 128 + l * 256);
        const unsigned total = list_plane(h[7], (LAS unsigned short*)(scr + 2), LIST_CAP - 4, lane);
        if (total <= LIST_CAP - 4) {
          scr[lane & 1] = lane & 1 ? e1 : e0;
          LDSWAIT();
          h[7] = *(volatile LAS v4u*)(scr + 4 * (lane & 15));
          hd = base | total << 8 | ((total + 35) >> 5) * 4 << 24;
          e0 = OFF_YP + (idx * NAP + yi) * 8192; e1 = e0 + 4096;
        } else {
          unpack8<true>(h, base);
        }
      }
      v4u t0, t1;
#pragma unroll
      for (int q = 0; q < 4; ++q) {
        t0[q] = (hd >> 17) & 1 ? half_offset(e0, q, lane) : e0 + q * 1024 + lane * 16;
        t1[q] = (hd >> 17) & 1 ? half_offset(e1, q, lane) : e1 + q * 1024 + lane * 16;
      }
      if (lane >= (hd >> 24)) t1[3] = NOWHERE;
      if (k >= 2) need(&rg->done, (k - 2) * NL + l + 1);
      LAS v4u* sl = (LAS v4u*)(lds + (2 * l + (k & 1)) * SLOT);
      sl[512 + lane] = t0;
      sl[576 + lane] = t1;
      meta[2 * l + (k & 1)] = v2u{LANE(df, k), hd};
#pragma unroll
      for (int q = 0; q < 8; ++q) sl[q * 64 + lane] = h[q];
      LDSWAIT();
      rg->full[l] = k + 1;
    }
    STAMP(1);
    return;
  }

  char* marker = p.out[idx] + (size_t)panel_row(p, 0) * SH * 4;
  auto await = [&](unsigned off, unsigned& hdr, unsigned& spins, int nap) {
    for (spins = 1;; ++spins) {
      const v2u f = ldflag(rl, lane == 0 ? off : lane == 1 && !(spins & 15) ? OFF_TMO : NOWHERE);
      hdr = UNI(f.y);
      if ((int)(UNI(f.x) - call) >= 0) return true;
      if (LANE(f.x, 1) == call || NOW() - tw > DEADLINE) {
        st_sys(myws + OFF_TMO, call);
        st_sys(marker, QNAN);
        return false;
      }
      if (nap) __builtin_amdgcn_s_sleep(16);
      CBAR();
    }
  };
  const int sw = wave * (NC + NU) + j, sx = sw - 3 * P_SI;
  const int lk = grp != idx ? grp : sx < 0 ? (idx + 1 + sw % 3) & 3 : 4 + (sx & 3);
  const bool xl = lk >= 4;
  const int nu = xl ? NU : P_NUI, WU = xl ? WUX : WUI;
  const int me = grp != idx ? wave * nu + j - NC : nu * NWAVE + (xl ? sx >> 2 : sw / 3);
  if ((grp != idx && j - NC >= nu) || me >= WU) return;
  LAS unsigned* plane = (LAS unsigned*)(lds + wave * 1024);
  {
    const int n = xl ? NXY : NPI, first = NDRAW + me, cnt = (n - first + WU - 1) / WU;
    bool isw; int u, c;
    link_item(xl, min(first + lane * WU, n - 1), isw, u, c);
    unsigned o0, o1, fl;
    item_offsets(isw, stg, landing_panel(xl, isw, u, lk & 3, idx), c, o0, o1, fl);
    unsigned pend = NOWHERE, hdr = 0;
    bool have_next = false;
    unsigned long long tseen = tw;
    sleep_until<4>(tw, 200 + first * P_PACE / 2);
    TREC_DECL();
    for (int r = 0; r < cnt; ++r) {
      const unsigned fr = LANE(fl, r);
      *(LAS v4u*)(plane + lane * 4) = v4u{0, 0, 0, 0};
      if (!have_next) {
        DRAIN();
        flag(lane == 0, rl, pend, call);
        pend = NOWHERE;
        sleep_until<2>(tseen, WU * 25 / 2 - P_UGAP);
        unsigned spins;
        if (!await(fr + aoff, hdr, spins, 0)) return;
        tseen = NOW() - (spins == 1 ? 60 : 0);
      }
      CBAR();
      TREC(4 * r);
      const unsigned a0 = LANE(o0, r), a1 = LANE(o1, r);
      const bool whole = (hdr >> 16) & 1;
      const unsigned e2 = ld4(rl, lane * 4 + 3072, a1);
      CBAR();
      v4u h[8];
#pragma unroll
      for (int q = 0; q < 4; ++q) h[q] = ld16(rl, lane * 16 + q * 1024, a0);
#pragma unroll
      for (int q = 0; q < 3; ++q) h[4 + q] = ld16(rl, lane * 16 + q * 1024, a1);
      h[7] = ld16(rl, whole ? lane * 16 + 3072 : NOWHERE, a1);
      const v2u fn = ldflag(rl, lane == 0 ? LANE(fl, min(r + 1, cnt - 1)) + aoff : NOWHERE);
      CBAR();
      const v4u pl = unlist_plane(e2, 0, (hdr >> 8) & 255, plane, lane);
      DRAIN();
      flag(lane == 0, rl, pend, call);
      h[7] |= pl;
      unpack8<false>(h, hdr & 31);
      store_item(rl, a0, a1, h, lane);
      pend = fr;
      TREC(4 * r + 2);
      hdr = UNI(fn.y);
      have_next = (int)(UNI(fn.x) - call) >= 0;
    }
    DRAIN();
    flag(lane == 0, rl, pend, call);
    TREC_DUMP(8);
    STAMP(2);
  }
  const rsrc_t ro = mk(p.out[idx], OUT_BYTES);
  for (int yi = me; xl && yi < NYF + NYD; yi += WU) {
    *(LAS v4u*)(plane + lane * 4) = v4u{0, 0, 0, 0};
    unsigned hdr, spins;
    if (yi >= NYF) sleep_until<8>(tw, 5000);
    if (!await(OFF_YF + ((lk & 3) * 256 + yi) * 128, hdr, spins, yi < NYF - P_NRAWY - 16)) return;
    if (yi >= NYF || ((hdr >> 17) & 1)) continue;
    CBAR();
    const unsigned a0 = OFF_YP + ((lk & 3) * NAP + yi) * 8192;
    const unsigned e2 = ld4(rl, lane * 4 + 7168, a0);
    CBAR();
    v4u h[8];
#pragma unroll
    for (int q = 0; q < 7; ++q) h[q] = ld16(rl, lane * 16 + (q & 3) * 1024, a0 + (q >> 2) * 4096);
    CBAR();
    h[7] = unlist_plane(e2, 4, (hdr >> 8) & 255, plane, lane);
    unpack8<true>(h, hdr & 255);
#pragma unroll
    for (int q = 0; q < 8; ++q) st16(ro, half_offset(LANE(e2, q >> 2), q & 3, lane), h[q]);
  }
  DIAG_DRAIN();
  STAMP(1);
}

}

extern "C" void kernel_launch(int rank, int nranks, ncclComm_t comm, hipStream_t stream, void* const* d_in,
                              const int* in_sizes, int n_in, void* d_out, int out_size, const int* in_dtypes,
                              int out_dtype, void* const* d_out_all, void* const* d_ws_all, size_t ws_bytes) {
  using namespace agg;
  if (nranks != NR || n_in != 2 || ws_bytes < WS_USED || out_size != 4096 * SH || in_sizes[0] != SH * KD ||
      in_sizes[1] != KD * SH || (unsigned)rank >= (unsigned)NR) {
    fprintf(stderr, "ag_gemm: unexpected problem shape\n");
    return;
  }
  static const int kClass[2][4] = {{0, 2, 3, 6}, {1, 4, 5, 7}};
  Params p;
  memset(&p, 0, sizeof(p));
  int cls = 0;
  for (int c = 0; c < 2; ++c)
    for (int i = 0; i < 4; ++i) if (kClass[c][i] == rank) { cls = c; p.idx = i; }
  for (int r = 0; r < NR; ++r) {
    const int q = kClass[cls ^ (r >> 2)][r & 3];
    p.ranks |= (unsigned)q << (4 * r);
    p.out[r] = (char*)d_out_all[q];
    p.ws[r] = (char*)d_ws_all[q];
  }
  p.x = (const float*)d_in[0];
  p.w = (const float*)d_in[1];
  p.myws = (char*)d_ws_all[rank];
  HIP_CHECK(hipFuncSetAttribute((const void*)ag_gemm, hipFuncAttributeMaxDynamicSharedMemorySize, LDS_BYTES));
  ag_gemm<<<GRID, NTHR, LDS_BYTES, stream>>>(p);
  HIP_CHECK(hipGetLastError());
}
```

```cpp
#include <hip/hip_runtime.h>
#include <rccl/rccl.h>
#include <cstdio>
#include <cstring>
#include <type_traits>

#define HIP_CHECK(expr) do { hipError_t _e = (expr); if (_e != hipSuccess) { \
    fprintf(stderr, "HIP error %s at %s:%d\n", hipGetErrorString(_e), __FILE__, __LINE__); \
    return; } } while (0)

#define P_ND 66
#define P_NDRAW 16
#define P_PACE0 80
#define P_PACE 25
#define P_LEAD 330
#define P_HOLD 40
#define P_CSTEP 250
#define P_GSTART 950
#define P_FAST 16
#define P_FNAP 3
#define P_TDIR 4850
#define P_WIN 11
#define P_WIDE 16
#define P_LATE 105
#define P_RATE 503
#define P_NARROW 4
#define P_NUI 3
#define P_SI 8
#define P_NRAW 48
#define P_NRAWY 22
#define P_ROT1 15
#define P_ROT2 5
#define P_UGAP 140
#define P_LASTA 3
#define P_LASTB 1

namespace agg {

#define GAS __attribute__((address_space(1)))
#define LAS __attribute__((address_space(3)))
typedef unsigned v4u __attribute__((ext_vector_type(4)));
typedef unsigned v2u __attribute__((ext_vector_type(2)));
typedef unsigned short us2 __attribute__((ext_vector_type(2)));
typedef float f32x4 __attribute__((ext_vector_type(4)));
typedef float f32x2 __attribute__((ext_vector_type(2)));
typedef _Float16 h8 __attribute__((ext_vector_type(8)));
typedef GAS unsigned gu32;
typedef __amdgpu_buffer_rsrc_t rsrc_t;

constexpr int NR = 8;
constexpr int KD = 4096, SH = 512;
constexpr int PANEL = 131072;
constexpr int NCH = 16, CHUNK = 8192;
constexpr int NI = 23, NX = 32 - NI;
constexpr int NB1 = 8;
constexpr int NAP = 128, NBP = 64, PSENT = NAP + NBP, NPAN = PSENT + 16;
constexpr int GRID = 256, NWAVE = 7, NTHR = NWAVE * 64;
constexpr int NC = 4, NL = NWAVE - 1, WL = NC * NL;
constexpr int NU = 4;
constexpr int SX = ((NC + NU) * NWAVE - 3 * P_SI) / 4;
constexpr int WUI = P_NUI * NWAVE + P_SI, WUX = NU * NWAVE + SX;
constexpr int NG = 32 - NC - NU;
constexpr int ND = P_ND, NDRAW = P_NDRAW;
constexpr int NIN = NI * NCH, NXY = 128 + NX * NCH;
constexpr int NPI = NIN - P_NRAW;
constexpr int NCV = 5 * (128 - ND) + 384;
constexpr int LEN_A = NI - NB1 - 2, LEN_B = NX + 2;
constexpr int NHALF = 2 * (NI + 3 * NB1) + 12 * ((LEN_A - P_LASTA + 1) / 2) + 12 * ((NX + 1) / 2 + (3 - P_LASTB) / 2);
constexpr int NYF = NHALF / 2, NYD = 24;
static_assert(NG == 24 && NI == 23 && NHALF % 2 == 0 && NYF <= NAP, "the deal of tiles and streams");
static_assert(NG * 32 >= NCV && ND <= 3 * NG && NDRAW <= NG && NDRAW <= ND, "one item per converting wave");
static_assert(NIN <= 63 * WL && NXY + NYF <= 63 * WL && NIN <= 64 * WUI && SX >= 0, "one lane per item");
static_assert(NXY <= NPI && P_LASTA >= 1 && P_LASTA <= 4 && P_LASTB >= 1 && P_LASTB <= 2, "");

constexpr unsigned OFF_CNT = 0;
constexpr unsigned OFF_TMO = 16384;
constexpr unsigned OFF_DEAL = 18432;
constexpr unsigned OFF_YQ = 20480;
constexpr unsigned OFF_END = 21504;
constexpr unsigned OFF_BAR = 24576;
constexpr unsigned OFF_ROLE = 32768;
constexpr unsigned OFF_YF = 65536;
constexpr unsigned OFF_YL = 196608;
constexpr unsigned OFF_PF = 393216;
constexpr unsigned PF_BYTES = NPAN * NCH * 128;
constexpr unsigned OFF_AF = 1u << 20;
constexpr unsigned OFF_YS = 2u << 20;
constexpr unsigned OFF_YP = OFF_YS + (4u << 20);
constexpr unsigned OFF_EXP = OFF_YP + (4u << 20);
constexpr unsigned OFF_STG = OFF_EXP + (8u << 20);
constexpr unsigned STG_BYTES = (unsigned)NPAN * PANEL;
constexpr unsigned OFF_DBG = OFF_STG + 2 * STG_BYTES;
constexpr unsigned WS_USED = OFF_DBG + (1u << 20);
static_assert(OFF_YF + 1024 * 128 <= OFF_YL && OFF_YL + 1024 * 128 <= OFF_PF && OFF_PF + PF_BYTES <= OFF_AF &&
              OFF_AF + 2 * PF_BYTES <= OFF_YS, "layout");
constexpr unsigned OUT_BYTES = 4096u * SH * 4u;
constexpr unsigned NOWHERE = 0xf0000000u;
constexpr unsigned long long DEADLINE = 50000000ull;
constexpr int AUX_SYS = 17;
constexpr unsigned QNAN = 0x7fc00000u;

constexpr int LDS_RED = 65536;
constexpr int LDS_CTL = 2 * LDS_RED;
constexpr int LDS_SIMG = LDS_CTL + 64;
constexpr int LDS_BYTES = LDS_SIMG + 16384;

struct Params {
  const float* x;
  const float* w;
  char* myws;
  int idx;
  unsigned ranks;
  char* out[NR];
  char* ws[NR];
};

#define DRAIN() asm volatile("s_waitcnt vmcnt(0)" ::: "memory")
#define LDSWAIT() asm volatile("s_waitcnt lgkmcnt(0)" ::: "memory")
#define CBAR() asm volatile("" ::: "memory")
#define NOW() __builtin_amdgcn_s_memrealtime()
#define UNI(v) ((unsigned)__builtin_amdgcn_readfirstlane(v))
#define LANE(v, l) ((unsigned)__builtin_amdgcn_readlane(v, l))

__device__ __forceinline__ rsrc_t mk(const void* p, unsigned bytes) {
  return __builtin_amdgcn_make_buffer_rsrc((void*)p, 0, (int)bytes, 0x00020000);
}
__device__ __forceinline__ unsigned long long uniform64(const void* q) {
  const unsigned long long v = (unsigned long long)q;
  return (unsigned long long)UNI((unsigned)(v >> 32)) << 32 | UNI((unsigned)v);
}
__device__ __forceinline__ unsigned pk(float a, float b) {
  typedef _Float16 h2 __attribute__((ext_vector_type(2)));
  h2 v = {(_Float16)a, (_Float16)b};
  return __builtin_bit_cast(unsigned, v);
}
__device__ __forceinline__ void st_sys(char* p, unsigned v) {
  __hip_atomic_store((gu32*)p, v, __ATOMIC_RELAXED, __HIP_MEMORY_SCOPE_SYSTEM);
}
__device__ __forceinline__ void st16(rsrc_t r, unsigned voff, v4u v) {
  __builtin_amdgcn_raw_buffer_store_b128(v, r, (int)voff, 0, AUX_SYS);
}
__device__ __forceinline__ v4u ld16(rsrc_t r, unsigned voff, unsigned soff) {
  return __builtin_amdgcn_raw_buffer_load_b128(r, (int)voff, (int)soff, AUX_SYS);
}
__device__ __forceinline__ unsigned ld4(rsrc_t r, unsigned voff, unsigned soff = 0) {
  return __builtin_amdgcn_raw_buffer_load_b32(r, (int)voff, (int)soff, AUX_SYS);
}
__device__ __forceinline__ void flag(bool on, rsrc_t r, unsigned off, unsigned call, unsigned hdr = 0) {
  __builtin_amdgcn_raw_buffer_store_b64(v2u{call, hdr}, r, (int)(on ? off : NOWHERE), 0, AUX_SYS);
}
__device__ __forceinline__ v2u ldflag(rsrc_t r, unsigned off) {
  return __builtin_amdgcn_raw_buffer_load_b64(r, (int)off, 0, AUX_SYS);
}
__device__ __forceinline__ void store_item(rsrc_t r, unsigned o0, unsigned o1, const v4u (&h)[8], int lane, unsigned nl = 64) {
#pragma unroll
  for (int s = 0; s < 4; ++s) st16(r, o0 + lane * 16 + s * 1024, h[s]);
#pragma unroll
  for (int s = 0; s < 3; ++s) st16(r, o1 + lane * 16 + s * 1024, h[4 + s]);
  st16(r, lane < nl ? o1 + lane * 16 + 3072 : NOWHERE, h[7]);
}
template <int S> __device__ __forceinline__ void sleep_until(unsigned long long t0, int ticks) {
  while ((int)(NOW() - t0) < ticks) __builtin_amdgcn_s_sleep(S);
}

#define STAMP(slot) do {} while (0)
#define TREC_DECL() do {} while (0)
#define TREC(i) do {} while (0)
#define TREC_DUMP(from) do {} while (0)
#define DIAG_DRAIN() do {} while (0)

constexpr int LIST_CAP = 128;
__device__ __forceinline__ unsigned bfi(unsigned m, unsigned a, unsigned b) { return (a & m) | (b & ~m); }
template <bool F32> __device__ __forceinline__ unsigned add_base(unsigned v, unsigned b) {
  return F32 ? v + b : __builtin_bit_cast(unsigned, (us2)(__builtin_bit_cast(us2, v) + __builtin_bit_cast(us2, b)));
}
template <bool F32> __device__ __forceinline__ void pack8(v4u (&h)[8], unsigned base) {
  constexpr unsigned TOP = F32 ? 0xF0000000u : 0xC000C000u;
  constexpr int B = F32 ? 4 : 2;
  const unsigned nb = F32 ? (0u - base) << 24 : ((32u - base) & 31u) * 0x08000800u;
#pragma unroll
  for (int d = 0; d < 4; ++d) {
    unsigned t[8], pl = 0;
#pragma unroll
    for (int s = 0; s < 8; ++s) {
      t[s] = add_base<F32>(__builtin_amdgcn_alignbit(h[s][d], h[s][d], 31), nb);
      pl |= (t[s] & TOP) >> (B * s);
    }
#pragma unroll
    for (int s = 0; s < 7; ++s) h[s][d] = bfi(TOP, t[7] << (B * (7 - s)), t[s]);
    h[7][d] = pl;
  }
}
template <bool F32> __device__ __forceinline__ void unpack8(v4u (&h)[8], unsigned base) {
  constexpr unsigned TOP = F32 ? 0xF0000000u : 0xC000C000u;
  constexpr int B = F32 ? 4 : 2;
  const unsigned pb = F32 ? base << 24 : base * 0x08000800u;
#pragma unroll
  for (int d = 0; d < 4; ++d) {
    const unsigned pl = h[7][d];
    unsigned t7 = (pl << (B * 7)) & TOP;
#pragma unroll
    for (int s = 0; s < 7; ++s) {
      t7 |= (h[s][d] & TOP) >> (B * (7 - s));
      const unsigned t = add_base<F32>(bfi(TOP, pl << (B * s), h[s][d]), pb);
      h[s][d] = __builtin_amdgcn_alignbit(t, t, 1);
    }
    t7 = add_base<F32>(t7, pb);
    h[7][d] = __builtin_amdgcn_alignbit(t7, t7, 1);
  }
}
template <int CTRL> __device__ __forceinline__ unsigned dpp0(unsigned v) {
  return (unsigned)__builtin_amdgcn_update_dpp(0, (int)v, CTRL, 0xf, 0xf, true);
}
__device__ __forceinline__ int top_exponent(float ss, float scale) {
  ss += __builtin_bit_cast(float, dpp0<0x111>(__builtin_bit_cast(unsigned, ss)));
  ss += __builtin_bit_cast(float, dpp0<0x112>(__builtin_bit_cast(unsigned, ss)));
  ss += __builtin_bit_cast(float, dpp0<0x114>(__builtin_bit_cast(unsigned, ss)));
  ss += __builtin_bit_cast(float, dpp0<0x118>(__builtin_bit_cast(unsigned, ss)));
  const unsigned sb = __builtin_bit_cast(unsigned, ss);
  const float tot = (__builtin_bit_cast(float, LANE(sb, 15)) + __builtin_bit_cast(float, LANE(sb, 31))) +
                    (__builtin_bit_cast(float, LANE(sb, 47)) + __builtin_bit_cast(float, LANE(sb, 63)));
  return (int)((UNI(__builtin_bit_cast(unsigned, __builtin_amdgcn_sqrtf(tot * scale))) >> 23) & 255);
}
__device__ __forceinline__ float sumsq(f32x4 z) { return z.x * z.x + z.y * z.y + z.z * z.z + z.w * z.w; }
__device__ __forceinline__ unsigned list_plane(const v4u& pl, LAS unsigned short* list, unsigned cap, int lane) {
  unsigned nz[4], cnt = 0;
#pragma unroll
  for (int d = 0; d < 4; ++d) {
    unsigned z = pl[d];
    z |= z >> 1; z |= z >> 2;
    nz[d] = z & 0x11111111u;
    cnt += __builtin_popcount(nz[d]);
  }
  unsigned inc = cnt;
  inc += dpp0<0x111>(inc); inc += dpp0<0x112>(inc); inc += dpp0<0x114>(inc); inc += dpp0<0x118>(inc);
  const unsigned r0 = LANE(inc, 15), r1 = r0 + LANE(inc, 31), r2 = r1 + LANE(inc, 47), total = r2 + LANE(inc, 63);
  if (total <= cap) {
    unsigned pos = inc - cnt + (lane < 16 ? 0 : lane < 32 ? r0 : lane < 48 ? r1 : r2);
#pragma unroll
    for (int d = 0; d < 4; ++d) {
      unsigned m = nz[d];
      while (m) {
        const unsigned b = __builtin_ctz(m);
        list[pos++] = (unsigned short)(((lane * 4 + d) << 7) | (b << 2) | ((pl[d] >> b) & 15));
        m &= m - 1;
      }
    }
  }
  return total;
}
__device__ __forceinline__ v4u unlist_plane(unsigned e2, unsigned skip, unsigned count, LAS unsigned* plane, int lane) {
  LDSWAIT();
#pragma unroll
  for (int k = 0; k < 2; ++k) {
    const unsigned e = (e2 >> (16 * k)) & 0xffffu;
    __hip_atomic_fetch_or(plane + ((e >> 7) & 255), 2u * lane + k - skip < count ? (e & 15) << ((e >> 2) & 28) : 0u,
                          __ATOMIC_RELAXED, __HIP_MEMORY_SCOPE_WORKGROUP);
  }
  LDSWAIT();
  return *(volatile LAS v4u*)(plane + lane * 4);
}

__device__ __forceinline__ void item_offsets(bool isw, unsigned stg, unsigned pan, int c, unsigned& o0, unsigned& o1,
                                             unsigned& fl) {
  o0 = stg + pan * PANEL + c * (isw ? 4096 : CHUNK);
  o1 = o0 + (isw ? PANEL : 4096);
  fl = OFF_PF + (isw ? (pan + (c & 1)) * NCH + (c >> 1) : pan * NCH + c) * 128;
}
__device__ __forceinline__ void link_item(bool cross, int i, bool& isw, int& u, int& c) {
  static_assert(NB1 == 8, "as many items as w");
  isw = cross && i < 128;
  u = i >= 128 ? (i >> 4) - (cross ? 8 : 0) : cross ? i & 3 : i & 7;
  c = i >= 128 ? i & 15 : cross ? i >> 2 : i >> 3;
}
__device__ __forceinline__ unsigned landing_panel(bool cross, bool isw, int u, int from, int to) {
  return !cross ? NI + 3 * u + ((from - to - 1) & 3) : isw ? NAP + 32 + from * 8 + 2 * u : 4 * NI + from * NX + u;
}
__device__ __forceinline__ unsigned panel_row(const Params& p, int a) {
  const int idx = p.idx & 3, q = a - NI, z = a - 4 * NI;
  const int role = a < NI || a >= PSENT ? idx : z < 0 ? (idx + 1 + q % 3) & 3 : 4 + z / NX;
  const int g = a < NI ? a : a >= PSENT ? NI + a - PSENT : z < 0 ? q / 3 : NI + z % NX;
  return ((p.ranks >> (4 * role)) & 7) * SH + g * 16;
}
__device__ __forceinline__ void load_inputs(const Params& p, bool isw, int g, int c, int lane, f32x4 (&v)[16]) {
  if (isw) {
    const GAS f32x2* src =
        (const GAS f32x2*)(p.w + (size_t)((c & 31) * 128 + 4 * (lane >> 4)) * SH + (g & 15) * 32 + 2 * (lane & 15));
#pragma unroll
    for (int q = 0; q < 16; ++q) {
      const f32x2 a = __builtin_nontemporal_load(src + ((q >> 1) * 16 + (q & 1) * 2) * (SH / 2));
      const f32x2 b = __builtin_nontemporal_load(src + ((q >> 1) * 16 + (q & 1) * 2 + 1) * (SH / 2));
      v[q] = f32x4{a.x, b.x, a.y, b.y};
    }
  } else {
    const GAS f32x4* src =
        (const GAS f32x4*)(p.x + (size_t)((g & 31) * 16 + (lane >> 2)) * KD + (c & 15) * 256 + 4 * (lane & 3));
#pragma unroll
    for (int q = 0; q < 16; ++q) v[q] = src[q * 4];
  }
}
__device__ __forceinline__ void to_fragments(bool isw, const f32x4 (&v)[16], LAS v4u* scr, int lane, v4u (&h)[8]) {
  if (isw) {
#pragma unroll
    for (int s = 0; s < 4; ++s) {
      h[s] = v4u{pk(v[4 * s].x, v[4 * s].y), pk(v[4 * s + 1].x, v[4 * s + 1].y), pk(v[4 * s + 2].x, v[4 * s + 2].y),
                 pk(v[4 * s + 3].x, v[4 * s + 3].y)};
      h[4 + s] = v4u{pk(v[4 * s].z, v[4 * s].w), pk(v[4 * s + 1].z, v[4 * s + 1].w), pk(v[4 * s + 2].z, v[4 * s + 2].w),
                     pk(v[4 * s + 3].z, v[4 * s + 3].w)};
    }
  } else {
#pragma unroll
    for (int s = 0; s < 8; ++s)
      scr[s * 64 + lane] = v4u{pk(v[2 * s].x, v[2 * s].y), pk(v[2 * s].z, v[2 * s].w), pk(v[2 * s + 1].x, v[2 * s + 1].y),
                               pk(v[2 * s + 1].z, v[2 * s + 1].w)};
    LDSWAIT();
#pragma unroll
    for (int s = 0; s < 8; ++s) h[s] = scr[s * 64 + (lane & 15) * 4 + (lane >> 4)];
    LDSWAIT();
  }
}
__device__ __forceinline__ unsigned half_offset(unsigned e, int q, int lane) {
  const unsigned o = e & 512 ? (q * 8 + (lane >> 3)) * (SH * 4) + (lane & 7) * 16
                             : (q * 4 + (lane >> 4)) * (SH * 4) + (lane & 15) * 16;
  return (e & 1024) && q >= 2 ? NOWHERE : (e >> 16 << 7) + o;
}
__device__ __forceinline__ unsigned queue_places(rsrc_t rl, bool on, int owner, int n) {
  return (unsigned)__builtin_amdgcn_raw_ptr_buffer_atomic_add_i32(n, rl, on ? OFF_YQ + owner * 128 : NOWHERE, 0, 16);
}

struct Ctl { unsigned prog, bad, full[2], fre[2], conv, sfull[2], sfre[2], clear; };
struct Job {
  int a0, mt;
  int b0, bs;
  int kind, j;
};
__device__ __forceinline__ Job make_job(int gg, int jg) {
  const int r = jg >> 2;
  Job q;
  q.b0 = (gg & 6) * 8 + (jg & 3) * 4; q.bs = q.b0 + (r & 1) * 2; q.j = r >> 1;
  if (gg & 1) { q.a0 = 4 * r; q.mt = r == 5 ? 3 : 4; q.kind = gg & 4 ? 0 : 3; }
  else if (gg & 4) { q.a0 = NI + 4 * r; q.mt = 4; q.kind = 2; }
  else { q.a0 = r < 4 ? NI + 6 * r : PSENT + 5 * (r - 4); q.mt = r < 4 ? 6 : 9 - r; q.kind = 1; }
  return q;
}
__device__ __forceinline__ int stream_len(const Job& q) {
  return q.kind == 0 ? LEN_A : q.kind == 1 ? LEN_A + 1 : q.kind == 2 ? LEN_B : 4 * NX / 3 + 1;
}
__device__ __forceinline__ int stream_panel(const Job& q, int i) {
  const int z = 3 * i + q.j;
  if (i == stream_len(q) - 1) return NI + 3 * (NI - 2 + (q.kind >> 1)) + q.j;
  if (q.kind < 2) return NI + 3 * NB1 + z;
  if (q.kind == 3) return 4 * NI + (z & 3) * NX + (z >> 2);
  return i < NX ? 4 * NI + (((q.bs >> 3) + 1 + q.j) & 3) * NX + i : NI + 3 * (NI - 3) + q.j;
}

__device__ __forceinline__ void need(volatile LAS unsigned* word, unsigned& have, unsigned n) {
  while (have < n) {
    have = UNI(*word);
    if (have < n) __builtin_amdgcn_s_sleep(1);
  }
  CBAR();
}
__device__ __forceinline__ void need(volatile LAS unsigned* word, unsigned n) { unsigned h = 0; need(word, h, n); }
__device__ __forceinline__ void arrive(volatile LAS unsigned* word, int lane) {
  LDSWAIT();
  if (lane == 0) __hip_atomic_fetch_add((LAS unsigned*)word, 1u, __ATOMIC_RELAXED, __HIP_MEMORY_SCOPE_WORKGROUP);
}
__device__ __forceinline__ f32x4 mfma(v4u b, v4u a, f32x4 c) {
  return __builtin_amdgcn_mfma_f32_16x16x32_f16(__builtin_bit_cast(h8, b), __builtin_bit_cast(h8, a), c, 0, 0, 0);
}

template <int MT>
__device__ __forceinline__ void gemm_tile(rsrc_t rs, LAS char* lds, const Job& job, unsigned& have, unsigned stg,
                                          int wave, int lane) {
  volatile LAS Ctl* ctl = (volatile LAS Ctl*)(lds + LDS_CTL);
  const unsigned ab = stg + job.a0 * PANEL + wave * 1024, bb = stg + (NAP + job.b0) * PANEL + wave * 1024;
  const int voff = lane * 16;
  f32x4 acc[MT][4];
#pragma unroll
  for (int m = 0; m < MT; ++m)
#pragma unroll
    for (int n = 0; n < 4; ++n) acc[m][n] = f32x4{0.f, 0.f, 0.f, 0.f};
  v4u ra[2][MT], rb[2][4];
#define LOADSTEP(d, i) do { \
    _Pragma("unroll") for (int m = 0; m < MT; ++m) ra[d][m] = ld16(rs, voff, ab + m * PANEL + (i) * 4096); \
    _Pragma("unroll") for (int n = 0; n < 4; ++n) rb[d][n] = ld16(rs, voff, bb + n * PANEL + (i) * 4096); \
    __builtin_amdgcn_sched_barrier(0); } while (0)
#define MMASTEP(d) do { \
    _Pragma("unroll") for (int m = 0; m < MT; ++m) \
    _Pragma("unroll") for (int n = 0; n < 4; ++n) acc[m][n] = mfma(rb[d][n], ra[d][m], acc[m][n]); \
    __builtin_amdgcn_sched_barrier(0); } while (0)
  need(&ctl->prog, have, 1);
  LOADSTEP(0, 0);
  LOADSTEP(1, 1);
#pragma unroll 1
  for (int c = 1; c < NCH; ++c) {
    MMASTEP(0);
    need(&ctl->prog, have, c + 1);
    LOADSTEP(0, 2 * c);
    MMASTEP(1);
    LOADSTEP(1, 2 * c + 1);
  }
  MMASTEP(0);
  MMASTEP(1);
#undef LOADSTEP
#undef MMASTEP
  asm volatile("" : "+v"(lane));
  need(&ctl->conv, NWAVE);
  constexpr int M1 = (MT + 1) / 2;
#pragma unroll
  for (int part = 0; part < 2; ++part) {
    const int m0 = part ? M1 : 0, mn = part ? MT - M1 : M1;
    LAS f32x4* red = (LAS f32x4*)(lds + part * LDS_RED) + wave * 1024;
#pragma unroll
    for (int m = 0; m < mn; ++m)
#pragma unroll
      for (int n = 0; n < 4; n += 2) {
        const int r = lane & 15, c4 = n * 4 + 2 * (lane >> 4);
        const f32x4 e = acc[m0 + m][n], o = acc[m0 + m][n + 1];
        red[(m * 16 + r) * 16 + (c4 ^ r)] = f32x4{e.x, o.x, e.y, o.y};
        red[(m * 16 + r) * 16 + ((c4 + 1) ^ r)] = f32x4{e.z, o.z, e.w, o.w};
      }
    arrive(&ctl->full[part], lane);
  }
}

__device__ __forceinline__ void finish_tile(const Params& p, rsrc_t rl, LAS char* lds, const Job& job, int part,
                                            unsigned call, int lane) {
  volatile LAS Ctl* ctl = (volatile LAS Ctl*)(lds + LDS_CTL);
  const LAS f32x4* red = (const LAS f32x4*)(lds + part * LDS_RED);
  const int idx = p.idx & 3, grp = job.b0 >> 3, m1 = (job.mt + 1) / 2;
  const int a0 = job.a0 + (part ? m1 : 0), mn = part ? job.mt - m1 : m1;
  const bool staged = grp >= 4;
  const rsrc_t ro = staged ? rl : mk(p.out[idx], OUT_BYTES);
  const unsigned ent = ((a0 + lane) * 4 + ((job.b0 >> 1) & 2)) % 512 |
                       (panel_row(p, a0 + min(lane, 2)) * SH + (staged ? idx : grp) * 128 + (job.b0 & 7) * 16) * 4 >> 7 << 16;
  need(&ctl->full[part], 4);
  STAMP(10);
  const bool bad = ctl->bad != 0;
  const unsigned ticket = queue_places(rl, staged && lane == 0, grp & 3, mn);
  for (int m = 0; m < mn; ++m) {
    const unsigned e = LANE(ent, m);
#pragma unroll
    for (int q = 0; q < 4; ++q) {
      const int r = q * 4 + (lane >> 4), li = (m * 16 + r) * 16 + ((lane & 15) ^ r);
      f32x4 s = (red[li] + red[1024 + li]) + (red[2048 + li] + red[3072 + li]);
      if (bad) s = __builtin_bit_cast(f32x4, v4u{QNAN, QNAN, QNAN, QNAN});
      st16(ro, staged ? OFF_YS + ((grp & 3) * 512 + (e & 511)) * 2048 + q * 1024 + lane * 16 : half_offset(e, q, lane),
           __builtin_bit_cast(v4u, s));
    }
  }
  LDSWAIT();
  ctl->fre[part] = 1;
  if (staged) {
    DRAIN();
    const unsigned pos = UNI(ticket) - (call - 1) * NHALF + lane;
    flag(lane < mn && pos < 256, rl, OFF_YL + ((grp & 3) * 256 + pos) * 128, call, ent);
  }
  STAMP(11);
}

__device__ __forceinline__ void gemm_stream(rsrc_t rs, LAS char* lds, const Job& job, unsigned& have, unsigned stg,
                                            int wave, int lane) {
  constexpr int D = 16;
  volatile LAS Ctl* ctl = (volatile LAS Ctl*)(lds + LDS_CTL);
  const int G = stream_len(job);
  const unsigned bb = stg + (NAP + job.bs) * PANEL + wave * 1024, voff = lane * 16;
  unsigned ab = stg + stream_panel(job, 0) * PANEL + wave * 1024, base = NCH;
  LAS v4u* bl = (LAS v4u*)(lds + wave * 32768) + lane;
  v4u br[32], ra[D];
#pragma unroll
  for (int s = 0; s < 32; ++s) {
    br[s] = ld16(rs, voff, bb + s * 4096);
    if (s >= 16) ra[s - 16] = ld16(rs, voff, bb + PANEL + (s - 16) * 4096);
    __builtin_amdgcn_sched_barrier(0);
  }
  need(&ctl->fre[0], 1);
  need(&ctl->fre[1], 1);
#pragma unroll
  for (int d = 0; d < D; ++d) {
    bl[d * 64] = ra[d];
    ra[d] = ld16(rs, voff, bb + PANEL + (16 + d) * 4096);
    __builtin_amdgcn_sched_barrier(0);
  }
#pragma unroll
  for (int d = 0; d < D; ++d) {
    bl[(16 + d) * 64] = ra[d];
    if ((d & 1) == 0) need(&ctl->prog, have, base + (d >> 1) + 1);
    ra[d] = ld16(rs, voff, ab + d * 4096);
    __builtin_amdgcn_sched_barrier(0);
  }
  bool primed = true;
#pragma unroll 1
  for (int i = 0; i < G; ++i) {
    if (!primed) {
#pragma unroll
      for (int d = 0; d < D; ++d) {
        if ((d & 1) == 0) need(&ctl->prog, have, base + (d >> 1) + 1);
        ra[d] = ld16(rs, voff, ab + d * 4096);
        __builtin_amdgcn_sched_barrier(0);
      }
    }
    const bool more = i + 1 < G;
    const unsigned an = stg + stream_panel(job, i + more) * PANEL + wave * 1024;
    unsigned vnext = NOWHERE;
    f32x4 e = {0.f, 0.f, 0.f, 0.f}, o = {0.f, 0.f, 0.f, 0.f};
    v4u q[4];
    q[0] = bl[0]; q[1] = bl[64]; q[2] = bl[128];
#pragma unroll
    for (int s = 0; s < 32; ++s) {
      if (s + 3 < 32) q[(s + 3) & 3] = bl[(s + 3) * 64];
      e = mfma(br[s], ra[s % D], e);
      o = mfma(q[s & 3], ra[s % D], o);
      __builtin_amdgcn_sched_barrier(0);
      const int n = s + D;
      if (n < 32 && (n & 1) == 0) need(&ctl->prog, have, base + (n >> 1) + 1);
      if (n == 32) {
        if (have < base + NCH + D / 2) have = UNI(ctl->prog);
        CBAR();
        primed = more && have >= base + NCH + D / 2;
        if (primed) vnext = voff;
      }
      ra[s % D] = ld16(rs, n < 32 ? voff : vnext, (n < 32 ? ab : an) + (n & 31) * 4096);
      __builtin_amdgcn_sched_barrier(0);
    }
    need(&ctl->sfre[i & 1], (unsigned)i >> 1);
    LAS f32x4* red = (LAS f32x4*)(lds + LDS_SIMG + (i & 1) * 8192) + wave * 128;
    const int row = lane & 15, c4 = 2 * (lane >> 4);
    red[row * 8 + (c4 ^ (row & 7))] = f32x4{e.x, o.x, e.y, o.y};
    red[row * 8 + ((c4 + 1) ^ (row & 7))] = f32x4{e.z, o.z, e.w, o.w};
    arrive(&ctl->sfull[i & 1], lane);
    ab = an;
    base += NCH;
  }
}

__device__ __forceinline__ void take_panel(LAS char* lds, int i, int lane, f32x4& s0, f32x4& s1) {
  volatile LAS Ctl* ctl = (volatile LAS Ctl*)(lds + LDS_CTL);
  const LAS f32x4* red = (const LAS f32x4*)(lds + LDS_SIMG + (i & 1) * 8192) + ((lane & 56) + ((lane ^ (lane >> 3)) & 7));
  need(&ctl->sfull[i & 1], 4u * ((i >> 1) + 1));
  s0 = (red[0] + red[128]) + (red[256] + red[384]);
  s1 = (red[64] + red[192]) + (red[320] + red[448]);
  if (ctl->bad) s0 = s1 = __builtin_bit_cast(f32x4, v4u{QNAN, QNAN, QNAN, QNAN});
  asm volatile("" : "+v"(s0), "+v"(s1));
  ctl->sfre[i & 1] = (i >> 1) + 1;
}
__device__ __forceinline__ void finish_stream(const Params& p, rsrc_t rl, LAS char* lds, const Job& job, int par,
                                              int link, unsigned call, unsigned long long tw, int lane) {
  volatile LAS Ctl* ctl = (volatile LAS Ctl*)(lds + LDS_CTL);
  const int idx = p.idx & 3, grp = job.bs >> 3, b = job.bs & 7, G = stream_len(job);
  const bool staged = grp >= 4;
  const int ns = !staged ? G : G - (job.kind ? P_LASTB : P_LASTA), nx = job.kind == 2 ? NX : 0;
  const unsigned col = ((staged ? idx : grp) * 128 + b * 16) * 4;
  const rsrc_t ro = staged ? rl : mk(p.out[idx], OUT_BYTES);
  TREC_DECL();
  int unit = 0;
  for (int i = 0; i < ns; ++unit) {
    const int np = staged ? min(2, (i < nx ? nx : ns) - i) : 1;
    if ((unit & 1) == par) {
      const int a = stream_panel(job, i), a2 = stream_panel(job, i + np - 1);
      const unsigned slot = np == 2 ? (b & 4 ? a2 : a) * 4 + (b & 2) : a * 4 + (b >> 1);
      const unsigned ent = slot % 512 | (np == 2 ? 512 : 1536) | (panel_row(p, a) * SH * 4 + col) >> 7 << 16;
      const unsigned vo = staged ? OFF_YS + ((grp & 3) * 512 + slot) * 2048 + lane * 16 : half_offset(ent, 0, lane);
      const unsigned qs = staged ? 1024 : 8 * SH * 4;
      unsigned ticket = 0;
      for (int e = 0; e < np; ++e) {
        f32x4 s0, s1;
        take_panel(lds, i + e, lane, s0, s1);
        TREC(32 + i + e);
        ticket = queue_places(rl, staged && e == np - 1 && lane == 0, grp & 3, 1);
        st16(ro, vo + 2 * e * qs, __builtin_bit_cast(v4u, s0));
        st16(ro, vo + (2 * e + 1) * qs, __builtin_bit_cast(v4u, s1));
      }
      if (staged) {
        DRAIN();
        const unsigned pos = UNI(ticket) - (call - 1) * NHALF;
        flag(lane == 0 && pos < 256, rl, OFF_YL + ((grp & 3) * 256 + pos) * 128, call, ent);
      }
    }
    i += np;
  }
  if (staged && (unit & 1) != par) {
    sleep_until<8>(tw, P_TDIR);
    while (link != idx && NOW() - tw <= DEADLINE) {
      CBAR();
      const unsigned long long m = __ballot(ld4(rl, lane < NC ? OFF_END + (link * NC + lane) * 128 : lane == 63 ? OFF_TMO : NOWHERE) == call);
      if ((m & 15) == 15 || (m >> 63)) break;
      __builtin_amdgcn_s_sleep(4);
    }
    ctl->clear = 1;
  } else if (staged) {
    constexpr int NE = P_LASTA > P_LASTB ? P_LASTA : P_LASTB;
    const rsrc_t rp = mk(p.out[grp], OUT_BYTES), rf = mk(p.ws[grp], WS_USED);
    f32x4 s[2 * NE];
    unsigned vo[NE];
#pragma unroll
    for (int e = 0; e < NE; ++e) {
      const int i = G - NE + e;
      vo[e] = i >= ns ? (panel_row(p, stream_panel(job, i)) + (lane >> 3)) * SH * 4 + col + (lane & 7) * 16 : NOWHERE;
      asm volatile("" : "+v"(vo[e]));
    }
#pragma unroll
    for (int e = 0; e < NE; ++e) {
      s[2 * e] = s[2 * e + 1] = f32x4{0.f, 0.f, 0.f, 0.f};
      if (G - NE + e < ns) continue;
      take_panel(lds, G - NE + e, lane, s[2 * e], s[2 * e + 1]);
      TREC(32 + G - NE + e);
    }
    need(&ctl->clear, 1);
#pragma unroll
    for (int q = 0; q < 2 * NE; ++q) st16(rp, vo[q >> 1] + (q & 1) * (8 * SH * 4), __builtin_bit_cast(v4u, s[q]));
    DRAIN();
    flag(lane == 0, rf, OFF_YF + (idx * 256 + NYF + (job.kind >> 1) * 12 + job.j * 4 + (b >> 1)) * 128, call);
  }
  DIAG_DRAIN();
  STAMP(1);
  TREC(55);
  TREC_DUMP(40);
}

__device__ __forceinline__ unsigned atomic_add(rsrc_t r, unsigned off, int v, int lane) {
  return (unsigned)__builtin_amdgcn_raw_ptr_buffer_atomic_add_i32(v, r, lane == 0 ? off : NOWHERE, 0, 16);
}
__device__ __forceinline__ void redeal(char* myws, volatile LAS unsigned* verdict, unsigned want, unsigned call, int lane) {
  const rsrc_t rl = mk(myws, WS_USED);
  const unsigned long long t0 = NOW();
  gu32* deal = (gu32*)(myws + OFF_DEAL);
  unsigned turn = 0;
  if (lane == 0) {
    __hip_atomic_fetch_max(deal, call << 8, __ATOMIC_RELAXED, __HIP_MEMORY_SCOPE_AGENT);
    __hip_atomic_fetch_max(deal + 32, call << 8, __ATOMIC_RELAXED, __HIP_MEMORY_SCOPE_AGENT);
    DRAIN();
    turn = __hip_atomic_fetch_add(deal, 1u, __ATOMIC_RELAXED, __HIP_MEMORY_SCOPE_AGENT) & 255;
  }
  turn = UNI(turn);
  bool late = false;
  unsigned n[4];
  while (true) {
    CBAR();
    unsigned sum = 0;
    for (int q = 0; q < 4; ++q) sum += n[q] = ld4(rl, OFF_ROLE + (q * 64 + lane) * 128) - (call - 1);
    for (int d = 32; d; d >>= 1) sum += __shfl_xor(sum, d);
    if (UNI(sum) == GRID || (late = NOW() - t0 > DEADLINE)) break;
  }
  unsigned role = want, left = 0;
  for (int q = 0; q < 4; ++q)
    for (unsigned long long m = __ballot(n[q] == 0); m; m &= m - 1, ++left)
      if (left == turn) role = q * 64 + __builtin_ctzll(m);
  *verdict = role + 1;
  atomic_add(rl, OFF_DEAL + 128, 1, lane);
  while (UNI(ld4(rl, OFF_DEAL + 128)) != (call << 8) + left && !(late = late || NOW() - t0 > DEADLINE)) CBAR();
  atomic_add(rl, OFF_ROLE + want * 128, -1, lane);
  atomic_add(rl, OFF_ROLE + role * 128, 1, lane);
  if (late) st_sys(myws + OFF_TMO, call);
}

template <bool TRIAL>
__device__ __forceinline__ void first_item(const Params& p, LAS char* lds, rsrc_t rl, rsrc_t rd, int role, unsigned before,
                                           unsigned last_call, unsigned long long tw, int lane) {
  const int grp = role & 7, jg = (role >> 3) - NC - NU, idx = p.idx & 3;
  const bool cross = grp >= 4;
  const int u = cross ? jg & 3 : jg & 7, c = cross ? jg >> 2 : jg >> 3;
  f32x4 v[16];
  v4u h[8];
  if (cross) {
    load_inputs(p, true, (grp - 4) * 4 + u, c, lane, v);
    if (TRIAL) __builtin_amdgcn_s_barrier();
    to_fragments(true, v, (LAS v4u*)(lds + LDS_SIMG), lane, h);
  } else {
    load_inputs(p, false, u, c, lane, v);
    if (TRIAL) __builtin_amdgcn_s_barrier();
    to_fragments(false, v, (LAS v4u*)(lds + LDS_SIMG), lane, h);
  }
  const unsigned call = UNI(last_call) + 1u, stg = OFF_STG + (call & 1u) * STG_BYTES;
  unsigned o0, o1, fl, kl;
  item_offsets(cross, stg, landing_panel(cross, cross, u, idx, grp & 3), c, o0, o1, fl);
  STAMP(7);
  if (TRIAL) {
    if (UNI(before) != call - 1) return;
    *(volatile LAS unsigned*)(lds + LDS_CTL + 60) = role + 1;
  }
  sleep_until<1>(tw, P_PACE0 + jg * P_PACE / 2);
  store_item(rd, o0, o1, h, lane);
  STAMP(6);
  item_offsets(cross, stg, NAP + (grp - 4) * 8 + 2 * u, c, o0, o1, kl);
  if (cross) store_item(rl, o0, o1, h, lane);
  DRAIN();
  flag(lane == 0, rd, fl, call);
  flag(lane == 0 && cross, rl, kl, call);
}

__global__ __launch_bounds__(NTHR, 2) void ag_gemm(const Params p) {
  extern __shared__ __attribute__((aligned(16))) char smem[];
  LAS char* lds = (LAS char*)smem;
  const int lane = threadIdx.x & 63;
  const int wave = UNI(threadIdx.x >> 6);
#define FAKE_XCC(x) (x)
  const int blk = blockIdx.x, want = (blk & ~7) | (FAKE_XCC(__builtin_amdgcn_s_getreg(20 | 31 << 11)) & 7);
  asm volatile("" :: "s"(p.x), "s"(p.w), "s"(p.myws), "s"(p.idx), "s"(p.ws[want & 7]));
  const int idx = p.idx & 3;
  char* myws = p.myws;
  const rsrc_t rl = mk(myws, WS_USED);
  STAMP(0);
  char* counter = myws + OFF_CNT + (blk * 8 + wave) * 4;
  const unsigned last_call = __hip_atomic_load((gu32*)counter, __ATOMIC_RELAXED, __HIP_MEMORY_SCOPE_SYSTEM);
  const unsigned long long tw = NOW();
  volatile LAS unsigned* verdict = (volatile LAS unsigned*)(lds + LDS_CTL + 60);
  auto opens = [&](int role) { return (unsigned)((role >> 3) - NC - NU) < (unsigned)NDRAW && (role & 7) != idx; };
  const bool claims = wave == ((unsigned)((blk >> 3) - NC - NU) < (unsigned)NDRAW ? 6 : 3);
  unsigned before = 0;
  if (claims) before = atomic_add(rl, OFF_ROLE + want * 128, 1, lane);
  if (wave == 6 && opens(want)) {
    first_item<true>(p, lds, rl, mk(p.ws[want & 7], WS_USED), want, before, last_call, tw, lane);
  } else {
    if (threadIdx.x < 16) ((volatile LAS unsigned*)(lds + LDS_CTL))[threadIdx.x] = 0;
    LDSWAIT();
    __builtin_amdgcn_s_barrier();
  }
  if (claims) {
    if (__builtin_expect(UNI(before) == UNI(last_call), 1)) *verdict = want + 1;
    else redeal(myws, verdict, want, UNI(last_call) + 1u, lane);
  }
  const bool direct = wave >= 4;
  LAS v4u* scr = (LAS v4u*)(lds + wave * 8192);
  unsigned role = want;
  bool work, isw;
  int leave, li, g, c, u;
  f32x4 v[16];
  __builtin_amdgcn_s_setprio(3);
  while (true) {
    const int grp = role & 7, jg = (role >> 3) - NC - NU;
    const int di = wave == 6 ? jg : wave == 5 ? NG + (jg + P_ROT1) % NG : 2 * NG + (jg + P_ROT2) % NG;
    const int it = wave * (NG * 8) + jg * 8 + grp;
    leave = P_PACE0 + di * P_PACE / 2;
    work = jg >= 0 && (direct ? (grp == idx || di >= NDRAW) && di < ND : it < NCV);
    if (work) {
      const int k = it / 5 + ND, m = direct ? grp - 4 : it % 5, xi = k < 128 ? k : it - (NCV - 512);
      const bool xc = direct ? grp >= 4 : k < 128 ? m < 4 : xi >= NIN;
      li = direct ? di : xi >= NIN ? xi - NIN + 128 : xi;
      link_item(xc, li, isw, u, c);
      g = isw ? m * 4 + u : xc ? NI + u : u;
      sleep_until<2>(tw, direct ? leave - P_LEAD : it * P_CSTEP >> 8);
      load_inputs(p, isw, g, c, lane, v);
    }
    unsigned got = 0;
    need(verdict, got, 1);
    got = (got - 1) & 255;
    if (__builtin_expect(got == role, 1)) break;
    role = got;
    if (wave == 6 && opens(role)) first_item<false>(p, lds, rl, mk(p.ws[role & 7], WS_USED), role, 0, last_call, tw, lane);
  }
  const int grp = role & 7, j = role >> 3, jg = j - NC - NU;
  const bool cross = grp >= 4;
  const rsrc_t rd = mk(p.ws[grp], WS_USED);
  const unsigned call = UNI(last_call) + 1u, stg = OFF_STG + (call & 1u) * STG_BYTES;

  if (jg >= 0) {
    volatile LAS Ctl* ctl = (volatile LAS Ctl*)(lds + LDS_CTL);
    if (work) {
      const bool keeper = cross || grp == idx;
      v4u h[8];
      to_fragments(isw, v, scr, lane, h);
      unsigned o0, o1, fl, hdr = 0, kl = NOWHERE;
      item_offsets(isw, stg, isw ? NAP + 2 * g : g < NI ? g : PSENT + u, c, o0, o1, fl);
      const rsrc_t rs = mk(direct ? p.ws[grp] : myws, WS_USED);
      STAMP(7);
      if (!direct || keeper) store_item(rl, o0, o1, h, lane);
      if (li < NPI && !(direct && grp == idx)) {
        const float ss = sumsq(v[0]) + sumsq(v[4]) + sumsq(v[8]) + sumsq(v[12]);
        const unsigned base = (unsigned)min(max(top_exponent(ss, 6.76f / 1024.f) - 127 + 15 - 7, 0), 24);
        pack8<false>(h, base);
        const unsigned total = list_plane(h[7], (LAS unsigned short*)scr, LIST_CAP, lane);
        const bool whole = total > (unsigned)LIST_CAP;
        if (!whole) {
          LDSWAIT();
          h[7] = *(volatile LAS v4u*)(scr + (lane & 15));
        }
        const unsigned nl = whole ? 64 : max((total + 31) >> 5, 1u) * 4;
        hdr = base | (whole ? 1u << 16 : total << 8) | nl << 24;
        if (direct) {
          if (keeper) kl = fl;
          item_offsets(isw, stg, landing_panel(cross, isw, u, idx, grp & 3), c, o0, o1, fl);
          fl += OFF_AF + (call & 1u) * PF_BYTES - OFF_PF;
          sleep_until<1>(tw, leave);
        } else {
          o0 = OFF_EXP + (isw ? 512 + g * 32 + c : g * 16 + c) * 8192;
          o1 = o0 + 4096;
        }
        store_item(rs, o0, o1, h, lane, nl);
      }
      STAMP(6);
      DRAIN();
      flag(lane == 0, rs, fl, call, hdr);
      flag(lane == 0, rl, kl, call);
    }
    STAMP(2);
    if (lane == 0) st_sys(counter, call);
    arrive(&ctl->conv, lane);
    const Job job = make_job(grp ^ 4, jg);
    if (wave < 4) {
      __builtin_amdgcn_s_setprio(0);
      unsigned have = 0;
      STAMP(8);
      switch (job.mt) {
        case 3: gemm_tile<3>(rl, lds, job, have, stg, wave, lane); break;
        case 4: gemm_tile<4>(rl, lds, job, have, stg, wave, lane); break;
        case 5: gemm_tile<5>(rl, lds, job, have, stg, wave, lane); break;
        default: gemm_tile<6>(rl, lds, job, have, stg, wave, lane); break;
      }
      STAMP(16);
      have = UNI(have);
      gemm_stream(rl, lds, job, have, stg, wave, lane);
      STAMP(1);
    } else if (wave > 4) {
      finish_tile(p, rl, lds, job, wave - 5, call, lane);
      finish_stream(p, rl, lds, job, wave - 5, grp, call, tw, lane);
    } else {
      const int np = job.mt + 4, l2 = np <= 8 ? 3 : 4, q = lane & ((1 << l2) - 1);
      const unsigned fo = OFF_PF + (q < job.mt ? job.a0 + q : NAP + job.b0 + q - job.mt) * (NCH * 128);
      const unsigned total = (stream_len(job) + 1) * NCH;
      unsigned pub = 0, win = 64, idle = 0;
      sleep_until<8>(tw, P_GSTART);
      TREC_DECL();
      auto give_up = [&]() {
        st_sys(myws + OFF_TMO, call);
        ctl->bad = 1;
        LDSWAIT();
        ctl->prog = 0xffffffffu;
      };
      while (pub <= total - P_FAST) {
        CBAR();
        const bool tile = pub < NCH;
        const unsigned most = tile ? 63 >> l2 : 32, ahead = tile ? lane >> l2 : lane, c = pub + ahead;
        win = min(win, most);
        const bool act = ahead < win && (tile ? q < np && c < NCH : c < total);
        const unsigned v = ld4(rl, !act ? (lane == 63 ? OFF_TMO : NOWHERE) : tile ? fo + c * 128
                                   : OFF_PF + (stream_panel(job, (c >> 4) - 1) * NCH + (c & 15)) * 128);
        if (LANE(v, 63) == call || NOW() - tw > DEADLINE) { give_up(); return; }
        const unsigned n = (unsigned)__builtin_ctzll(__ballot(act ? (int)(v - call) < 0 : ahead >= win)) >> (tile ? l2 : 0);
        idle = n ? 0 : idle + 1;
        win = n >= win ? 64 : tile ? n + 1 : idle > 2 ? 2 : 8;
        pub = min(pub + n, tile ? (unsigned)NCH : total);
        ctl->prog = pub;
        if (n && pub + NCH > total) TREC(32 + pub + NCH - total);
        if (tile) {
          if (pub == NCH) { win = 64; need(&ctl->full[0], 1); }
          else if (!n) __builtin_amdgcn_s_sleep(8);
        }
        while (pub < total && pub >= NCH + 40 + 4 * UNI(ctl->sfull[0] + ctl->sfull[1])) __builtin_amdgcn_s_sleep(4);
      }
      const unsigned fl = lane < NCH ? OFF_PF + (stream_panel(job, stream_len(job) - 1) * NCH + lane) * 128
                                     : lane == 63 ? OFF_TMO : NOWHERE;
      unsigned v[4];
#pragma unroll
      for (int d = 0; d < 4; ++d) {
        v[d] = ld4(rl, pub < total ? fl : NOWHERE);
        CBAR();
        __builtin_amdgcn_s_sleep(P_FNAP);
      }
      while (pub < total) {
        bool late = NOW() - tw > DEADLINE;
#pragma unroll
        for (int d = 0; d < 4; ++d) {
          const unsigned was = pub;
          pub = total - NCH + __builtin_ctzll(__ballot(lane >= NCH || (lane + total >= pub + NCH && (int)(v[d] - call) < 0)));
          late |= LANE(v[d], 63) == call;
          ctl->prog = pub;
          if (pub > was) TREC(32 + pub + NCH - total);
          v[d] = ld4(rl, lane + total >= pub + NCH ? fl : NOWHERE);
          CBAR();
          __builtin_amdgcn_s_sleep(P_FNAP);
        }
        if (late) { give_up(); return; }
      }
      TREC_DUMP(40);
    }
    return;
  }

  __builtin_amdgcn_s_setprio(0);
  const unsigned aoff = OFF_AF + (call & 1u) * PF_BYTES - OFF_PF;
  if (lane == 0) st_sys(counter, call);
  if (j < NC && grp != idx) {
    struct Ring { unsigned full[NL], done; };
    volatile LAS Ring* rg = (volatile LAS Ring*)(lds + LDS_CTL);
    constexpr int SLOT = 10240;
    volatile LAS v2u* meta = (volatile LAS v2u*)(lds + 2 * NL * SLOT);
    const int n = cross ? NXY + NYF : NIN;
    if (wave == 0) {
      __builtin_amdgcn_s_setprio(3);
      const unsigned long long peer_out = uniform64(p.out[grp]), peer_ws = uniform64(p.ws[grp]);
      unsigned pl[3] = {NOWHERE, NOWHERE, NOWHERE}, ph[3] = {0, 0, 0};
      const int cnt = (n - ND - j + NC - 1) / NC, ry = ((cross ? NXY : n) - ND - j + NC - 1) / NC;
      unsigned lane0 = lane == 0 ? ~0u : 0u, lane1 = lane == 1 ? ~0u : 0u;
      asm volatile("" : "+v"(lane0), "+v"(lane1));
      int r = 0, l = 0, k = 0;
      TREC_DECL();
      auto flush = [&]() {
        DRAIN();
        flag(lane < 3, rd, bfi(lane0, pl[0], bfi(lane1, pl[1], pl[2])), call, bfi(lane0, ph[0], bfi(lane1, ph[1], ph[2])));
        pl[0] = pl[1] = pl[2] = NOWHERE;
      };
      auto rounds = [&](int end, auto wide, auto yphase) {
        constexpr int W = wide() ? P_WIDE : P_WIN;
        static_assert(W <= (wide() ? 20 : 12) && !(wide() && yphase()), "window");
        for (; r < end; ++r) {
          LAS char* sl = lds + (2 * l + (k & 1)) * SLOT + lane * 16;
          unsigned got;
          v2u m;
          v4u a0, a1, v[8];
#define READ_SLOT() do { \
          const unsigned fu = rg->full[l]; \
          m = meta[2 * l + (k & 1)]; \
          a0 = *(volatile LAS v4u*)(sl + 8192); a1 = *(volatile LAS v4u*)(sl + 9216); \
          _Pragma("unroll") for (int q = 0; q < 8; ++q) v[q] = *(volatile LAS v4u*)(sl + q * 1024); \
          got = UNI(fu); } while (0)
          READ_SLOT();
          if (__builtin_expect(got <= (unsigned)k, 0)) {
            flush();
            while (UNI(rg->full[l]) <= (unsigned)k) __builtin_amdgcn_s_sleep(0);
            READ_SLOT();
          }
#undef READ_SLOT
          if (got == 0xffffffffu) { r = cnt; rg->done = 0xffffffffu; break; }
          const unsigned hd = UNI(m.y);
          const bool to_out = yphase() && ((hd >> 17) & 1);
          const rsrc_t rs = mk((char*)(to_out ? peer_out : peer_ws), to_out ? OUT_BYTES : WS_USED);
          const unsigned fo = wide() ? bfi(lane0, pl[0], NOWHERE) : bfi(lane0, pl[1], bfi(lane1, pl[0], NOWHERE));
          const v2u fv = {call, wide() ? ph[0] : bfi(lane0, ph[1], ph[0])};
#pragma unroll
          for (int q = 0; q < 8; ++q) asm volatile("" : "+v"(v[q]));
          rg->done = r + 1;
          asm volatile("s_waitcnt vmcnt(%0)" :: "n"(W) : "memory");
#pragma unroll
          for (int q = 0; q < 4; ++q) st16(rs, a0[q], v[q]);
          asm volatile("s_waitcnt vmcnt(%0)" :: "n"(W) : "memory");
          if (yphase() && (((hd | ph[2]) >> 18) & 1)) asm volatile("s_waitcnt vmcnt(6)" ::: "memory");
          __builtin_amdgcn_raw_buffer_store_b64(fv, rd, (int)fo, 0, AUX_SYS);
          CBAR();
#pragma unroll
          for (int q = 0; q < 4; ++q) st16(rs, a1[q], v[4 + q]);
          CBAR();
          if (!(r & 1)) TREC(r >> 1);
          if (r >= cnt - 6) TREC(48 + r - cnt + 6);
          pl[0] = wide() ? pl[1] : NOWHERE; ph[0] = ph[1]; pl[1] = pl[2]; ph[1] = ph[2];
          pl[2] = UNI(m.x); ph[2] = hd;
          if (++l == NL) { l = 0; ++k; }
        }
      };
      sleep_until<2>(tw, P_PACE0 + ND * P_PACE / 2 + P_HOLD);
      while (r < ry) {
        const int raw = cross ? 0 : max(r - (NPI - ND) / NC, 0);
        if (r < ry - P_NARROW && (int)(NOW() - tw) > P_LATE + ND * P_PACE / 2 + r * P_RATE / 10 + raw * 7)
          rounds(min(r + 2, ry), std::true_type(), std::false_type());
        else rounds(min(r + 2, ry), std::false_type(), std::false_type());
      }
      rounds(cnt, std::false_type(), std::true_type());
      if (!cross && pl[0] == NOWHERE && pl[1] != NOWHERE) {
        asm volatile("s_waitcnt vmcnt(8)" ::: "memory");
        flag(lane == 0, rd, pl[1], call, ph[1]);
        TREC(54);
        asm volatile("s_waitcnt vmcnt(1)" ::: "memory");
        flag(lane == 0, rd, pl[2], call, ph[2]);
      } else {
        flush();
      }
      flag(!cross && lane == 0, rl, OFF_END + (grp * NC + j) * 128, call);
      STAMP(1);
      TREC_DUMP(8);
      return;
    }
    const int l = wave - 1, first = ND + l * NC + j, cnt = (n - first + WL - 1) / WL;
    const int i = min(first + lane * WL, n - 1);
    const bool isy = cross && i >= NXY, packed = !isy && i < NPI;
    unsigned s0, s1, sf, d0, d1, df, hdv = 64u << 24;
    {
      bool isw; int u, c;
      link_item(cross, i, isw, u, c);
      const int g = isw ? (grp - 4) * 4 + u : cross ? NI + u : u;
      item_offsets(isw, stg, isw ? NAP + 2 * g : cross ? PSENT + u : u, c, s0, s1, sf);
      item_offsets(isw, stg, landing_panel(cross, isw, u, idx, grp & 3), c, d0, d1, df);
      if (packed) {
        s0 = OFF_EXP + (isw ? 512 + g * 32 + c : g * 16 + c) * 8192; s1 = s0 + 4096;
        df += aoff;
      }
      if (isy) {
        sf = OFF_YL + ((grp & 3) * 128 + i - NXY) * 256;
        df = OFF_YF + (idx * 256 + i - NXY) * 128;
      }
      if (lane == 63) sf = OFF_TMO;
    }
    unsigned long long ready = 0;
    for (int k = 0; k < cnt; ++k) {
      while (!((ready >> k) & 1)) {
        CBAR();
        const bool ask = ((lane >= k && lane < min(cnt, k < 2 ? k + 3 : 63)) || lane == 63) && !((ready >> lane) & 1);
        const bool two = isy && lane < 63;
        const v2u f = ldflag(rl, ask ? sf : NOWHERE);
        const v2u g = ldflag(rl, ask && two ? sf + 128 : NOWHERE);
        const bool ok = f.x == call && (!two || g.x == call);
        if (ok && packed) hdv = f.y;
        if (ok && isy) { d0 = f.y; d1 = g.y; }
        ready |= __ballot(ok);
        if ((ready >> 63) || NOW() - tw > DEADLINE) {
          st_sys(myws + OFF_TMO, call);
          rg->full[l] = 0xffffffffu;
          return;
        }
      }
      CBAR();
      const int yi = cross ? first + k * WL - NXY : -1;
      unsigned hd = LANE(hdv, k), a0 = LANE(s0, k), a1 = LANE(s1, k), e0 = LANE(d0, k), e1 = LANE(d1, k);
      if (yi >= 0) {
        a0 = OFF_YS + ((grp & 3) * 512 + (e0 & 511)) * 2048;
        a1 = OFF_YS + ((grp & 3) * 512 + (e1 & 511)) * 2048;
        hd |= 1u << 17 | ((e0 | e1) & 1024) << 8;
      }
      v4u h[8];
#pragma unroll
      for (int q = 0; q < 4; ++q) h[q] = ld16(rl, lane * 16 + q * 1024, a0);
#pragma unroll
      for (int q = 0; q < 3; ++q) h[4 + q] = ld16(rl, lane * 16 + q * 1024, a1);
      h[7] = ld16(rl, lane < (hd >> 24) ? lane * 16 + 3072 : NOWHERE, a1);
      if (yi >= 0 && yi < NYF - P_NRAWY && !((hd >> 18) & 1)) {
        const float ss = sumsq(__builtin_bit_cast(f32x4, h[0])) + sumsq(__builtin_bit_cast(f32x4, h[2])) +
                         sumsq(__builtin_bit_cast(f32x4, h[4])) + sumsq(__builtin_bit_cast(f32x4, h[6]));
        const unsigned base = (unsigned)min(max(top_exponent(ss, 20.f / 1024.f) - 15, 0), 240);
        pack8<true>(h, base);
        LAS unsigned* scr = (LAS unsigned*)(lds + 2 * NL * SLOT + 128 + l * 256);
        const unsigned total = list_plane(h[7], (LAS unsigned short*)(scr + 2), LIST_CAP - 4, lane);
        if (total <= LIST_CAP - 4) {
          scr[lane & 1] = lane & 1 ? e1 : e0;
          LDSWAIT();
          h[7] = *(volatile LAS v4u*)(scr + 4 * (lane & 15));
          hd = base | total << 8 | ((total + 35) >> 5) * 4 << 24;
          e0 = OFF_YP + (idx * NAP + yi) * 8192; e1 = e0 + 4096;
        } else {
          unpack8<true>(h, base);
        }
      }
      v4u t0, t1;
#pragma unroll
      for (int q = 0; q < 4; ++q) {
        t0[q] = (hd >> 17) & 1 ? half_offset(e0, q, lane) : e0 + q * 1024 + lane * 16;
        t1[q] = (hd >> 17) & 1 ? half_offset(e1, q, lane) : e1 + q * 1024 + lane * 16;
      }
      if (lane >= (hd >> 24)) t1[3] = NOWHERE;
      if (k >= 2) need(&rg->done, (k - 2) * NL + l + 1);
      LAS v4u* sl = (LAS v4u*)(lds + (2 * l + (k & 1)) * SLOT);
      sl[512 + lane] = t0;
      sl[576 + lane] = t1;
      meta[2 * l + (k & 1)] = v2u{LANE(df, k), hd};
#pragma unroll
      for (int q = 0; q < 8; ++q) sl[q * 64 + lane] = h[q];
      LDSWAIT();
      rg->full[l] = k + 1;
    }
    STAMP(1);
    return;
  }

  char* marker = p.out[idx] + (size_t)panel_row(p, 0) * SH * 4;
  auto await = [&](unsigned off, unsigned& hdr, unsigned& spins, int nap) {
    for (spins = 1;; ++spins) {
      const v2u f = ldflag(rl, lane == 0 ? off : lane == 1 && !(spins & 15) ? OFF_TMO : NOWHERE);
      hdr = UNI(f.y);
      if ((int)(UNI(f.x) - call) >= 0) return true;
      if (LANE(f.x, 1) == call || NOW() - tw > DEADLINE) {
        st_sys(myws + OFF_TMO, call);
        st_sys(marker, QNAN);
        return false;
      }
      if (nap) __builtin_amdgcn_s_sleep(16);
      CBAR();
    }
  };
  const int sw = wave * (NC + NU) + j, sx = sw - 3 * P_SI;
  const int lk = grp != idx ? grp : sx < 0 ? (idx + 1 + sw % 3) & 3 : 4 + (sx & 3);
  const bool xl = lk >= 4;
  const int nu = xl ? NU : P_NUI, WU = xl ? WUX : WUI;
  const int me = grp != idx ? wave * nu + j - NC : nu * NWAVE + (xl ? sx >> 2 : sw / 3);
  if ((grp != idx && j - NC >= nu) || me >= WU) return;
  LAS unsigned* plane = (LAS unsigned*)(lds + wave * 1024);
  {
    const int n = xl ? NXY : NPI, first = NDRAW + me, cnt = (n - first + WU - 1) / WU;
    bool isw; int u, c;
    link_item(xl, min(first + lane * WU, n - 1), isw, u, c);
    unsigned o0, o1, fl;
    item_offsets(isw, stg, landing_panel(xl, isw, u, lk & 3, idx), c, o0, o1, fl);
    unsigned pend = NOWHERE, hdr = 0;
    bool have_next = false;
    unsigned long long tseen = tw;
    sleep_until<4>(tw, 200 + first * P_PACE / 2);
    TREC_DECL();
    for (int r = 0; r < cnt; ++r) {
      const unsigned fr = LANE(fl, r);
      *(LAS v4u*)(plane + lane * 4) = v4u{0, 0, 0, 0};
      if (!have_next) {
        DRAIN();
        flag(lane == 0, rl, pend, call);
        pend = NOWHERE;
        sleep_until<2>(tseen, WU * 25 / 2 - P_UGAP);
        unsigned spins;
        if (!await(fr + aoff, hdr, spins, 0)) return;
        tseen = NOW() - (spins == 1 ? 60 : 0);
      }
      CBAR();
      TREC(4 * r);
      const unsigned a0 = LANE(o0, r), a1 = LANE(o1, r);
      const bool whole = (hdr >> 16) & 1;
      const unsigned e2 = ld4(rl, lane * 4 + 3072, a1);
      CBAR();
      v4u h[8];
#pragma unroll
      for (int q = 0; q < 4; ++q) h[q] = ld16(rl, lane * 16 + q * 1024, a0);
#pragma unroll
      for (int q = 0; q < 3; ++q) h[4 + q] = ld16(rl, lane * 16 + q * 1024, a1);
      h[7] = ld16(rl, whole ? lane * 16 + 3072 : NOWHERE, a1);
      const v2u fn = ldflag(rl, lane == 0 ? LANE(fl, min(r + 1, cnt - 1)) + aoff : NOWHERE);
      CBAR();
      const v4u pl = unlist_plane(e2, 0, (hdr >> 8) & 255, plane, lane);
      DRAIN();
      flag(lane == 0, rl, pend, call);
      h[7] |= pl;
      unpack8<false>(h, hdr & 31);
      store_item(rl, a0, a1, h, lane);
      pend = fr;
      TREC(4 * r + 2);
      hdr = UNI(fn.y);
      have_next = (int)(UNI(fn.x) - call) >= 0;
    }
    DRAIN();
    flag(lane == 0, rl, pend, call);
    TREC_DUMP(8);
    STAMP(2);
  }
  const rsrc_t ro = mk(p.out[idx], OUT_BYTES);
  for (int yi = me; xl && yi < NYF + NYD; yi += WU) {
    *(LAS v4u*)(plane + lane * 4) = v4u{0, 0, 0, 0};
    unsigned hdr, spins;
    if (yi >= NYF) sleep_until<8>(tw, 5000);
    if (!await(OFF_YF + ((lk & 3) * 256 + yi) * 128, hdr, spins, yi < NYF - P_NRAWY - 16)) return;
    if (yi >= NYF || ((hdr >> 17) & 1)) continue;
    CBAR();
    const unsigned a0 = OFF_YP + ((lk & 3) * NAP + yi) * 8192;
    const unsigned e2 = ld4(rl, lane * 4 + 7168, a0);
    CBAR();
    v4u h[8];
#pragma unroll
    for (int q = 0; q < 7; ++q) h[q] = ld16(rl, lane * 16 + (q & 3) * 1024, a0 + (q >> 2) * 4096);
    CBAR();
    h[7] = unlist_plane(e2, 4, (hdr >> 8) & 255, plane, lane);
    unpack8<true>(h, hdr & 255);
#pragma unroll
    for (int q = 0; q < 8; ++q) st16(ro, half_offset(LANE(e2, q >> 2), q & 3, lane), h[q]);
  }
  DIAG_DRAIN();
  STAMP(1);
}

}

extern "C" void kernel_launch(int rank, int nranks, ncclComm_t comm, hipStream_t stream, void* const* d_in,
                              const int* in_sizes, int n_in, void* d_out, int out_size, const int* in_dtypes,
                              int out_dtype, void* const* d_out_all, void* const* d_ws_all, size_t ws_bytes) {
  using namespace agg;
  if (nranks != NR || n_in != 2 || ws_bytes < WS_USED || out_size != 4096 * SH || in_sizes[0] != SH * KD ||
      in_sizes[1] != KD * SH || (unsigned)rank >= (unsigned)NR) {
    fprintf(stderr, "ag_gemm: unexpected problem shape\n");
    return;
  }
  static const int kClass[2][4] = {{0, 2, 3, 6}, {1, 4, 5, 7}};
  Params p;
  memset(&p, 0, sizeof(p));
  int cls = 0;
  for (int c = 0; c < 2; ++c)
    for (int i = 0; i < 4; ++i) if (kClass[c][i] == rank) { cls = c; p.idx = i; }
  for (int r = 0; r < NR; ++r) {
    const int q = kClass[cls ^ (r >> 2)][r & 3];
    p.ranks |= (unsigned)q << (4 * r);
    p.out[r] = (char*)d_out_all[q];
    p.ws[r] = (char*)d_ws_all[q];
  }
  p.x = (const float*)d_in[0];
  p.w = (const float*)d_in[1];
  p.myws = (char*)d_ws_all[rank];
  HIP_CHECK(hipFuncSetAttribute((const void*)ag_gemm, hipFuncAttributeMaxDynamicSharedMemorySize, LDS_BYTES));
  ag_gemm<<<GRID, NTHR, LDS_BYTES, stream>>>(p);
  HIP_CHECK(hipGetLastError());
}
```
